```python
import numpy as np

kernel_name = "gpu_gemm_a2a_m4096_k4096_n16384_f32_q8"


ABSMAX_RTOL = 0.02

COMPUTE_DTYPE = "float32"
EPILOGUE = "quant_int8"
IN_QUANT = None
OUT_QUANT = "int8"
IN_DTYPES = {"x": "float32", "w_mat": "float32"}
WORLD_SIZE = 8

def setup_inputs():

    x = np.random.randn(4096, 4096).astype(np.float32)
    w_mat = (np.random.randn(4096, 16384) / np.sqrt(4096)).astype(np.float32)
    return {"x": x, "w_mat": w_mat}

def _quant_dequant(y, amax):
    scale = amax / 127.0
    q = np.clip(np.round(y / scale), -127.0, 127.0)
    return (q * scale).astype(np.float32)
def _epilogue(y):
    return _quant_dequant(y, np.abs(y).max())

def reference(x, w_mat):
    y = (x @ w_mat).astype(np.float32)
    return _epilogue(y)
```

```cpp
#include <hip/hip_runtime.h>
#include <rccl/rccl.h>
#include <cstdio>
#include <cstdint>

#define HIP_CHECK(expr) do { hipError_t _e = (expr); if (_e != hipSuccess) { \
    fprintf(stderr, "HIP error %s at %s:%d\n", hipGetErrorString(_e), __FILE__, __LINE__); \
    return; } } while (0)

#define LAS __attribute__((address_space(3)))
typedef float f32x4 __attribute__((ext_vector_type(4)));
typedef float f32x2 __attribute__((ext_vector_type(2)));
typedef __bf16 bf16x8 __attribute__((ext_vector_type(8)));
typedef __bf16 bf16x2 __attribute__((ext_vector_type(2)));
typedef unsigned u32x4 __attribute__((ext_vector_type(4)));
typedef int i32x4 __attribute__((ext_vector_type(4)));
typedef unsigned long long u64;

constexpr int NR = 8;
constexpr int KD = 4096, ND = 16384;
constexpr int ML = 512;
constexpr int NL = ND / NR;
constexpr int NWG = 256, NTHR = 256;
constexpr int NSTEP = KD / 64;
constexpr int LDS_BYTES = 4 * 32768;

constexpr size_t WS_CALL = 0;
constexpr size_t WS_TMO = 128;
constexpr size_t WS_AMAX = 4096;
constexpr size_t WS_DONE = WS_AMAX + 2 * NR * NWG * 8;
constexpr size_t WS_STAMP = WS_DONE + NR * 32 * 128;
constexpr size_t WS_XT = 1 << 20;
constexpr size_t WS_END = WS_XT + (size_t)ML * KD * 2;
static_assert(WS_STAMP + NWG * 16 * 8 <= WS_XT, "control block overlaps x");
constexpr u64 DEADLINE_TICKS = 50000000ull;

struct Params {
  const float* x;
  const float* w;
  char* out[NR];
  char* ws[NR];
  int rank;
  int phase;
};

#define W_POLICY ""
#define STAMP(i) do {} while (0)
#define STAMPC(i) do {} while (0)
#define STAMP_DECL do {} while (0)
#define STAMP_OUT() do {} while (0)

__device__ __forceinline__ unsigned pk_bf16(float lo, float hi) {
  f32x2 v = {lo, hi};
  return __builtin_bit_cast(unsigned, __builtin_convertvector(v, bf16x2));
}

#define WSH 16
#define NT_STEPS 8
#define KBLK(s) ((s) < NSTEP - NT_STEPS ? ((s) ^ c.alt) : (s))
#define NT_ROWS 0
#define ALT 0
__device__ __forceinline__ void w_load(f32x4& d, unsigned voff, i32x4 rsrc, unsigned soff, int j) {
  if (j >= 8 - NT_ROWS) asm volatile("buffer_load_dwordx4 %0, %1, %2, %3 offen nt" : "=v"(d) : "v"(voff), "s"(rsrc), "s"(soff) : "memory");
  else asm volatile("buffer_load_dwordx4 %0, %1, %2, %3 offen " W_POLICY : "=v"(d) : "v"(voff), "s"(rsrc), "s"(soff) : "memory");
}
__device__ __forceinline__ void set_m0(unsigned m0v) { asm volatile("s_mov_b32 m0, %0" :: "s"(m0v) : "memory"); }
template <int IMM>
__device__ __forceinline__ void x_load(unsigned voff, const char* sbase) {
  asm volatile("global_load_lds_dwordx4 %0, %1 offset:%2" :: "v"(voff), "s"(sbase), "i"(IMM) : "memory");
}
#define XPF 0
__device__ __forceinline__ void x_prefetch(f32x2& d, unsigned voff, const char* sbase) {
  asm volatile("global_load_dwordx2 %0, %1, %2" : "+v"(d) : "v"(voff), "s"(sbase) : "memory");
}
template <int N0>
__device__ __forceinline__ void wait_w(f32x4 (&r)[8]) {
  constexpr int N = XPF ? (N0 == 40 ? 43 : N0 == 32 ? 34 : N0 == 16 ? 17 : 0) : N0;
  asm volatile("s_waitcnt vmcnt(%8) lgkmcnt(0)\n\ts_barrier"
               : "+v"(r[0]), "+v"(r[1]), "+v"(r[2]), "+v"(r[3]), "+v"(r[4]), "+v"(r[5]), "+v"(r[6]), "+v"(r[7])
               : "n"(N) : "memory");
}
__device__ __forceinline__ void cvt_w(u32x4& a, const f32x4 (&r)[8], int t) {
  a = u32x4{pk_bf16(r[0][t], r[1][t]), pk_bf16(r[2][t], r[3][t]), pk_bf16(r[4][t], r[5][t]), pk_bf16(r[6][t], r[7][t])};
}
__device__ __forceinline__ void mfma(f32x4& c, u32x4 a, u32x4 b) {
  asm volatile("v_mfma_f32_16x16x32_bf16 %0, %1, %2, %0" : "+a"(c) : "v"(a), "v"(b));
}
#define SCHED() __builtin_amdgcn_sched_barrier(0)

struct Ctx {
  i32x4 wrsrc;
  unsigned wvoff, xvoff0, xvoff1, m0base, pfoff;
  const char* xbase;
  unsigned wk;
  int alt;
  LAS const char* lrd0;
  LAS const char* lrd1;
};

template <int SLOT>
__device__ __forceinline__ u32x4 x_frag(const Ctx& c, int mt) {
  return *(LAS const u32x4*)((SLOT < 2 ? c.lrd0 : c.lrd1) + (SLOT & 1) * 32768 + mt * 1024);
}
template <int SL, bool LX, bool LW, int VM, bool NEXT, bool NT = false>
__device__ __forceinline__ void kstep(const Ctx& c, int s, f32x4 (&acc)[16][4], f32x4 (&wr)[4][8], u32x4 (&a)[2][4], u32x4 (&xf)[8], f32x2& pf) {
  constexpr int CUR = SL & 1, NXT = CUR ^ 1;
  const char* xs = c.xbase + (size_t)KBLK(s + 3) * 32768;
  const unsigned m0x = c.m0base + ((SL + 3) & 3) * 32768;
  const unsigned wso = ((unsigned)KBLK(s + 4) * 64u + c.wk * 32u) << WSH;
#pragma unroll
  for (int mt = 0; mt < 12; ++mt) {
    mfma(acc[mt][0], a[CUR][0], xf[mt & 7]);
    xf[(mt + 4) & 7] = x_frag<SL>(c, mt + 4);
    if (LX && mt == 0) set_m0(m0x);
    if (LX && mt == 2) set_m0(m0x + 4096);
    SCHED();
    mfma(acc[mt][1], a[CUR][1], xf[mt & 7]);
    if (LX && mt == 0) x_load<0>(c.xvoff0, xs);
    if (LX && mt == 1) x_load<2048>(c.xvoff0, xs);
    if (LX && mt == 2) x_load<0>(c.xvoff1, xs);
    if (LX && mt == 3) x_load<2048>(c.xvoff1, xs);
    if (LW && mt >= 4) w_load(wr[SL][(mt - 4) & 7], c.wvoff, c.wrsrc, wso + ((unsigned)(mt - 4) << WSH), NT ? 8 : mt - 4);
    SCHED();
    mfma(acc[mt][2], a[CUR][2], xf[mt & 7]);
    if (XPF && LW && mt == 11) x_prefetch(pf, c.pfoff, c.xbase + (size_t)KBLK((s + 3 + XPF) & 63) * 32768);
    SCHED();
    mfma(acc[mt][3], a[CUR][3], xf[mt & 7]);
    if (LX && mt == 0) x_load<1024>(c.xvoff0, xs);
    if (LX && mt == 1) x_load<3072>(c.xvoff0, xs);
    if (LX && mt == 2) x_load<1024>(c.xvoff1, xs);
    if (LX && mt == 3) x_load<3072>(c.xvoff1, xs);
    SCHED();
  }
  if (NEXT) {
    wait_w<VM>(wr[(SL + 1) & 3]);
    SCHED();
  }
#pragma unroll
  for (int mt = 12; mt < 16; ++mt) {
    const f32x4 (&r)[8] = wr[(SL + 1) & 3];
    const int t = mt - 12;
    mfma(acc[mt][0], a[CUR][0], xf[mt & 7]);
    if (NEXT) xf[(mt + 4) & 7] = x_frag<(SL + 1) & 3>(c, t);
    SCHED();
    mfma(acc[mt][1], a[CUR][1], xf[mt & 7]);
    if (NEXT) { a[NXT][t][0] = pk_bf16(r[0][t], r[1][t]); a[NXT][t][1] = pk_bf16(r[2][t], r[3][t]); }
    SCHED();
    mfma(acc[mt][2], a[CUR][2], xf[mt & 7]);
    if (NEXT) { a[NXT][t][2] = pk_bf16(r[4][t], r[5][t]); a[NXT][t][3] = pk_bf16(r[6][t], r[7][t]); }
    SCHED();
    mfma(acc[mt][3], a[CUR][3], xf[mt & 7]);
    SCHED();
  }
}

__global__ void __launch_bounds__(NTHR, 1) fused_kernel(Params p) {
  extern __shared__ __attribute__((aligned(1024))) char smem[];
  const int tid = threadIdx.x, lane = tid & 63;
  const int wave = __builtin_amdgcn_readfirstlane(tid >> 6);
  char* ws = p.ws[p.rank];

  if (p.phase == 0) {
    for (int it = blockIdx.x * NTHR + tid; it < ML * KD / 8; it += gridDim.x * NTHR) {
      const int l = it & 63, mt = (it >> 6) & 15, ks = (it >> 10) & 127, mh = it >> 17;
      const float* src = p.x + (size_t)(mh * 256 + (mt ^ ((ks & 1) << 3)) * 16 + (l & 15)) * KD + ks * 32 + (l >> 4) * 8;
      const f32x4 v0 = *(const f32x4*)src, v1 = *(const f32x4*)(src + 4);
      *(u32x4*)(ws + WS_XT + (size_t)it * 16) =
          u32x4{pk_bf16(v0[0], v0[1]), pk_bf16(v0[2], v0[3]), pk_bf16(v1[0], v1[1]), pk_bf16(v1[2], v1[3])};
    }
    if (blockIdx.x == 0 && tid == 0) {
      unsigned* ctr = (unsigned*)(ws + WS_CALL);
      __hip_atomic_fetch_add(ctr, 1u, __ATOMIC_RELAXED, __HIP_MEMORY_SCOPE_AGENT);
    }
    return;
  }

  STAMP_DECL;
  STAMP(0);
  STAMPC(8);
  const int b = blockIdx.x, xcd = b & 7, idx = b >> 3;
  const int mh = idx & 1, ntile = xcd * 16 + (idx >> 1);
  const int wn = wave & 1, wk = wave >> 1;
  const int q = lane >> 4, i16 = lane & 15;

  Ctx c;
  {
    const u64 wb = (u64)(p.w + ntile * 128 + wn * 64);
    c.wrsrc = i32x4{(int)(unsigned)wb, (int)(unsigned)(wb >> 32), (int)0x10000000u, 0x00020000};
    c.wrsrc[0] = __builtin_amdgcn_readfirstlane(c.wrsrc[0]);
    c.wrsrc[1] = __builtin_amdgcn_readfirstlane(c.wrsrc[1]);
  }
  c.wvoff = ((unsigned)(q * 8) << WSH) + i16 * 16;
  c.xvoff0 = wave * 8192 + lane * 16;
  c.xvoff1 = c.xvoff0 + 4096;
  c.m0base = (unsigned)(size_t)(LAS char*)smem + wave * 8192;
  c.xbase = ws + WS_XT + (size_t)mh * (2u << 20);
  c.wk = wk;
  c.pfoff = ((idx >> 1) * 4 + wave) * 512 + lane * 8;
  c.alt = mh * ALT;
  c.lrd0 = (LAS const char*)smem + wk * 16384 + lane * 16;
  c.lrd1 = c.lrd0 + 65536;
  asm volatile("" : "+v"(c.lrd1));

  f32x4 acc[16][4];
  f32x4 wr[4][8];
  u32x4 a[2][4], xf[8];
  f32x2 pf = {0.f, 0.f};

  const unsigned call = *(const __attribute__((address_space(4))) unsigned*)(ws + WS_CALL);
#pragma unroll
  for (int s = 0; s < 4; ++s) {
    if (s > 0) {
      const char* xs = c.xbase + (size_t)((s - 1) ^ c.alt) * 32768;
      const unsigned m0x = c.m0base + (s - 1) * 32768;
      set_m0(m0x);
      asm volatile("s_nop 0");
      x_load<0>(c.xvoff0, xs); x_load<1024>(c.xvoff0, xs); x_load<2048>(c.xvoff0, xs); x_load<3072>(c.xvoff0, xs);
      set_m0(m0x + 4096);
      asm volatile("s_nop 0");
      x_load<0>(c.xvoff1, xs); x_load<1024>(c.xvoff1, xs); x_load<2048>(c.xvoff1, xs); x_load<3072>(c.xvoff1, xs);
    }
#pragma unroll
    for (int j = 0; j < 8; ++j) w_load(wr[s][j], c.wvoff, c.wrsrc, ((unsigned)((s ^ c.alt) * 64 + j) + c.wk * 32u) << WSH, j);
    if (XPF) x_prefetch(pf, c.pfoff, c.xbase + (size_t)((s - 1 + XPF) ^ c.alt) * 32768);
  }
#pragma unroll
  for (int m = 0; m < 16; ++m)
#pragma unroll
    for (int t = 0; t < 4; ++t) {
      acc[m][t] = f32x4{0.f, 0.f, 0.f, 0.f};
      asm volatile("" : "+a"(acc[m][t]));
    }
  __builtin_amdgcn_s_waitcnt(0xC07F);
  wait_w<40>(wr[0]);
  SCHED();
#pragma unroll
  for (int t = 0; t < 4; ++t) {
    xf[t] = x_frag<0>(c, t);
    cvt_w(a[0][t], wr[0], t);
  }
  asm volatile("s_nop 1" : "+v"(a[0][0]), "+v"(a[0][1]), "+v"(a[0][2]), "+v"(a[0][3]));
  SCHED();
  STAMP(1);

  for (int s = 0; s < NSTEP - 4 - NT_STEPS; s += 4) {
    kstep<0, true, true, 40, true>(c, s, acc, wr, a, xf, pf);
    kstep<1, true, true, 40, true>(c, s + 1, acc, wr, a, xf, pf);
    kstep<2, true, true, 40, true>(c, s + 2, acc, wr, a, xf, pf);
    kstep<3, true, true, 40, true>(c, s + 3, acc, wr, a, xf, pf);
  }
  for (int s = NSTEP - 4 - NT_STEPS; s < NSTEP - 4; s += 4) {
    kstep<0, true, true, 40, true, true>(c, s, acc, wr, a, xf, pf);
    kstep<1, true, true, 40, true, true>(c, s + 1, acc, wr, a, xf, pf);
    kstep<2, true, true, 40, true, true>(c, s + 2, acc, wr, a, xf, pf);
    kstep<3, true, true, 40, true, true>(c, s + 3, acc, wr, a, xf, pf);
  }
  kstep<0, true, false, 32, true>(c, NSTEP - 4, acc, wr, a, xf, pf);
  kstep<1, false, false, 16, true>(c, NSTEP - 3, acc, wr, a, xf, pf);
  kstep<2, false, false, 0, true>(c, NSTEP - 2, acc, wr, a, xf, pf);
  kstep<3, false, false, 0, false>(c, NSTEP - 1, acc, wr, a, xf, pf);
  asm volatile("s_nop 7\n\ts_nop 7" :: "v"(pf) : "memory");
#pragma unroll
  for (int m = 0; m < 16; ++m)
#pragma unroll
    for (int t = 0; t < 4; ++t) asm volatile("" : "+a"(acc[m][t]));
  SCHED();
  STAMP(2);
  STAMPC(9);

  __syncthreads();
  f32x4 y[8][4];
  {
    LAS char* mine = (LAS char*)smem + wave * 32768 + lane * 16;
    LAS const char* theirs = (LAS const char*)smem + (wave ^ 2) * 32768 + lane * 16;
#pragma unroll
    for (int m = 0; m < 8; ++m)
#pragma unroll
      for (int t = 0; t < 4; ++t) *(LAS f32x4*)(mine + (m * 4 + t) * 1024) = acc[8 + m][t];
    __syncthreads();
#pragma unroll
    for (int m = 0; m < 8; ++m)
#pragma unroll
      for (int t = 0; t < 4; ++t) y[m][t] = acc[m][t] + *(LAS const f32x4*)(theirs + (m * 4 + t) * 1024);
  }
  float am = 0.f;
#pragma unroll
  for (int m = 0; m < 8; ++m)
#pragma unroll
    for (int t = 0; t < 4; ++t)
#pragma unroll
      for (int r = 0; r < 4; ++r) am = fmaxf(am, fabsf(y[m][t][r]));
#pragma unroll
  for (int o = 32; o >= 1; o >>= 1) am = fmaxf(am, __shfl_xor(am, o));
  __syncthreads();
  LAS float* red = (LAS float*)smem;
  if (lane == 0) red[wave] = am;
  __syncthreads();
  STAMP(3);

  const size_t par_off = WS_AMAX + (size_t)(call & 1) * NR * NWG * 8;
  unsigned* tmo = (unsigned*)(ws + WS_TMO);
  if (tid < NR) {
    const float wam = fmaxf(fmaxf(red[0], red[1]), fmaxf(red[2], red[3]));
    u64* slot = (u64*)(p.ws[tid] + par_off) + p.rank * NWG + b;
    __hip_atomic_store(slot, ((u64)call << 32) | __builtin_bit_cast(unsigned, wam), __ATOMIC_RELAXED, __HIP_MEMORY_SCOPE_SYSTEM);
  }
  float gam = 0.f;
  bool ok = true;
  {
    const u64* slots = (const u64*)(ws + par_off) + tid;
    const u64 t0 = __builtin_amdgcn_s_memrealtime();
    for (;;) {
      bool all = true;
      float mx = 0.f;
      u64 v[NR];
#pragma unroll
      for (int r = 0; r < NR; ++r) v[r] = __hip_atomic_load(slots + r * NWG, __ATOMIC_RELAXED, __HIP_MEMORY_SCOPE_SYSTEM);
#pragma unroll
      for (int r = 0; r < NR; ++r) {
        all &= (unsigned)(v[r] >> 32) == call;
        mx = fmaxf(mx, __builtin_bit_cast(float, (unsigned)v[r]));
      }
      if (__all(all)) { gam = mx; break; }
      if (__hip_atomic_load(tmo, __ATOMIC_RELAXED, __HIP_MEMORY_SCOPE_AGENT) == call ||
          __builtin_amdgcn_s_memrealtime() - t0 > DEADLINE_TICKS) {
        __hip_atomic_store(tmo, call, __ATOMIC_RELAXED, __HIP_MEMORY_SCOPE_AGENT);
        ok = false;
        break;
      }
      __builtin_amdgcn_s_sleep(4);
    }
  }
#pragma unroll
  for (int o = 32; o >= 1; o >>= 1) gam = fmaxf(gam, __shfl_xor(gam, o));
  if (lane == 0) { red[8 + wave] = gam; red[16 + wave] = ok ? 0.f : 1.f; }
  __syncthreads();
  gam = fmaxf(fmaxf(red[8], red[9]), fmaxf(red[10], red[11]));
  ok = (red[16] + red[17] + red[18] + red[19]) == 0.f;
  STAMP(4);

  const float inv = gam > 0.f ? 127.0f / gam : 0.f;
  const unsigned dest = (unsigned)ntile >> 4;
  if (dest < (unsigned)NR) {
    char* ob = p.out[dest] + (size_t)(p.rank * ML + mh * 256 + wk * 128 + i16) * NL + (ntile & 15) * 128 + wn * 64 + q * 16;
#pragma unroll
    for (int m = 0; m < 8; ++m) {
      u32x4 o;
#pragma unroll
      for (int r = 0; r < 4; ++r) {
        unsigned e[4];
#pragma unroll
        for (int t = 0; t < 4; ++t) {
          const float s = __builtin_amdgcn_fmed3f(y[m][t][r] * inv, -127.0f, 127.0f);
          e[t] = __builtin_bit_cast(unsigned, s + 12582912.0f);
        }
        const unsigned lo = __builtin_amdgcn_perm(e[1], e[0], 0x0c0c0400u);
        const unsigned hi = __builtin_amdgcn_perm(e[3], e[2], 0x04000c0cu);
        o[r] = ok ? (lo | hi) : 0x80808080u;
      }
      asm volatile("global_store_dwordx4 %0, %1, off sc0 sc1\n\ts_nop 1" :: "v"(ob + (size_t)m * 16 * NL), "v"(o) : "memory");
    }
  }
  asm volatile("s_waitcnt vmcnt(0)" ::: "memory");
  __syncthreads();
  STAMP(5);
  if (tid == 0) {
    if (dest < (unsigned)NR) {
      __builtin_amdgcn_fence(__ATOMIC_RELEASE, "");
      asm volatile("s_waitcnt vmcnt(0)" ::: "memory");
      unsigned* flag = (unsigned*)(p.ws[dest] + WS_DONE + (size_t)(p.rank * 32 + idx) * 128);
      __hip_atomic_store(flag, call, __ATOMIC_RELAXED, __HIP_MEMORY_SCOPE_SYSTEM);
    }
    STAMP(6);
    const unsigned* flag = (const unsigned*)(ws + WS_DONE + (size_t)b * 128);
    const u64 t0 = __builtin_amdgcn_s_memrealtime();
    while (__hip_atomic_load(flag, __ATOMIC_RELAXED, __HIP_MEMORY_SCOPE_SYSTEM) < call) {
      if (__hip_atomic_load(tmo, __ATOMIC_RELAXED, __HIP_MEMORY_SCOPE_AGENT) == call ||
          __builtin_amdgcn_s_memrealtime() - t0 > DEADLINE_TICKS) {
        __hip_atomic_store(tmo, call, __ATOMIC_RELAXED, __HIP_MEMORY_SCOPE_AGENT);
        p.out[p.rank][(size_t)(p.rank * ML) * NL] = (char)-128;
        break;
      }
      __builtin_amdgcn_s_sleep(2);
    }
    STAMP(7);
    STAMP_OUT();
  }
}

extern "C" void kernel_launch(int rank, int nranks, ncclComm_t comm,
                              hipStream_t stream,
                              void* const* d_in, const int* in_sizes, int n_in,
                              void* d_out, int out_size,
                              const int* in_dtypes, int out_dtype,
                              void* const* d_out_all,
                              void* const* d_ws_all, size_t ws_bytes) {
  (void)comm; (void)in_dtypes; (void)out_dtype; (void)d_out;
  if (nranks != NR || n_in != 2 || in_sizes[0] != ML * KD || in_sizes[1] != KD * ND || out_size != NR * ML * NL ||
      ws_bytes < WS_END || (unsigned)rank >= (unsigned)NR) {
    fprintf(stderr, "unexpected problem shape\n");
    return;
  }
  HIP_CHECK(hipFuncSetAttribute((const void*)fused_kernel, hipFuncAttributeMaxDynamicSharedMemorySize, LDS_BYTES));
  Params p;
  p.x = (const float*)d_in[0];
  p.w = (const float*)d_in[1];
  for (int r = 0; r < NR; ++r) { p.out[r] = (char*)d_out_all[r]; p.ws[r] = (char*)d_ws_all[r]; }
  p.rank = rank;
  p.phase = 0;
  fused_kernel<<<NWG, NTHR, 0, stream>>>(p);
  p.phase = 1;
  fused_kernel<<<NWG, NTHR, LDS_BYTES, stream>>>(p);
  HIP_CHECK(hipGetLastError());
}
```

```cpp
#include <hip/hip_runtime.h>
#include <rccl/rccl.h>
#include <cstdio>
#include <cstdint>
#include <cstdlib>

#define HIP_CHECK(expr) do { hipError_t _e = (expr); if (_e != hipSuccess) { \
    fprintf(stderr, "HIP error %s at %s:%d\n", hipGetErrorString(_e), __FILE__, __LINE__); \
    return; } } while (0)

#define LAS __attribute__((address_space(3)))
typedef float f32x4 __attribute__((ext_vector_type(4)));
typedef float f32x2 __attribute__((ext_vector_type(2)));
typedef __bf16 bf16x2 __attribute__((ext_vector_type(2)));
typedef unsigned u32x4 __attribute__((ext_vector_type(4)));
typedef int i32x4 __attribute__((ext_vector_type(4)));
typedef unsigned long long u64;

constexpr int NR = 8;
constexpr int KD = 4096, ND = 16384;
constexpr int ML = 512;
constexpr int NL = ND / NR;
constexpr int NWG = 256, NTHR = 256;
constexpr int NBLK = KD / 64;
constexpr int NS = 64;
constexpr int KQ = NBLK / 4;
#define NE 9
static_assert(NE * 7 <= NS && NS - NE * 7 <= 16, "early tiles: NE per peer, the rest local");
#define MID 2
#define PACE_TABLE {600, 170, 3320, 100}, {600, 179, 3464, 100}, {600, 188, 3608, 100}, {600, 197, 3752, 100}
constexpr int LDS_X = 4 * 32768;
constexpr int LDS_BYTES = LDS_X + 16384;

constexpr size_t WS_CALL = 0;
constexpr size_t WS_TMO = 128;
constexpr size_t WS_START = 1024;
constexpr size_t WS_GMAX = 2048;
constexpr size_t WS_AMAX = 4096;
constexpr size_t WS_LFLAG = WS_AMAX + NWG * 8;
constexpr size_t WS_EFLAG = WS_LFLAG + NWG * 128;
constexpr size_t WS_PFLAG = WS_EFLAG + NWG * 4 * 128;
constexpr size_t WS_STAMP = WS_PFLAG + 2 * NWG * 4 * 128;
constexpr size_t WS_XT = 1 << 20;
constexpr size_t XT_BYTES = (size_t)ML * KD * 2;
constexpr size_t WS_PART = WS_XT + XT_BYTES;
constexpr size_t WS_STAGE = WS_PART + (size_t)NWG * 4 * 24576;
constexpr size_t WS_END = WS_STAGE + (size_t)NWG * 4 * 8192;
static_assert(WS_STAMP + 65536 + NWG * 512 <= WS_XT, "control block overlaps x");
constexpr u64 DEADLINE_TICKS = 50000000ull;

struct Params {
  const float* x;
  const float* w;
  char* out[NR];
  char* ws[NR];
  int rank;
  int phase;
  int pace[4][4];
};

#define W_POLICY "nt"
#define XPF 1
#define PL_POL "sc1"
#define PS_POL "sc1"
#define GSLEEP 8
#define PSLEEP 6
#define STAMP(i) do {} while (0)
#define STAMPC(i) do {} while (0)
#define STAMP_DECL do {} while (0)
#define STAMP_OUT() do {} while (0)

__device__ __forceinline__ unsigned pk_bf16(float lo, float hi) {
  f32x2 v = {lo, hi};
  return __builtin_bit_cast(unsigned, __builtin_convertvector(v, bf16x2));
}
__device__ __forceinline__ i32x4 make_rsrc(const void* base, unsigned bytes) {
  const u64 a = (u64)base;
  return i32x4{__builtin_amdgcn_readfirstlane((int)(unsigned)a), __builtin_amdgcn_readfirstlane((int)(unsigned)(a >> 32)), (int)bytes, 0x00020000};
}

__device__ __forceinline__ void w_load(f32x4& d, unsigned voff, i32x4 rsrc, unsigned soff) {
  asm volatile("buffer_load_dwordx4 %0, %1, %2, %3 offen " W_POLICY : "=v"(d) : "v"(voff), "s"(rsrc), "s"(soff) : "memory");
}
__device__ __forceinline__ void set_m0(unsigned m0v) { asm volatile("s_mov_b32 m0, %0" :: "s"(m0v) : "memory"); }
template <int IMM>
__device__ __forceinline__ void x_load(unsigned voff, i32x4 rsrc, unsigned soff) {
  asm volatile("buffer_load_dwordx4 %0, %1, %2 offen offset:%3 lds" :: "v"(voff), "s"(rsrc), "s"(soff), "i"(IMM) : "memory");
}
template <int N>
__device__ __forceinline__ void wait_bar() { asm volatile("s_waitcnt vmcnt(%0) lgkmcnt(0)\n\ts_barrier" :: "n"(N) : "memory"); }
template <int N>
__device__ __forceinline__ void wait_bar_w(f32x4 (&r)[8]) {
  asm volatile("s_waitcnt vmcnt(%8) lgkmcnt(0)\n\ts_barrier"
               : "+v"(r[0]), "+v"(r[1]), "+v"(r[2]), "+v"(r[3]), "+v"(r[4]), "+v"(r[5]), "+v"(r[6]), "+v"(r[7])
               : "n"(N) : "memory");
}
__device__ __forceinline__ void mfma(f32x4& c, u32x4 a, u32x4 b) {
  asm volatile("v_mfma_f32_16x16x32_bf16 %0, %1, %2, %0" : "+a"(c) : "v"(a), "v"(b));
}
__device__ __forceinline__ void mfma_v(f32x4& c, u32x4 a, u32x4 b) {
  asm volatile("v_mfma_f32_16x16x32_bf16 %0, %1, %2, %0" : "+v"(c) : "v"(a), "v"(b));
}
__device__ __forceinline__ void acc_zero(f32x4& c, u32x4 z) {
  asm volatile("v_mfma_f32_16x16x32_bf16 %0, %1, %1, 0" : "+a"(c) : "v"(z));
}
#define SCHED() __builtin_amdgcn_sched_barrier(0)
__device__ __forceinline__ void acc_fence(f32x4 (&acc)[16][4]) {
  asm volatile("s_nop 7\n\ts_nop 7" ::: "memory");
#pragma unroll
  for (int m = 0; m < 16; ++m)
#pragma unroll
    for (int t = 0; t < 4; ++t) asm volatile("" : "+a"(acc[m][t]));
  SCHED();
}
__device__ __forceinline__ u32x4 pack_piece(f32x4 (&acc)[16][4], int m, int u) {
  asm volatile("" : "+a"(acc[m][2 * u]), "+a"(acc[m][2 * u + 1]));
  const f32x4 lo = acc[m][2 * u], hi = acc[m][2 * u + 1];
  return u32x4{pk_bf16(lo[0], lo[1]), pk_bf16(lo[2], lo[3]), pk_bf16(hi[0], hi[1]), pk_bf16(hi[2], hi[3])};
}
constexpr float QBIAS = 12582912.0f;
struct Quant { float inv, lo, hi; };
__device__ __forceinline__ unsigned quant4(float v0, float v1, float v2, float v3, const Quant& qz) {
  const float inv = qz.inv, lo = qz.lo, hi = qz.hi;
  const unsigned e0 = __builtin_bit_cast(unsigned, __builtin_amdgcn_fmed3f(__builtin_fmaf(v0, inv, QBIAS), lo, hi));
  const unsigned e1 = __builtin_bit_cast(unsigned, __builtin_amdgcn_fmed3f(__builtin_fmaf(v1, inv, QBIAS), lo, hi));
  const unsigned e2 = __builtin_bit_cast(unsigned, __builtin_amdgcn_fmed3f(__builtin_fmaf(v2, inv, QBIAS), lo, hi));
  const unsigned e3 = __builtin_bit_cast(unsigned, __builtin_amdgcn_fmed3f(__builtin_fmaf(v3, inv, QBIAS), lo, hi));
  return __builtin_amdgcn_perm(e1, e0, 0x0c0c0400u) | __builtin_amdgcn_perm(e3, e2, 0x04000c0cu);
}

struct Ctx {
  i32x4 wrsrc, xrsrc;
  unsigned wvoff[8], xvoff0, xvoff1, m0base;
  unsigned xs;
  unsigned ws1, ws2;
  i32x4 srsrc;
  unsigned svoff, sphase;
  unsigned due, step;
  LAS const char* lrd0;
  LAS const char* lrd1;
  LAS char* wmine;
  LAS const char* wother;
};

template <int SLOT>
__device__ __forceinline__ u32x4 x_frag(const Ctx& c, int mt) {
  return *(LAS const u32x4*)((SLOT < 2 ? c.lrd0 : c.lrd1) + (SLOT & 1) * 32768 + mt * 1024);
}
template <int G>
__device__ __forceinline__ void x_piece(const Ctx& c, i32x4 xr, unsigned xso) {
  x_load<(G & 3) * 1024>((G & 4) ? c.xvoff1 : c.xvoff0, xr, (G & 8) ? xso + 32768 : xso);
}
template <int SL, bool SENDS>
__device__ __forceinline__ void kstep(Ctx& c, int v, f32x4 (&acc)[16][4], f32x4 (&wr)[2][8], u32x4 (&a)[2][4], u32x4 (&xf)[8],
                                      u32x4 snd, unsigned soff) {
  constexpr int E = SL & 1, WS = SL >> 1;
  u64 now;
  i32x4 xr = c.xrsrc, wrs = c.wrsrc;
  xr[2] = v + 1 < 2 * KQ ? c.xrsrc[2] : 0;
  wrs[2] = v + 2 < 2 * KQ ? c.wrsrc[2] : 0;
  const unsigned xso = c.xs + (unsigned)((v + 1) & (KQ - 1)) * 65536u;
  const unsigned wso = (v + 2 < KQ ? c.ws1 : c.ws2) + ((unsigned)(v + 2) << 22);
  const unsigned m0x = c.m0base + ((SL + 2) & 3) * 32768;
#pragma unroll
  for (int mt = 0; mt < 16; ++mt) {
    const f32x4 (&r)[8] = wr[WS ^ 1];
    const int t = mt - 12;
    if (SENDS && mt == 4 && E == 1) {
      i32x4 sr = c.srsrc;
      sr[2] = c.sphase == (unsigned)WS ? c.srsrc[2] : 0;
      asm volatile("buffer_store_dwordx4 %0, %1, %2, %3 offen sc0 sc1" :: "v"(snd), "v"(c.svoff), "s"(sr), "s"(soff) : "memory");
      SCHED();
    }
    if (mt == 12) {
      if (E == 0) wait_bar<20 + SENDS>();
      else {
        asm volatile("s_waitcnt lgkmcnt(0)" : "+s"(now) :: "memory");
        while ((int)((unsigned)now - c.due) < 0) { __builtin_amdgcn_s_sleep(1); now = __builtin_amdgcn_s_memrealtime(); }
        c.due += c.step;
        wait_bar_w<16 + SENDS>(wr[WS ^ 1]);
      }
      SCHED();
    }
    mfma(acc[mt][0], a[E][0], xf[mt & 7]);
    if (mt == 11 && E == 1) asm volatile("s_memrealtime %0" : "=s"(now) :: "memory");
    if (mt < 12) xf[(mt + 4) & 7] = x_frag<SL>(c, mt + 4);
    else xf[(mt + 4) & 7] = x_frag<(SL + 1) & 3>(c, t);
    if (E == 0 && (mt & 3) == 0) set_m0(m0x + (mt >> 3) * 32768 + ((mt >> 2) & 1) * 4096);
    SCHED();
    mfma(acc[mt][1], a[E][1], xf[mt & 7]);
    if (E == 0) {
      switch (mt) {
        case 0: x_piece<0>(c, xr, xso); break;   case 1: x_piece<1>(c, xr, xso); break;
        case 2: x_piece<2>(c, xr, xso); break;   case 3: x_piece<3>(c, xr, xso); break;
        case 4: x_piece<4>(c, xr, xso); break;   case 5: x_piece<5>(c, xr, xso); break;
        case 6: x_piece<6>(c, xr, xso); break;   case 7: x_piece<7>(c, xr, xso); break;
        case 8: x_piece<8>(c, xr, xso); break;   case 9: x_piece<9>(c, xr, xso); break;
        case 10: x_piece<10>(c, xr, xso); break; case 11: x_piece<11>(c, xr, xso); break;
        case 12: x_piece<12>(c, xr, xso); break; case 13: x_piece<13>(c, xr, xso); break;
        case 14: x_piece<14>(c, xr, xso); break; default: x_piece<15>(c, xr, xso); break;
      }
    } else {
      if (mt < 4) w_load(wr[WS][(mt * 2) & 7], c.wvoff[(mt * 2) & 7], wrs, wso);
      if (mt >= 12) { a[0][t & 3][0] = pk_bf16(r[0][t & 3], r[1][t & 3]); a[0][t & 3][1] = pk_bf16(r[2][t & 3], r[3][t & 3]); }
    }
    SCHED();
    mfma(acc[mt][2], a[E][2], xf[mt & 7]);
    if (E == 0 && mt >= 12) a[1][t & 3] = *(LAS const u32x4*)(c.wother + (t & 3) * 1024);
    if (E == 1 && mt >= 12) { a[0][t & 3][2] = pk_bf16(r[4][t & 3], r[5][t & 3]); a[0][t & 3][3] = pk_bf16(r[6][t & 3], r[7][t & 3]); }
    SCHED();
    mfma(acc[mt][3], a[E][3], xf[mt & 7]);
    if (E == 1 && mt < 4) w_load(wr[WS][(mt * 2 + 1) & 7], c.wvoff[(mt * 2 + 1) & 7], wrs, wso);
    if (E == 1 && mt >= 12) *(LAS u32x4*)(c.wmine + (t & 3) * 1024) = a[0][t & 3];
    SCHED();
  }
}

template <int SCOPE, int SLEEP>
__device__ __forceinline__ bool wait_flag(const unsigned* flag, unsigned call, unsigned* tmo) {
  const u64 t0 = __builtin_amdgcn_s_memrealtime();
  unsigned polls = 0;
  while (__any(__hip_atomic_load(flag, __ATOMIC_RELAXED, SCOPE) < call)) {
    if (((++polls & 63u) == 0 && __hip_atomic_load(tmo, __ATOMIC_RELAXED, __HIP_MEMORY_SCOPE_AGENT) == call) ||
        __builtin_amdgcn_s_memrealtime() - t0 > DEADLINE_TICKS) {
      __hip_atomic_store(tmo, call, __ATOMIC_RELAXED, __HIP_MEMORY_SCOPE_AGENT);
      return false;
    }
    __builtin_amdgcn_s_sleep(SLEEP);
  }
  return true;
}

template <int N, int STRIDE, int SCOPE, int SLEEP>
__device__ __forceinline__ float sweep_max(const u64* slots, unsigned call, unsigned* tmo) {
  const u64 t0 = __builtin_amdgcn_s_memrealtime();
  for (unsigned polls = 1;; ++polls) {
    bool all = true, bad = false;
    float mx = 0.f;
    u64 v[N];
#pragma unroll
    for (int r = 0; r < N; ++r) v[r] = __hip_atomic_load(slots + r * STRIDE, __ATOMIC_RELAXED, SCOPE);
#pragma unroll
    for (int r = 0; r < N; ++r) {
      const float f = __builtin_bit_cast(float, (unsigned)v[r]);
      all &= (unsigned)(v[r] >> 32) == call;
      bad |= f < 0.f;
      mx = fmaxf(mx, f);
    }
    if (__all(all)) {
#pragma unroll
      for (int o = 32; o >= 1; o >>= 1) mx = fmaxf(mx, __shfl_xor(mx, o));
      return __any(bad) ? -1.0f : mx;
    }
    if (((polls & 63u) == 0 && __hip_atomic_load(tmo, __ATOMIC_RELAXED, __HIP_MEMORY_SCOPE_AGENT) == call) ||
        __builtin_amdgcn_s_memrealtime() - t0 > DEADLINE_TICKS) {
      __hip_atomic_store(tmo, call, __ATOMIC_RELAXED, __HIP_MEMORY_SCOPE_AGENT);
      return -1.0f;
    }
    __builtin_amdgcn_s_sleep(SLEEP);
  }
}

__device__ __forceinline__ void early_tile(int j, int& pd, int& ct) { pd = j < 7 * NE ? j % 7 + 1 : 0; ct = j < 7 * NE ? j / 7 : j - 7 * NE; }
__device__ __forceinline__ void late_tile(int j, int& pd, int& ct) {
  pd = j < 7 * (16 - NE) ? j % 7 + 1 : 0;
  ct = j < 7 * (16 - NE) ? NE + j / 7 : j - 7 * (16 - NE) + (NS - 7 * NE);
}
template <bool FIRST>
__device__ __forceinline__ void publish(f32x4 (&acc)[16][4], const char* part, int unit, int wave, unsigned voff) {
  u32x4 z = {0u, 0u, 0u, 0u};
  asm volatile("s_nop 1" : "+v"(z));
#pragma unroll
  for (int g = 1; g < 4; ++g) {
    const i32x4 prs = make_rsrc(part + (size_t)((FIRST ? unit ^ g : unit) * 4 + wave) * 24576 + (g - 1) * 8192, 8192);
#pragma unroll
    for (int j = 0; j < 8; ++j) {
      const int m = 4 * g + (j >> 1), u = j & 1;
      const u32x4 d = pack_piece(acc, m, u);
      asm volatile("buffer_store_dwordx4 %0, %1, %2, 0 offen offset:%3 " PS_POL "\n\ts_nop 1"
                   :: "v"(d), "v"((j & 4) ? voff + 4096 : voff), "s"(prs), "i"((j & 3) * 1024) : "memory");
      if (FIRST) { acc_zero(acc[m][2 * u], z); acc_zero(acc[m][2 * u + 1], z); }
    }
  }
}
template <bool IN_ACC>
__device__ __forceinline__ void collect(f32x4 (&dst)[16][4], const char* part, int unit, int wave, unsigned voff, int lane) {
  const i32x4 prs = make_rsrc(part + (size_t)((unit & ~3) * 4 + wave) * 24576, 3 * 4 * 24576 + 24576);
  unsigned so[6];
#pragma unroll
  for (int i = 0; i < 6; ++i) so[i] = (unsigned)((IN_ACC ? (unit ^ (1 + (i >> 1))) : unit) & 3) * (4 * 24576) + i * 4096;
  u32x4 pv[12];
#pragma unroll
  for (int j = 0; j < 12; ++j)
    asm volatile("buffer_load_dwordx4 %0, %1, %2, %3 offen offset:%4 " PL_POL : "=v"(pv[j]) : "v"(voff), "s"(prs), "s"(so[j >> 2]), "i"((j & 3) * 1024) : "memory");
  const int i16 = lane & 15, q = lane >> 4;
  const unsigned one = (i16 >> 2) == q ? ((i16 & 1) ? 0x3F800000u : 0x00003F80u) : 0u;
  const unsigned e0 = (i16 & 2) ? 0u : one, e1 = (i16 & 2) ? one : 0u;
  u32x4 sel0 = {e0, e1, 0u, 0u}, sel1 = {0u, 0u, e0, e1};
  asm volatile("s_nop 1" : "+v"(sel0), "+v"(sel1));
#pragma unroll
  for (int j = 0; j < 24; ++j) {
    const int m = (j & 7) >> 1, u = j & 1;
    asm volatile("s_waitcnt vmcnt(%1)" : "+v"(pv[j % 12]) : "n"(j < 12 ? 11 : 23 - j) : "memory");
    if (IN_ACC) { mfma(dst[m][2 * u], sel0, pv[j % 12]); mfma(dst[m][2 * u + 1], sel1, pv[j % 12]); }
    else { mfma_v(dst[m][2 * u], sel0, pv[j % 12]); mfma_v(dst[m][2 * u + 1], sel1, pv[j % 12]); }
    if (j < 12)
      asm volatile("buffer_load_dwordx4 %0, %1, %2, %3 offen offset:%4 " PL_POL : "+v"(pv[j]) : "v"(voff), "s"(prs), "s"(so[(j + 12) >> 2]), "i"((j & 3) * 1024) : "memory");
  }
  asm volatile("s_nop 7\n\ts_nop 7" ::: "memory");
#pragma unroll
  for (int m = 0; m < 4; ++m)
#pragma unroll
    for (int t = 0; t < 4; ++t) {
      if (IN_ACC) asm volatile("" : "+a"(dst[m][t]));
      else asm volatile("" : "+v"(dst[m][t]));
    }
  SCHED();
}

__global__ void __launch_bounds__(NTHR, 1) fused_kernel(Params p) {
  extern __shared__ __attribute__((aligned(1024))) char smem[];
  const int tid = threadIdx.x, lane = tid & 63;
  const int wave = __builtin_amdgcn_readfirstlane(tid >> 6);
  char* ws = p.ws[p.rank];

  if (p.phase == 0) {
    for (int it = blockIdx.x * NTHR + tid; it < ML * KD / 8; it += gridDim.x * NTHR) {
      const int l = it & 63, mt = (it >> 6) & 31, st = it >> 11;
      const float* src = p.x + (size_t)((mt ^ ((st >> 5) << 2)) * 16 + (l & 15)) * KD + (st ^ (mt >> 4)) * 32 + (l >> 4) * 8;
      const f32x4 v0 = *(const f32x4*)src, v1 = *(const f32x4*)(src + 4);
      *(u32x4*)(ws + WS_XT + (size_t)it * 16) =
          u32x4{pk_bf16(v0[0], v0[1]), pk_bf16(v0[2], v0[3]), pk_bf16(v1[0], v1[1]), pk_bf16(v1[2], v1[3])};
    }
    if (blockIdx.x == 0 && tid == 0) {
      unsigned* ctr = (unsigned*)(ws + WS_CALL);
      __hip_atomic_fetch_add(ctr, 1u, __ATOMIC_RELAXED, __HIP_MEMORY_SCOPE_AGENT);
    }
    return;
  }

  STAMP_DECL;
  STAMP(0);
  STAMPC(8);
  const int b = blockIdx.x, xcd = b & 7, idx = b >> 3;
  const int kq = xcd & 3, grp = idx & 3, slot = grp * 16 + (xcd >> 2) * 8 + (idx >> 2), unit = slot * 4 + kq;
  const unsigned t_start = (unsigned)__builtin_amdgcn_s_memrealtime();
  int pdE, ctE, pdL, ctL;
  early_tile(slot, pdE, ctE);
  late_tile(slot, pdL, ctL);
  const int tileE = ((p.rank + pdE) & 7) * 16 + ctE, tileL = ((p.rank + pdL) & 7) * 16 + ctL;
  const int wn = wave & 1, wm = wave >> 1;
  const int q = lane >> 4, i16 = lane & 15;
  unsigned* tmo = (unsigned*)(ws + WS_TMO);

  Ctx c;
  c.wrsrc = make_rsrc(p.w, (unsigned)((size_t)KD * ND * 4));
  c.xrsrc = make_rsrc(ws + WS_XT, (unsigned)XT_BYTES);
#pragma unroll
  for (int j = 0; j < 8; ++j) {
    c.wvoff[j] = ((unsigned)(wm * 32 + q * 8 + j) << 16) + wn * 256 + i16 * 16;
    asm volatile("" : "+v"(c.wvoff[j]));
  }
  c.xvoff0 = wave * 8192 + lane * 16;
  c.xvoff1 = c.xvoff0 + 4096;
  asm volatile("" : "+v"(c.xvoff1));
  c.m0base = (unsigned)(size_t)(LAS char*)smem + wave * 8192;
  c.xs = (unsigned)(kq * KQ) * 65536u;
  c.ws1 = ((unsigned)(kq * KQ) << 22) + (unsigned)tileE * 512u;
  c.ws2 = ((unsigned)((kq - 1) * KQ) << 22) + (unsigned)tileL * 512u;
  const unsigned destE = (unsigned)(p.rank + pdE) & 7u, destL = (unsigned)(p.rank + pdL) & 7u;
  char* wsE = p.ws[destE];
  c.srsrc = make_rsrc(wsE + WS_STAGE + (size_t)(unit * 4 + wave) * 8192, 8192);
  c.svoff = lane * 16;
  c.due = t_start + p.pace[grp][0];
  c.step = p.pace[grp][1];
  const unsigned due2 = t_start + p.pace[grp][2], step2 = p.pace[grp][3];
  c.sphase = (idx + wave) & 1;
  c.lrd0 = (LAS const char*)smem + wm * 16384 + lane * 16;
  c.lrd1 = c.lrd0 + 65536;
  asm volatile("" : "+v"(c.lrd1));
  c.wmine = (LAS char*)smem + LDS_X + wave * 4096 + lane * 16;
  c.wother = (LAS const char*)smem + LDS_X + (wave ^ 2) * 4096 + lane * 16;

  f32x4 acc[16][4];
  f32x4 wr[2][8];
  u32x4 a[2][4], xf[8];

  const unsigned call = *(const __attribute__((address_space(4))) unsigned*)(ws + WS_CALL);
#pragma unroll
  for (int j = 0; j < 8; ++j) w_load(wr[0][j], c.wvoff[j], c.wrsrc, c.ws1);
  {
    const unsigned xso = c.xs;
    set_m0(c.m0base); asm volatile("s_nop 0");
    x_piece<0>(c, c.xrsrc, xso); x_piece<1>(c, c.xrsrc, xso); x_piece<2>(c, c.xrsrc, xso); x_piece<3>(c, c.xrsrc, xso);
    set_m0(c.m0base + 4096); asm volatile("s_nop 0");
    x_piece<4>(c, c.xrsrc, xso); x_piece<5>(c, c.xrsrc, xso); x_piece<6>(c, c.xrsrc, xso); x_piece<7>(c, c.xrsrc, xso);
    set_m0(c.m0base + 32768); asm volatile("s_nop 0");
    x_piece<8>(c, c.xrsrc, xso); x_piece<9>(c, c.xrsrc, xso); x_piece<10>(c, c.xrsrc, xso); x_piece<11>(c, c.xrsrc, xso);
    set_m0(c.m0base + 32768 + 4096); asm volatile("s_nop 0");
    x_piece<12>(c, c.xrsrc, xso); x_piece<13>(c, c.xrsrc, xso); x_piece<14>(c, c.xrsrc, xso); x_piece<15>(c, c.xrsrc, xso);
  }
#pragma unroll
  for (int j = 0; j < 8; ++j) w_load(wr[1][j], c.wvoff[j], c.wrsrc, c.ws1 + (1u << 22));
  constexpr int NPF = XPF ? KQ / 2 : 0;
  u32x4 pf = {0u, 0u, 0u, 0u};
  if (XPF) {
    const i32x4 pr = c.xrsrc;
    const unsigned pv = c.xs + (idx * 4 + wave) * 1024 + lane * 16;
#pragma unroll
    for (int j = 0; j < NPF; ++j) {
      const unsigned v = pv + j * 131072;
      asm volatile("buffer_load_dwordx4 %0, %1, %2, 0 offen" : "+v"(pf) : "v"(v), "s"(pr) : "memory");
    }
  }
  __builtin_amdgcn_s_waitcnt(0xC07F);
  wait_bar_w<16 + NPF>(wr[0]);
  SCHED();
#pragma unroll
  for (int t = 0; t < 4; ++t) {
    xf[t] = x_frag<0>(c, t);
    a[0][t] = u32x4{pk_bf16(wr[0][0][t], wr[0][1][t]), pk_bf16(wr[0][2][t], wr[0][3][t]), pk_bf16(wr[0][4][t], wr[0][5][t]), pk_bf16(wr[0][6][t], wr[0][7][t])};
    *(LAS u32x4*)(c.wmine + t * 1024) = a[0][t];
  }
  unsigned* pflag = (unsigned*)(ws + WS_PFLAG);
  if (b == 0 && tid < NR)
    __hip_atomic_store((unsigned*)(p.ws[tid] + WS_START + p.rank * 128), call, __ATOMIC_RELAXED, __HIP_MEMORY_SCOPE_SYSTEM);
#pragma unroll
  for (int m = 0; m < 16; ++m)
#pragma unroll
    for (int t = 0; t < 4; ++t) {
      acc[m][t] = f32x4{0.f, 0.f, 0.f, 0.f};
      asm volatile("" : "+a"(acc[m][t]));
    }
  u32x4 snd[8];
  asm volatile("s_nop 1" : "+v"(a[0][0]), "+v"(a[0][1]), "+v"(a[0][2]), "+v"(a[0][3]));
  __builtin_amdgcn_s_waitcnt(0xC07F);
  SCHED();
  STAMP(1);

#define TRACE_PT()
#pragma unroll 1
  for (int v = 0; v < KQ; v += 2) {
    TRACE_PT()
    kstep<0, false>(c, v, acc, wr, a, xf, pf, 0);
    kstep<1, false>(c, v, acc, wr, a, xf, pf, 0);
    kstep<2, false>(c, v + 1, acc, wr, a, xf, pf, 0);
    kstep<3, false>(c, v + 1, acc, wr, a, xf, pf, 0);
  }
  f32x4 own[16][4];
  acc_fence(acc);
  {
    u32x4 z = {0u, 0u, 0u, 0u};
    asm volatile("s_nop 1" : "+v"(z));
#pragma unroll
    for (int m = 0; m < 4; ++m)
#pragma unroll
      for (int t = 0; t < 4; ++t) {
        own[m][t] = acc[m][t];
        asm volatile("" : "+v"(own[m][t]));
        acc_zero(acc[m][t], z);
      }
  }
  publish<true>(acc, ws + WS_PART, unit, wave, c.svoff);
  c.due = due2;
  c.step = step2;
  STAMP(3);
#pragma unroll 1
  for (int v = KQ; v < KQ + MID; v += 2) {
    TRACE_PT()
    kstep<0, false>(c, v, acc, wr, a, xf, pf, 0);
    kstep<1, false>(c, v, acc, wr, a, xf, pf, 0);
    if (v == KQ && lane == 0) __hip_atomic_store(pflag + (unit * 4 + wave) * 32, call, __ATOMIC_RELAXED, __HIP_MEMORY_SCOPE_AGENT);
    SCHED();
    kstep<2, false>(c, v + 1, acc, wr, a, xf, pf, 0);
    kstep<3, false>(c, v + 1, acc, wr, a, xf, pf, 0);
  }
  float am = 0.f;
  bool ok;
  {
    const int g = lane & 3;
    const unsigned* f = g ? pflag + ((unit ^ g) * 4 + wave) * 32 : (const unsigned*)(ws + WS_START + destE * 128);
    ok = wait_flag<__HIP_MEMORY_SCOPE_SYSTEM, PSLEEP>(f, call, tmo);
    collect<false>(own, ws + WS_PART, unit, wave, c.svoff, lane);
#pragma unroll
    for (int j = 0; j < 8; ++j) {
      const f32x4 lo = own[j >> 1][(j & 1) * 2], hi = own[j >> 1][(j & 1) * 2 + 1];
      am = fmaxf(fmaxf(am, fabsf(lo[0])), fabsf(lo[1])); am = fmaxf(fmaxf(am, fabsf(lo[2])), fabsf(lo[3]));
      am = fmaxf(fmaxf(am, fabsf(hi[0])), fabsf(hi[1])); am = fmaxf(fmaxf(am, fabsf(hi[2])), fabsf(hi[3]));
      snd[j] = u32x4{pk_bf16(lo[0], lo[1]), pk_bf16(lo[2], lo[3]), pk_bf16(hi[0], hi[1]), pk_bf16(hi[2], hi[3])};
      asm volatile("" : "+v"(snd[j]), "+v"(am));
    }
    __builtin_amdgcn_s_waitcnt(0xC07F);
    SCHED();
  }
  STAMP(6);
  static_assert(KQ - MID == 14, "seven pairs of blocks for pieces 1..7");
  asm volatile("buffer_store_dwordx4 %0, %1, %2, 0 offen sc0 sc1" :: "v"(snd[0]), "v"(c.svoff), "s"(c.srsrc) : "memory");
#pragma unroll 1
  for (int v = KQ + MID; v < 2 * KQ; v += 2) {
    TRACE_PT()
    const unsigned soff = (unsigned)(v - (KQ + MID - 2)) * 512u;
    kstep<0, true>(c, v, acc, wr, a, xf, snd[1], soff);
    kstep<1, true>(c, v, acc, wr, a, xf, snd[1], soff);
    kstep<2, true>(c, v + 1, acc, wr, a, xf, snd[1], soff);
    kstep<3, true>(c, v + 1, acc, wr, a, xf, snd[1], soff);
#pragma unroll
    for (int j = 1; j < 7; ++j) snd[j] = snd[j + 1];
  }
  acc_fence(acc);
  STAMP(2);
  STAMPC(9);

  publish<false>(acc, ws + WS_PART, unit, wave, c.svoff);
  asm volatile("s_waitcnt vmcnt(0)"
               : "+v"(wr[0][0]), "+v"(wr[0][1]), "+v"(wr[0][2]), "+v"(wr[0][3]), "+v"(wr[0][4]), "+v"(wr[0][5]), "+v"(wr[0][6]), "+v"(wr[0][7]),
                 "+v"(wr[1][0]), "+v"(wr[1][1]), "+v"(wr[1][2]), "+v"(wr[1][3]), "+v"(wr[1][4]), "+v"(wr[1][5]), "+v"(wr[1][6]), "+v"(wr[1][7]), "+v"(pf)
               :: "memory");
  if (lane == 0) {
    __hip_atomic_store(pflag + ((NWG + unit) * 4 + wave) * 32, call, __ATOMIC_RELAXED, __HIP_MEMORY_SCOPE_AGENT);
    __hip_atomic_store((unsigned*)(wsE + WS_EFLAG + (size_t)(unit * 4 + wave) * 128), call, __ATOMIC_RELAXED, __HIP_MEMORY_SCOPE_SYSTEM);
  }
  ok &= wait_flag<__HIP_MEMORY_SCOPE_AGENT, PSLEEP>(pflag + ((NWG + (unit ^ (lane & 3))) * 4 + wave) * 32, call, tmo);
  collect<true>(acc, ws + WS_PART, unit, wave, c.svoff, lane);
  f32x4 y[4][4];
#pragma unroll
  for (int m = 0; m < 4; ++m)
#pragma unroll
    for (int t = 0; t < 4; ++t) {
      y[m][t] = acc[m][t];
      am = fmaxf(fmaxf(am, fabsf(y[m][t][0])), fabsf(y[m][t][1]));
      am = fmaxf(fmaxf(am, fabsf(y[m][t][2])), fabsf(y[m][t][3]));
    }
#pragma unroll
  for (int o = 32; o >= 1; o >>= 1) am = fmaxf(am, __shfl_xor(am, o));
  LAS float* red = (LAS float*)(smem + LDS_X);
  __syncthreads();
  if (lane == 0) red[wave] = ok ? am : -1.0f;
  __syncthreads();
  const size_t gmax_off = WS_GMAX + (size_t)(call & 1) * 128;
  if (tid == 0) {
    const float r0 = red[0], r1 = red[1], r2 = red[2], r3 = red[3];
    const float wam = fminf(fminf(r0, r1), fminf(r2, r3)) < 0.f ? -1.0f : fmaxf(fmaxf(r0, r1), fmaxf(r2, r3));
    __hip_atomic_store((u64*)(ws + WS_AMAX) + unit, ((u64)call << 32) | __builtin_bit_cast(unsigned, wam), __ATOMIC_RELAXED, __HIP_MEMORY_SCOPE_AGENT);
  }
  if (unit == 0 && wave == 1) {
    const float rm = sweep_max<4, 64, __HIP_MEMORY_SCOPE_AGENT, 1>((const u64*)(ws + WS_AMAX) + lane, call, tmo);
    if (lane < NR)
      __hip_atomic_store((u64*)(p.ws[lane] + gmax_off) + p.rank, ((u64)call << 32) | __builtin_bit_cast(unsigned, rm), __ATOMIC_RELAXED, __HIP_MEMORY_SCOPE_SYSTEM);
  }
  STAMP(5);

  ok &= wait_flag<__HIP_MEMORY_SCOPE_SYSTEM, 16>((const unsigned*)(ws + WS_EFLAG + (size_t)(unit * 4 + wave) * 128), call, tmo);
  u32x4 st[8];
  {
    const i32x4 rs = make_rsrc(ws + WS_STAGE + (size_t)(unit * 4 + wave) * 8192, 8192);
#pragma unroll
    for (int j = 0; j < 8; ++j)
      asm volatile("buffer_load_dwordx4 %0, %1, %2, %3 offen offset:%4 sc0 sc1" : "=v"(st[j]) : "v"(c.svoff), "s"(rs), "s"((j >> 2) * 4096), "i"((j & 3) * 1024) : "memory");
    asm volatile("s_waitcnt vmcnt(0)" : "+v"(st[0]), "+v"(st[1]), "+v"(st[2]), "+v"(st[3]), "+v"(st[4]), "+v"(st[5]), "+v"(st[6]), "+v"(st[7]) :: "memory");
  }
  float gam = ok ? 0.f : -1.0f;
  if (wave == 0 && ok) gam = sweep_max<1, 0, __HIP_MEMORY_SCOPE_SYSTEM, GSLEEP>((const u64*)(ws + gmax_off) + (lane & 7), call, tmo);
  if (lane == 0) red[8 + wave] = gam;
  __syncthreads();
  Quant qz;
  {
    const float r0 = red[8], r1 = red[9], r2 = red[10], r3 = red[11];
    const bool good = fminf(fminf(r0, r1), fminf(r2, r3)) >= 0.f;
    qz.inv = (good && r0 > 0.f) ? 127.0f / r0 : 0.f;
    qz.lo = good ? QBIAS - 127.0f : QBIAS - 128.0f;
    qz.hi = good ? QBIAS + 127.0f : QBIAS - 128.0f;
    asm volatile("" : "+v"(qz.lo), "+v"(qz.hi));
  }
  STAMP(4);

  const int row0 = wm * 256 + kq * 64 + i16, col0 = wn * 64 + q * 16;
  {
    char* ob = p.out[destL] + (size_t)(p.rank * ML + row0) * NL + ctL * 128 + col0;
#pragma unroll
    for (int m = 0; m < 4; ++m) {
      u32x4 o;
#pragma unroll
      for (int r = 0; r < 4; ++r) o[r] = quant4(y[m][0][r], y[m][1][r], y[m][2][r], y[m][3][r], qz);
      asm volatile("global_store_dwordx4 %0, %1, off sc0 sc1\n\ts_nop 1" :: "v"(ob + (size_t)m * 16 * NL), "v"(o) : "memory");
    }
  }
  {
    const int srank = (p.rank - pdE) & 7;
    char* ob = p.out[p.rank] + (size_t)(srank * ML + row0) * NL + ctE * 128 + col0;
#pragma unroll
    for (int m = 0; m < 4; ++m) {
      const u32x4 s0 = st[2 * m], s1 = st[2 * m + 1];
      u32x4 o;
#pragma unroll
      for (int r = 0; r < 4; ++r) {
        const int h = r >> 1;
        const unsigned w0 = s0[h], w1 = s0[2 + h], w2 = s1[h], w3 = s1[2 + h];
        const float v0 = __builtin_bit_cast(float, (r & 1) ? (w0 & 0xFFFF0000u) : (w0 << 16));
        const float v1 = __builtin_bit_cast(float, (r & 1) ? (w1 & 0xFFFF0000u) : (w1 << 16));
        const float v2 = __builtin_bit_cast(float, (r & 1) ? (w2 & 0xFFFF0000u) : (w2 << 16));
        const float v3 = __builtin_bit_cast(float, (r & 1) ? (w3 & 0xFFFF0000u) : (w3 << 16));
        o[r] = quant4(v0, v1, v2, v3, qz);
      }
      *(u32x4*)(ob + (size_t)m * 16 * NL) = o;
    }
  }
  asm volatile("s_waitcnt vmcnt(0)" ::: "memory");
  __syncthreads();
  if (wave == 0) {
    if (lane == 0)
      __hip_atomic_store((unsigned*)(p.ws[destL] + WS_LFLAG + (size_t)unit * 128), call, __ATOMIC_RELAXED, __HIP_MEMORY_SCOPE_SYSTEM);
    if (!wait_flag<__HIP_MEMORY_SCOPE_SYSTEM, 2>((const unsigned*)(ws + WS_LFLAG + (size_t)unit * 128), call, tmo) && lane == 0)
      p.out[p.rank][(size_t)(((p.rank - pdE) & 7) * ML + kq * 64) * NL + ctE * 128] = (char)-128;
    STAMP(7);
    STAMP_OUT();
  }
}

extern "C" void kernel_launch(int rank, int nranks, ncclComm_t comm,
                              hipStream_t stream,
                              void* const* d_in, const int* in_sizes, int n_in,
                              void* d_out, int out_size,
                              const int* in_dtypes, int out_dtype,
                              void* const* d_out_all,
                              void* const* d_ws_all, size_t ws_bytes) {
  (void)comm; (void)in_dtypes; (void)out_dtype; (void)d_out;
  if (nranks != NR || n_in != 2 || in_sizes[0] != ML * KD || in_sizes[1] != KD * ND || out_size != NR * ML * NL ||
      ws_bytes < WS_END || (unsigned)rank >= (unsigned)NR) {
    fprintf(stderr, "unexpected problem shape\n");
    return;
  }
  HIP_CHECK(hipFuncSetAttribute((const void*)fused_kernel, hipFuncAttributeMaxDynamicSharedMemorySize, LDS_BYTES));
  Params p;
  p.x = (const float*)d_in[0];
  p.w = (const float*)d_in[1];
  for (int r = 0; r < NR; ++r) { p.out[r] = (char*)d_out_all[r]; p.ws[r] = (char*)d_ws_all[r]; }
  p.rank = rank;
  static const int pace[4][4] = {PACE_TABLE};
  for (int i = 0; i < 16; ++i) p.pace[i >> 2][i & 3] = pace[i >> 2][i & 3];
  p.phase = 0;
  fused_kernel<<<NWG, NTHR, 0, stream>>>(p);
  p.phase = 1;
  fused_kernel<<<NWG, NTHR, LDS_BYTES, stream>>>(p);
  HIP_CHECK(hipGetLastError());
}
```

```cpp
#include <hip/hip_runtime.h>
#include <rccl/rccl.h>
#include <cstdio>
#include <cstdint>
#include <cstdlib>

#define HIP_CHECK(expr) do { hipError_t _e = (expr); if (_e != hipSuccess) { \
    fprintf(stderr, "HIP error %s at %s:%d\n", hipGetErrorString(_e), __FILE__, __LINE__); \
    return; } } while (0)

#define LAS __attribute__((address_space(3)))
typedef float f32x4 __attribute__((ext_vector_type(4)));
typedef float f32x2 __attribute__((ext_vector_type(2)));
typedef __bf16 bf16x2 __attribute__((ext_vector_type(2)));
typedef unsigned u32x4 __attribute__((ext_vector_type(4)));
typedef int i32x4 __attribute__((ext_vector_type(4)));
typedef unsigned long long u64;

constexpr int NR = 8;
constexpr int KD = 4096, ND = 16384;
constexpr int ML = 512;
constexpr int NL = ND / NR;
constexpr int NWG = 256, NTHR = 256;
constexpr int NBLK = KD / 64;
constexpr int NT = ND / 128;
#define NE 6
constexpr int NEARLY = 7 * NE;
static_assert(NE >= 0 && NE <= 7, "early tiles are the odd column tiles below 15");
constexpr int KH = NBLK / 2;
#define PACE_ARGS 220, 2200, 150, 4400, 145, 140, 700, 100
constexpr int LDS_X = 4 * 32768;
constexpr int LDS_BYTES = LDS_X + 32768;
constexpr size_t HALF_B = 65536;

constexpr size_t WS_TMO = 128;
constexpr size_t WS_START = 1024;
constexpr size_t WS_GMAX = 2048;
constexpr size_t WS_AMAX = 4096;
constexpr size_t WS_XFLAG = WS_AMAX + NWG * 8;
constexpr size_t WS_LFLAG = WS_XFLAG + NWG * 128;
constexpr size_t WS_EFLAG = WS_LFLAG + NWG * 128;
constexpr size_t WS_PFLAG = WS_EFLAG + NWG * 4 * 128;
constexpr size_t WS_STAMP = WS_PFLAG + NWG * 4 * 128;
constexpr size_t WS_XT = 1 << 20;
constexpr size_t XT_BYTES = (size_t)ML * KD * 2, W_BYTES = (size_t)KD * ND * 4;
constexpr size_t WS_PART = WS_XT + XT_BYTES;
constexpr size_t WS_STAGE = WS_PART + NWG * HALF_B;
constexpr size_t WS_END = WS_STAGE + 2 * 49 * HALF_B;
static_assert(WS_STAMP + 65536 + NWG * 512 <= WS_XT, "control block overlaps x");
constexpr u64 DEADLINE_TICKS = 50000000ull;

struct Params {
  const float* x;
  const float* w;
  char* myws;
  char* out[NR];
  char* ws[NR];
  int rank;
  int first, slack;
  unsigned short step[2][66];
};

#define SEND_WIN 2
#define STAMP(i) do {} while (0)
#define STAMPC(i) do {} while (0)
#define STAMP_DECL do {} while (0)
#define STAMP_OUT() do {} while (0)

__device__ __forceinline__ unsigned pk_bf16(float lo, float hi) {
  f32x2 v = {lo, hi};
  return __builtin_bit_cast(unsigned, __builtin_convertvector(v, bf16x2));
}
__device__ __forceinline__ i32x4 make_rsrc(const void* base, unsigned bytes) {
  const u64 a = (u64)base;
  return i32x4{__builtin_amdgcn_readfirstlane((int)(unsigned)a), __builtin_amdgcn_readfirstlane((int)(unsigned)(a >> 32)), (int)bytes, 0x00020000};
}

__device__ __forceinline__ void w_load(f32x4& d, unsigned voff, i32x4 rsrc, unsigned soff) {
  asm volatile("buffer_load_dwordx4 %0, %1, %2, %3 offen nt" : "=v"(d) : "v"(voff), "s"(rsrc), "s"(soff) : "memory");
}
__device__ __forceinline__ void set_m0(unsigned m0v) { asm volatile("s_mov_b32 m0, %0" :: "s"(m0v) : "memory"); }
template <int IMM>
__device__ __forceinline__ void x_load(unsigned voff, i32x4 rsrc, unsigned soff) {
  asm volatile("buffer_load_dwordx4 %0, %1, %2 offen offset:%3 sc1 lds" :: "v"(voff), "s"(rsrc), "s"(soff), "i"(IMM) : "memory");
}
template <int N>
__device__ __forceinline__ void wait_bar() { asm volatile("s_waitcnt vmcnt(%0) lgkmcnt(0)\n\ts_barrier" :: "n"(N) : "memory"); }
__device__ __forceinline__ void mfma(f32x4& c, u32x4 a, u32x4 b) {
  asm volatile("v_mfma_f32_16x16x32_bf16 %0, %1, %2, %0" : "+a"(c) : "v"(a), "v"(b));
}
#define SCHED() __builtin_amdgcn_sched_barrier(0)
__device__ __forceinline__ void acc_fence(f32x4 (&acc)[16][4]) {
  asm volatile("s_nop 7\n\ts_nop 7" ::: "memory");
#pragma unroll
  for (int m = 0; m < 16; ++m)
#pragma unroll
    for (int t = 0; t < 4; ++t) asm volatile("" : "+a"(acc[m][t]));
  SCHED();
}
__device__ __forceinline__ u32x4 pack_piece(f32x4 (&acc)[16][4], int j) {
  const int m = j >> 1, u = j & 1;
  asm volatile("" : "+a"(acc[m][2 * u]), "+a"(acc[m][2 * u + 1]));
  const f32x4 lo = acc[m][2 * u], hi = acc[m][2 * u + 1];
  return u32x4{pk_bf16(lo[0], lo[1]), pk_bf16(lo[2], lo[3]), pk_bf16(hi[0], hi[1]), pk_bf16(hi[2], hi[3])};
}
constexpr float QBIAS = 12582912.0f;
struct Quant { float inv, lo, hi; };
__device__ __forceinline__ unsigned quant4(float v0, float v1, float v2, float v3, const Quant& qz) {
  const unsigned e0 = __builtin_bit_cast(unsigned, __builtin_amdgcn_fmed3f(__builtin_fmaf(v0, qz.inv, QBIAS), qz.lo, qz.hi));
  const unsigned e1 = __builtin_bit_cast(unsigned, __builtin_amdgcn_fmed3f(__builtin_fmaf(v1, qz.inv, QBIAS), qz.lo, qz.hi));
  const unsigned e2 = __builtin_bit_cast(unsigned, __builtin_amdgcn_fmed3f(__builtin_fmaf(v2, qz.inv, QBIAS), qz.lo, qz.hi));
  const unsigned e3 = __builtin_bit_cast(unsigned, __builtin_amdgcn_fmed3f(__builtin_fmaf(v3, qz.inv, QBIAS), qz.lo, qz.hi));
  return __builtin_amdgcn_perm(e1, e0, 0x0c0c0400u) | __builtin_amdgcn_perm(e3, e2, 0x04000c0cu);
}

template <int SCOPE, int SLEEP, int N = 1, int STRIDE = 0>
__device__ __forceinline__ bool wait_flag(const unsigned* flag, unsigned call, unsigned* tmo) {
  const u64 t0 = __builtin_amdgcn_s_memrealtime();
  for (unsigned polls = 1;; ++polls) {
    unsigned v[N];
    bool pend = false;
#pragma unroll
    for (int i = 0; i < N; ++i) v[i] = __hip_atomic_load(flag + i * STRIDE, __ATOMIC_RELAXED, SCOPE);
#pragma unroll
    for (int i = 0; i < N; ++i) pend |= v[i] < call;
    if (!__any(pend)) return true;
    if (((polls & 63u) == 0 && __hip_atomic_load(tmo, __ATOMIC_RELAXED, __HIP_MEMORY_SCOPE_AGENT) == call) ||
        __builtin_amdgcn_s_memrealtime() - t0 > DEADLINE_TICKS) {
      __hip_atomic_store(tmo, call, __ATOMIC_RELAXED, __HIP_MEMORY_SCOPE_AGENT);
      return false;
    }
    __builtin_amdgcn_s_sleep(SLEEP);
  }
}

struct Ctx {
  i32x4 wrsrc, xrsrc;
  unsigned wvoff[8], xvoff0, xvoff1, m0base;
  int left;
  unsigned xso, wso;
  unsigned due, slack;
  int steps;
  i32x4 frsrc;
  const char* fbase;
  unsigned fvoff, fso, call;
  unsigned* tmo;
  bool ok;
  LAS const char* lrd0;
  LAS const char* lrd1;
  LAS char* wmine;
  LAS const char* wother;
};

template <int SLOT>
__device__ __forceinline__ u32x4 x_frag(const Ctx& c, int mt) {
  return *(LAS const u32x4*)((SLOT < 2 ? c.lrd0 : c.lrd1) + (SLOT & 1) * 32768 + mt * 1024);
}
template <int G>
__device__ __forceinline__ void x_piece(const Ctx& c, i32x4 xr, unsigned xso) {
  x_load<(G & 3) * 1024>((G & 4) ? c.xvoff1 : c.xvoff0, xr, (G & 8) ? xso + 32768 : xso);
}
template <int SL>
__device__ __forceinline__ void kstep(Ctx& c, f32x4 (&acc)[16][4], f32x4 (&wr)[2][8], u32x4 (&a)[2][4], u32x4 (&xf)[8]) {
  constexpr int E = SL & 1, WS = SL >> 1, NX = (SL + 1) & 3;
  u64 now;
  unsigned lateby, fl;
  const i32x4 xr = c.xrsrc, wrs = c.wrsrc;
  const unsigned xso = c.xso, wso = c.wso;
  const unsigned m0x = c.m0base + ((SL + 2) & 3) * 32768;
  f32x4 (&r)[8] = wr[WS ^ 1];
#pragma unroll
  for (int g = 0; g < 16; ++g) {
    if (E == 1 && g == 12) {
      asm volatile("s_waitcnt lgkmcnt(0)" : "+s"(now) :: "memory");
      lateby = (unsigned)now - c.due;
      while ((int)((unsigned)now - c.due) < 0) { __builtin_amdgcn_s_sleep(1); now = __builtin_amdgcn_s_memrealtime(); }
      asm volatile("s_waitcnt vmcnt(8)" : "+v"(fl) :: "memory");
      if (__any(fl < c.call)) {
        c.ok &= wait_flag<__HIP_MEMORY_SCOPE_AGENT, 1>((const unsigned*)(c.fbase + c.fso + c.fvoff), c.call, c.tmo);
        __builtin_amdgcn_s_waitcnt(0xC07F);
      }
      wait_bar<8>();
      SCHED();
    }
    mfma(acc[g][0], a[E][0], xf[g & 7]);
    SCHED();
    if (g < 9) xf[(g + 7) & 7] = x_frag<SL>(c, g + 7);
    else if (E == 0) xf[(g + 7) & 7] = x_frag<NX>(c, g - 9);
    else if (g >= 12) xf[(2 * (g - 12)) & 7] = x_frag<NX>(c, 2 * (g - 12));
    if (E == 1 && g == 0) asm volatile("buffer_load_dword %0, %1, %2, %3 offen sc1" : "=v"(fl) : "v"(c.fvoff), "s"(c.frsrc), "s"(c.fso) : "memory");
    if (E == 1 && g == 9) *(LAS u32x4*)(c.wmine + (WS ^ 1) * 16384 + 1024) = a[0][1];
    if (E == 1 && g == 10) *(LAS u32x4*)(c.wmine + (WS ^ 1) * 16384 + 3072) = a[0][3];
    SCHED();
    mfma(acc[g][1], a[E][1], xf[g & 7]);
    SCHED();
    if (E == 0) {
      switch (g) {
        case 0: x_piece<0>(c, xr, xso); break;   case 1: x_piece<1>(c, xr, xso); break;
        case 2: x_piece<2>(c, xr, xso); break;   case 3: x_piece<3>(c, xr, xso); break;
        case 4: x_piece<4>(c, xr, xso); break;   case 5: x_piece<5>(c, xr, xso); break;
        case 6: x_piece<6>(c, xr, xso); break;   case 7: x_piece<7>(c, xr, xso); break;
        case 8: x_piece<8>(c, xr, xso); break;   case 9: x_piece<9>(c, xr, xso); break;
        case 10: x_piece<10>(c, xr, xso); break; case 11: x_piece<11>(c, xr, xso); break;
        case 12: x_piece<12>(c, xr, xso); break; case 13: x_piece<13>(c, xr, xso); break;
        case 14: x_piece<14>(c, xr, xso); break; default: x_piece<15>(c, xr, xso); break;
      }
    } else {
      if (g < 8) w_load(wr[WS][g & 7], c.wvoff[g & 7], wrs, wso);
      if (g == 5)
        asm volatile("s_waitcnt vmcnt(23)" : "+v"(r[0]), "+v"(r[1]), "+v"(r[2]), "+v"(r[3]), "+v"(r[4]), "+v"(r[5]), "+v"(r[6]), "+v"(r[7]) :: "memory");
      if (g == 8) *(LAS u32x4*)(c.wmine + (WS ^ 1) * 16384) = a[0][0];
      if (g == 9) { c.left -= 1; c.xso += 65536u; asm volatile("" : "+s"(c.left), "+s"(c.xso)); }
      if (g == 10) { c.xrsrc[2] = c.left > 1 ? (int)XT_BYTES : 0; asm volatile("" : "+s"(c.xrsrc)); }
      if (g == 11) { c.wrsrc[2] = c.left > 2 ? (int)W_BYTES : 0; asm volatile("" : "+s"(c.wrsrc)); }
      if (g == 12) { lateby = (int)lateby > (int)c.slack ? lateby - c.slack : 0u; asm volatile("" : "+s"(lateby)); }
      if (g == 13) { c.due += lateby; lateby = __builtin_amdgcn_readlane(c.steps, c.left); asm volatile("" : "+s"(c.due), "+s"(lateby)); }
      if (g == 14) { c.due += lateby; asm volatile("" : "+s"(c.due)); }
      if (g == 15) { c.fso = c.fso + 512u < KH * 512u ? c.fso + 512u : c.fso; asm volatile("" : "+s"(c.fso)); }
    }
    SCHED();
    mfma(acc[g][2], a[E][2], xf[g & 7]);
    SCHED();
    if (E == 0 && g < 4) a[1][g] = *(LAS const u32x4*)(c.wother + WS * 16384 + g * 1024);
    if (E == 1 && g >= 5 && g < 9) { const int t = (g - 5) & 3; a[0][t][0] = pk_bf16(r[0][t], r[1][t]); a[0][t][1] = pk_bf16(r[2][t], r[3][t]); }
    if (E == 1 && g == 9) *(LAS u32x4*)(c.wmine + (WS ^ 1) * 16384 + 2048) = a[0][2];
    if (E == 1 && g == 10) asm volatile("s_memrealtime %0" : "=s"(now) :: "memory");
    if (E == 1 && g >= 12 && g < 15) xf[(2 * (g - 12) + 1) & 7] = x_frag<NX>(c, 2 * (g - 12) + 1);
    SCHED();
    mfma(acc[g][3], a[E][3], xf[g & 7]);
    SCHED();
    if (E == 1 && g >= 5 && g < 9) { const int t = (g - 5) & 3; a[0][t][2] = pk_bf16(r[4][t], r[5][t]); a[0][t][3] = pk_bf16(r[6][t], r[7][t]); }
    if (E == 0 && (g & 3) == 3 && g < 15) set_m0(m0x + ((g + 1) >> 3) * 32768 + (((g + 1) >> 2) & 1) * 4096);
    if (E == 1 && g == 11) { c.wso += 1u << 22; asm volatile("" : "+s"(c.wso)); }
    if (E == 1 && g == 15) set_m0(c.m0base + ((SL + 3) & 3) * 32768);
    SCHED();
  }
}

template <int N, int STRIDE, int SCOPE, int SLEEP>
__device__ __forceinline__ float sweep_max(const u64* slots, unsigned call, unsigned* tmo) {
  const u64 t0 = __builtin_amdgcn_s_memrealtime();
  for (unsigned polls = 1;; ++polls) {
    bool all = true, bad = false;
    float mx = 0.f;
    u64 v[N];
#pragma unroll
    for (int r = 0; r < N; ++r) v[r] = __hip_atomic_load(slots + r * STRIDE, __ATOMIC_RELAXED, SCOPE);
#pragma unroll
    for (int r = 0; r < N; ++r) {
      const float f = __builtin_bit_cast(float, (unsigned)v[r]);
      all &= (unsigned)(v[r] >> 32) == call;
      bad |= f < 0.f;
      mx = fmaxf(mx, f);
    }
    if (__all(all)) {
#pragma unroll
      for (int o = 32; o >= 1; o >>= 1) mx = fmaxf(mx, __shfl_xor(mx, o));
      return __any(bad) ? -1.0f : mx;
    }
    if (((polls & 63u) == 0 && __hip_atomic_load(tmo, __ATOMIC_RELAXED, __HIP_MEMORY_SCOPE_AGENT) == call) ||
        __builtin_amdgcn_s_memrealtime() - t0 > DEADLINE_TICKS) {
      __hip_atomic_store(tmo, call, __ATOMIC_RELAXED, __HIP_MEMORY_SCOPE_AGENT);
      return -1.0f;
    }
    __builtin_amdgcn_s_sleep(SLEEP);
  }
}
__device__ __forceinline__ Quant global_scale(const u64* gmax, unsigned call, unsigned* tmo, LAS float* red, bool ok, int wave, int lane) {
  float gam = ok ? 0.f : -1.0f;
  if (wave == 0 && ok) gam = sweep_max<1, 0, __HIP_MEMORY_SCOPE_SYSTEM, 6>(gmax + (lane & 7), call, tmo);
  if (lane == 0) red[wave] = gam;
  __syncthreads();
  const float r0 = red[0], r1 = red[1], r2 = red[2], r3 = red[3];
  const bool good = fminf(fminf(r0, r1), fminf(r2, r3)) >= 0.f;
  Quant qz;
  qz.inv = (good && r0 > 0.f) ? 127.0f / r0 : 0.f;
  qz.lo = good ? QBIAS - 127.0f : QBIAS - 128.0f;
  qz.hi = good ? QBIAS + 127.0f : QBIAS - 128.0f;
  asm volatile("" : "+v"(qz.lo), "+v"(qz.hi));
  return qz;
}

__device__ __forceinline__ void tile_of(int j, int& pd, int& ct) {
  const int l = j - NEARLY, cl = l / 7;
  if (l < 0) { pd = j % 7 + 1; ct = 2 * (j / 7) + 1; }
  else if (l < 7 * (16 - NE)) { pd = l % 7 + 1; ct = cl < 8 ? 2 * cl : 2 * (NE + cl - 8) + 1; }
  else { pd = 0; ct = l - 7 * (16 - NE); }
}

__global__ void __launch_bounds__(NTHR, 1) fused_kernel(Params p) {
  extern __shared__ __attribute__((aligned(1024))) char smem[];
  const int tid = threadIdx.x, lane = tid & 63;
  const int wave = __builtin_amdgcn_readfirstlane(tid >> 6);
  char* ws = p.myws;

  STAMP_DECL;
  STAMP(0);
  STAMPC(8);
  const int b = blockIdx.x, xcd = b & 7, idx = b >> 3;
  const int kh = xcd & 1, slot = idx * 4 + (xcd >> 1), unit = slot * 2 + kh;
  const bool early = slot < NEARLY;
  const unsigned t_start = (unsigned)__builtin_amdgcn_s_memrealtime();
  int pd, ct;
  tile_of(slot, pd, ct);
  const unsigned dest = (unsigned)(p.rank + pd) & 7u;
  const int k0 = kh * KH;
  const int wn = wave & 1, wm = wave >> 1;
  const int q = lane >> 4, i16 = lane & 15;
  const unsigned lane16 = lane * 16;
  unsigned* tmo = (unsigned*)(ws + WS_TMO);

  Ctx c;
  c.wrsrc = make_rsrc(p.w, (unsigned)W_BYTES);
  c.xrsrc = make_rsrc(ws + WS_XT, (unsigned)XT_BYTES);
#pragma unroll
  for (int j = 0; j < 8; ++j) {
    c.wvoff[j] = ((unsigned)(wm * 32 + q * 8 + j) << 16) + wn * 256 + i16 * 16;
    asm volatile("" : "+v"(c.wvoff[j]));
  }
  c.xvoff0 = wave * 8192 + lane16;
  c.xvoff1 = c.xvoff0 + 4096;
  asm volatile("" : "+v"(c.xvoff1));
  c.m0base = (unsigned)(size_t)(LAS char*)smem + wave * 8192;
  constexpr int nblk = KH;
  const unsigned wsb = ((unsigned)k0 << 22) + (dest * 16 + ct) * 512u;
  c.left = nblk;
  c.xso = (unsigned)(k0 + 1) * 65536u;
  c.wso = wsb + (2u << 22);
  const int phase = early ? (b * 37 & 255) : idx < 16 ? (b * 37 & 63) : 64 + (b * 37 & 255) * 3 / 4;
  c.due = t_start + p.first + phase * p.step[!early][0] / 256;
  c.slack = p.slack;
  c.steps = p.step[!early][(nblk - lane) & 63];
  int sprev = p.step[!early][(nblk - 1 - lane) & 63];
  c.lrd0 = (LAS const char*)smem + wm * 16384 + lane16;
  c.lrd1 = c.lrd0 + 65536;
  asm volatile("" : "+v"(c.lrd1));
  c.wmine = (LAS char*)smem + LDS_X + wave * 4096 + lane16;
  c.wother = (LAS const char*)smem + LDS_X + (wave ^ 2) * 4096 + lane16;

  f32x4 acc[16][4];
  f32x4 wr[2][8];
  u32x4 a[2][4], xf[8];

  unsigned* xflag = (unsigned*)(ws + WS_XFLAG);
  const int hs = kh * NT + slot;
  unsigned prev;
  asm volatile("global_load_dword %0, %1, off sc1" : "=v"(prev) : "v"(xflag + hs * 32) : "memory");
  {
    const int st = hs >> 1, h = hs & 1;
    const i32x4 xin = make_rsrc(p.x, (unsigned)((size_t)ML * KD * 4));
    f32x4 raw[8];
#pragma unroll
    for (int j = 0; j < 4; ++j) {
      const unsigned o = (unsigned)((h * 16 + ((wave * 4 + j) ^ (kh << 3))) * 16 + i16) * (KD * 4) + ((st ^ h) * 32 + q * 8) * 4;
      asm volatile("buffer_load_dwordx4 %0, %2, %3, 0 offen\n\tbuffer_load_dwordx4 %1, %2, %3, 0 offen offset:16"
                   : "=&v"(raw[2 * j]), "=&v"(raw[2 * j + 1]) : "v"(o), "s"(xin) : "memory");
    }
    const unsigned so = (unsigned)(hs * 16384 + wave * 4096);
    unsigned warm;
    asm volatile("buffer_load_dword %0, %1, %2, %3 offen sc1" : "=v"(warm) : "v"(0), "s"(c.xrsrc), "s"(so) : "memory");
#pragma unroll
    for (int m = 0; m < 16; ++m)
#pragma unroll
      for (int t = 0; t < 4; ++t) {
        acc[m][t] = f32x4{0.f, 0.f, 0.f, 0.f};
        asm volatile("" : "+a"(acc[m][t]));
      }
    asm volatile("s_waitcnt vmcnt(0)" : "+v"(c.steps), "+v"(sprev), "+v"(prev), "+v"(warm), "+v"(raw[0]), "+v"(raw[1]), "+v"(raw[2]), "+v"(raw[3]), "+v"(raw[4]), "+v"(raw[5]), "+v"(raw[6]), "+v"(raw[7]) :: "memory");
#pragma unroll
    for (int j = 0; j < 4; ++j) {
      const f32x4 v0 = raw[2 * j], v1 = raw[2 * j + 1];
      const u32x4 d = {pk_bf16(v0[0], v0[1]), pk_bf16(v0[2], v0[3]), pk_bf16(v1[0], v1[1]), pk_bf16(v1[2], v1[3])};
      asm volatile("buffer_store_dwordx4 %0, %1, %2, %3 offen offset:%4 sc1\n\ts_nop 1" :: "v"(d), "v"(lane16), "s"(c.xrsrc), "s"(so), "i"(j * 1024) : "memory");
    }
  }
  c.steps += phase * (c.steps - sprev) / 256;
  const unsigned call = __builtin_amdgcn_readfirstlane(prev) + 1;
  wait_bar<0>();
  if (tid == 0) __hip_atomic_store(xflag + hs * 32, call, __ATOMIC_RELAXED, __HIP_MEMORY_SCOPE_AGENT);
  if (b == 0 && wave == 1 && lane < NR)
    __hip_atomic_store((unsigned*)(p.ws[lane] + WS_START + p.rank * 128), call, __ATOMIC_RELAXED, __HIP_MEMORY_SCOPE_SYSTEM);
#pragma unroll
  for (int j = 0; j < 8; ++j) w_load(wr[0][j], c.wvoff[j], c.wrsrc, wsb);
  c.fbase = (const char*)(xflag + kh * NT * 32);
  c.frsrc = make_rsrc(c.fbase, NT * 128);
  c.fvoff = (lane & 3) * 128;
  c.fso = 2 * 512;
  c.call = call;
  c.tmo = tmo;
  c.ok = true;
  if (idx >= 16) {
    const int pw = (idx - 16) * 4 + wave;
    c.ok = wait_flag<__HIP_MEMORY_SCOPE_AGENT, 2>((const unsigned*)(c.fbase + (lane < 32 ? lane * 512 + (pw >> 4) * 128 : (lane & 7) * 128)), call, tmo);
    const unsigned pv = (unsigned)k0 * 65536u + pw * 1024 + lane16;
    u32x4 pf = {0u, 0u, 0u, 0u};
#pragma unroll
    for (int j = 2; j < KH; ++j) {
      const unsigned v = pv + j * 65536;
      asm volatile("buffer_load_dwordx4 %0, %1, %2, 0 offen sc1" : "+v"(pf) : "v"(v), "s"(c.xrsrc) : "memory");
    }
    asm volatile("s_waitcnt vmcnt(0)" : "+v"(pf) :: "memory");
  } else if (wave == 0) {
    c.ok = wait_flag<__HIP_MEMORY_SCOPE_AGENT, 1>((const unsigned*)(c.fbase + (lane & 7) * 128), call, tmo);
  }
  asm volatile("s_waitcnt vmcnt(0)" : "+v"(wr[0][0]), "+v"(wr[0][1]), "+v"(wr[0][2]), "+v"(wr[0][3]), "+v"(wr[0][4]), "+v"(wr[0][5]), "+v"(wr[0][6]), "+v"(wr[0][7]) :: "memory");
#pragma unroll
  for (int t = 0; t < 4; ++t) {
    a[0][t] = u32x4{pk_bf16(wr[0][0][t], wr[0][1][t]), pk_bf16(wr[0][2][t], wr[0][3][t]), pk_bf16(wr[0][4][t], wr[0][5][t]), pk_bf16(wr[0][6][t], wr[0][7][t])};
    *(LAS u32x4*)(c.wmine + t * 1024) = a[0][t];
  }
  wait_bar<0>();
  STAMP(10);
  {
    const unsigned xso = (unsigned)k0 * 65536u;
    set_m0(c.m0base); asm volatile("s_nop 0");
    x_piece<0>(c, c.xrsrc, xso); x_piece<1>(c, c.xrsrc, xso); x_piece<2>(c, c.xrsrc, xso); x_piece<3>(c, c.xrsrc, xso);
    set_m0(c.m0base + 4096); asm volatile("s_nop 0");
    x_piece<4>(c, c.xrsrc, xso); x_piece<5>(c, c.xrsrc, xso); x_piece<6>(c, c.xrsrc, xso); x_piece<7>(c, c.xrsrc, xso);
    set_m0(c.m0base + 32768); asm volatile("s_nop 0");
    x_piece<8>(c, c.xrsrc, xso); x_piece<9>(c, c.xrsrc, xso); x_piece<10>(c, c.xrsrc, xso); x_piece<11>(c, c.xrsrc, xso);
    set_m0(c.m0base + 32768 + 4096); asm volatile("s_nop 0");
    x_piece<12>(c, c.xrsrc, xso); x_piece<13>(c, c.xrsrc, xso); x_piece<14>(c, c.xrsrc, xso); x_piece<15>(c, c.xrsrc, xso);
  }
#pragma unroll
  for (int j = 0; j < 8; ++j) w_load(wr[1][j], c.wvoff[j], c.wrsrc, wsb + (1u << 22));
  __builtin_amdgcn_s_waitcnt(0xC07F);
  wait_bar<8>();
  SCHED();
  set_m0(c.m0base + 2 * 32768);
#pragma unroll
  for (int t = 0; t < 7; ++t) xf[t] = x_frag<0>(c, t);
  SCHED();
  STAMP(1);

  do {
    kstep<0>(c, acc, wr, a, xf);
    kstep<1>(c, acc, wr, a, xf);
    kstep<2>(c, acc, wr, a, xf);
    kstep<3>(c, acc, wr, a, xf);
  } while (c.left > 0);
  acc_fence(acc);
  bool ok = c.ok;
  STAMP(2);
  STAMPC(9);

  LAS float* red = (LAS float*)(smem + LDS_X);
  const size_t gmax_off = WS_GMAX + (size_t)(call & 1) * 128;
  const size_t wave_off = (size_t)unit * HALF_B + wave * 16384;
  unsigned* pflag = (unsigned*)(ws + WS_PFLAG) + wave * 32;
  {
    const i32x4 prs = make_rsrc(ws + WS_PART + wave_off, 16384);
#pragma unroll
    for (int j = 0; j < 16; ++j) {
      const u32x4 d = pack_piece(acc, 16 + j);
      asm volatile("buffer_store_dwordx4 %0, %1, %2, %3 offen offset:%4 sc1\n\ts_nop 1"
                   :: "v"(d), "v"(lane16), "s"(prs), "s"((j >> 2) * 4096), "i"((j & 3) * 1024) : "memory");
    }
  }
  asm volatile("s_waitcnt vmcnt(0)"
               : "+v"(wr[0][0]), "+v"(wr[0][1]), "+v"(wr[0][2]), "+v"(wr[0][3]), "+v"(wr[0][4]), "+v"(wr[0][5]), "+v"(wr[0][6]), "+v"(wr[0][7]),
                 "+v"(wr[1][0]), "+v"(wr[1][1]), "+v"(wr[1][2]), "+v"(wr[1][3]), "+v"(wr[1][4]), "+v"(wr[1][5]), "+v"(wr[1][6]), "+v"(wr[1][7])
               :: "memory");
  if (lane == 0) __hip_atomic_store(pflag + unit * 128, call, __ATOMIC_RELAXED, __HIP_MEMORY_SCOPE_AGENT);
  ok &= wait_flag<__HIP_MEMORY_SCOPE_SYSTEM, 2>((lane & 1) ? (const unsigned*)(ws + WS_START + dest * 128) : pflag + (unit ^ 1) * 128, call, tmo);
  f32x4 y[8][4];
  {
    const i32x4 prs = make_rsrc(ws + WS_PART + (wave_off ^ HALF_B), 16384);
    u32x4 pv[16];
#pragma unroll
    for (int j = 0; j < 16; ++j)
      asm volatile("buffer_load_dwordx4 %0, %1, %2, %3 offen offset:%4 sc1" : "=v"(pv[j]) : "v"(lane16), "s"(prs), "s"((j >> 2) * 4096), "i"((j & 3) * 1024) : "memory");
    const unsigned one = (i16 >> 2) == q ? ((i16 & 1) ? 0x3F800000u : 0x00003F80u) : 0u;
    const unsigned e0 = (i16 & 2) ? 0u : one, e1 = (i16 & 2) ? one : 0u;
    u32x4 sel0 = {e0, e1, 0u, 0u}, sel1 = {0u, 0u, e0, e1};
    asm volatile("s_nop 1" : "+v"(sel0), "+v"(sel1));
    float am = 0.f;
#pragma unroll
    for (int j = 0; j < 16 + 4; ++j) {
      if (j < 16) {
        asm volatile("s_waitcnt vmcnt(%1)" : "+v"(pv[j]) : "n"(15 - j) : "memory");
        mfma(acc[j >> 1][(j & 1) * 2], sel0, pv[j]);
        mfma(acc[j >> 1][(j & 1) * 2 + 1], sel1, pv[j]);
      }
      if (j == 16) asm volatile("s_nop 7\n\ts_nop 7" ::: "memory");
      if (j >= 4) {
        const int m = (j - 4) >> 1, u = (j - 4) & 1;
        asm volatile("" : "+a"(acc[m][2 * u]), "+a"(acc[m][2 * u + 1]), "+v"(am));
        y[m][2 * u] = acc[m][2 * u]; y[m][2 * u + 1] = acc[m][2 * u + 1];
        asm volatile("" : "+v"(y[m][2 * u]), "+v"(y[m][2 * u + 1]));
        const f32x4 lo = y[m][2 * u], hi = y[m][2 * u + 1];
        am = fmaxf(fmaxf(am, fabsf(lo[0])), fabsf(lo[1])); am = fmaxf(fmaxf(am, fabsf(lo[2])), fabsf(lo[3]));
        am = fmaxf(fmaxf(am, fabsf(hi[0])), fabsf(hi[1])); am = fmaxf(fmaxf(am, fabsf(hi[2])), fabsf(hi[3]));
        asm volatile("" : "+v"(am));
      }
    }
#pragma unroll
    for (int o = 32; o >= 1; o >>= 1) am = fmaxf(am, __shfl_xor(am, o));
    __syncthreads();
    if (lane == 0) red[wave] = ok ? am : -1.0f;
    __syncthreads();
    if (tid == 0) {
      const float r0 = red[0], r1 = red[1], r2 = red[2], r3 = red[3];
      const float wam = fminf(fminf(r0, r1), fminf(r2, r3)) < 0.f ? -1.0f : fmaxf(fmaxf(r0, r1), fmaxf(r2, r3));
      __hip_atomic_store((u64*)(ws + WS_AMAX) + unit, ((u64)call << 32) | __builtin_bit_cast(unsigned, wam), __ATOMIC_RELAXED, __HIP_MEMORY_SCOPE_AGENT);
    }
  }
  STAMP(3);
  const int row0 = wm * 256 + kh * 128 + i16, col0 = ct * 128 + wn * 64 + q * 16;
  const int srank = (p.rank - pd) & 7;

  if (early) {
    char* dws = p.ws[dest];
    const i32x4 srs = make_rsrc(dws + WS_STAGE + wave_off, ok ? 16384 : 0);
#pragma unroll
    for (int j = 0; j < 16; ++j) {
      const f32x4 lo = y[j >> 1][(j & 1) * 2], hi = y[j >> 1][(j & 1) * 2 + 1];
      const u32x4 d = {pk_bf16(lo[0], lo[1]), pk_bf16(lo[2], lo[3]), pk_bf16(hi[0], hi[1]), pk_bf16(hi[2], hi[3])};
      asm volatile("s_waitcnt vmcnt(%5)\n\tbuffer_store_dwordx4 %0, %1, %2, %3 offen offset:%4 sc0 sc1\n\ts_nop 1"
                   :: "v"(d), "v"(lane16), "s"(srs), "s"((j >> 2) * 4096), "i"((j & 3) * 1024), "n"(SEND_WIN - 1) : "memory");
    }
    asm volatile("s_waitcnt vmcnt(0)" ::: "memory");
    if (lane == 0)
      __hip_atomic_store((unsigned*)(dws + WS_EFLAG + (size_t)(unit * 4 + wave) * 128), call, __ATOMIC_RELAXED, __HIP_MEMORY_SCOPE_SYSTEM);
    STAMP(5);
    ok &= wait_flag<__HIP_MEMORY_SCOPE_SYSTEM, 8>((const unsigned*)(ws + WS_EFLAG + (size_t)(unit * 4 + wave) * 128), call, tmo);
    u32x4 st[16];
    {
      const i32x4 rs = make_rsrc(ws + WS_STAGE + wave_off, 16384);
#pragma unroll
      for (int j = 0; j < 16; ++j)
        asm volatile("buffer_load_dwordx4 %0, %1, %2, %3 offen offset:%4 sc0 sc1" : "=v"(st[j]) : "v"(lane16), "s"(rs), "s"((j >> 2) * 4096), "i"((j & 3) * 1024) : "memory");
      asm volatile("s_waitcnt vmcnt(0)" ::: "memory");
#pragma unroll
      for (int j = 0; j < 16; ++j) asm volatile("" : "+v"(st[j]));
    }
    STAMP(6);
    const Quant qz = global_scale((const u64*)(ws + gmax_off), call, tmo, red + 8, ok, wave, lane);
    STAMP(4);
    char* ob = p.out[p.rank] + (size_t)(srank * ML + row0) * NL + col0;
#pragma unroll
    for (int m = 0; m < 8; ++m) {
      const u32x4 s0 = st[2 * m], s1 = st[2 * m + 1];
      u32x4 o;
#pragma unroll
      for (int r = 0; r < 4; ++r) {
        const int h = r >> 1;
        const unsigned w0 = s0[h], w1 = s0[2 + h], w2 = s1[h], w3 = s1[2 + h];
        const float v0 = __builtin_bit_cast(float, (r & 1) ? (w0 & 0xFFFF0000u) : (w0 << 16));
        const float v1 = __builtin_bit_cast(float, (r & 1) ? (w1 & 0xFFFF0000u) : (w1 << 16));
        const float v2 = __builtin_bit_cast(float, (r & 1) ? (w2 & 0xFFFF0000u) : (w2 << 16));
        const float v3 = __builtin_bit_cast(float, (r & 1) ? (w3 & 0xFFFF0000u) : (w3 << 16));
        o[r] = quant4(v0, v1, v2, v3, qz);
      }
      *(u32x4*)(ob + (size_t)m * 16 * NL) = o;
    }
    STAMP(7);
    STAMP_OUT();
    return;
  }

  if (unit == NWG - 1 && wave == 1) {
    const float rm = sweep_max<4, 64, __HIP_MEMORY_SCOPE_AGENT, 1>((const u64*)(ws + WS_AMAX) + lane, call, tmo);
    if (lane < NR)
      __hip_atomic_store((u64*)(p.ws[lane] + gmax_off) + p.rank, ((u64)call << 32) | __builtin_bit_cast(unsigned, rm), __ATOMIC_RELAXED, __HIP_MEMORY_SCOPE_SYSTEM);
  }
  const Quant qz = global_scale((const u64*)(ws + gmax_off), call, tmo, red + 8, ok, wave, lane);
  STAMP(4);
  {
    char* ob = p.out[dest] + (size_t)(p.rank * ML + row0) * NL + col0;
#pragma unroll
    for (int m = 0; m < 8; ++m) {
      u32x4 o;
#pragma unroll
      for (int r = 0; r < 4; ++r) o[r] = quant4(y[m][0][r], y[m][1][r], y[m][2][r], y[m][3][r], qz);
      asm volatile("global_store_dwordx4 %0, %1, off sc0 sc1\n\ts_nop 1" :: "v"(ob + (size_t)m * 16 * NL), "v"(o) : "memory");
    }
  }
  asm volatile("s_waitcnt vmcnt(0)" ::: "memory");
  __syncthreads();
  STAMP(5);
  if (wave == 0) {
    if (lane == 0)
      __hip_atomic_store((unsigned*)(p.ws[dest] + WS_LFLAG + (size_t)unit * 128), call, __ATOMIC_RELAXED, __HIP_MEMORY_SCOPE_SYSTEM);
    if (!wait_flag<__HIP_MEMORY_SCOPE_SYSTEM, 2>((const unsigned*)(ws + WS_LFLAG + (size_t)unit * 128), call, tmo) && lane == 0)
      p.out[p.rank][(size_t)(srank * ML + kh * 128) * NL + ct * 128] = (char)-128;
    STAMP(7);
    STAMP_OUT();
  }
}

static void make_pace(Params& p, const int* pa) {
  const double T0 = pa[0], TH = pa[1], T1 = pa[2], TR = pa[3], TE = pa[4], TC = pa[5], A = pa[6];
  p.first = pa[6]; p.slack = pa[7];
  const double ende = A + KH * TE;
  double t = A;
  for (int v = 0; v < 66; ++v) {
    double done = 0, u = t;
    while (done < 1.0) {
      const double env = u < TH ? T0 : u < TR ? T0 + (T1 - T0) * (u - TH) / (TR - TH) : T1;
      double r = (NWG / env - (u < ende ? 2 * NEARLY / TE : 0.0)) / (NWG - 2 * NEARLY);
      if (r > 1.0 / TC) r = 1.0 / TC;
      if (r < 0.001) r = 0.001;
      done += r; u += 1.0;
    }
    p.step[0][v] = (unsigned short)TE;
    p.step[1][v] = (unsigned short)(u - t);
    if (v) t = u; else t += (u - t) * 0.5;
  }
}

extern "C" void kernel_launch(int rank, int nranks, ncclComm_t comm,
                              hipStream_t stream,
                              void* const* d_in, const int* in_sizes, int n_in,
                              void* d_out, int out_size,
                              const int* in_dtypes, int out_dtype,
                              void* const* d_out_all,
                              void* const* d_ws_all, size_t ws_bytes) {
  (void)comm; (void)in_dtypes; (void)out_dtype; (void)d_out;
  if (nranks != NR || n_in != 2 || in_sizes[0] != ML * KD || in_sizes[1] != KD * ND || out_size != NR * ML * NL ||
      ws_bytes < WS_END || (unsigned)rank >= (unsigned)NR) {
    fprintf(stderr, "unexpected problem shape\n");
    return;
  }
  HIP_CHECK(hipFuncSetAttribute((const void*)fused_kernel, hipFuncAttributeMaxDynamicSharedMemorySize, LDS_BYTES));
  Params p;
  p.x = (const float*)d_in[0];
  p.w = (const float*)d_in[1];
  for (int r = 0; r < NR; ++r) { p.out[r] = (char*)d_out_all[r]; p.ws[r] = (char*)d_ws_all[r]; }
  p.rank = rank;
  p.myws = (char*)d_ws_all[rank];
  int pa[8] = {PACE_ARGS};
  make_pace(p, pa);
  fused_kernel<<<NWG, NTHR, LDS_BYTES, stream>>>(p);
  HIP_CHECK(hipGetLastError());
}
```

```cpp
#include <hip/hip_runtime.h>
#include <rccl/rccl.h>
#include <cstdio>
#include <cstdint>
#include <cstdlib>
#include <cstddef>

#define HIP_CHECK(expr) do { hipError_t _e = (expr); if (_e != hipSuccess) { \
    fprintf(stderr, "HIP error %s at %s:%d\n", hipGetErrorString(_e), __FILE__, __LINE__); \
    return; } } while (0)

#define LAS __attribute__((address_space(3)))
#define GAS __attribute__((address_space(1)))
typedef float f32x4 __attribute__((ext_vector_type(4)));
typedef float f32x2 __attribute__((ext_vector_type(2)));
typedef __bf16 bf16x2 __attribute__((ext_vector_type(2)));
typedef unsigned u32x4 __attribute__((ext_vector_type(4)));
typedef int i32x4 __attribute__((ext_vector_type(4)));
typedef int i32x2 __attribute__((ext_vector_type(2)));
typedef unsigned long long u64;

constexpr int NR = 8;
constexpr int KD = 4096, ND = 16384;
constexpr int ML = 512;
constexpr int NL = ND / NR;
constexpr int NWG = 256, NTHR = 256;
constexpr int NBLK = KD / 64;
constexpr int NT = ND / 128;
#define NE 9
constexpr int NEARLY = 7 * NE;
static_assert(NE >= 8 && NE <= 12, "the workgroups that prefetch x belong to early tiles that start late");
constexpr int KH = NBLK / 2;
#define PACE_ARGS 220, 2100, 150, 4300, 130, 150, 140, 600, 240, 100, 2, 240
constexpr int LDS_X = 4 * 32768;
constexpr int LDS_BYTES = LDS_X + 32768;
constexpr size_t HALF_B = 65536;

constexpr size_t WS_TMO = 128;
constexpr size_t WS_START = 1024;
constexpr size_t WS_GMAX = 2048;
constexpr size_t WS_AMAX = 4096;
constexpr size_t WS_XFLAG = WS_AMAX + NWG * 8;
constexpr size_t WS_LFLAG = WS_XFLAG + NWG * 128;
constexpr size_t WS_EFLAG = WS_LFLAG + NWG * 128;
constexpr size_t WS_PFLAG = WS_EFLAG + NWG * 4 * 128;
constexpr size_t WS_STAMP = WS_PFLAG + NWG * 4 * 128;
constexpr size_t WS_XT = 1 << 20;
constexpr size_t XT_BYTES = (size_t)ML * KD * 2, W_BYTES = (size_t)KD * ND * 4;
constexpr size_t WS_PART = WS_XT + XT_BYTES;
constexpr size_t WS_STAGE = WS_PART + NWG * HALF_B;
constexpr size_t WS_END = WS_STAGE + 2 * NEARLY * HALF_B;
static_assert(WS_STAMP + 65536 + NWG * 512 <= WS_XT, "control block overlaps x");
constexpr u64 DEADLINE_TICKS = 50000000ull;

struct Params {
  const float* x;
  const float* w;
  char* myws;
  int rank;
  int slack;
  char* out[NR];
  char* ws[NR];
  int nwide;
  int start[NE + 1], te[NE];
  int step[KH + 1];
};

#define SEND_WIN 3
#define STAMP(i) do {} while (0)
#define STAMPC(i) do {} while (0)
#define STAMP_DECL do {} while (0)
#define STAMP_OUT() do {} while (0)

__device__ __forceinline__ unsigned pk_bf16(float lo, float hi) {
  f32x2 v = {lo, hi};
  return __builtin_bit_cast(unsigned, __builtin_convertvector(v, bf16x2));
}
__device__ __forceinline__ float wave_max(float f) {
  int v = __builtin_bit_cast(int, f);
  v = max(v, __builtin_amdgcn_update_dpp(0, v, 0x111, 0xf, 0xf, true));
  v = max(v, __builtin_amdgcn_update_dpp(0, v, 0x112, 0xf, 0xf, true));
  v = max(v, __builtin_amdgcn_update_dpp(0, v, 0x114, 0xf, 0xf, true));
  v = max(v, __builtin_amdgcn_update_dpp(0, v, 0x118, 0xf, 0xf, true));
  const int r = max(max(__builtin_amdgcn_readlane(v, 15), __builtin_amdgcn_readlane(v, 31)), max(__builtin_amdgcn_readlane(v, 47), __builtin_amdgcn_readlane(v, 63)));
  return __builtin_bit_cast(float, r);
}
__device__ __forceinline__ i32x4 make_rsrc(const void* base, unsigned bytes) {
  const u64 a = (u64)base;
  return i32x4{__builtin_amdgcn_readfirstlane((int)(unsigned)a), __builtin_amdgcn_readfirstlane((int)(unsigned)(a >> 32)), (int)bytes, 0x00020000};
}

__device__ __forceinline__ void w_load(f32x4& d, unsigned voff, i32x4 rsrc, unsigned soff) {
  asm volatile("buffer_load_dwordx4 %0, %1, %2, %3 offen nt" : "=v"(d) : "v"(voff), "s"(rsrc), "s"(soff) : "memory");
}
__device__ __forceinline__ void set_m0(unsigned m0v) { asm volatile("s_mov_b32 m0, %0" :: "s"(m0v) : "memory"); }
template <int IMM>
__device__ __forceinline__ void x_load(unsigned voff, i32x4 rsrc, unsigned soff) {
  asm volatile("buffer_load_dwordx4 %0, %1, %2 offen offset:%3 sc1 lds" :: "v"(voff), "s"(rsrc), "s"(soff), "i"(IMM) : "memory");
}
template <int N>
__device__ __forceinline__ void wait_bar() { asm volatile("s_waitcnt vmcnt(%0) lgkmcnt(0)\n\ts_barrier" :: "n"(N) : "memory"); }
__device__ __forceinline__ void mfma(f32x4& c, u32x4 a, u32x4 b) {
  asm volatile("v_mfma_f32_16x16x32_bf16 %0, %1, %2, %0" : "+a"(c) : "v"(a), "v"(b));
}
#define SCHED() __builtin_amdgcn_sched_barrier(0)
__device__ __forceinline__ void acc_fence(f32x4 (&acc)[16][4]) {
  asm volatile("s_nop 7\n\ts_nop 7" ::: "memory");
#pragma unroll
  for (int m = 0; m < 16; ++m)
#pragma unroll
    for (int t = 0; t < 4; ++t) asm volatile("" : "+a"(acc[m][t]));
  SCHED();
}
__device__ __forceinline__ u32x4 pack_piece(f32x4 (&acc)[16][4], int j) {
  const int m = j >> 1, u = j & 1;
  asm volatile("" : "+a"(acc[m][2 * u]), "+a"(acc[m][2 * u + 1]));
  const f32x4 lo = acc[m][2 * u], hi = acc[m][2 * u + 1];
  return u32x4{pk_bf16(lo[0], lo[1]), pk_bf16(lo[2], lo[3]), pk_bf16(hi[0], hi[1]), pk_bf16(hi[2], hi[3])};
}
constexpr float QBIAS = 12582912.0f;
struct Quant { float inv, lo, hi; };
__device__ __forceinline__ unsigned quant4(float v0, float v1, float v2, float v3, const Quant& qz) {
  const unsigned e0 = __builtin_bit_cast(unsigned, __builtin_amdgcn_fmed3f(__builtin_fmaf(v0, qz.inv, QBIAS), qz.lo, qz.hi));
  const unsigned e1 = __builtin_bit_cast(unsigned, __builtin_amdgcn_fmed3f(__builtin_fmaf(v1, qz.inv, QBIAS), qz.lo, qz.hi));
  const unsigned e2 = __builtin_bit_cast(unsigned, __builtin_amdgcn_fmed3f(__builtin_fmaf(v2, qz.inv, QBIAS), qz.lo, qz.hi));
  const unsigned e3 = __builtin_bit_cast(unsigned, __builtin_amdgcn_fmed3f(__builtin_fmaf(v3, qz.inv, QBIAS), qz.lo, qz.hi));
  return __builtin_amdgcn_perm(e1, e0, 0x0c0c0400u) | __builtin_amdgcn_perm(e3, e2, 0x04000c0cu);
}

template <int SCOPE, int SLEEP, int N = 1, int STRIDE = 0>
__device__ __forceinline__ bool wait_flag(const unsigned* flag, unsigned call, unsigned* tmo) {
  const u64 t0 = __builtin_amdgcn_s_memrealtime();
  for (unsigned polls = 1;; ++polls) {
    unsigned v[N];
    bool pend = false;
#pragma unroll
    for (int i = 0; i < N; ++i) v[i] = __hip_atomic_load(flag + i * STRIDE, __ATOMIC_RELAXED, SCOPE);
#pragma unroll
    for (int i = 0; i < N; ++i) pend |= v[i] < call;
    if (!__any(pend)) return true;
    if (((polls & 63u) == 0 && __hip_atomic_load(tmo, __ATOMIC_RELAXED, __HIP_MEMORY_SCOPE_AGENT) == call) ||
        __builtin_amdgcn_s_memrealtime() - t0 > DEADLINE_TICKS) {
      __hip_atomic_store(tmo, call, __ATOMIC_RELAXED, __HIP_MEMORY_SCOPE_AGENT);
      return false;
    }
    __builtin_amdgcn_s_sleep(SLEEP);
  }
}

struct Ctx {
  i32x4 wrsrc, xrsrc;
  unsigned wvoff[8], xvoff0, xvoff1, m0base;
  int left;
  unsigned xso, wso;
  unsigned due, slack;
  int steps;
  i32x4 frsrc;
  const char* fbase;
  unsigned fvoff, fso, call;
  unsigned* tmo;
  bool ok;
  LAS const char* lrd0;
  LAS const char* lrd1;
  LAS char* wmine;
  LAS const char* wother;
};

template <int SLOT>
__device__ __forceinline__ u32x4 x_frag(const Ctx& c, int mt) {
  return *(LAS const u32x4*)((SLOT < 2 ? c.lrd0 : c.lrd1) + (SLOT & 1) * 32768 + mt * 1024);
}
template <int G>
__device__ __forceinline__ void x_piece(const Ctx& c, i32x4 xr, unsigned xso) {
  x_load<(G & 3) * 1024>((G & 4) ? c.xvoff1 : c.xvoff0, xr, (G & 8) ? xso + 32768 : xso);
}
template <int SL>
__device__ __forceinline__ void kstep(Ctx& c, f32x4 (&acc)[16][4], f32x4 (&wr)[2][8], u32x4 (&a)[2][4], u32x4 (&xf)[8]) {
  constexpr int E = SL & 1, WS = SL >> 1, NX = (SL + 1) & 3;
  u64 now;
  unsigned lateby, fl;
  const i32x4 xr = c.xrsrc, wrs = c.wrsrc;
  const unsigned xso = c.xso, wso = c.wso;
  const unsigned m0x = c.m0base + ((SL + 2) & 3) * 32768;
  f32x4 (&r)[8] = wr[WS ^ 1];
#pragma unroll
  for (int g = 0; g < 16; ++g) {
    if (E == 1 && g == 12) {
      asm volatile("s_waitcnt lgkmcnt(0)" : "+s"(now) :: "memory");
      lateby = (unsigned)now - c.due;
      while ((int)((unsigned)now - c.due) < 0) { __builtin_amdgcn_s_sleep(1); now = __builtin_amdgcn_s_memrealtime(); }
      asm volatile("s_waitcnt vmcnt(8)" : "+v"(fl) :: "memory");
      if (__any(fl < c.call)) {
        c.ok &= wait_flag<__HIP_MEMORY_SCOPE_AGENT, 1>((const unsigned*)(c.fbase + c.fso + c.fvoff), c.call, c.tmo);
        __builtin_amdgcn_s_waitcnt(0xC07F);
      }
      wait_bar<8>();
      SCHED();
    }
    mfma(acc[g][0], a[E][0], xf[g & 7]);
    SCHED();
    if (g < 9) xf[(g + 7) & 7] = x_frag<SL>(c, g + 7);
    else if (E == 0) xf[(g + 7) & 7] = x_frag<NX>(c, g - 9);
    else if (g >= 12) xf[(2 * (g - 12)) & 7] = x_frag<NX>(c, 2 * (g - 12));
    if (E == 1 && g == 0) asm volatile("buffer_load_dword %0, %1, %2, %3 offen sc1" : "=v"(fl) : "v"(c.fvoff), "s"(c.frsrc), "s"(c.fso) : "memory");
    if (E == 1 && g == 9) *(LAS u32x4*)(c.wmine + (WS ^ 1) * 16384 + 1024) = a[0][1];
    if (E == 1 && g == 10) *(LAS u32x4*)(c.wmine + (WS ^ 1) * 16384 + 3072) = a[0][3];
    SCHED();
    mfma(acc[g][1], a[E][1], xf[g & 7]);
    SCHED();
    if (E == 0) {
      switch (g) {
        case 0: x_piece<0>(c, xr, xso); break;   case 1: x_piece<1>(c, xr, xso); break;
        case 2: x_piece<2>(c, xr, xso); break;   case 3: x_piece<3>(c, xr, xso); break;
        case 4: x_piece<4>(c, xr, xso); break;   case 5: x_piece<5>(c, xr, xso); break;
        case 6: x_piece<6>(c, xr, xso); break;   case 7: x_piece<7>(c, xr, xso); break;
        case 8: x_piece<8>(c, xr, xso); break;   case 9: x_piece<9>(c, xr, xso); break;
        case 10: x_piece<10>(c, xr, xso); break; case 11: x_piece<11>(c, xr, xso); break;
        case 12: x_piece<12>(c, xr, xso); break; case 13: x_piece<13>(c, xr, xso); break;
        case 14: x_piece<14>(c, xr, xso); break; default: x_piece<15>(c, xr, xso); break;
      }
    } else {
      if (g < 8) w_load(wr[WS][g & 7], c.wvoff[g & 7], wrs, wso);
      if (g == 5)
        asm volatile("s_waitcnt vmcnt(23)" : "+v"(r[0]), "+v"(r[1]), "+v"(r[2]), "+v"(r[3]), "+v"(r[4]), "+v"(r[5]), "+v"(r[6]), "+v"(r[7]) :: "memory");
      if (g == 8) *(LAS u32x4*)(c.wmine + (WS ^ 1) * 16384) = a[0][0];
      if (g == 9) { c.left -= 1; c.xso += 65536u; asm volatile("" : "+s"(c.left), "+s"(c.xso)); }
      if (g == 10) { c.xrsrc[2] = c.left > 1 ? (int)XT_BYTES : 0; asm volatile("" : "+s"(c.xrsrc)); }
      if (g == 11) { c.wrsrc[2] = c.left > 2 ? (int)W_BYTES : 0; asm volatile("" : "+s"(c.wrsrc)); }
      if (g == 12) { lateby = (int)lateby > (int)c.slack ? lateby - c.slack : 0u; asm volatile("" : "+s"(lateby)); }
      if (g == 13) { c.due += lateby; lateby = __builtin_amdgcn_readlane(c.steps, c.left); asm volatile("" : "+s"(c.due), "+s"(lateby)); }
      if (g == 14) { c.due += lateby; asm volatile("" : "+s"(c.due)); }
      if (g == 15) { c.fso = c.fso + 512u < KH * 512u ? c.fso + 512u : c.fso; asm volatile("" : "+s"(c.fso)); }
    }
    SCHED();
    mfma(acc[g][2], a[E][2], xf[g & 7]);
    SCHED();
    if (E == 0 && g < 4) a[1][g] = *(LAS const u32x4*)(c.wother + WS * 16384 + g * 1024);
    if (E == 1 && g >= 5 && g < 9) { const int t = (g - 5) & 3; a[0][t][0] = pk_bf16(r[0][t], r[1][t]); a[0][t][1] = pk_bf16(r[2][t], r[3][t]); }
    if (E == 1 && g == 9) *(LAS u32x4*)(c.wmine + (WS ^ 1) * 16384 + 2048) = a[0][2];
    if (E == 1 && g == 10) asm volatile("s_memrealtime %0" : "=s"(now) :: "memory");
    if (E == 1 && g >= 12 && g < 15) xf[(2 * (g - 12) + 1) & 7] = x_frag<NX>(c, 2 * (g - 12) + 1);
    SCHED();
    mfma(acc[g][3], a[E][3], xf[g & 7]);
    SCHED();
    if (E == 1 && g >= 5 && g < 9) { const int t = (g - 5) & 3; a[0][t][2] = pk_bf16(r[4][t], r[5][t]); a[0][t][3] = pk_bf16(r[6][t], r[7][t]); }
    if (E == 0 && (g & 3) == 3 && g < 15) set_m0(m0x + ((g + 1) >> 3) * 32768 + (((g + 1) >> 2) & 1) * 4096);
    if (E == 1 && g == 11) { c.wso += 1u << 22; asm volatile("" : "+s"(c.wso)); }
    if (E == 1 && g == 15) set_m0(c.m0base + ((SL + 3) & 3) * 32768);
    SCHED();
  }
}

template <int N, int STRIDE, int SCOPE, int SLEEP>
__device__ __forceinline__ float sweep_max(const u64* slots, unsigned call, unsigned* tmo) {
  const u64 t0 = __builtin_amdgcn_s_memrealtime();
  for (unsigned polls = 1;; ++polls) {
    bool all = true, bad = false;
    float mx = 0.f;
    u64 v[N];
#pragma unroll
    for (int r = 0; r < N; ++r) v[r] = __hip_atomic_load(slots + r * STRIDE, __ATOMIC_RELAXED, SCOPE);
#pragma unroll
    for (int r = 0; r < N; ++r) {
      const float f = __builtin_bit_cast(float, (unsigned)v[r]);
      all &= (unsigned)(v[r] >> 32) == call;
      bad |= f < 0.f;
      mx = fmaxf(mx, f);
    }
    if (__all(all)) {
      return __any(bad) ? -1.0f : wave_max(mx);
    }
    if (((polls & 63u) == 0 && __hip_atomic_load(tmo, __ATOMIC_RELAXED, __HIP_MEMORY_SCOPE_AGENT) == call) ||
        __builtin_amdgcn_s_memrealtime() - t0 > DEADLINE_TICKS) {
      __hip_atomic_store(tmo, call, __ATOMIC_RELAXED, __HIP_MEMORY_SCOPE_AGENT);
      return -1.0f;
    }
    __builtin_amdgcn_s_sleep(SLEEP);
  }
}
template <int SLEEP>
__device__ __forceinline__ Quant global_scale(const u64* gmax, unsigned call, unsigned* tmo, LAS float* red, bool ok, int wave, int lane) {
  float gam = ok ? 0.f : -1.0f;
  if (wave == 0 && ok) gam = sweep_max<1, 0, __HIP_MEMORY_SCOPE_SYSTEM, SLEEP>(gmax + (lane & 7), call, tmo);
  if (lane == 0) red[wave] = gam;
  __syncthreads();
  const float r0 = red[0], r1 = red[1], r2 = red[2], r3 = red[3];
  const bool good = fminf(fminf(r0, r1), fminf(r2, r3)) >= 0.f;
  Quant qz;
  qz.inv = (good && r0 > 0.f) ? 127.0f / r0 : 0.f;
  qz.lo = good ? QBIAS - 127.0f : QBIAS - 128.0f;
  qz.hi = good ? QBIAS + 127.0f : QBIAS - 128.0f;
  asm volatile("" : "+v"(qz.lo), "+v"(qz.hi));
  return qz;
}

__device__ __forceinline__ void tile_of(int j, int& pd, int& ct) {
  const int k = j / 7;
  pd = j < 112 ? j % 7 + 1 : 0;
  ct = j >= 112 ? j - 112 : k < 7 ? 2 * k + 1 : k < 15 ? 2 * (k - 7) : 15;
}

__global__ void __launch_bounds__(NTHR, 1) fused_kernel(Params p) {
  extern __shared__ __attribute__((aligned(1024))) char smem[];
  const int tid = threadIdx.x, lane = tid & 63;
  const int wave = __builtin_amdgcn_readfirstlane(tid >> 6);
  char* ws = p.myws;

  STAMP_DECL;
  STAMP(0);
  STAMPC(8);
  const int b = blockIdx.x, xcd = b & 7, idx = b >> 3;
  const int kh = xcd & 1, slot = idx * 4 + (xcd >> 1), unit = slot * 2 + kh;
  const int wn = wave & 1, wm = wave >> 1;
  const int q = lane >> 4, i16 = lane & 15;
  const unsigned lane16 = lane * 16;
  const unsigned t_start = (unsigned)__builtin_amdgcn_s_memrealtime();

  Ctx c;
  c.xrsrc = make_rsrc(ws + WS_XT, (unsigned)XT_BYTES);
  unsigned* xflag = (unsigned*)(ws + WS_XFLAG);
  const int hs = kh * NT + slot;
  const unsigned xto = (unsigned)(hs * 16384 + wave * 4096);
  f32x4 raw[8];
  unsigned prev, warm;
  {
    const int st = hs >> 1, h = hs & 1;
    const i32x4 xin = make_rsrc(p.x, (unsigned)((size_t)ML * KD * 4));
#pragma unroll
    for (int j = 0; j < 4; ++j) {
      const unsigned o = (unsigned)((h * 16 + ((wave * 4 + j) ^ (kh << 3))) * 16 + i16) * (KD * 4) + ((st ^ h) * 32 + q * 8) * 4;
      asm volatile("buffer_load_dwordx4 %0, %2, %3, 0 offen\n\tbuffer_load_dwordx4 %1, %2, %3, 0 offen offset:16"
                   : "=&v"(raw[2 * j]), "=&v"(raw[2 * j + 1]) : "v"(o), "s"(xin) : "memory");
    }
    asm volatile("global_load_dword %0, %1, off sc1" : "=v"(prev) : "v"(xflag + hs * 32) : "memory");
    asm volatile("buffer_load_dword %0, %1, %2, %3 offen sc1" : "=v"(warm) : "v"(0), "s"(c.xrsrc), "s"(xto) : "memory");
  }

  const bool early = slot < NEARLY;
  int pd, ct;
  tile_of(slot, pd, ct);
  const int rank = p.rank;
  const unsigned dest = (unsigned)(rank + pd) & 7u;
  const int k0 = kh * KH;
  const int colw = ct * 128 + wn * 64;
  unsigned* tmo = (unsigned*)(ws + WS_TMO);
  GAS char* dws = (GAS char*)p.ws[dest];
  GAS char* dout = (GAS char*)p.out[early ? rank : dest];
  GAS char* myout = (GAS char*)p.out[rank];
  asm volatile("" : "+s"(dws), "+s"(dout), "+s"(myout));
  GAS char* anyws;
  i32x2 stp;
  asm volatile("global_load_dwordx2 %0, %2, %4 offset:%5\n\tglobal_load_dwordx2 %1, %3, %4 offset:%6"
               : "=&v"(anyws), "=&v"(stp) : "v"((lane & 7) * 8), "v"(((KH - 1 - lane) & 31) * 4), "s"(__builtin_amdgcn_kernarg_segment_ptr()),
                 "i"(offsetof(Params, ws)), "i"(offsetof(Params, step)) : "memory");

  c.wrsrc = make_rsrc(p.w, (unsigned)W_BYTES);
#pragma unroll
  for (int j = 0; j < 8; ++j) {
    c.wvoff[j] = ((unsigned)(wm * 32 + q * 8 + j) << 16) + (colw + i16 * 4) * 4;
    asm volatile("" : "+v"(c.wvoff[j]));
  }
  c.xvoff0 = wave * 8192 + lane16;
  c.xvoff1 = c.xvoff0 + 4096;
  asm volatile("" : "+v"(c.xvoff1));
  c.m0base = (unsigned)(size_t)(LAS char*)smem + wave * 8192;
  constexpr int nblk = KH;
  const unsigned wsb = ((unsigned)k0 << 22) + dest * (NL * 4u);
  c.left = nblk;
  c.xso = (unsigned)(k0 + 1) * 65536u;
  c.wso = wsb + (2u << 22);
  const int phase = b * 37 & 255, cls = early ? slot / 7 : NE, te = p.te[early ? cls : 0];
  c.due = t_start + p.start[cls] + phase * (early ? te : p.step[0]) / 256;
  c.slack = p.slack;
  int wide = p.nwide - cls;
  asm volatile("" : "+s"(wide));
  c.lrd0 = (LAS const char*)smem + wm * 16384 + lane16;
  c.lrd1 = c.lrd0 + 65536;
  asm volatile("" : "+v"(c.lrd1));
  c.wmine = (LAS char*)smem + LDS_X + wave * 4096 + lane16;
  c.wother = (LAS const char*)smem + LDS_X + (wave ^ 2) * 4096 + lane16;

  f32x4 acc[16][4];
  f32x4 wr[2][8];
  u32x4 a[2][4], xf[8];
#pragma unroll
  for (int m = 0; m < 16; ++m)
#pragma unroll
    for (int t = 0; t < 4; ++t) {
      acc[m][t] = f32x4{0.f, 0.f, 0.f, 0.f};
      asm volatile("" : "+a"(acc[m][t]));
    }
  asm volatile("s_waitcnt vmcnt(2)" : "+v"(prev), "+v"(warm), "+v"(raw[0]), "+v"(raw[1]), "+v"(raw[2]), "+v"(raw[3]), "+v"(raw[4]), "+v"(raw[5]), "+v"(raw[6]), "+v"(raw[7]) :: "memory");
  STAMP(11);
#pragma unroll
  for (int j = 0; j < 4; ++j) {
    const f32x4 v0 = raw[2 * j], v1 = raw[2 * j + 1];
    const u32x4 d = {pk_bf16(v0[0], v0[1]), pk_bf16(v0[2], v0[3]), pk_bf16(v1[0], v1[1]), pk_bf16(v1[2], v1[3])};
    asm volatile("buffer_store_dwordx4 %0, %1, %2, %3 offen offset:%4 sc1\n\ts_nop 1" :: "v"(d), "v"(lane16), "s"(c.xrsrc), "s"(xto), "i"(j * 1024) : "memory");
  }
  const unsigned call = __builtin_amdgcn_readfirstlane(prev) + 1;
  asm volatile("s_waitcnt vmcnt(0)\n\ts_barrier" : "+v"(anyws), "+v"(stp) :: "memory");
  c.steps = early ? te : stp[1] + phase * (stp[1] - stp[0]) / 256;
  STAMP(12);
  if (tid == 0) __hip_atomic_store(xflag + hs * 32, call, __ATOMIC_RELAXED, __HIP_MEMORY_SCOPE_AGENT);
  if (b == 0 && wave == 1 && lane < NR)
    __hip_atomic_store((GAS unsigned*)(anyws + WS_START + rank * 128), call, __ATOMIC_RELAXED, __HIP_MEMORY_SCOPE_SYSTEM);
#pragma unroll
  for (int j = 0; j < 8; ++j) w_load(wr[0][j], c.wvoff[j], c.wrsrc, wsb);
  c.fbase = (const char*)(xflag + kh * NT * 32);
  c.frsrc = make_rsrc(c.fbase, NT * 128);
  c.fvoff = (lane & 3) * 128;
  c.fso = 2 * 512;
  c.call = call;
  c.tmo = tmo;
  c.ok = true;
  if ((unsigned)(idx - 6) < 8u) {
    const int pw = (idx - 6) * 4 + wave;
    c.ok = wait_flag<__HIP_MEMORY_SCOPE_AGENT, 2>((const unsigned*)(c.fbase + (lane < 32 ? lane * 512 + (pw >> 3) * 128 : (lane & 7) * 128)), call, tmo);
    const unsigned pv = (unsigned)k0 * 65536u + pw * 2048 + lane16;
    u32x4 pf = {0u, 0u, 0u, 0u};
#pragma unroll
    for (int j = 2; j < KH; ++j) {
      const unsigned v = pv + j * 65536;
      asm volatile("buffer_load_dwordx4 %0, %1, %2, 0 offen sc1\n\tbuffer_load_dwordx4 %0, %1, %2, 0 offen offset:1024 sc1" : "+v"(pf) : "v"(v), "s"(c.xrsrc) : "memory");
    }
    asm volatile("s_waitcnt vmcnt(0)" : "+v"(pf) :: "memory");
  } else if (wave == 0) {
    c.ok = wait_flag<__HIP_MEMORY_SCOPE_AGENT, 1>((const unsigned*)(c.fbase + (lane & 7) * 128), call, tmo);
  }
  asm volatile("s_waitcnt vmcnt(0)" : "+v"(wr[0][0]), "+v"(wr[0][1]), "+v"(wr[0][2]), "+v"(wr[0][3]), "+v"(wr[0][4]), "+v"(wr[0][5]), "+v"(wr[0][6]), "+v"(wr[0][7]) :: "memory");
#pragma unroll
  for (int t = 0; t < 4; ++t) {
    a[0][t] = u32x4{pk_bf16(wr[0][0][t], wr[0][1][t]), pk_bf16(wr[0][2][t], wr[0][3][t]), pk_bf16(wr[0][4][t], wr[0][5][t]), pk_bf16(wr[0][6][t], wr[0][7][t])};
    *(LAS u32x4*)(c.wmine + t * 1024) = a[0][t];
  }
  wait_bar<0>();
  {
    const unsigned xso = (unsigned)k0 * 65536u;
    set_m0(c.m0base); asm volatile("s_nop 0");
    x_piece<0>(c, c.xrsrc, xso); x_piece<1>(c, c.xrsrc, xso); x_piece<2>(c, c.xrsrc, xso); x_piece<3>(c, c.xrsrc, xso);
    set_m0(c.m0base + 4096); asm volatile("s_nop 0");
    x_piece<4>(c, c.xrsrc, xso); x_piece<5>(c, c.xrsrc, xso); x_piece<6>(c, c.xrsrc, xso); x_piece<7>(c, c.xrsrc, xso);
    set_m0(c.m0base + 32768); asm volatile("s_nop 0");
    x_piece<8>(c, c.xrsrc, xso); x_piece<9>(c, c.xrsrc, xso); x_piece<10>(c, c.xrsrc, xso); x_piece<11>(c, c.xrsrc, xso);
    set_m0(c.m0base + 32768 + 4096); asm volatile("s_nop 0");
    x_piece<12>(c, c.xrsrc, xso); x_piece<13>(c, c.xrsrc, xso); x_piece<14>(c, c.xrsrc, xso); x_piece<15>(c, c.xrsrc, xso);
  }
#pragma unroll
  for (int j = 0; j < 8; ++j) w_load(wr[1][j], c.wvoff[j], c.wrsrc, wsb + (1u << 22));
  __builtin_amdgcn_s_waitcnt(0xC07F);
  wait_bar<8>();
  STAMP(10);
  SCHED();
  set_m0(c.m0base + 2 * 32768);
#pragma unroll
  for (int t = 0; t < 7; ++t) xf[t] = x_frag<0>(c, t);
  SCHED();
  while ((int)((unsigned)__builtin_amdgcn_s_memrealtime() + 100u - c.due) < 0) __builtin_amdgcn_s_sleep(4);
  __builtin_amdgcn_s_waitcnt(0xC07F);
  STAMP(1);

  do {
    kstep<0>(c, acc, wr, a, xf);
    kstep<1>(c, acc, wr, a, xf);
    kstep<2>(c, acc, wr, a, xf);
    kstep<3>(c, acc, wr, a, xf);
  } while (c.left > 0);
  acc_fence(acc);
  bool ok = c.ok;
  STAMP(2);
  STAMPC(9);

  LAS float* red = (LAS float*)(smem + LDS_X + 16384);
  const size_t gmax_off = WS_GMAX + (size_t)(call & 1) * 128;
  const size_t wave_off = (size_t)unit * HALF_B + wave * 16384;
  unsigned* pflag = (unsigned*)(ws + WS_PFLAG) + wave * 32;
  {
    const i32x4 prs = make_rsrc(ws + WS_PART + wave_off, 16384);
#pragma unroll
    for (int j = 0; j < 16; ++j) {
      const u32x4 d = pack_piece(acc, 16 + j);
      asm volatile("buffer_store_dwordx4 %0, %1, %2, %3 offen offset:%4 sc1\n\ts_nop 1"
                   :: "v"(d), "v"(lane16), "s"(prs), "s"((j >> 2) * 4096), "i"((j & 3) * 1024) : "memory");
    }
  }
  asm volatile("s_waitcnt vmcnt(0)"
               : "+v"(wr[0][0]), "+v"(wr[0][1]), "+v"(wr[0][2]), "+v"(wr[0][3]), "+v"(wr[0][4]), "+v"(wr[0][5]), "+v"(wr[0][6]), "+v"(wr[0][7]),
                 "+v"(wr[1][0]), "+v"(wr[1][1]), "+v"(wr[1][2]), "+v"(wr[1][3]), "+v"(wr[1][4]), "+v"(wr[1][5]), "+v"(wr[1][6]), "+v"(wr[1][7])
               :: "memory");
  if (lane == 0) __hip_atomic_store(pflag + unit * 128, call, __ATOMIC_RELAXED, __HIP_MEMORY_SCOPE_AGENT);
  ok &= wait_flag<__HIP_MEMORY_SCOPE_SYSTEM, 2>((lane & 1) ? (const unsigned*)(ws + WS_START + dest * 128) : pflag + (unit ^ 1) * 128, call, tmo);
  f32x4 y[8][4];
  {
    const i32x4 prs = make_rsrc(ws + WS_PART + (wave_off ^ HALF_B), 16384);
    u32x4 pv[16];
#pragma unroll
    for (int j = 0; j < 16; ++j)
      asm volatile("buffer_load_dwordx4 %0, %1, %2, %3 offen offset:%4 sc1" : "=v"(pv[j]) : "v"(lane16), "s"(prs), "s"((j >> 2) * 4096), "i"((j & 3) * 1024) : "memory");
    const unsigned one = (i16 >> 2) == q ? ((i16 & 1) ? 0x3F800000u : 0x00003F80u) : 0u;
    const unsigned e0 = (i16 & 2) ? 0u : one, e1 = (i16 & 2) ? one : 0u;
    u32x4 sel0 = {e0, e1, 0u, 0u}, sel1 = {0u, 0u, e0, e1};
    asm volatile("s_nop 1" : "+v"(sel0), "+v"(sel1));
    float am = 0.f;
#pragma unroll
    for (int j = 0; j < 16 + 4; ++j) {
      if (j < 16) {
        asm volatile("s_waitcnt vmcnt(%1)" : "+v"(pv[j]) : "n"(15 - j) : "memory");
        mfma(acc[j >> 1][(j & 1) * 2], sel0, pv[j]);
        mfma(acc[j >> 1][(j & 1) * 2 + 1], sel1, pv[j]);
      }
      if (j == 16) asm volatile("s_nop 7\n\ts_nop 7" ::: "memory");
      if (j >= 4) {
        const int m = (j - 4) >> 1, u = (j - 4) & 1;
        asm volatile("" : "+a"(acc[m][2 * u]), "+a"(acc[m][2 * u + 1]), "+v"(am));
        y[m][2 * u] = acc[m][2 * u]; y[m][2 * u + 1] = acc[m][2 * u + 1];
        asm volatile("" : "+v"(y[m][2 * u]), "+v"(y[m][2 * u + 1]));
        const f32x4 lo = y[m][2 * u], hi = y[m][2 * u + 1];
        am = fmaxf(fmaxf(am, fabsf(lo[0])), fabsf(lo[1])); am = fmaxf(fmaxf(am, fabsf(lo[2])), fabsf(lo[3]));
        am = fmaxf(fmaxf(am, fabsf(hi[0])), fabsf(hi[1])); am = fmaxf(fmaxf(am, fabsf(hi[2])), fabsf(hi[3]));
        asm volatile("" : "+v"(am));
      }
    }
    am = wave_max(am);
    if (lane == 0) red[wave] = ok ? am : -1.0f;
    __syncthreads();
    if (tid == 0) {
      const float r0 = red[0], r1 = red[1], r2 = red[2], r3 = red[3];
      const float wam = fminf(fminf(r0, r1), fminf(r2, r3)) < 0.f ? -1.0f : fmaxf(fmaxf(r0, r1), fmaxf(r2, r3));
      __hip_atomic_store((u64*)(ws + WS_AMAX) + unit, ((u64)call << 32) | __builtin_bit_cast(unsigned, wam), __ATOMIC_RELAXED, __HIP_MEMORY_SCOPE_AGENT);
    }
  }
  STAMP(3);
  const int row0 = wm * 256 + kh * 128 + i16, col0 = colw + q * 16;
  const int srank = (rank - pd) & 7;

  if (early) {
    const i32x4 srs = make_rsrc((const void*)(dws + WS_STAGE + wave_off), ok ? 16384 : 0);
#pragma unroll
    for (int j = 0; j < 16; ++j) {
      const f32x4 lo = y[j >> 1][(j & 1) * 2], hi = y[j >> 1][(j & 1) * 2 + 1];
      const u32x4 d = {pk_bf16(lo[0], lo[1]), pk_bf16(lo[2], lo[3]), pk_bf16(hi[0], hi[1]), pk_bf16(hi[2], hi[3])};
      if (wide > 0) asm volatile("s_waitcnt vmcnt(%0)" :: "n"(2 * SEND_WIN - 1) : "memory");
      else asm volatile("s_waitcnt vmcnt(%0)" :: "n"(SEND_WIN - 1) : "memory");
      asm volatile("buffer_store_dwordx4 %0, %1, %2, %3 offen offset:%4 sc0 sc1\n\ts_nop 1"
                   :: "v"(d), "v"(lane16), "s"(srs), "s"((j >> 2) * 4096), "i"((j & 3) * 1024) : "memory");
    }
    asm volatile("s_waitcnt vmcnt(0)" ::: "memory");
    if (lane == 0)
      __hip_atomic_store((GAS unsigned*)(dws + WS_EFLAG + (size_t)(unit * 4 + wave) * 128), call, __ATOMIC_RELAXED, __HIP_MEMORY_SCOPE_SYSTEM);
    STAMP(5);
    ok &= wait_flag<__HIP_MEMORY_SCOPE_SYSTEM, 8>((const unsigned*)(ws + WS_EFLAG + (size_t)(unit * 4 + wave) * 128), call, tmo);
    u32x4 st[16];
    {
      const i32x4 rs = make_rsrc(ws + WS_STAGE + wave_off, 16384);
#pragma unroll
      for (int j = 0; j < 16; ++j)
        asm volatile("buffer_load_dwordx4 %0, %1, %2, %3 offen offset:%4 sc0 sc1" : "=v"(st[j]) : "v"(lane16), "s"(rs), "s"((j >> 2) * 4096), "i"((j & 3) * 1024) : "memory");
      asm volatile("s_waitcnt vmcnt(0)" ::: "memory");
#pragma unroll
      for (int j = 0; j < 16; ++j) asm volatile("" : "+v"(st[j]));
    }
    STAMP(6);
    const Quant qz = global_scale<8>((const u64*)(ws + gmax_off), call, tmo, red + 8, ok, wave, lane);
    STAMP(4);
    GAS char* ob = dout + (size_t)(srank * ML + row0) * NL + col0;
#pragma unroll
    for (int m = 0; m < 8; ++m) {
      const u32x4 s0 = st[2 * m], s1 = st[2 * m + 1];
      u32x4 o;
#pragma unroll
      for (int r = 0; r < 4; ++r) {
        const int h = r >> 1;
        const unsigned w0 = s0[h], w1 = s0[2 + h], w2 = s1[h], w3 = s1[2 + h];
        const float v0 = __builtin_bit_cast(float, (r & 1) ? (w0 & 0xFFFF0000u) : (w0 << 16));
        const float v1 = __builtin_bit_cast(float, (r & 1) ? (w1 & 0xFFFF0000u) : (w1 << 16));
        const float v2 = __builtin_bit_cast(float, (r & 1) ? (w2 & 0xFFFF0000u) : (w2 << 16));
        const float v3 = __builtin_bit_cast(float, (r & 1) ? (w3 & 0xFFFF0000u) : (w3 << 16));
        o[r] = quant4(v0, v1, v2, v3, qz);
      }
      asm volatile("global_store_dwordx4 %0, %1, off sc0 sc1\n\ts_nop 1" :: "v"(ob + (size_t)m * 16 * NL), "v"(o) : "memory");
    }
    STAMP(7);
    STAMP_OUT();
    return;
  }

  if (unit == NWG - 1 && wave == 1) {
    const float rm = sweep_max<4, 64, __HIP_MEMORY_SCOPE_AGENT, 1>((const u64*)(ws + WS_AMAX) + lane, call, tmo);
    if (lane < NR)
      __hip_atomic_store((GAS u64*)(anyws + gmax_off) + rank, ((u64)call << 32) | __builtin_bit_cast(unsigned, rm), __ATOMIC_RELAXED, __HIP_MEMORY_SCOPE_SYSTEM);
  }
  const Quant qz = global_scale<2>((const u64*)(ws + gmax_off), call, tmo, red + 8, ok, wave, lane);
  STAMP(4);
  {
    GAS char* ob = dout + (size_t)(rank * ML + row0) * NL + col0;
#pragma unroll
    for (int m = 0; m < 8; ++m) {
      u32x4 o;
#pragma unroll
      for (int r = 0; r < 4; ++r) o[r] = quant4(y[m][0][r], y[m][1][r], y[m][2][r], y[m][3][r], qz);
      asm volatile("global_store_dwordx4 %0, %1, off sc0 sc1\n\ts_nop 1" :: "v"(ob + (size_t)m * 16 * NL), "v"(o) : "memory");
    }
  }
  asm volatile("s_waitcnt vmcnt(0)" ::: "memory");
  __syncthreads();
  STAMP(5);
  if (wave == 0) {
    if (lane == 0)
      __hip_atomic_store((GAS unsigned*)(dws + WS_LFLAG + (size_t)unit * 128), call, __ATOMIC_RELAXED, __HIP_MEMORY_SCOPE_SYSTEM);
    if (!wait_flag<__HIP_MEMORY_SCOPE_SYSTEM, 2>((const unsigned*)(ws + WS_LFLAG + (size_t)unit * 128), call, tmo) && lane == 0)
      myout[(size_t)(srank * ML + kh * 128) * NL + colw] = (char)-128;
    STAMP(7);
    STAMP_OUT();
  }
}

static void make_pace(Params& p, const int* pa) {
  const double T0 = pa[0], TH = pa[1], T1 = pa[2], TR = pa[3], TC = pa[6];
  const int TE = pa[4], TS = pa[5], A = pa[7], DS = pa[8];
  p.slack = pa[9];
  p.nwide = pa[10];
  for (int k = 0; k < NE; ++k) { p.te[k] = k < 7 ? TE : TS; p.start[k] = A + (k * DS > pa[11] ? k * DS - pa[11] : 0) - KH * (p.te[k] - TE); }
  p.start[NE] = A;
  double t = A;
  for (int v = 0; v <= KH; ++v) {
    double done = 0, u = t;
    while (done < 1.0) {
      double r = NWG / (u < TH ? T0 : u < TR ? T0 + (T1 - T0) * (u - TH) / (TR - TH) : T1);
      for (int k = 0; k < NE; ++k) if (u >= p.start[k] - p.te[k] / 2 && u < p.start[k] + (KH - 0.5) * p.te[k]) r -= 14.0 / p.te[k];
      r /= NWG - 2 * NEARLY;
      if (r > 1.0 / TC) r = 1.0 / TC;
      if (r < 0.001) r = 0.001;
      done += r; u += 1.0;
    }
    p.step[v] = (int)(u - t);
    if (v) t = u; else t += (u - t) * 0.5;
  }
}

extern "C" void kernel_launch(int rank, int nranks, ncclComm_t comm,
                              hipStream_t stream,
                              void* const* d_in, const int* in_sizes, int n_in,
                              void* d_out, int out_size,
                              const int* in_dtypes, int out_dtype,
                              void* const* d_out_all,
                              void* const* d_ws_all, size_t ws_bytes) {
  (void)comm; (void)in_dtypes; (void)out_dtype; (void)d_out;
  if (nranks != NR || n_in != 2 || in_sizes[0] != ML * KD || in_sizes[1] != KD * ND || out_size != NR * ML * NL ||
      ws_bytes < WS_END || (unsigned)rank >= (unsigned)NR) {
    fprintf(stderr, "unexpected problem shape\n");
    return;
  }
  HIP_CHECK(hipFuncSetAttribute((const void*)fused_kernel, hipFuncAttributeMaxDynamicSharedMemorySize, LDS_BYTES));
  Params p;
  p.x = (const float*)d_in[0];
  p.w = (const float*)d_in[1];
  for (int r = 0; r < NR; ++r) { p.out[r] = (char*)d_out_all[r]; p.ws[r] = (char*)d_ws_all[r]; }
  p.rank = rank;
  p.myws = (char*)d_ws_all[rank];
  int pa[12] = {PACE_ARGS};
  make_pace(p, pa);
  fused_kernel<<<NWG, NTHR, LDS_BYTES, stream>>>(p);
  HIP_CHECK(hipGetLastError());
}
```

```cpp
#include <hip/hip_runtime.h>
#include <rccl/rccl.h>
#include <cstdio>
#include <cstdint>
#include <cstdlib>
#include <cstddef>

#define HIP_CHECK(expr) do { hipError_t _e = (expr); if (_e != hipSuccess) { \
    fprintf(stderr, "HIP error %s at %s:%d\n", hipGetErrorString(_e), __FILE__, __LINE__); \
    return; } } while (0)

#define LAS __attribute__((address_space(3)))
#define GAS __attribute__((address_space(1)))
typedef float f32x4 __attribute__((ext_vector_type(4)));
typedef float f32x2 __attribute__((ext_vector_type(2)));
typedef __bf16 bf16x2 __attribute__((ext_vector_type(2)));
typedef unsigned u32x4 __attribute__((ext_vector_type(4)));
typedef int i32x4 __attribute__((ext_vector_type(4)));
typedef int i32x2 __attribute__((ext_vector_type(2)));
typedef unsigned long long u64;

constexpr int NR = 8;
constexpr int KD = 4096, ND = 16384;
constexpr int ML = 512;
constexpr int NL = ND / NR;
constexpr int NWG = 256, NTHR = 256;
constexpr int NBLK = KD / 64;
constexpr int NT = ND / 128;
#define NE 9
constexpr int NEARLY = 7 * NE;
constexpr int PF0 = 4;
static_assert(NE >= 7 && NE <= 12, "the workgroups that prefetch x belong to early tiles that start late");
constexpr int KH = NBLK / 2;
#define PACE_ARGS 220, 2100, 150, 4300, 125, 150, 120, 600, 240, 100, 2, 100, 118, 170
constexpr int LDS_X = 4 * 32768;
constexpr int LDS_BYTES = LDS_X + 32768;
constexpr size_t HALF_B = 65536;

constexpr size_t WS_TMO = 128;
constexpr size_t WS_START = 1024;
constexpr size_t WS_GMAX = 2048;
constexpr size_t WS_AMAX = 4096;
constexpr size_t WS_XFLAG = WS_AMAX + NWG * 8;
constexpr size_t WS_EFLAG = WS_XFLAG + NWG * 128;
constexpr size_t WS_PFLAG = WS_EFLAG + NWG * 4 * 128;
constexpr size_t WS_STAMP = WS_PFLAG + NWG * 4 * 128;
constexpr size_t WS_XT = 1 << 20;
constexpr size_t XT_BYTES = (size_t)ML * KD * 2, W_BYTES = (size_t)KD * ND * 4;
constexpr size_t WS_PART = WS_XT + XT_BYTES;
constexpr size_t WS_STAGE = WS_PART + NWG * HALF_B;
constexpr size_t WS_END = WS_STAGE + 2 * NEARLY * HALF_B;
static_assert(WS_STAMP + 65536 + NWG * 512 <= WS_XT, "control block overlaps x");
constexpr u64 DEADLINE_TICKS = 50000000ull;

struct Params {
  const float* x;
  const float* w;
  char* myws;
  int rank;
  int slack;
  char* out[NR];
  char* ws[NR];
  int nwide;
  int start[NE + 1], te[NE];
  int step[KH + 1];
};

#define SEND_WIN 3
#define STAMP(i) do {} while (0)
#define STAMPC(i) do {} while (0)
#define STAMP_DECL do {} while (0)
#define STAMP_OUT() do {} while (0)

__device__ __forceinline__ unsigned pk_bf16(float lo, float hi) {
  f32x2 v = {lo, hi};
  return __builtin_bit_cast(unsigned, __builtin_convertvector(v, bf16x2));
}
__device__ __forceinline__ float wave_max(float f) {
  int v = __builtin_bit_cast(int, f);
  v = max(v, __builtin_amdgcn_update_dpp(0, v, 0x111, 0xf, 0xf, true));
  v = max(v, __builtin_amdgcn_update_dpp(0, v, 0x112, 0xf, 0xf, true));
  v = max(v, __builtin_amdgcn_update_dpp(0, v, 0x114, 0xf, 0xf, true));
  v = max(v, __builtin_amdgcn_update_dpp(0, v, 0x118, 0xf, 0xf, true));
  const int r = max(max(__builtin_amdgcn_readlane(v, 15), __builtin_amdgcn_readlane(v, 31)), max(__builtin_amdgcn_readlane(v, 47), __builtin_amdgcn_readlane(v, 63)));
  return __builtin_bit_cast(float, r);
}
__device__ __forceinline__ i32x4 make_rsrc(const void* base, unsigned bytes) {
  const u64 a = (u64)base;
  return i32x4{__builtin_amdgcn_readfirstlane((int)(unsigned)a), __builtin_amdgcn_readfirstlane((int)(unsigned)(a >> 32)), (int)bytes, 0x00020000};
}

__device__ __forceinline__ void w_load(f32x4& d, unsigned voff, i32x4 rsrc, unsigned soff) {
  asm volatile("buffer_load_dwordx4 %0, %1, %2, %3 offen nt" : "=v"(d) : "v"(voff), "s"(rsrc), "s"(soff) : "memory");
}
__device__ __forceinline__ void set_m0(unsigned m0v) { asm volatile("s_mov_b32 m0, %0" :: "s"(m0v) : "memory"); }
template <int IMM>
__device__ __forceinline__ void x_load(unsigned voff, i32x4 rsrc, unsigned soff) {
  asm volatile("buffer_load_dwordx4 %0, %1, %2 offen offset:%3 sc1 lds" :: "v"(voff), "s"(rsrc), "s"(soff), "i"(IMM) : "memory");
}
template <int N>
__device__ __forceinline__ void wait_bar() { asm volatile("s_waitcnt vmcnt(%0) lgkmcnt(0)\n\ts_barrier" :: "n"(N) : "memory"); }
__device__ __forceinline__ void mfma(f32x4& c, u32x4 a, u32x4 b) {
  asm volatile("v_mfma_f32_16x16x32_bf16 %0, %1, %2, %0" : "+a"(c) : "v"(a), "v"(b));
}
#define SCHED() __builtin_amdgcn_sched_barrier(0)
__device__ __forceinline__ void acc_fence(f32x4 (&acc)[16][4]) {
  asm volatile("s_nop 7\n\ts_nop 7" ::: "memory");
#pragma unroll
  for (int m = 0; m < 16; ++m)
#pragma unroll
    for (int t = 0; t < 4; ++t) asm volatile("" : "+a"(acc[m][t]));
  SCHED();
}
__device__ __forceinline__ u32x4 pack_piece(f32x4 (&acc)[16][4], int j) {
  const int m = j >> 1, u = j & 1;
  asm volatile("" : "+a"(acc[m][2 * u]), "+a"(acc[m][2 * u + 1]));
  const f32x4 lo = acc[m][2 * u], hi = acc[m][2 * u + 1];
  return u32x4{pk_bf16(lo[0], lo[1]), pk_bf16(lo[2], lo[3]), pk_bf16(hi[0], hi[1]), pk_bf16(hi[2], hi[3])};
}
__device__ __forceinline__ void send_piece(const f32x4 (&y)[8][4], int j, i32x4 srs, unsigned lane16) {
  const f32x4 lo = y[j >> 1][(j & 1) * 2], hi = y[j >> 1][(j & 1) * 2 + 1];
  const u32x4 d = {pk_bf16(lo[0], lo[1]), pk_bf16(lo[2], lo[3]), pk_bf16(hi[0], hi[1]), pk_bf16(hi[2], hi[3])};
  asm volatile("buffer_store_dwordx4 %0, %1, %2, %3 offen sc0 sc1\n\ts_nop 1" :: "v"(d), "v"(lane16), "s"(srs), "s"(j * 1024) : "memory");
}
constexpr float QBIAS = 12582912.0f;
struct Quant { float inv, lo, hi; };
__device__ __forceinline__ unsigned quant4(float v0, float v1, float v2, float v3, const Quant& qz) {
  const unsigned e0 = __builtin_bit_cast(unsigned, __builtin_amdgcn_fmed3f(__builtin_fmaf(v0, qz.inv, QBIAS), qz.lo, qz.hi));
  const unsigned e1 = __builtin_bit_cast(unsigned, __builtin_amdgcn_fmed3f(__builtin_fmaf(v1, qz.inv, QBIAS), qz.lo, qz.hi));
  const unsigned e2 = __builtin_bit_cast(unsigned, __builtin_amdgcn_fmed3f(__builtin_fmaf(v2, qz.inv, QBIAS), qz.lo, qz.hi));
  const unsigned e3 = __builtin_bit_cast(unsigned, __builtin_amdgcn_fmed3f(__builtin_fmaf(v3, qz.inv, QBIAS), qz.lo, qz.hi));
  return __builtin_amdgcn_perm(e1, e0, 0x0c0c0400u) | __builtin_amdgcn_perm(e3, e2, 0x04000c0cu);
}

template <int SCOPE, int SLEEP, int N = 1, int STRIDE = 0>
__device__ __forceinline__ bool wait_flag(const unsigned* flag, unsigned call, unsigned* tmo) {
  const u64 t0 = __builtin_amdgcn_s_memrealtime();
  for (unsigned polls = 1;; ++polls) {
    unsigned v[N];
    bool pend = false;
#pragma unroll
    for (int i = 0; i < N; ++i) v[i] = __hip_atomic_load(flag + i * STRIDE, __ATOMIC_RELAXED, SCOPE);
#pragma unroll
    for (int i = 0; i < N; ++i) pend |= v[i] < call;
    if (!__any(pend)) return true;
    if (((polls & 63u) == 0 && __hip_atomic_load(tmo, __ATOMIC_RELAXED, __HIP_MEMORY_SCOPE_AGENT) == call) ||
        __builtin_amdgcn_s_memrealtime() - t0 > DEADLINE_TICKS) {
      __hip_atomic_store(tmo, call, __ATOMIC_RELAXED, __HIP_MEMORY_SCOPE_AGENT);
      return false;
    }
    __builtin_amdgcn_s_sleep(SLEEP);
  }
}

struct Ctx {
  i32x4 wrsrc, xrsrc;
  unsigned wvoff[8], xvoff0, xvoff1, m0base;
  int left;
  unsigned xso, wso;
  unsigned due, slack;
  int steps;
  i32x4 frsrc;
  const char* fbase;
  unsigned fvoff, fso, lane128, call, fl;
  unsigned* tmo;
  bool ok;
  LAS const char* lrd0;
  LAS const char* lrd1;
  LAS char* wmine;
  LAS const char* wother;
};

template <int SLOT>
__device__ __forceinline__ u32x4 x_frag(const Ctx& c, int mt) {
  return *(LAS const u32x4*)((SLOT < 2 ? c.lrd0 : c.lrd1) + (SLOT & 1) * 32768 + mt * 1024);
}
template <int G>
__device__ __forceinline__ void x_piece(const Ctx& c, i32x4 xr, unsigned xso) {
  x_load<(G & 3) * 1024>((G & 4) ? c.xvoff1 : c.xvoff0, xr, (G & 8) ? xso + 32768 : xso);
}
template <int SL, bool CHK>
__device__ __forceinline__ void kstep(Ctx& c, f32x4 (&acc)[16][4], f32x4 (&wr)[2][8], u32x4 (&a)[2][4], u32x4 (&xf)[8]) {
  constexpr int E = SL & 1, WS = SL >> 1, NX = (SL + 1) & 3;
  u64 now;
  unsigned lateby;
  const i32x4 xr = c.xrsrc, wrs = c.wrsrc;
  const unsigned xso = c.xso, wso = c.wso;
  const unsigned m0x = c.m0base + ((SL + 2) & 3) * 32768;
  f32x4 (&r)[8] = wr[WS ^ 1];
#pragma unroll
  for (int g = 0; g < 16; ++g) {
    if (E == 1 && g == 12) {
      asm volatile("s_waitcnt lgkmcnt(0)" : "+s"(now) :: "memory");
      lateby = (unsigned)now - c.due;
      while ((int)((unsigned)now - c.due) < 0) { __builtin_amdgcn_s_sleep(1); now = __builtin_amdgcn_s_memrealtime(); }
      if (CHK) {
        asm volatile("s_waitcnt vmcnt(8)" : "+v"(c.fl) :: "memory");
        if (__any(c.fl < c.call)) {
          c.ok &= wait_flag<__HIP_MEMORY_SCOPE_AGENT, 1>((const unsigned*)(c.fbase + c.fso + c.fvoff), c.call, c.tmo);
          __builtin_amdgcn_s_waitcnt(0xC07F);
        }
      }
      wait_bar<8>();
      SCHED();
    }
    mfma(acc[g][0], a[E][0], xf[g & 7]);
    SCHED();
    if (g < 9) xf[(g + 7) & 7] = x_frag<SL>(c, g + 7);
    else if (E == 0) xf[(g + 7) & 7] = x_frag<NX>(c, g - 9);
    else if (g >= 12) xf[(2 * (g - 12)) & 7] = x_frag<NX>(c, 2 * (g - 12));
    if (CHK && E == 0 && g == 0) asm volatile("buffer_load_dword %0, %1, %2, %3 offen sc1" : "=v"(c.fl) : "v"(c.fvoff), "s"(c.frsrc), "s"(c.fso) : "memory");
    if (E == 1 && g == 9) *(LAS u32x4*)(c.wmine + (WS ^ 1) * 16384 + 1024) = a[0][1];
    if (E == 1 && g == 10) *(LAS u32x4*)(c.wmine + (WS ^ 1) * 16384 + 3072) = a[0][3];
    SCHED();
    mfma(acc[g][1], a[E][1], xf[g & 7]);
    SCHED();
    if (E == 0) {
      switch (g) {
        case 0: x_piece<0>(c, xr, xso); break;   case 1: x_piece<1>(c, xr, xso); break;
        case 2: x_piece<2>(c, xr, xso); break;   case 3: x_piece<3>(c, xr, xso); break;
        case 4: x_piece<4>(c, xr, xso); break;   case 5: x_piece<5>(c, xr, xso); break;
        case 6: x_piece<6>(c, xr, xso); break;   case 7: x_piece<7>(c, xr, xso); break;
        case 8: x_piece<8>(c, xr, xso); break;   case 9: x_piece<9>(c, xr, xso); break;
        case 10: x_piece<10>(c, xr, xso); break; case 11: x_piece<11>(c, xr, xso); break;
        case 12: x_piece<12>(c, xr, xso); break; case 13: x_piece<13>(c, xr, xso); break;
        case 14: x_piece<14>(c, xr, xso); break; default: x_piece<15>(c, xr, xso); break;
      }
    } else {
      if (g < 8) w_load(wr[WS][g & 7], c.wvoff[g & 7], wrs, wso);
      if (g == 5)
        asm volatile("s_waitcnt vmcnt(%8)" : "+v"(r[0]), "+v"(r[1]), "+v"(r[2]), "+v"(r[3]), "+v"(r[4]), "+v"(r[5]), "+v"(r[6]), "+v"(r[7]) : "n"(22 + CHK) : "memory");
      if (g == 8) *(LAS u32x4*)(c.wmine + (WS ^ 1) * 16384) = a[0][0];
      if (g == 9) { c.left -= 1; c.xso += 65536u; asm volatile("" : "+s"(c.left), "+s"(c.xso)); }
      if (g == 10) { c.xrsrc[2] = c.left > 1 ? (int)XT_BYTES : 0; asm volatile("" : "+s"(c.xrsrc)); }
      if (g == 11) { c.wrsrc[2] = c.left > 2 ? (int)W_BYTES : 0; asm volatile("" : "+s"(c.wrsrc)); }
      if (g == 12) { lateby = (int)lateby > (int)c.slack ? lateby - c.slack : 0u; asm volatile("" : "+s"(lateby)); }
      if (g == 13) { c.due += lateby; lateby = __builtin_amdgcn_readlane(c.steps, c.left); asm volatile("" : "+s"(c.due), "+s"(lateby)); }
      if (g == 14) { c.due += lateby; asm volatile("" : "+s"(c.due)); }
      if (CHK && g == 15) {
        c.fso = c.left > KH - 2 ? c.fso + 512u : c.left == KH - 3 ? 8192u : 0u;
        c.fvoff = c.lane128 & (c.left > KH - 2 ? 0x180u : 0x1F80u);
        asm volatile("" : "+s"(c.fso), "+v"(c.fvoff));
      }
    }
    SCHED();
    mfma(acc[g][2], a[E][2], xf[g & 7]);
    SCHED();
    if (E == 0 && g < 4) a[1][g] = *(LAS const u32x4*)(c.wother + WS * 16384 + g * 1024);
    if (E == 1 && g >= 5 && g < 9) { const int t = (g - 5) & 3; a[0][t][0] = pk_bf16(r[0][t], r[1][t]); a[0][t][1] = pk_bf16(r[2][t], r[3][t]); }
    if (E == 1 && g == 9) *(LAS u32x4*)(c.wmine + (WS ^ 1) * 16384 + 2048) = a[0][2];
    if (E == 1 && g == 10) asm volatile("s_memrealtime %0" : "=s"(now) :: "memory");
    if (E == 1 && g >= 12 && g < 15) xf[(2 * (g - 12) + 1) & 7] = x_frag<NX>(c, 2 * (g - 12) + 1);
    SCHED();
    mfma(acc[g][3], a[E][3], xf[g & 7]);
    SCHED();
    if (E == 1 && g >= 5 && g < 9) { const int t = (g - 5) & 3; a[0][t][2] = pk_bf16(r[4][t], r[5][t]); a[0][t][3] = pk_bf16(r[6][t], r[7][t]); }
    if (E == 0 && (g & 3) == 3 && g < 15) set_m0(m0x + ((g + 1) >> 3) * 32768 + (((g + 1) >> 2) & 1) * 4096);
    if (E == 1 && g == 11) { c.wso += 1u << 22; asm volatile("" : "+s"(c.wso)); }
    if (E == 1 && g == 15) set_m0(c.m0base + ((SL + 3) & 3) * 32768);
    SCHED();
  }
}

template <int N, int STRIDE, int SCOPE, int SLEEP>
__device__ __forceinline__ float sweep_max(const u64* slots, unsigned call, unsigned* tmo) {
  const u64 t0 = __builtin_amdgcn_s_memrealtime();
  for (unsigned polls = 1;; ++polls) {
    bool all = true, bad = false;
    float mx = 0.f;
    u64 v[N];
#pragma unroll
    for (int r = 0; r < N; ++r) v[r] = __hip_atomic_load(slots + r * STRIDE, __ATOMIC_RELAXED, SCOPE);
#pragma unroll
    for (int r = 0; r < N; ++r) {
      const float f = __builtin_bit_cast(float, (unsigned)v[r]);
      all &= (unsigned)(v[r] >> 32) == call;
      bad |= f < 0.f;
      mx = fmaxf(mx, f);
    }
    if (__all(all)) {
      return __any(bad) ? -1.0f : wave_max(mx);
    }
    if (((polls & 63u) == 0 && __hip_atomic_load(tmo, __ATOMIC_RELAXED, __HIP_MEMORY_SCOPE_AGENT) == call) ||
        __builtin_amdgcn_s_memrealtime() - t0 > DEADLINE_TICKS) {
      __hip_atomic_store(tmo, call, __ATOMIC_RELAXED, __HIP_MEMORY_SCOPE_AGENT);
      return -1.0f;
    }
    __builtin_amdgcn_s_sleep(SLEEP);
  }
}
template <int SLEEP>
__device__ __forceinline__ Quant global_scale(const u64* gmax, unsigned call, unsigned* tmo, LAS float* red, bool ok, int wave, int lane) {
  float gam = ok ? 0.f : -1.0f;
  if (wave == 0 && ok) gam = sweep_max<1, 0, __HIP_MEMORY_SCOPE_SYSTEM, SLEEP>(gmax + (lane & 7), call, tmo);
  if (lane == 0) red[wave] = gam;
  __syncthreads();
  const float r0 = red[0], r1 = red[1], r2 = red[2], r3 = red[3];
  const bool good = fminf(fminf(r0, r1), fminf(r2, r3)) >= 0.f;
  Quant qz;
  qz.inv = (good && r0 > 0.f) ? 127.0f / r0 : 0.f;
  qz.lo = good ? QBIAS - 127.0f : QBIAS - 128.0f;
  qz.hi = good ? QBIAS + 127.0f : QBIAS - 128.0f;
  asm volatile("" : "+v"(qz.lo), "+v"(qz.hi));
  return qz;
}

__device__ __forceinline__ int line_of(int ct, int s) {
  const int n = 2 * ct + (s >> 1);
  return ct < 12 ? 2 * (n + (n + 2) / 3) + (s & 1) : (ct & 1) * 32 + 8 * s + (ct >> 1) - 4;
}
__device__ __forceinline__ void tile_of(int j, int& pd, int& ct) {
  const int k = j / 7;
  pd = j < 112 ? j % 7 + 1 : 0;
  ct = j >= 112 ? j - 112 : k < 2 ? 12 + k : k < NE - 2 ? 14 - NE + k : k < NE ? 16 - NE + k : k - NE;
}

__global__ void __launch_bounds__(NTHR, 1) fused_kernel(Params p) {
  extern __shared__ __attribute__((aligned(1024))) char smem[];
  const int tid = threadIdx.x, lane = tid & 63;
  const int wave = __builtin_amdgcn_readfirstlane(tid >> 6);
  char* ws = p.myws;

  STAMP_DECL;
  STAMP(0);
  STAMPC(8);
  const int b = blockIdx.x, xcd = b & 7, idx = b >> 3;
  const int kh = xcd & 1, slot = idx * 4 + (xcd >> 1), unit = slot * 2 + kh;
  const int wn = wave & 1, wm = wave >> 1;
  const int q = lane >> 4, i16 = lane & 15;
  const unsigned lane16 = lane * 16;
  const unsigned t_start = (unsigned)__builtin_amdgcn_s_memrealtime();

  Ctx c;
  c.xrsrc = make_rsrc(ws + WS_XT, (unsigned)XT_BYTES);
  unsigned* xflag = (unsigned*)(ws + WS_XFLAG);
  const int hs = kh * NT + slot;
  const unsigned xto = (unsigned)(hs * 16384 + wave * 4096);
  f32x4 raw[8];
  unsigned prev, warm;
  {
    const int st = hs >> 1, h = hs & 1;
    const i32x4 xin = make_rsrc(p.x, (unsigned)((size_t)ML * KD * 4));
#pragma unroll
    for (int j = 0; j < 4; ++j) {
      const unsigned o = (unsigned)((h * 16 + ((wave * 4 + j) ^ (kh << 3))) * 16 + i16) * (KD * 4) + ((st ^ h) * 32 + q * 8) * 4;
      asm volatile("buffer_load_dwordx4 %0, %2, %3, 0 offen\n\tbuffer_load_dwordx4 %1, %2, %3, 0 offen offset:16"
                   : "=&v"(raw[2 * j]), "=&v"(raw[2 * j + 1]) : "v"(o), "s"(xin) : "memory");
    }
    asm volatile("global_load_dword %0, %1, off sc1" : "=v"(prev) : "v"(xflag + hs * 32) : "memory");
    asm volatile("buffer_load_dword %0, %1, %2, %3 offen sc1" : "=v"(warm) : "v"(0), "s"(c.xrsrc), "s"(xto) : "memory");
  }

  const bool early = slot < NEARLY;
  int pd, ct;
  tile_of(slot, pd, ct);
  const int rank = p.rank;
  const unsigned dest = (unsigned)(rank + pd) & 7u;
  const int k0 = kh * KH;
  const int ln0 = line_of(ct, 2 * wn), ln1 = line_of(ct, 2 * wn + 1);
  unsigned* tmo = (unsigned*)(ws + WS_TMO);
  GAS char* dws = (GAS char*)p.ws[dest];
  GAS char* dout = (GAS char*)p.out[early ? rank : dest];
  GAS char* myout = (GAS char*)p.out[rank];
  asm volatile("" : "+s"(dws), "+s"(dout), "+s"(myout));
  GAS char* anyws;
  i32x2 stp;
  asm volatile("global_load_dwordx2 %0, %2, %4 offset:%5\n\tglobal_load_dwordx2 %1, %3, %4 offset:%6"
               : "=&v"(anyws), "=&v"(stp) : "v"((lane & 7) * 8), "v"(((KH - 1 - lane) & 31) * 4), "s"(__builtin_amdgcn_kernarg_segment_ptr()),
                 "i"(offsetof(Params, ws)), "i"(offsetof(Params, step)) : "memory");

  c.wrsrc = make_rsrc(p.w, (unsigned)W_BYTES);
#pragma unroll
  for (int j = 0; j < 8; ++j) {
    c.wvoff[j] = ((unsigned)(wm * 32 + q * 8 + j) << 16) + (i16 < 8 ? ln0 : ln1) * 128 + (i16 & 7) * 16;
    asm volatile("" : "+v"(c.wvoff[j]));
  }
  c.xvoff0 = wave * 8192 + lane16;
  c.xvoff1 = c.xvoff0 + 4096;
  asm volatile("" : "+v"(c.xvoff1));
  c.m0base = (unsigned)(size_t)(LAS char*)smem + wave * 8192;
  constexpr int nblk = KH;
  const unsigned wsb = ((unsigned)k0 << 22) + dest * (NL * 4u);
  c.left = nblk;
  c.xso = (unsigned)(k0 + 1) * 65536u;
  c.wso = wsb + (2u << 22);
  const bool steady = early || ct == 11 || ct >= 14;
  const int cls = early ? slot / 7 : steady ? NE - 1 : NE, te = p.te[steady ? cls : 0], phase = cls < p.nwide ? 0 : (slot * 74 + kh * 8) & 255;
  c.due = t_start + p.start[cls] + phase * (steady ? te : p.step[0]) / 256;
  c.slack = p.slack;
  int wide = p.nwide - cls;
  asm volatile("" : "+s"(wide));
  c.lrd0 = (LAS const char*)smem + wm * 16384 + lane16;
  c.lrd1 = c.lrd0 + 65536;
  asm volatile("" : "+v"(c.lrd1));
  c.wmine = (LAS char*)smem + LDS_X + wave * 4096 + lane16;
  c.wother = (LAS const char*)smem + LDS_X + (wave ^ 2) * 4096 + lane16;

  f32x4 acc[16][4];
  f32x4 wr[2][8];
  u32x4 a[2][4], xf[8];
#pragma unroll
  for (int m = 0; m < 16; ++m)
#pragma unroll
    for (int t = 0; t < 4; ++t) {
      acc[m][t] = f32x4{0.f, 0.f, 0.f, 0.f};
      asm volatile("" : "+a"(acc[m][t]));
    }
  asm volatile("s_waitcnt vmcnt(2)" : "+v"(prev), "+v"(warm), "+v"(raw[0]), "+v"(raw[1]), "+v"(raw[2]), "+v"(raw[3]), "+v"(raw[4]), "+v"(raw[5]), "+v"(raw[6]), "+v"(raw[7]) :: "memory");
  STAMP(11);
#pragma unroll
  for (int j = 0; j < 4; ++j) {
    const f32x4 v0 = raw[2 * j], v1 = raw[2 * j + 1];
    const u32x4 d = {pk_bf16(v0[0], v0[1]), pk_bf16(v0[2], v0[3]), pk_bf16(v1[0], v1[1]), pk_bf16(v1[2], v1[3])};
    asm volatile("buffer_store_dwordx4 %0, %1, %2, %3 offen offset:%4 sc1\n\ts_nop 1" :: "v"(d), "v"(lane16), "s"(c.xrsrc), "s"(xto), "i"(j * 1024) : "memory");
  }
  const unsigned call = __builtin_amdgcn_readfirstlane(prev) + 1;
  asm volatile("s_waitcnt vmcnt(0)\n\ts_barrier" : "+v"(anyws), "+v"(stp) :: "memory");
  c.steps = steady ? te : stp[1] + phase * (stp[1] - stp[0]) / 256;
  STAMP(12);
  if (tid == 0) __hip_atomic_store(xflag + hs * 32, call, __ATOMIC_RELAXED, __HIP_MEMORY_SCOPE_AGENT);
  if (b == 0 && wave == 1 && lane < NR)
    __hip_atomic_store((GAS unsigned*)(anyws + WS_START + rank * 128), call, __ATOMIC_RELAXED, __HIP_MEMORY_SCOPE_SYSTEM);
#pragma unroll
  for (int j = 0; j < 8; ++j) w_load(wr[0][j], c.wvoff[j], c.wrsrc, wsb);
  c.fbase = (const char*)(xflag + kh * NT * 32);
  c.frsrc = make_rsrc(c.fbase, NT * 128);
  c.lane128 = lane * 128;
  c.fvoff = (lane & 3) * 128;
  c.fso = 2 * 512;
  c.call = call;
  c.tmo = tmo;
  c.ok = true;
  if ((unsigned)(idx - PF0) < 8u) {
    const int pw = (idx - PF0) * 4 + wave;
    c.ok = wait_flag<__HIP_MEMORY_SCOPE_AGENT, 2>((const unsigned*)(c.fbase + (lane < 32 ? lane * 512 + (pw >> 3) * 128 : (lane & 7) * 128)), call, tmo);
    const unsigned pv = (unsigned)k0 * 65536u + pw * 2048 + lane16;
    u32x4 pf = {0u, 0u, 0u, 0u};
#pragma unroll
    for (int j = 2; j < KH; ++j) {
      const unsigned v = pv + j * 65536;
      asm volatile("buffer_load_dwordx4 %0, %1, %2, 0 offen sc1\n\tbuffer_load_dwordx4 %0, %1, %2, 0 offen offset:1024 sc1" : "+v"(pf) : "v"(v), "s"(c.xrsrc) : "memory");
    }
    asm volatile("s_waitcnt vmcnt(0)" : "+v"(pf) :: "memory");
  } else if (wave == 0) {
    c.ok = wait_flag<__HIP_MEMORY_SCOPE_AGENT, 1>((const unsigned*)(c.fbase + (lane & 7) * 128), call, tmo);
  }
  asm volatile("s_waitcnt vmcnt(0)" : "+v"(wr[0][0]), "+v"(wr[0][1]), "+v"(wr[0][2]), "+v"(wr[0][3]), "+v"(wr[0][4]), "+v"(wr[0][5]), "+v"(wr[0][6]), "+v"(wr[0][7]) :: "memory");
#pragma unroll
  for (int t = 0; t < 4; ++t) {
    a[0][t] = u32x4{pk_bf16(wr[0][0][t], wr[0][1][t]), pk_bf16(wr[0][2][t], wr[0][3][t]), pk_bf16(wr[0][4][t], wr[0][5][t]), pk_bf16(wr[0][6][t], wr[0][7][t])};
    *(LAS u32x4*)(c.wmine + t * 1024) = a[0][t];
  }
  wait_bar<0>();
  {
    const unsigned xso = (unsigned)k0 * 65536u;
    set_m0(c.m0base); asm volatile("s_nop 0");
    x_piece<0>(c, c.xrsrc, xso); x_piece<1>(c, c.xrsrc, xso); x_piece<2>(c, c.xrsrc, xso); x_piece<3>(c, c.xrsrc, xso);
    set_m0(c.m0base + 4096); asm volatile("s_nop 0");
    x_piece<4>(c, c.xrsrc, xso); x_piece<5>(c, c.xrsrc, xso); x_piece<6>(c, c.xrsrc, xso); x_piece<7>(c, c.xrsrc, xso);
    set_m0(c.m0base + 32768); asm volatile("s_nop 0");
    x_piece<8>(c, c.xrsrc, xso); x_piece<9>(c, c.xrsrc, xso); x_piece<10>(c, c.xrsrc, xso); x_piece<11>(c, c.xrsrc, xso);
    set_m0(c.m0base + 32768 + 4096); asm volatile("s_nop 0");
    x_piece<12>(c, c.xrsrc, xso); x_piece<13>(c, c.xrsrc, xso); x_piece<14>(c, c.xrsrc, xso); x_piece<15>(c, c.xrsrc, xso);
  }
#pragma unroll
  for (int j = 0; j < 8; ++j) w_load(wr[1][j], c.wvoff[j], c.wrsrc, wsb + (1u << 22));
  __builtin_amdgcn_s_waitcnt(0xC07F);
  wait_bar<8>();
  STAMP(10);
  SCHED();
  set_m0(c.m0base + 2 * 32768);
#pragma unroll
  for (int t = 0; t < 7; ++t) xf[t] = x_frag<0>(c, t);
  SCHED();
  while ((int)((unsigned)__builtin_amdgcn_s_memrealtime() + 100u - c.due) < 0) __builtin_amdgcn_s_sleep(4);
  __builtin_amdgcn_s_waitcnt(0xC07F);
  STAMP(1);

#define TRACE_PT()
  do {
    TRACE_PT()
    kstep<0, true>(c, acc, wr, a, xf);
    kstep<1, true>(c, acc, wr, a, xf);
    kstep<2, true>(c, acc, wr, a, xf);
    kstep<3, true>(c, acc, wr, a, xf);
  } while (c.left > KH - 4);
  do {
    TRACE_PT()
    kstep<0, false>(c, acc, wr, a, xf);
    kstep<1, false>(c, acc, wr, a, xf);
    kstep<2, false>(c, acc, wr, a, xf);
    kstep<3, false>(c, acc, wr, a, xf);
  } while (c.left > 0);
  acc_fence(acc);
  bool ok = c.ok;
  STAMP(2);
  STAMPC(9);

  LAS float* red = (LAS float*)(smem + LDS_X + 16384);
  const size_t gmax_off = WS_GMAX + (size_t)(call & 1) * 128;
  const size_t wave_off = (size_t)unit * HALF_B + wave * 16384;
  unsigned* pflag = (unsigned*)(ws + WS_PFLAG) + wave * 32;
  {
    const i32x4 prs = make_rsrc(ws + WS_PART + wave_off, 16384);
#pragma unroll
    for (int j = 0; j < 16; ++j) {
      const u32x4 d = pack_piece(acc, 16 + j);
      asm volatile("buffer_store_dwordx4 %0, %1, %2, %3 offen offset:%4 sc1\n\ts_nop 1"
                   :: "v"(d), "v"(lane16), "s"(prs), "s"((j >> 2) * 4096), "i"((j & 3) * 1024) : "memory");
    }
  }
  asm volatile("s_waitcnt vmcnt(0)"
               : "+v"(wr[0][0]), "+v"(wr[0][1]), "+v"(wr[0][2]), "+v"(wr[0][3]), "+v"(wr[0][4]), "+v"(wr[0][5]), "+v"(wr[0][6]), "+v"(wr[0][7]),
                 "+v"(wr[1][0]), "+v"(wr[1][1]), "+v"(wr[1][2]), "+v"(wr[1][3]), "+v"(wr[1][4]), "+v"(wr[1][5]), "+v"(wr[1][6]), "+v"(wr[1][7])
               :: "memory");
  if (lane == 0) __hip_atomic_store(pflag + unit * 128, call, __ATOMIC_RELAXED, __HIP_MEMORY_SCOPE_AGENT);
  ok &= wait_flag<__HIP_MEMORY_SCOPE_SYSTEM, 2>((lane & 1) ? (const unsigned*)(ws + WS_START + dest * 128) : pflag + (unit ^ 1) * 128, call, tmo);
  const i32x4 srs = make_rsrc((const void*)(dws + WS_STAGE + wave_off), ok ? 16384 : 0);
  f32x4 y[8][4];
  {
    const i32x4 prs = make_rsrc(ws + WS_PART + (wave_off ^ HALF_B), 16384);
    u32x4 pv[16];
#pragma unroll
    for (int j = 0; j < 16; ++j)
      asm volatile("buffer_load_dwordx4 %0, %1, %2, %3 offen offset:%4 sc1" : "=v"(pv[j]) : "v"(lane16), "s"(prs), "s"((j >> 2) * 4096), "i"((j & 3) * 1024) : "memory");
    const unsigned one = (i16 >> 2) == q ? ((i16 & 1) ? 0x3F800000u : 0x00003F80u) : 0u;
    const unsigned e0 = (i16 & 2) ? 0u : one, e1 = (i16 & 2) ? one : 0u;
    u32x4 sel0 = {e0, e1, 0u, 0u}, sel1 = {0u, 0u, e0, e1};
    asm volatile("s_nop 1" : "+v"(sel0), "+v"(sel1));
    float am = 0.f;
#pragma unroll
    for (int j = 0; j < 16 + 4; ++j) {
      if (j < 16) {
        asm volatile("s_waitcnt vmcnt(%1)" : "+v"(pv[j]) : "n"(15 - j) : "memory");
        mfma(acc[j >> 1][(j & 1) * 2], sel0, pv[j]);
        mfma(acc[j >> 1][(j & 1) * 2 + 1], sel1, pv[j]);
      }
      if (j == 16) asm volatile("s_nop 7\n\ts_nop 7" ::: "memory");
      if (j >= 4) {
        const int m = (j - 4) >> 1, u = (j - 4) & 1;
        asm volatile("" : "+a"(acc[m][2 * u]), "+a"(acc[m][2 * u + 1]), "+v"(am));
        y[m][2 * u] = acc[m][2 * u]; y[m][2 * u + 1] = acc[m][2 * u + 1];
        asm volatile("" : "+v"(y[m][2 * u]), "+v"(y[m][2 * u + 1]));
        const f32x4 lo = y[m][2 * u], hi = y[m][2 * u + 1];
        am = fmaxf(fmaxf(am, fabsf(lo[0])), fabsf(lo[1])); am = fmaxf(fmaxf(am, fabsf(lo[2])), fabsf(lo[3]));
        am = fmaxf(fmaxf(am, fabsf(hi[0])), fabsf(hi[1])); am = fmaxf(fmaxf(am, fabsf(hi[2])), fabsf(hi[3]));
        asm volatile("" : "+v"(am));
      }
    }
    if (early) {
#pragma unroll
      for (int j = 0; j < SEND_WIN; ++j) send_piece(y, j, srs, lane16);
    }
    am = wave_max(am);
    if (lane == 0) red[wave] = ok ? am : -1.0f;
    __syncthreads();
    if (tid == 0) {
      const float r0 = red[0], r1 = red[1], r2 = red[2], r3 = red[3];
      const float wam = fminf(fminf(r0, r1), fminf(r2, r3)) < 0.f ? -1.0f : fmaxf(fmaxf(r0, r1), fmaxf(r2, r3));
      __hip_atomic_store((u64*)(ws + WS_AMAX) + unit, ((u64)call << 32) | __builtin_bit_cast(unsigned, wam), __ATOMIC_RELAXED, __HIP_MEMORY_SCOPE_AGENT);
    }
  }
  STAMP(3);
  const int row0 = wm * 256 + kh * 128 + i16, col0 = (q < 2 ? ln0 : ln1) * 32 + (q & 1) * 16;
  const int srank = (rank - pd) & 7;

  if (early) {
#pragma unroll
    for (int j = SEND_WIN; j < 16; ++j) {
      if (wide > 0) asm volatile("s_waitcnt vmcnt(%0)" :: "n"(2 * SEND_WIN - 1) : "memory");
      else asm volatile("s_waitcnt vmcnt(%0)" :: "n"(SEND_WIN - 1) : "memory");
      send_piece(y, j, srs, lane16);
    }
    asm volatile("s_waitcnt vmcnt(0)" ::: "memory");
    if (lane == 0)
      __hip_atomic_store((GAS unsigned*)(dws + WS_EFLAG + (size_t)(unit * 4 + wave) * 128), call, __ATOMIC_RELAXED, __HIP_MEMORY_SCOPE_SYSTEM);
    STAMP(5);
    ok &= wait_flag<__HIP_MEMORY_SCOPE_SYSTEM, 8>((const unsigned*)(ws + WS_EFLAG + (size_t)(unit * 4 + wave) * 128), call, tmo);
    u32x4 st[16];
    {
      const i32x4 rs = make_rsrc(ws + WS_STAGE + wave_off, 16384);
#pragma unroll
      for (int j = 0; j < 16; ++j)
        asm volatile("buffer_load_dwordx4 %0, %1, %2, %3 offen offset:%4 sc0 sc1" : "=v"(st[j]) : "v"(lane16), "s"(rs), "s"((j >> 2) * 4096), "i"((j & 3) * 1024) : "memory");
      asm volatile("s_waitcnt vmcnt(0)" ::: "memory");
#pragma unroll
      for (int j = 0; j < 16; ++j) asm volatile("" : "+v"(st[j]));
    }
    STAMP(6);
    const Quant qz = global_scale<8>((const u64*)(ws + gmax_off), call, tmo, red + 8, ok, wave, lane);
    STAMP(4);
    GAS char* ob = dout + (size_t)(srank * ML + row0) * NL + col0;
#pragma unroll
    for (int m = 0; m < 8; ++m) {
      const u32x4 s0 = st[2 * m], s1 = st[2 * m + 1];
      u32x4 o;
#pragma unroll
      for (int r = 0; r < 4; ++r) {
        const int h = r >> 1;
        const unsigned w0 = s0[h], w1 = s0[2 + h], w2 = s1[h], w3 = s1[2 + h];
        const float v0 = __builtin_bit_cast(float, (r & 1) ? (w0 & 0xFFFF0000u) : (w0 << 16));
        const float v1 = __builtin_bit_cast(float, (r & 1) ? (w1 & 0xFFFF0000u) : (w1 << 16));
        const float v2 = __builtin_bit_cast(float, (r & 1) ? (w2 & 0xFFFF0000u) : (w2 << 16));
        const float v3 = __builtin_bit_cast(float, (r & 1) ? (w3 & 0xFFFF0000u) : (w3 << 16));
        o[r] = quant4(v0, v1, v2, v3, qz);
      }
      asm volatile("global_store_dwordx4 %0, %1, off sc0 sc1\n\ts_nop 1" :: "v"(ob + (size_t)m * 16 * NL), "v"(o) : "memory");
    }
    STAMP(7);
    STAMP_OUT();
    return;
  }

  if (unit == NWG - 1 && wave == 1) {
    const float rm = sweep_max<4, 64, __HIP_MEMORY_SCOPE_AGENT, 1>((const u64*)(ws + WS_AMAX) + lane, call, tmo);
    if (lane < NR)
      __hip_atomic_store((GAS u64*)(anyws + gmax_off) + rank, ((u64)call << 32) | __builtin_bit_cast(unsigned, rm), __ATOMIC_RELAXED, __HIP_MEMORY_SCOPE_SYSTEM);
  }
  const Quant qz = global_scale<2>((const u64*)(ws + gmax_off), call, tmo, red + 8, ok, wave, lane);
  STAMP(4);
  {
    GAS char* ob = dout + (size_t)(rank * ML + row0) * NL + col0;
#pragma unroll
    for (int m = 0; m < 8; ++m) {
      u32x4 o;
#pragma unroll
      for (int r = 0; r < 4; ++r) o[r] = quant4(y[m][0][r], y[m][1][r], y[m][2][r], y[m][3][r], qz);
      asm volatile("global_store_dwordx4 %0, %1, off sc0 sc1\n\ts_nop 1" :: "v"(ob + (size_t)m * 16 * NL), "v"(o) : "memory");
    }
  }
  asm volatile("s_waitcnt vmcnt(0)" ::: "memory");
  STAMP(5);
  if (lane == 0)
    __hip_atomic_store((GAS unsigned*)(dws + WS_EFLAG + (size_t)(unit * 4 + wave) * 128), call, __ATOMIC_RELAXED, __HIP_MEMORY_SCOPE_SYSTEM);
  if (!wait_flag<__HIP_MEMORY_SCOPE_SYSTEM, 2>((const unsigned*)(ws + WS_EFLAG + (size_t)(unit * 4 + wave) * 128), call, tmo) && lane == 0)
    myout[(size_t)(srank * ML + row0) * NL + col0] = (char)-128;
  STAMP(7);
  STAMP_OUT();
}

static void make_pace(Params& p, const int* pa) {
  const double T0 = pa[0], TH = pa[1], T1 = pa[2], TR = pa[3], TC = pa[6];
  const int TE = pa[4], TS = pa[5], A = pa[7], DS = pa[8];
  p.slack = pa[9];
  p.nwide = pa[10];
  for (int k = 0; k < NE; ++k) { p.te[k] = k < NE - 3 ? TE : k == NE - 3 ? TS : pa[13]; p.start[k] = A + (k * DS > pa[11] ? k * DS - pa[11] : 0) - KH * (p.te[k] - TE); if (p.start[k] < A) p.start[k] = A; }
  for (int k = 0; k < pa[10]; ++k) p.te[k] = pa[12];
  p.start[NE] = A;
  double t = A;
  for (int v = 0; v <= KH; ++v) {
    double done = 0, u = t;
    while (done < 1.0) {
      double r = NWG / (u < TH ? T0 : u < TR ? T0 + (T1 - T0) * (u - TH) / (TR - TH) : T1);
      for (int k = 0; k < NE; ++k) if (u >= p.start[k] - p.te[k] / 2 && u < p.start[k] + (KH - 0.5) * p.te[k]) r -= (k == NE - 1 ? 20.0 : 14.0) / p.te[k];
      r /= NWG - 2 * NEARLY - 6;
      if (r > 1.0 / TC) r = 1.0 / TC;
      if (r < 0.001) r = 0.001;
      done += r; u += 1.0;
    }
    p.step[v] = (int)(u - t);
    if (v) t = u; else t += (u - t) * 0.5;
  }
}

extern "C" void kernel_launch(int rank, int nranks, ncclComm_t comm,
                              hipStream_t stream,
                              void* const* d_in, const int* in_sizes, int n_in,
                              void* d_out, int out_size,
                              const int* in_dtypes, int out_dtype,
                              void* const* d_out_all,
                              void* const* d_ws_all, size_t ws_bytes) {
  (void)comm; (void)in_dtypes; (void)out_dtype; (void)d_out;
  if (nranks != NR || n_in != 2 || in_sizes[0] != ML * KD || in_sizes[1] != KD * ND || out_size != NR * ML * NL ||
      ws_bytes < WS_END || (unsigned)rank >= (unsigned)NR) {
    fprintf(stderr, "unexpected problem shape\n");
    return;
  }
  HIP_CHECK(hipFuncSetAttribute((const void*)fused_kernel, hipFuncAttributeMaxDynamicSharedMemorySize, LDS_BYTES));
  Params p;
  p.x = (const float*)d_in[0];
  p.w = (const float*)d_in[1];
  for (int r = 0; r < NR; ++r) { p.out[r] = (char*)d_out_all[r]; p.ws[r] = (char*)d_ws_all[r]; }
  p.rank = rank;
  p.myws = (char*)d_ws_all[rank];
  int pa[14] = {PACE_ARGS};
  make_pace(p, pa);
  fused_kernel<<<NWG, NTHR, LDS_BYTES, stream>>>(p);
  HIP_CHECK(hipGetLastError());
}
```

```cpp
#include <hip/hip_runtime.h>
#include <rccl/rccl.h>
#include <cstdio>
#include <cstdint>
#include <cstdlib>
#include <cstddef>

#define HIP_CHECK(expr) do { hipError_t _e = (expr); if (_e != hipSuccess) { \
    fprintf(stderr, "HIP error %s at %s:%d\n", hipGetErrorString(_e), __FILE__, __LINE__); \
    return; } } while (0)

#define LAS __attribute__((address_space(3)))
#define GAS __attribute__((address_space(1)))
typedef float f32x4 __attribute__((ext_vector_type(4)));
typedef float f32x2 __attribute__((ext_vector_type(2)));
typedef __bf16 bf16x2 __attribute__((ext_vector_type(2)));
typedef unsigned u32x4 __attribute__((ext_vector_type(4)));
typedef int i32x4 __attribute__((ext_vector_type(4)));
typedef int i32x2 __attribute__((ext_vector_type(2)));
typedef unsigned long long u64;

constexpr int NR = 8;
constexpr int KD = 4096, ND = 16384;
constexpr int ML = 512;
constexpr int NL = ND / NR;
constexpr int NWG = 256, NTHR = 256;
constexpr int NBLK = KD / 64;
constexpr int NT = ND / 128;
#define NE 9
constexpr int NEARLY = 7 * NE;
constexpr int PF0 = 4;
static_assert(NE >= 7 && NE <= 12, "the workgroups that prefetch x belong to early tiles that start late");
constexpr int KH = NBLK / 2;
#define PACE_ARGS 212, 2250, 160, 4200, 125, 150, 120, 600, 245, 150, 2, 100, 118, 170
constexpr int LDS_X = 4 * 32768;
constexpr int LDS_BYTES = LDS_X + 32768;
#define PC 2048
constexpr int FS = 256, WAVE_B = 16 * PC;
constexpr size_t HALF_B = 4 * WAVE_B;

constexpr size_t WS_TMO = 128;
constexpr size_t WS_GMAX = 512;
constexpr size_t WS_START = 1024;
constexpr size_t WS_AMAX = 4096;
constexpr size_t WS_XFLAG = WS_AMAX + NWG / 16 * FS;
constexpr size_t WS_EFLAG = WS_XFLAG + NWG * FS;
constexpr size_t WS_PFLAG = WS_EFLAG + NWG * 4 * FS;
constexpr size_t WS_STAMP = WS_PFLAG + NWG * 4 * FS;
constexpr size_t WS_XT = 1 << 20;
constexpr size_t XT_BYTES = (size_t)ML * KD * 2, W_BYTES = (size_t)KD * ND * 4;
constexpr size_t WS_PART = WS_XT + XT_BYTES;
constexpr size_t WS_STAGE = WS_PART + NWG * HALF_B;
constexpr size_t WS_END = WS_STAGE + 2 * NEARLY * HALF_B;
static_assert(WS_STAMP + 65536 + NWG * 512 <= WS_XT, "control block overlaps x");
constexpr u64 DEADLINE_TICKS = 50000000ull;

struct Params {
  const float* x;
  const float* w;
  char* myws;
  int rank;
  int slack;
  char* out[NR];
  char* ws[NR];
  int nwide;
  int start[NE + 1], te[NE];
  int step[KH + 1];
};

#define ESLEEP 32
#define LSLEEP 4
#define AGG0 2
#define LATE_WIN 2
#define SEND_WIN 3
#define STAMP(i) do {} while (0)
#define STAMPC(i) do {} while (0)
#define STAMP_DECL do {} while (0)
#define STAMP_OUT() do {} while (0)

__device__ __forceinline__ unsigned pk_bf16(float lo, float hi) {
  f32x2 v = {lo, hi};
  return __builtin_bit_cast(unsigned, __builtin_convertvector(v, bf16x2));
}
__device__ __forceinline__ float wave_max(float f) {
  int v = __builtin_bit_cast(int, f);
  v = max(v, __builtin_amdgcn_update_dpp(0, v, 0x111, 0xf, 0xf, true));
  v = max(v, __builtin_amdgcn_update_dpp(0, v, 0x112, 0xf, 0xf, true));
  v = max(v, __builtin_amdgcn_update_dpp(0, v, 0x114, 0xf, 0xf, true));
  v = max(v, __builtin_amdgcn_update_dpp(0, v, 0x118, 0xf, 0xf, true));
  const int r = max(max(__builtin_amdgcn_readlane(v, 15), __builtin_amdgcn_readlane(v, 31)), max(__builtin_amdgcn_readlane(v, 47), __builtin_amdgcn_readlane(v, 63)));
  return __builtin_bit_cast(float, r);
}
__device__ __forceinline__ i32x4 make_rsrc(const void* base, unsigned bytes) {
  const u64 a = (u64)base;
  return i32x4{__builtin_amdgcn_readfirstlane((int)(unsigned)a), __builtin_amdgcn_readfirstlane((int)(unsigned)(a >> 32)), (int)bytes, 0x00020000};
}

__device__ __forceinline__ void w_load(f32x4& d, unsigned voff, i32x4 rsrc, unsigned soff) {
  asm volatile("buffer_load_dwordx4 %0, %1, %2, %3 offen nt" : "=v"(d) : "v"(voff), "s"(rsrc), "s"(soff) : "memory");
}
__device__ __forceinline__ void set_m0(unsigned m0v) { asm volatile("s_mov_b32 m0, %0" :: "s"(m0v) : "memory"); }
template <int IMM>
__device__ __forceinline__ void x_load(unsigned voff, i32x4 rsrc, unsigned soff) {
  asm volatile("buffer_load_dwordx4 %0, %1, %2 offen offset:%3 sc1 lds" :: "v"(voff), "s"(rsrc), "s"(soff), "i"(IMM) : "memory");
}
template <int N>
__device__ __forceinline__ void wait_bar() { asm volatile("s_waitcnt vmcnt(%0) lgkmcnt(0)\n\ts_barrier" :: "n"(N) : "memory"); }
__device__ __forceinline__ void mfma(f32x4& c, u32x4 a, u32x4 b) {
  asm volatile("v_mfma_f32_16x16x32_bf16 %0, %1, %2, %0" : "+a"(c) : "v"(a), "v"(b));
}
#define SCHED() __builtin_amdgcn_sched_barrier(0)
__device__ __forceinline__ void send_piece(const f32x4 (&y)[8][4], int j, i32x4 srs, unsigned lane16) {
  const f32x4 lo = y[j >> 1][(j & 1) * 2], hi = y[j >> 1][(j & 1) * 2 + 1];
  const u32x4 d = {pk_bf16(lo[0], lo[1]), pk_bf16(lo[2], lo[3]), pk_bf16(hi[0], hi[1]), pk_bf16(hi[2], hi[3])};
  asm volatile("buffer_store_dwordx4 %0, %1, %2, %3 offen sc0 sc1\n\ts_nop 1" :: "v"(d), "v"(lane16), "s"(srs), "s"(j * PC) : "memory");
}
constexpr float QBIAS = 12582912.0f;
struct Quant { float inv, lo, hi; };
__device__ __forceinline__ unsigned quant4(float v0, float v1, float v2, float v3, const Quant& qz) {
  const unsigned e0 = __builtin_bit_cast(unsigned, __builtin_amdgcn_fmed3f(__builtin_fmaf(v0, qz.inv, QBIAS), qz.lo, qz.hi));
  const unsigned e1 = __builtin_bit_cast(unsigned, __builtin_amdgcn_fmed3f(__builtin_fmaf(v1, qz.inv, QBIAS), qz.lo, qz.hi));
  const unsigned e2 = __builtin_bit_cast(unsigned, __builtin_amdgcn_fmed3f(__builtin_fmaf(v2, qz.inv, QBIAS), qz.lo, qz.hi));
  const unsigned e3 = __builtin_bit_cast(unsigned, __builtin_amdgcn_fmed3f(__builtin_fmaf(v3, qz.inv, QBIAS), qz.lo, qz.hi));
  return __builtin_amdgcn_perm(e1, e0, 0x0c0c0400u) | __builtin_amdgcn_perm(e3, e2, 0x04000c0cu);
}

template <int SCOPE, int SLEEP, int N = 1, int STRIDE = 0>
__device__ __forceinline__ bool wait_flag(const unsigned* flag, unsigned call, unsigned* tmo) {
  const u64 t0 = __builtin_amdgcn_s_memrealtime();
  for (unsigned polls = 1;; ++polls) {
    unsigned v[N];
    bool pend = false;
#pragma unroll
    for (int i = 0; i < N; ++i) v[i] = __hip_atomic_load(flag + i * STRIDE, __ATOMIC_RELAXED, SCOPE);
#pragma unroll
    for (int i = 0; i < N; ++i) pend |= v[i] < call;
    if (!__any(pend)) return true;
    if (((polls & 63u) == 0 && __hip_atomic_load(tmo, __ATOMIC_RELAXED, __HIP_MEMORY_SCOPE_AGENT) == call) ||
        __builtin_amdgcn_s_memrealtime() - t0 > DEADLINE_TICKS) {
      __hip_atomic_store(tmo, call, __ATOMIC_RELAXED, __HIP_MEMORY_SCOPE_AGENT);
      return false;
    }
    __builtin_amdgcn_s_sleep(SLEEP);
  }
}

struct Ctx {
  i32x4 wrsrc, xrsrc;
  unsigned wvoff[8], xvoff0, xvoff1, m0base;
  int left;
  unsigned xso, wso;
  unsigned due, slack;
  int steps;
  i32x4 frsrc;
  const char* fbase;
  unsigned fvoff, fso, call, fl;
  unsigned* tmo;
  bool ok;
  LAS const char* lrd0;
  LAS const char* lrd1;
  LAS char* wmine;
  LAS const char* wother;
};

template <int SLOT>
__device__ __forceinline__ u32x4 x_frag(const Ctx& c, int mt) {
  return *(LAS const u32x4*)((SLOT < 2 ? c.lrd0 : c.lrd1) + (SLOT & 1) * 32768 + mt * 1024);
}
template <int G>
__device__ __forceinline__ void x_piece(const Ctx& c, i32x4 xr, unsigned xso) {
  x_load<(G & 3) * 1024>((G & 4) ? c.xvoff1 : c.xvoff0, xr, (G & 8) ? xso + 32768 : xso);
}
template <int SL, bool CHK>
__device__ __forceinline__ void kstep(Ctx& c, f32x4 (&acc)[16][4], f32x4 (&wr)[2][8], u32x4 (&a)[2][4], u32x4 (&xf)[8]) {
  constexpr int E = SL & 1, WS = SL >> 1, NX = (SL + 1) & 3;
  u64 now;
  unsigned lateby;
  const i32x4 xr = c.xrsrc, wrs = c.wrsrc;
  const unsigned xso = c.xso, wso = c.wso;
  const unsigned m0x = c.m0base + ((SL + 2) & 3) * 32768;
  f32x4 (&r)[8] = wr[WS ^ 1];
#pragma unroll
  for (int g = 0; g < 16; ++g) {
    if (E == 1 && g == 12) {
      asm volatile("s_waitcnt lgkmcnt(0)" : "+s"(now) :: "memory");
      lateby = (unsigned)now - c.due;
      while ((int)((unsigned)now - c.due) < 0) { __builtin_amdgcn_s_sleep(1); now = __builtin_amdgcn_s_memrealtime(); }
      if (CHK) {
        asm volatile("s_waitcnt vmcnt(8)" : "+v"(c.fl) :: "memory");
        if (__any(c.fl < c.call)) {
          c.ok &= wait_flag<__HIP_MEMORY_SCOPE_AGENT, 1>((const unsigned*)(c.fbase + c.fso + c.fvoff), c.call, c.tmo);
          __builtin_amdgcn_s_waitcnt(0xC07F);
        }
      }
      wait_bar<8>();
      SCHED();
    }
    mfma(acc[g][0], a[E][0], xf[g & 7]);
    SCHED();
    if (g < 9) xf[(g + 7) & 7] = x_frag<SL>(c, g + 7);
    else if (E == 0) xf[(g + 7) & 7] = x_frag<NX>(c, g - 9);
    else if (g >= 12) xf[(2 * (g - 12)) & 7] = x_frag<NX>(c, 2 * (g - 12));
    if (CHK && E == 0 && g == 0) asm volatile("buffer_load_dword %0, %1, %2, %3 offen sc1" : "=v"(c.fl) : "v"(c.fvoff), "s"(c.frsrc), "s"(c.fso) : "memory");
    if (E == 1 && g == 9) *(LAS u32x4*)(c.wmine + (WS ^ 1) * 16384 + 1024) = a[0][1];
    if (E == 1 && g == 10) *(LAS u32x4*)(c.wmine + (WS ^ 1) * 16384 + 3072) = a[0][3];
    SCHED();
    mfma(acc[g][1], a[E][1], xf[g & 7]);
    SCHED();
    if (E == 0) {
      switch (g) {
        case 0: x_piece<0>(c, xr, xso); break;   case 1: x_piece<1>(c, xr, xso); break;
        case 2: x_piece<2>(c, xr, xso); break;   case 3: x_piece<3>(c, xr, xso); break;
        case 4: x_piece<4>(c, xr, xso); break;   case 5: x_piece<5>(c, xr, xso); break;
        case 6: x_piece<6>(c, xr, xso); break;   case 7: x_piece<7>(c, xr, xso); break;
        case 8: x_piece<8>(c, xr, xso); break;   case 9: x_piece<9>(c, xr, xso); break;
        case 10: x_piece<10>(c, xr, xso); break; case 11: x_piece<11>(c, xr, xso); break;
        case 12: x_piece<12>(c, xr, xso); break; case 13: x_piece<13>(c, xr, xso); break;
        case 14: x_piece<14>(c, xr, xso); break; default: x_piece<15>(c, xr, xso); break;
      }
    } else {
      if (g < 8) w_load(wr[WS][g & 7], c.wvoff[g & 7], wrs, wso);
      if (g == 5)
        asm volatile("s_waitcnt vmcnt(%8)" : "+v"(r[0]), "+v"(r[1]), "+v"(r[2]), "+v"(r[3]), "+v"(r[4]), "+v"(r[5]), "+v"(r[6]), "+v"(r[7]) : "n"(22 + CHK) : "memory");
      if (g == 8) *(LAS u32x4*)(c.wmine + (WS ^ 1) * 16384) = a[0][0];
      if (g == 9) { c.left -= 1; c.xso += 65536u; asm volatile("" : "+s"(c.left), "+s"(c.xso)); }
      if (g == 10) { c.xrsrc[2] = c.left > 1 ? (int)XT_BYTES : 0; asm volatile("" : "+s"(c.xrsrc)); }
      if (g == 11) { c.wrsrc[2] = c.left > 2 ? (int)W_BYTES : 0; asm volatile("" : "+s"(c.wrsrc)); }
      if (g == 12) { lateby = (int)lateby > (int)c.slack ? lateby - c.slack : 0u; asm volatile("" : "+s"(lateby)); }
      if (g == 13) { c.due += lateby; lateby = __builtin_amdgcn_readlane(c.steps, c.left); asm volatile("" : "+s"(c.due), "+s"(lateby)); }
      if (g == 14) { c.due += lateby; asm volatile("" : "+s"(c.due)); }
    }
    SCHED();
    mfma(acc[g][2], a[E][2], xf[g & 7]);
    SCHED();
    if (E == 0 && g < 4) a[1][g] = *(LAS const u32x4*)(c.wother + WS * 16384 + g * 1024);
    if (E == 1 && g >= 5 && g < 9) { const int t = (g - 5) & 3; a[0][t][0] = pk_bf16(r[0][t], r[1][t]); a[0][t][1] = pk_bf16(r[2][t], r[3][t]); }
    if (E == 1 && g == 9) *(LAS u32x4*)(c.wmine + (WS ^ 1) * 16384 + 2048) = a[0][2];
    if (E == 1 && g == 10) asm volatile("s_memrealtime %0" : "=s"(now) :: "memory");
    if (E == 1 && g >= 12 && g < 15) xf[(2 * (g - 12) + 1) & 7] = x_frag<NX>(c, 2 * (g - 12) + 1);
    SCHED();
    mfma(acc[g][3], a[E][3], xf[g & 7]);
    SCHED();
    if (E == 1 && g >= 5 && g < 9) { const int t = (g - 5) & 3; a[0][t][2] = pk_bf16(r[4][t], r[5][t]); a[0][t][3] = pk_bf16(r[6][t], r[7][t]); }
    if (E == 0 && (g & 3) == 3 && g < 15) set_m0(m0x + ((g + 1) >> 3) * 32768 + (((g + 1) >> 2) & 1) * 4096);
    if (E == 1 && g == 11) { c.wso += 1u << 22; asm volatile("" : "+s"(c.wso)); }
    if (E == 1 && g == 15) set_m0(c.m0base + ((SL + 3) & 3) * 32768);
    SCHED();
  }
}

__device__ __forceinline__ void last_block(const Ctx& c, f32x4 (&acc)[16][4], u32x4 (&a)[2][4], u32x4 (&xf)[8], i32x4 prs, unsigned pov) {
  f32x4 lo, hi;
#pragma unroll
  for (int i = 0; i < 32; ++i) {
    const int e = (i >> 3) & 1, m = (i & 7) + (i >> 4) * 8;
    const int i7 = i + 7, e7 = (i7 >> 3) & 1, m7 = (i7 & 7) + (i7 >> 4) * 8;
    const int j = i - 16;
    mfma(acc[m][0], a[e][0], xf[i & 7]);
    SCHED();
    if (i7 < 32) xf[i7 & 7] = e7 ? x_frag<3>(c, m7) : x_frag<2>(c, m7);
    if (j >= 0) {
      asm volatile("" : "+a"(acc[j >> 1][(j & 1) * 2]));
      lo = acc[j >> 1][(j & 1) * 2];
      asm volatile("" : "+v"(lo));
    }
    SCHED();
    mfma(acc[m][1], a[e][1], xf[i & 7]);
    SCHED();
    if (i < 4) a[1][i] = *(LAS const u32x4*)(c.wother + 16384 + i * 1024);
    if (j >= 0) {
      asm volatile("" : "+a"(acc[j >> 1][(j & 1) * 2 + 1]));
      hi = acc[j >> 1][(j & 1) * 2 + 1];
      asm volatile("" : "+v"(hi));
    }
    SCHED();
    mfma(acc[m][2], a[e][2], xf[i & 7]);
    SCHED();
    u32x4 d;
    if (j >= 0) { d = u32x4{pk_bf16(lo[0], lo[1]), pk_bf16(lo[2], lo[3]), pk_bf16(hi[0], hi[1]), pk_bf16(hi[2], hi[3])}; asm volatile("" : "+v"(d)); }
    SCHED();
    mfma(acc[m][3], a[e][3], xf[i & 7]);
    SCHED();
    if (j >= 0)
      asm volatile("buffer_store_dwordx4 %0, %1, %2, %3 offen offset:%4 sc1\n\ts_nop 1"
                   :: "v"(d), "v"(pov), "s"(prs), "s"(j * PC & ~4095), "i"(j * PC & 4095) : "memory");
    SCHED();
  }
}

template <int N, int STRIDE, int SCOPE, int SLEEP>
__device__ __forceinline__ float sweep_max(const u64* slots, unsigned call, unsigned* tmo) {
  const u64 t0 = __builtin_amdgcn_s_memrealtime();
  for (unsigned polls = 1;; ++polls) {
    bool all = true, bad = false;
    float mx = 0.f;
    u64 v[N];
#pragma unroll
    for (int r = 0; r < N; ++r) v[r] = __hip_atomic_load(slots + r * STRIDE, __ATOMIC_RELAXED, SCOPE);
#pragma unroll
    for (int r = 0; r < N; ++r) {
      const float f = __builtin_bit_cast(float, (unsigned)v[r]);
      all &= (unsigned)(v[r] >> 32) == call;
      bad |= f < 0.f;
      mx = fmaxf(mx, f);
    }
    if (__all(all)) {
      return __any(bad) ? -1.0f : wave_max(mx);
    }
    if (((polls & 63u) == 0 && __hip_atomic_load(tmo, __ATOMIC_RELAXED, __HIP_MEMORY_SCOPE_AGENT) == call) ||
        __builtin_amdgcn_s_memrealtime() - t0 > DEADLINE_TICKS) {
      __hip_atomic_store(tmo, call, __ATOMIC_RELAXED, __HIP_MEMORY_SCOPE_AGENT);
      return -1.0f;
    }
    __builtin_amdgcn_s_sleep(SLEEP);
  }
}
template <int SLEEP>
__device__ __forceinline__ Quant global_scale(const u64* gmax, unsigned call, unsigned* tmo, bool ok, int lane) {
  const float gam = ok ? sweep_max<1, 0, __HIP_MEMORY_SCOPE_SYSTEM, SLEEP>(gmax + (lane & 7), call, tmo) : -1.0f;
  Quant qz;
  qz.inv = gam > 0.f ? 127.0f / gam : 0.f;
  qz.lo = gam >= 0.f ? QBIAS - 127.0f : QBIAS - 128.0f;
  qz.hi = gam >= 0.f ? QBIAS + 127.0f : QBIAS - 128.0f;
  asm volatile("" : "+v"(qz.lo), "+v"(qz.hi));
  return qz;
}

__device__ __forceinline__ int line_of(int ct, int s) {
  return ct < 7 ? 8 * ct + 4 + s : ct < 11 ? 16 * (ct - 7) + (s >> 1) * 8 + (s & 1) : ct == 11 ? 60 + s : (ct & 1) * 32 + 8 * s + (ct >> 1) - 4;
}
__device__ __forceinline__ void tile_of(int j, int& pd, int& ct) {
  const int k = j / 7;
  pd = j < 112 ? j % 7 + 1 : 0;
  ct = j >= 112 ? j - 112 : k < 2 ? 12 + k : k < NE - 2 ? 14 - NE + k : k < NE ? 16 - NE + k : k - NE;
}

__global__ void __launch_bounds__(NTHR, 1) fused_kernel(Params p) {
  extern __shared__ __attribute__((aligned(1024))) char smem[];
  const int tid = threadIdx.x, lane = tid & 63;
  const int wave = __builtin_amdgcn_readfirstlane(tid >> 6);
  char* ws = p.myws;

  STAMP_DECL;
  STAMP(0);
  STAMPC(8);
  const int b = blockIdx.x, xcd = b & 7, idx = b >> 3;
  const int kh = xcd & 1, slot = idx * 4 + (xcd >> 1), unit = slot * 2 + kh;
  const int wn = wave & 1, wm = wave >> 1;
  const int q = lane >> 4, i16 = lane & 15;
  const unsigned lane16 = lane * 16;
  const unsigned t_start = (unsigned)__builtin_amdgcn_s_memrealtime();

  Ctx c;
  c.xrsrc = make_rsrc(ws + WS_XT, (unsigned)XT_BYTES);
  unsigned* xflag = (unsigned*)(ws + WS_XFLAG);
  const int hs = kh * NT + slot;
  const unsigned xto = (unsigned)(hs * 16384 + wave * 4096);
  f32x4 raw[8];
  unsigned prev, warm;
  {
    const int st = hs >> 1, h = hs & 1;
    const i32x4 xin = make_rsrc(p.x, (unsigned)((size_t)ML * KD * 4));
#pragma unroll
    for (int j = 0; j < 4; ++j) {
      const unsigned o = (unsigned)((h * 16 + ((wave * 4 + j) ^ ((kh ^ 1) << 3))) * 16 + i16) * (KD * 4) + ((st ^ h) * 32 + q * 8) * 4;
      asm volatile("buffer_load_dwordx4 %0, %2, %3, 0 offen\n\tbuffer_load_dwordx4 %1, %2, %3, 0 offen offset:16"
                   : "=&v"(raw[2 * j]), "=&v"(raw[2 * j + 1]) : "v"(o), "s"(xin) : "memory");
    }
    asm volatile("global_load_dword %0, %1, off sc1" : "=v"(prev) : "v"(xflag + hs * (FS / 4)) : "memory");
    asm volatile("buffer_load_dword %0, %1, %2, %3 offen sc1" : "=v"(warm) : "v"(0), "s"(c.xrsrc), "s"(xto) : "memory");
  }

  const bool early = slot < NEARLY;
  int pd, ct;
  tile_of(slot, pd, ct);
  const int rank = p.rank;
  const unsigned dest = (unsigned)(rank + pd) & 7u;
  const int k0 = kh * KH;
  const int ln0 = line_of(ct, 2 * wn), ln1 = line_of(ct, 2 * wn + 1);
  unsigned* tmo = (unsigned*)(ws + WS_TMO);
  GAS char* dws = (GAS char*)p.ws[dest];
  GAS char* dout = (GAS char*)p.out[early ? rank : dest];
  GAS char* myout = (GAS char*)p.out[rank];
  asm volatile("" : "+s"(dws), "+s"(dout), "+s"(myout));
  GAS char* anyws;
  i32x2 stp;
  asm volatile("global_load_dwordx2 %0, %2, %4 offset:%5\n\tglobal_load_dwordx2 %1, %3, %4 offset:%6"
               : "=&v"(anyws), "=&v"(stp) : "v"((lane & 7) * 8), "v"(((KH - 1 - lane) & 31) * 4), "s"(__builtin_amdgcn_kernarg_segment_ptr()),
                 "i"(offsetof(Params, ws)), "i"(offsetof(Params, step)) : "memory");

  c.wrsrc = make_rsrc(p.w, (unsigned)W_BYTES);
#pragma unroll
  for (int j = 0; j < 8; ++j) {
    c.wvoff[j] = ((unsigned)(wm * 32 + q * 8 + j) << 16) + (i16 < 8 ? ln0 : ln1) * 128 + (i16 & 7) * 16;
    asm volatile("" : "+v"(c.wvoff[j]));
  }
  c.xvoff0 = wave * 8192 + lane16;
  c.xvoff1 = c.xvoff0 + 4096;
  asm volatile("" : "+v"(c.xvoff1));
  c.m0base = (unsigned)(size_t)(LAS char*)smem + wave * 8192;
  constexpr int nblk = KH;
  const unsigned wsb = ((unsigned)k0 << 22) + dest * (NL * 4u);
  c.left = nblk;
  c.xso = (unsigned)(k0 + 1) * 65536u;
  c.wso = wsb + (2u << 22);
  const bool steady = early || ct == 11 || ct >= 14;
  const int cls = early ? slot / 7 : steady ? NE - 1 : NE, te = p.te[steady ? cls : 0], phase = cls < p.nwide ? 0 : (slot * 74 + kh * 8) & 255;
  c.due = t_start + p.start[cls] + phase * (steady ? te : p.step[0]) / 256;
  c.slack = p.slack;
  int wide = p.nwide - cls;
  asm volatile("" : "+s"(wide));
  c.lrd0 = (LAS const char*)smem + wm * 16384 + lane16;
  c.lrd1 = c.lrd0 + 65536;
  asm volatile("" : "+v"(c.lrd1));
  c.wmine = (LAS char*)smem + LDS_X + wave * 4096 + lane16;
  c.wother = (LAS const char*)smem + LDS_X + (wave ^ 2) * 4096 + lane16;

  f32x4 acc[16][4];
  f32x4 wr[2][8];
  u32x4 a[2][4], xf[8];
#pragma unroll
  for (int m = 0; m < 16; ++m)
#pragma unroll
    for (int t = 0; t < 4; ++t) {
      acc[m][t] = f32x4{0.f, 0.f, 0.f, 0.f};
      asm volatile("" : "+a"(acc[m][t]));
    }
  asm volatile("s_waitcnt vmcnt(2)" : "+v"(prev), "+v"(warm), "+v"(raw[0]), "+v"(raw[1]), "+v"(raw[2]), "+v"(raw[3]), "+v"(raw[4]), "+v"(raw[5]), "+v"(raw[6]), "+v"(raw[7]) :: "memory");
  STAMP(11);
#pragma unroll
  for (int j = 0; j < 4; ++j) {
    const f32x4 v0 = raw[2 * j], v1 = raw[2 * j + 1];
    const u32x4 d = {pk_bf16(v0[0], v0[1]), pk_bf16(v0[2], v0[3]), pk_bf16(v1[0], v1[1]), pk_bf16(v1[2], v1[3])};
    asm volatile("buffer_store_dwordx4 %0, %1, %2, %3 offen offset:%4 sc1\n\ts_nop 1" :: "v"(d), "v"(lane16), "s"(c.xrsrc), "s"(xto), "i"(j * 1024) : "memory");
  }
  const unsigned call = __builtin_amdgcn_readfirstlane(prev) + 1;
  asm volatile("s_waitcnt vmcnt(0)\n\ts_barrier" : "+v"(anyws), "+v"(stp) :: "memory");
  c.steps = steady ? te : stp[1] + phase * (stp[1] - stp[0]) / 256;
  STAMP(12);
  if (tid == 0) __hip_atomic_store(xflag + hs * (FS / 4), call, __ATOMIC_RELAXED, __HIP_MEMORY_SCOPE_AGENT);
  if (b == 0 && wave == 1 && lane < NR)
    __hip_atomic_store((GAS unsigned*)(anyws + WS_START + rank * FS), call, __ATOMIC_RELAXED, __HIP_MEMORY_SCOPE_SYSTEM);
#pragma unroll
  for (int j = 0; j < 8; ++j) w_load(wr[0][j], c.wvoff[j], c.wrsrc, wsb);
  c.fbase = (const char*)xflag + kh * NT * FS;
  c.frsrc = make_rsrc(c.fbase, NT * FS);
  c.fvoff = (lane & 3) * FS;
  c.fso = 8 * FS;
  c.call = call;
  c.tmo = tmo;
  c.ok = true;
  if ((unsigned)(idx - PF0) < 8u) {
    const int pw = (idx - PF0) * 4 + wave;
    c.ok = wait_flag<__HIP_MEMORY_SCOPE_AGENT, 2>((const unsigned*)(c.fbase + (lane < 32 ? lane * 4 + (pw >> 3) : lane & 7) * FS), call, tmo);
    const unsigned pv = (unsigned)k0 * 65536u + pw * 2048 + lane16;
    u32x4 pf = {0u, 0u, 0u, 0u};
#pragma unroll
    for (int j = 2; j < KH; ++j) {
      const unsigned v = pv + j * 65536;
      asm volatile("buffer_load_dwordx4 %0, %1, %2, 0 offen sc1\n\tbuffer_load_dwordx4 %0, %1, %2, 0 offen offset:1024 sc1" : "+v"(pf) : "v"(v), "s"(c.xrsrc) : "memory");
    }
    asm volatile("s_waitcnt vmcnt(0)" : "+v"(pf) :: "memory");
  } else if (wave == 0) {
    c.ok = wait_flag<__HIP_MEMORY_SCOPE_AGENT, 1>((const unsigned*)(c.fbase + (lane & 7) * FS), call, tmo);
  }
  asm volatile("s_waitcnt vmcnt(0)" : "+v"(wr[0][0]), "+v"(wr[0][1]), "+v"(wr[0][2]), "+v"(wr[0][3]), "+v"(wr[0][4]), "+v"(wr[0][5]), "+v"(wr[0][6]), "+v"(wr[0][7]) :: "memory");
#pragma unroll
  for (int t = 0; t < 4; ++t) {
    a[0][t] = u32x4{pk_bf16(wr[0][0][t], wr[0][1][t]), pk_bf16(wr[0][2][t], wr[0][3][t]), pk_bf16(wr[0][4][t], wr[0][5][t]), pk_bf16(wr[0][6][t], wr[0][7][t])};
    *(LAS u32x4*)(c.wmine + t * 1024) = a[0][t];
  }
  wait_bar<0>();
  {
    const unsigned xso = (unsigned)k0 * 65536u;
    set_m0(c.m0base); asm volatile("s_nop 0");
    x_piece<0>(c, c.xrsrc, xso); x_piece<1>(c, c.xrsrc, xso); x_piece<2>(c, c.xrsrc, xso); x_piece<3>(c, c.xrsrc, xso);
    set_m0(c.m0base + 4096); asm volatile("s_nop 0");
    x_piece<4>(c, c.xrsrc, xso); x_piece<5>(c, c.xrsrc, xso); x_piece<6>(c, c.xrsrc, xso); x_piece<7>(c, c.xrsrc, xso);
    set_m0(c.m0base + 32768); asm volatile("s_nop 0");
    x_piece<8>(c, c.xrsrc, xso); x_piece<9>(c, c.xrsrc, xso); x_piece<10>(c, c.xrsrc, xso); x_piece<11>(c, c.xrsrc, xso);
    set_m0(c.m0base + 32768 + 4096); asm volatile("s_nop 0");
    x_piece<12>(c, c.xrsrc, xso); x_piece<13>(c, c.xrsrc, xso); x_piece<14>(c, c.xrsrc, xso); x_piece<15>(c, c.xrsrc, xso);
  }
#pragma unroll
  for (int j = 0; j < 8; ++j) w_load(wr[1][j], c.wvoff[j], c.wrsrc, wsb + (1u << 22));
  __builtin_amdgcn_s_waitcnt(0xC07F);
  wait_bar<8>();
  STAMP(10);
  SCHED();
  set_m0(c.m0base + 2 * 32768);
#pragma unroll
  for (int t = 0; t < 7; ++t) xf[t] = x_frag<0>(c, t);
  SCHED();
  while ((int)((unsigned)__builtin_amdgcn_s_memrealtime() + 100u - c.due) < 0) __builtin_amdgcn_s_sleep(4);
  __builtin_amdgcn_s_waitcnt(0xC07F);
  STAMP(1);

#define TRACE_PT()
  TRACE_PT()
  kstep<0, true>(c, acc, wr, a, xf);
  kstep<1, true>(c, acc, wr, a, xf);
  c.fvoff = lane * FS;
  c.fso = 0;
  kstep<2, true>(c, acc, wr, a, xf);
  kstep<3, true>(c, acc, wr, a, xf);
  c.fso = 64 * FS;
  kstep<0, true>(c, acc, wr, a, xf);
  kstep<1, true>(c, acc, wr, a, xf);
  do {
    TRACE_PT()
    kstep<2, false>(c, acc, wr, a, xf);
    kstep<3, false>(c, acc, wr, a, xf);
    kstep<0, false>(c, acc, wr, a, xf);
    kstep<1, false>(c, acc, wr, a, xf);
  } while (c.left > 1);
  LAS float* red = (LAS float*)(smem + LDS_X);
  const size_t gmax_off = WS_GMAX + (size_t)(call & 1) * FS;
  const size_t wave_off = (size_t)unit * HALF_B + wave * WAVE_B;
  const unsigned pov = (PC > 1024 && (lane >> 3) == 3) ? lane16 + 5 * 128 : lane16;
  unsigned* pflag = (unsigned*)(ws + WS_PFLAG + wave * FS);
  last_block(c, acc, a, xf, make_rsrc(ws + WS_PART + wave_off, WAVE_B), pov);
  bool ok = c.ok;
  STAMP(2);
  STAMPC(9);
  asm volatile("s_waitcnt vmcnt(0)"
               : "+v"(wr[0][0]), "+v"(wr[0][1]), "+v"(wr[0][2]), "+v"(wr[0][3]), "+v"(wr[0][4]), "+v"(wr[0][5]), "+v"(wr[0][6]), "+v"(wr[0][7])
               :: "memory");
  if (lane == 0) __hip_atomic_store(pflag + unit * FS, call, __ATOMIC_RELAXED, __HIP_MEMORY_SCOPE_AGENT);
  ok &= wait_flag<__HIP_MEMORY_SCOPE_SYSTEM, 2>((lane & 1) ? (const unsigned*)(ws + WS_START + dest * FS) : pflag + (unit ^ 1) * FS, call, tmo);
  const i32x4 srs = make_rsrc((const void*)(dws + WS_STAGE + wave_off), ok ? WAVE_B : 0);
  f32x4 y[8][4];
  {
    const i32x4 prs = make_rsrc(ws + WS_PART + (wave_off ^ HALF_B), WAVE_B);
    u32x4 pv[16];
#pragma unroll
    for (int j = 0; j < 16; ++j)
      asm volatile("buffer_load_dwordx4 %0, %1, %2, %3 offen offset:%4 sc1" : "=v"(pv[j]) : "v"(pov), "s"(prs), "s"(j * PC & ~4095), "i"(j * PC & 4095) : "memory");
    const unsigned one = (i16 >> 2) == q ? ((i16 & 1) ? 0x3F800000u : 0x00003F80u) : 0u;
    const unsigned e0 = (i16 & 2) ? 0u : one, e1 = (i16 & 2) ? one : 0u;
    u32x4 sel0 = {e0, e1, 0u, 0u}, sel1 = {0u, 0u, e0, e1};
    asm volatile("s_nop 1" : "+v"(sel0), "+v"(sel1));
    float am = 0.f;
#pragma unroll
    for (int j = 0; j < 16 + 4; ++j) {
      if (j < 16) {
        asm volatile("s_waitcnt vmcnt(%1)" : "+v"(pv[j]) : "n"(15 - j) : "memory");
        mfma(acc[8 + (j >> 1)][(j & 1) * 2], sel0, pv[j]);
        mfma(acc[8 + (j >> 1)][(j & 1) * 2 + 1], sel1, pv[j]);
      }
      if (j == 16) asm volatile("s_nop 7\n\ts_nop 7" ::: "memory");
      if (j >= 4) {
        const int m = (j - 4) >> 1, u = (j - 4) & 1;
        asm volatile("" : "+a"(acc[8 + m][2 * u]), "+a"(acc[8 + m][2 * u + 1]), "+v"(am));
        y[m][2 * u] = acc[8 + m][2 * u]; y[m][2 * u + 1] = acc[8 + m][2 * u + 1];
        asm volatile("" : "+v"(y[m][2 * u]), "+v"(y[m][2 * u + 1]));
        const f32x4 lo = y[m][2 * u], hi = y[m][2 * u + 1];
        am = fmaxf(fmaxf(am, fabsf(lo[0])), fabsf(lo[1])); am = fmaxf(fmaxf(am, fabsf(lo[2])), fabsf(lo[3]));
        am = fmaxf(fmaxf(am, fabsf(hi[0])), fabsf(hi[1])); am = fmaxf(fmaxf(am, fabsf(hi[2])), fabsf(hi[3]));
        asm volatile("" : "+v"(am));
      }
    }
    if (early) {
#pragma unroll
      for (int j = 0; j < SEND_WIN; ++j) send_piece(y, j, srs, pov);
    }
    am = wave_max(am);
    if (lane == 0) red[wave] = ok ? am : -1.0f;
    __syncthreads();
    if (tid == 0) {
      const float r0 = red[0], r1 = red[1], r2 = red[2], r3 = red[3];
      const float wam = fminf(fminf(r0, r1), fminf(r2, r3)) < 0.f ? -1.0f : fmaxf(fmaxf(r0, r1), fmaxf(r2, r3));
      __hip_atomic_store((u64*)(ws + WS_AMAX + (unit >> 4) * FS) + (unit & 15), ((u64)call << 32) | __builtin_bit_cast(unsigned, wam), __ATOMIC_RELAXED, __HIP_MEMORY_SCOPE_AGENT);
    }
  }
  STAMP(3);
  const int row0 = wm * 256 + kh * 128 + i16, col0 = (q < 2 ? ln0 : ln1) * 32 + (q & 1) * 16;
  const int srank = (rank - pd) & 7;

  if (early) {
#pragma unroll
    for (int j = SEND_WIN; j < 16; ++j) {
      if (wide > 0) asm volatile("s_waitcnt vmcnt(%0)" :: "n"(2 * SEND_WIN - 1) : "memory");
      else asm volatile("s_waitcnt vmcnt(%0)" :: "n"(SEND_WIN - 1) : "memory");
      send_piece(y, j, srs, pov);
    }
    asm volatile("s_waitcnt vmcnt(0)" ::: "memory");
    if (lane == 0)
      __hip_atomic_store((GAS unsigned*)(dws + WS_EFLAG + (size_t)(unit * 4 + wave) * FS), call, __ATOMIC_RELAXED, __HIP_MEMORY_SCOPE_SYSTEM);
    STAMP(5);
    ok &= wait_flag<__HIP_MEMORY_SCOPE_SYSTEM, 8>((const unsigned*)(ws + WS_EFLAG + (size_t)(unit * 4 + wave) * FS), call, tmo);
    u32x4 st[16];
    {
      const i32x4 rs = make_rsrc(ws + WS_STAGE + wave_off, WAVE_B);
#pragma unroll
      for (int j = 0; j < 16; ++j)
        asm volatile("buffer_load_dwordx4 %0, %1, %2, %3 offen offset:%4 sc0 sc1" : "=v"(st[j]) : "v"(pov), "s"(rs), "s"(j * PC & ~4095), "i"(j * PC & 4095) : "memory");
      asm volatile("s_waitcnt vmcnt(0)" ::: "memory");
#pragma unroll
      for (int j = 0; j < 16; ++j) asm volatile("" : "+v"(st[j]));
    }
    STAMP(6);
    const Quant qz = global_scale<ESLEEP>((const u64*)(ws + gmax_off), call, tmo, ok, lane);
    STAMP(4);
    GAS char* ob = dout + (size_t)(srank * ML + row0) * NL + col0;
#pragma unroll
    for (int m = 0; m < 8; ++m) {
      const u32x4 s0 = st[2 * m], s1 = st[2 * m + 1];
      u32x4 o;
#pragma unroll
      for (int r = 0; r < 4; ++r) {
        const int h = r >> 1;
        const unsigned w0 = s0[h], w1 = s0[2 + h], w2 = s1[h], w3 = s1[2 + h];
        const float v0 = __builtin_bit_cast(float, (r & 1) ? (w0 & 0xFFFF0000u) : (w0 << 16));
        const float v1 = __builtin_bit_cast(float, (r & 1) ? (w1 & 0xFFFF0000u) : (w1 << 16));
        const float v2 = __builtin_bit_cast(float, (r & 1) ? (w2 & 0xFFFF0000u) : (w2 << 16));
        const float v3 = __builtin_bit_cast(float, (r & 1) ? (w3 & 0xFFFF0000u) : (w3 << 16));
        o[r] = quant4(v0, v1, v2, v3, qz);
      }
      asm volatile("global_store_dwordx4 %0, %1, off sc0 sc1\n\ts_nop 1" :: "v"(ob + (size_t)m * 16 * NL), "v"(o) : "memory");
    }
    STAMP(7);
    STAMP_OUT();
    return;
  }

  if (unit == NWG - 1 && wave >= AGG0) {
    const float rm = sweep_max<4, 4 * FS / 8, __HIP_MEMORY_SCOPE_AGENT, 1>((const u64*)(ws + WS_AMAX + (lane >> 4) * FS) + (lane & 15), call, tmo);
    if (lane < NR)
      __hip_atomic_store((GAS u64*)(anyws + gmax_off) + rank, ((u64)call << 32) | __builtin_bit_cast(unsigned, rm), __ATOMIC_RELAXED, __HIP_MEMORY_SCOPE_SYSTEM);
  }
  const Quant qz = global_scale<LSLEEP>((const u64*)(ws + gmax_off), call, tmo, ok, lane);
  STAMP(4);
  {
    GAS char* ob = dout + (size_t)(rank * ML + row0) * NL + col0;
#pragma unroll
    for (int m = 0; m < 8; ++m) {
      u32x4 o;
#pragma unroll
      for (int r = 0; r < 4; ++r) o[r] = quant4(y[m][0][r], y[m][1][r], y[m][2][r], y[m][3][r], qz);
      asm volatile("s_waitcnt vmcnt(%2)\n\tglobal_store_dwordx4 %0, %1, off sc0 sc1\n\ts_nop 1" :: "v"(ob + (size_t)m * 16 * NL), "v"(o), "n"(LATE_WIN - 1) : "memory");
    }
  }
  asm volatile("s_waitcnt vmcnt(0)" ::: "memory");
  STAMP(5);
  if (lane == 0)
    __hip_atomic_store((GAS unsigned*)(dws + WS_EFLAG + (size_t)(unit * 4 + wave) * FS), call, __ATOMIC_RELAXED, __HIP_MEMORY_SCOPE_SYSTEM);
  if (!wait_flag<__HIP_MEMORY_SCOPE_SYSTEM, 2>((const unsigned*)(ws + WS_EFLAG + (size_t)(unit * 4 + wave) * FS), call, tmo) && lane == 0)
    myout[(size_t)(srank * ML + row0) * NL + col0] = (char)-128;
  STAMP(7);
  STAMP_OUT();
}

static void make_pace(Params& p, const int* pa) {
  const double T0 = pa[0], TH = pa[1], T1 = pa[2], TR = pa[3], TC = pa[6];
  const int TE = pa[4], TS = pa[5], A = pa[7], DS = pa[8];
  p.slack = pa[9];
  p.nwide = pa[10];
  for (int k = 0; k < NE; ++k) { p.te[k] = k < NE - 3 ? TE : k == NE - 3 ? TS : pa[13]; p.start[k] = A + (k * DS > pa[11] ? k * DS - pa[11] : 0) - KH * (p.te[k] - TE); if (p.start[k] < A) p.start[k] = A; }
  for (int k = 0; k < pa[10]; ++k) p.te[k] = pa[12];
  p.start[NE] = A;
  double t = A;
  for (int v = 0; v <= KH; ++v) {
    double done = 0, u = t;
    while (done < 1.0) {
      double r = NWG / (u < TH ? T0 : u < TR ? T0 + (T1 - T0) * (u - TH) / (TR - TH) : T1);
      for (int k = 0; k < NE; ++k) if (u >= p.start[k] - p.te[k] / 2 && u < p.start[k] + (KH - 0.5) * p.te[k]) r -= (k == NE - 1 ? 20.0 : 14.0) / p.te[k];
      r /= NWG - 2 * NEARLY - 6;
      if (r > 1.0 / TC) r = 1.0 / TC;
      if (r < 0.001) r = 0.001;
      done += r; u += 1.0;
    }
    p.step[v] = (int)(u - t);
    if (v) t = u; else t += (u - t) * 0.5;
  }
}

extern "C" void kernel_launch(int rank, int nranks, ncclComm_t comm,
                              hipStream_t stream,
                              void* const* d_in, const int* in_sizes, int n_in,
                              void* d_out, int out_size,
                              const int* in_dtypes, int out_dtype,
                              void* const* d_out_all,
                              void* const* d_ws_all, size_t ws_bytes) {
  (void)comm; (void)in_dtypes; (void)out_dtype; (void)d_out;
  if (nranks != NR || n_in != 2 || in_sizes[0] != ML * KD || in_sizes[1] != KD * ND || out_size != NR * ML * NL ||
      ws_bytes < WS_END || (unsigned)rank >= (unsigned)NR) {
    fprintf(stderr, "unexpected problem shape\n");
    return;
  }
  HIP_CHECK(hipFuncSetAttribute((const void*)fused_kernel, hipFuncAttributeMaxDynamicSharedMemorySize, LDS_BYTES));
  Params p;
  p.x = (const float*)d_in[0];
  p.w = (const float*)d_in[1];
  for (int r = 0; r < NR; ++r) { p.out[r] = (char*)d_out_all[r]; p.ws[r] = (char*)d_ws_all[r]; }
  p.rank = rank;
  p.myws = (char*)d_ws_all[rank];
  int pa[14] = {PACE_ARGS};
  make_pace(p, pa);
  fused_kernel<<<NWG, NTHR, LDS_BYTES, stream>>>(p);
  HIP_CHECK(hipGetLastError());
}
```

```cpp
#include <hip/hip_runtime.h>
#include <rccl/rccl.h>
#include <cstdio>
#include <cstdint>
#include <cstdlib>
#include <cstddef>

#define HIP_CHECK(expr) do { hipError_t _e = (expr); if (_e != hipSuccess) { \
    fprintf(stderr, "HIP error %s at %s:%d\n", hipGetErrorString(_e), __FILE__, __LINE__); \
    return; } } while (0)

#define LAS __attribute__((address_space(3)))
#define GAS __attribute__((address_space(1)))
typedef float f32x4 __attribute__((ext_vector_type(4)));
typedef float f32x2 __attribute__((ext_vector_type(2)));
typedef __bf16 bf16x2 __attribute__((ext_vector_type(2)));
typedef unsigned u32x4 __attribute__((ext_vector_type(4)));
typedef int i32x4 __attribute__((ext_vector_type(4)));
typedef int i32x2 __attribute__((ext_vector_type(2)));
typedef unsigned long long u64;

constexpr int NR = 8;
constexpr int KD = 4096, ND = 16384;
constexpr int ML = 512;
constexpr int NL = ND / NR;
constexpr int NWG = 256, NTHR = 256;
constexpr int NBLK = KD / 64;
constexpr int NT = ND / 128;
#define NE 9
constexpr int NEARLY = 7 * NE;
#define PF0 4
#define PFL 3
static_assert(NE >= 7 && NE <= 12, "the workgroups that prefetch x belong to early tiles that start late");
constexpr int KH = NBLK / 2;
#define NH 2
#define PACE_ARGS 212, 2250, 160, 4200, 125, 150, 115, 600, 245, 200, 2, 100, 122, 175, 3
constexpr int LDS_X = 4 * 32768;
constexpr int LDS_BYTES = LDS_X + 32768;
#define PC 2048
constexpr int FS = 256, WAVE_B = 16 * PC;
constexpr size_t HALF_B = 4 * WAVE_B;

constexpr size_t WS_TMO = 128;
constexpr size_t WS_GMAX = 512;
constexpr size_t WS_START = 1024;
constexpr size_t WS_AMAX = 4096;
constexpr size_t WS_XFLAG = WS_AMAX + NWG / 16 * FS;
constexpr size_t WS_EFLAG = WS_XFLAG + NWG * FS;
constexpr size_t WS_PFLAG = WS_EFLAG + NWG * 4 * FS;
constexpr size_t WS_STAMP = WS_PFLAG + NWG * 4 * FS;
constexpr size_t WS_XT = 1 << 20;
constexpr size_t XT_BYTES = (size_t)ML * KD * 2, W_BYTES = (size_t)KD * ND * 4;
constexpr size_t WS_PART = WS_XT + XT_BYTES;
constexpr size_t WS_STAGE = WS_PART + NWG * HALF_B;
constexpr size_t WS_END = WS_STAGE + 2 * NEARLY * HALF_B;
static_assert(WS_STAMP + 65536 + NWG * 512 <= WS_XT, "control block overlaps x");
constexpr u64 DEADLINE_TICKS = 50000000ull;

struct Params {
  const float* x;
  const float* w;
  char* myws;
  int rank;
  int slack;
  char* out[NR];
  char* ws[NR];
  int nwide, wmask;
  int start[NE + 1], te[NE];
  int step[KH + 1];
};

#define ESLEEP 32
#define LSLEEP 4
#define AGG0 2
#define LATE_WIN 2
#define SEND_WIN 3
#define STAMP(i) do {} while (0)
#define STAMPC(i) do {} while (0)
#define STAMP_DECL do {} while (0)
#define STAMP_OUT() do {} while (0)

__device__ __forceinline__ unsigned pk_bf16(float lo, float hi) {
  f32x2 v = {lo, hi};
  return __builtin_bit_cast(unsigned, __builtin_convertvector(v, bf16x2));
}
__device__ __forceinline__ float wave_max(float f) {
  int v = __builtin_bit_cast(int, f);
  v = max(v, __builtin_amdgcn_update_dpp(0, v, 0x111, 0xf, 0xf, true));
  v = max(v, __builtin_amdgcn_update_dpp(0, v, 0x112, 0xf, 0xf, true));
  v = max(v, __builtin_amdgcn_update_dpp(0, v, 0x114, 0xf, 0xf, true));
  v = max(v, __builtin_amdgcn_update_dpp(0, v, 0x118, 0xf, 0xf, true));
  const int r = max(max(__builtin_amdgcn_readlane(v, 15), __builtin_amdgcn_readlane(v, 31)), max(__builtin_amdgcn_readlane(v, 47), __builtin_amdgcn_readlane(v, 63)));
  return __builtin_bit_cast(float, r);
}
__device__ __forceinline__ i32x4 make_rsrc(const void* base, unsigned bytes) {
  const u64 a = (u64)base;
  return i32x4{__builtin_amdgcn_readfirstlane((int)(unsigned)a), __builtin_amdgcn_readfirstlane((int)(unsigned)(a >> 32)), (int)bytes, 0x00020000};
}

__device__ __forceinline__ void w_load(f32x4& d, unsigned voff, i32x4 rsrc, unsigned soff) {
  asm volatile("buffer_load_dwordx4 %0, %1, %2, %3 offen nt" : "=v"(d) : "v"(voff), "s"(rsrc), "s"(soff) : "memory");
}
__device__ __forceinline__ void set_m0(unsigned m0v) { asm volatile("s_mov_b32 m0, %0" :: "s"(m0v) : "memory"); }
template <int IMM>
__device__ __forceinline__ void x_load(unsigned voff, i32x4 rsrc, unsigned soff) {
  asm volatile("buffer_load_dwordx4 %0, %1, %2 offen offset:%3 sc1 lds" :: "v"(voff), "s"(rsrc), "s"(soff), "i"(IMM) : "memory");
}
template <int N>
__device__ __forceinline__ void wait_bar() { asm volatile("s_waitcnt vmcnt(%0) lgkmcnt(0)\n\ts_barrier" :: "n"(N) : "memory"); }
__device__ __forceinline__ void mfma(f32x4& c, u32x4 a, u32x4 b) {
  asm volatile("v_mfma_f32_16x16x32_bf16 %0, %1, %2, %0" : "+a"(c) : "v"(a), "v"(b));
}
#define SCHED() __builtin_amdgcn_sched_barrier(0)
__device__ __forceinline__ void send_piece(const f32x4 (&y)[8][4], int j, i32x4 srs, unsigned lane16) {
  const f32x4 lo = y[j >> 1][(j & 1) * 2], hi = y[j >> 1][(j & 1) * 2 + 1];
  const u32x4 d = {pk_bf16(lo[0], lo[1]), pk_bf16(lo[2], lo[3]), pk_bf16(hi[0], hi[1]), pk_bf16(hi[2], hi[3])};
  asm volatile("buffer_store_dwordx4 %0, %1, %2, %3 offen sc0 sc1\n\ts_nop 1" :: "v"(d), "v"(lane16), "s"(srs), "s"(j * PC) : "memory");
}
constexpr float QBIAS = 12582912.0f;
struct Quant { float inv, lo, hi; };
__device__ __forceinline__ unsigned quant4(float v0, float v1, float v2, float v3, const Quant& qz) {
  const unsigned e0 = __builtin_bit_cast(unsigned, __builtin_amdgcn_fmed3f(__builtin_fmaf(v0, qz.inv, QBIAS), qz.lo, qz.hi));
  const unsigned e1 = __builtin_bit_cast(unsigned, __builtin_amdgcn_fmed3f(__builtin_fmaf(v1, qz.inv, QBIAS), qz.lo, qz.hi));
  const unsigned e2 = __builtin_bit_cast(unsigned, __builtin_amdgcn_fmed3f(__builtin_fmaf(v2, qz.inv, QBIAS), qz.lo, qz.hi));
  const unsigned e3 = __builtin_bit_cast(unsigned, __builtin_amdgcn_fmed3f(__builtin_fmaf(v3, qz.inv, QBIAS), qz.lo, qz.hi));
  return __builtin_amdgcn_perm(e1, e0, 0x0c0c0400u) | __builtin_amdgcn_perm(e3, e2, 0x04000c0cu);
}

template <int SCOPE, int SLEEP, int N = 1, int STRIDE = 0>
__device__ __forceinline__ bool wait_flag(const unsigned* flag, unsigned call, unsigned* tmo) {
  const u64 t0 = __builtin_amdgcn_s_memrealtime();
  for (unsigned polls = 1;; ++polls) {
    unsigned v[N];
    bool pend = false;
#pragma unroll
    for (int i = 0; i < N; ++i) v[i] = __hip_atomic_load(flag + i * STRIDE, __ATOMIC_RELAXED, SCOPE);
#pragma unroll
    for (int i = 0; i < N; ++i) pend |= v[i] < call;
    if (!__any(pend)) return true;
    if (((polls & 63u) == 0 && __hip_atomic_load(tmo, __ATOMIC_RELAXED, __HIP_MEMORY_SCOPE_AGENT) == call) ||
        __builtin_amdgcn_s_memrealtime() - t0 > DEADLINE_TICKS) {
      __hip_atomic_store(tmo, call, __ATOMIC_RELAXED, __HIP_MEMORY_SCOPE_AGENT);
      return false;
    }
    __builtin_amdgcn_s_sleep(SLEEP);
  }
}

struct Ctx {
  i32x4 wrsrc, xrsrc;
  unsigned wvoff[8], xvoff0, xvoff1, m0base;
  int wlim;
  int left;
  unsigned xso, wso;
  unsigned due, slack;
  int steps;
  i32x4 frsrc;
  const char* fbase;
  unsigned fvoff, fso, call, fl;
  unsigned* tmo;
  bool ok;
  LAS const char* lrd0;
  LAS const char* lrd1;
  LAS char* wmine;
  LAS const char* wother;
};

template <int SLOT>
__device__ __forceinline__ u32x4 x_frag(const Ctx& c, int mt) {
  return *(LAS const u32x4*)((SLOT < 2 ? c.lrd0 : c.lrd1) + (SLOT & 1) * 32768 + mt * 1024);
}
template <int G>
__device__ __forceinline__ void x_piece(const Ctx& c, i32x4 xr, unsigned xso) {
  x_load<(G & 3) * 1024>((G & 4) ? c.xvoff1 : c.xvoff0, xr, (G & 8) ? xso + 32768 : xso);
}
template <int SL, bool CHK, bool HELD = false>
__device__ __forceinline__ void kstep(Ctx& c, f32x4 (&acc)[16][4], f32x4 (&wr)[2][8], u32x4 (&a)[2][4], u32x4 (&xf)[8], const u32x4* held = nullptr) {
  constexpr int E = SL & 1, WS = SL >> 1, NX = (SL + 1) & 3;
  u64 now;
  unsigned lateby;
  const i32x4 xr = c.xrsrc, wrs = c.wrsrc;
  const unsigned xso = c.xso, wso = c.wso;
  const unsigned m0x = c.m0base + ((SL + 2) & 3) * 32768;
  f32x4 (&r)[8] = wr[WS ^ 1];
#pragma unroll
  for (int g = 0; g < 16; ++g) {
    if (E == 1 && g == 12) {
      if (CHK) {
        asm volatile("s_waitcnt vmcnt(8)" : "+v"(c.fl) :: "memory");
        if (__any(c.fl < c.call)) {
          c.ok &= wait_flag<__HIP_MEMORY_SCOPE_AGENT, 1>((const unsigned*)(c.fbase + c.fso + c.fvoff), c.call, c.tmo);
          __builtin_amdgcn_s_waitcnt(0xC07F);
        }
      }
      wait_bar<HELD ? 0 : 8>();
      SCHED();
    }
    mfma(acc[g][0], a[E][0], xf[g & 7]);
    SCHED();
    if (g < 9) xf[(g + 7) & 7] = x_frag<SL>(c, g + 7);
    else if (E == 0) xf[(g + 7) & 7] = x_frag<NX>(c, g - 9);
    else if (g >= 12) xf[(2 * (g - 12)) & 7] = x_frag<NX>(c, 2 * (g - 12));
    if (CHK && E == 0 && g == 0) asm volatile("buffer_load_dword %0, %1, %2, %3 offen sc1" : "=v"(c.fl) : "v"(c.fvoff), "s"(c.frsrc), "s"(c.fso) : "memory");
    if (E == 1 && g == 9) *(LAS u32x4*)(c.wmine + (WS ^ 1) * 16384 + 1024) = a[0][1];
    if (E == 1 && g == 10) *(LAS u32x4*)(c.wmine + (WS ^ 1) * 16384 + 3072) = a[0][3];
    if (E == 1 && g == 11) {
      asm volatile("s_waitcnt lgkmcnt(0)" : "+s"(now) :: "memory");
      lateby = (unsigned)now - c.due;
      asm volatile("" : "+s"(lateby));
    }
    SCHED();
    mfma(acc[g][1], a[E][1], xf[g & 7]);
    SCHED();
    if (E == 0) {
      switch (g) {
        case 0: x_piece<0>(c, xr, xso); break;   case 1: x_piece<1>(c, xr, xso); break;
        case 2: x_piece<2>(c, xr, xso); break;   case 3: x_piece<3>(c, xr, xso); break;
        case 4: x_piece<4>(c, xr, xso); break;   case 5: x_piece<5>(c, xr, xso); break;
        case 6: x_piece<6>(c, xr, xso); break;   case 7: x_piece<7>(c, xr, xso); break;
        case 8: x_piece<8>(c, xr, xso); break;   case 9: x_piece<9>(c, xr, xso); break;
        case 10: x_piece<10>(c, xr, xso); break; case 11: x_piece<11>(c, xr, xso); break;
        case 12: x_piece<12>(c, xr, xso); break; case 13: x_piece<13>(c, xr, xso); break;
        case 14: x_piece<14>(c, xr, xso); break; default: x_piece<15>(c, xr, xso); break;
      }
    } else {
      if (!HELD && g < 8) w_load(wr[WS][g & 7], c.wvoff[g & 7], wrs, wso);
      if (HELD && g >= 4 && g < 8) { a[0][g & 3] = held[g & 3]; asm volatile("" : "+v"(a[0][g & 3])); }
      if (!HELD && g == 5)
        asm volatile("s_waitcnt vmcnt(%8)" : "+v"(r[0]), "+v"(r[1]), "+v"(r[2]), "+v"(r[3]), "+v"(r[4]), "+v"(r[5]), "+v"(r[6]), "+v"(r[7]) : "n"(22 + CHK) : "memory");
      if (g == 8) *(LAS u32x4*)(c.wmine + (WS ^ 1) * 16384) = a[0][0];
      if (g == 9) { c.left -= 1; c.xso += 65536u; asm volatile("" : "+s"(c.left), "+s"(c.xso)); }
      if (g == 10) { c.xrsrc[2] = c.left > 1 ? (int)XT_BYTES : 0; asm volatile("" : "+s"(c.xrsrc)); }
      if (g == 11) { c.wrsrc[2] = c.left > c.wlim ? (int)W_BYTES : 0; asm volatile("" : "+s"(c.wrsrc)); }
      if (g == 12) { lateby = (int)lateby > (int)c.slack ? lateby - c.slack : 0u; asm volatile("" : "+s"(lateby)); }
      if (g == 13) { c.due += lateby; lateby = __builtin_amdgcn_readlane(c.steps, c.left); asm volatile("" : "+s"(c.due), "+s"(lateby)); }
      if (g == 14) { c.due += lateby; asm volatile("" : "+s"(c.due)); }
    }
    SCHED();
    mfma(acc[g][2], a[E][2], xf[g & 7]);
    SCHED();
    if (E == 0 && g < 4) a[1][g] = *(LAS const u32x4*)(c.wother + WS * 16384 + g * 1024);
    if (!HELD && E == 1 && g >= 5 && g < 9) { const int t = (g - 5) & 3; a[0][t][0] = pk_bf16(r[0][t], r[1][t]); a[0][t][1] = pk_bf16(r[2][t], r[3][t]); }
    if (E == 1 && g == 9) *(LAS u32x4*)(c.wmine + (WS ^ 1) * 16384 + 2048) = a[0][2];
    if (E == 1 && g == 11) {
      if (__builtin_expect((int)lateby < 0, 0))
        do __builtin_amdgcn_s_sleep(1); while ((int)((unsigned)__builtin_amdgcn_s_memrealtime() - c.due) < 0);
    }
    if (E == 1 && g >= 12 && g < 15) xf[(2 * (g - 12) + 1) & 7] = x_frag<NX>(c, 2 * (g - 12) + 1);
    SCHED();
    mfma(acc[g][3], a[E][3], xf[g & 7]);
    SCHED();
    if (!HELD && E == 1 && g >= 5 && g < 9) { const int t = (g - 5) & 3; a[0][t][2] = pk_bf16(r[4][t], r[5][t]); a[0][t][3] = pk_bf16(r[6][t], r[7][t]); }
    if (E == 0 && (g & 3) == 3 && g < 15) set_m0(m0x + ((g + 1) >> 3) * 32768 + (((g + 1) >> 2) & 1) * 4096);
    if (E == 1 && g == 9) asm volatile("s_memrealtime %0" : "=s"(now) :: "memory");
    if (E == 1 && g == 10) { c.wso += 1u << 22; asm volatile("" : "+s"(c.wso)); }
    if (E == 1 && g == 15) set_m0(c.m0base + ((SL + 3) & 3) * 32768);
    SCHED();
  }
}

__device__ __forceinline__ void last_block(const Ctx& c, f32x4 (&acc)[16][4], u32x4 (&a)[2][4], u32x4 (&xf)[8], i32x4 prs, unsigned pov) {
  f32x4 lo, hi;
#pragma unroll
  for (int i = 0; i < 32; ++i) {
    const int e = (i >> 3) & 1, m = (i & 7) + (i >> 4) * 8;
    const int i7 = i + 7, e7 = (i7 >> 3) & 1, m7 = (i7 & 7) + (i7 >> 4) * 8;
    const int j = i - 16;
    mfma(acc[m][0], a[e][0], xf[i & 7]);
    SCHED();
    if (i7 < 32) xf[i7 & 7] = e7 ? x_frag<3>(c, m7) : x_frag<2>(c, m7);
    if (j >= 0) {
      asm volatile("" : "+a"(acc[j >> 1][(j & 1) * 2]));
      lo = acc[j >> 1][(j & 1) * 2];
      asm volatile("" : "+v"(lo));
    }
    SCHED();
    mfma(acc[m][1], a[e][1], xf[i & 7]);
    SCHED();
    if (i < 4) a[1][i] = *(LAS const u32x4*)(c.wother + 16384 + i * 1024);
    if (j >= 0) {
      asm volatile("" : "+a"(acc[j >> 1][(j & 1) * 2 + 1]));
      hi = acc[j >> 1][(j & 1) * 2 + 1];
      asm volatile("" : "+v"(hi));
    }
    SCHED();
    mfma(acc[m][2], a[e][2], xf[i & 7]);
    SCHED();
    u32x4 d;
    if (j >= 0) { d = u32x4{pk_bf16(lo[0], lo[1]), pk_bf16(lo[2], lo[3]), pk_bf16(hi[0], hi[1]), pk_bf16(hi[2], hi[3])}; asm volatile("" : "+v"(d)); }
    SCHED();
    mfma(acc[m][3], a[e][3], xf[i & 7]);
    SCHED();
    if (j >= 0)
      asm volatile("buffer_store_dwordx4 %0, %1, %2, %3 offen offset:%4 sc1\n\ts_nop 1"
                   :: "v"(d), "v"(pov), "s"(prs), "s"(j * PC & ~4095), "i"(j * PC & 4095) : "memory");
    SCHED();
  }
}

template <int N, int STRIDE, int SCOPE, int SLEEP>
__device__ __forceinline__ float sweep_max(const u64* slots, unsigned call, unsigned* tmo) {
  const u64 t0 = __builtin_amdgcn_s_memrealtime();
  for (unsigned polls = 1;; ++polls) {
    bool all = true, bad = false;
    float mx = 0.f;
    u64 v[N];
#pragma unroll
    for (int r = 0; r < N; ++r) v[r] = __hip_atomic_load(slots + r * STRIDE, __ATOMIC_RELAXED, SCOPE);
#pragma unroll
    for (int r = 0; r < N; ++r) {
      const float f = __builtin_bit_cast(float, (unsigned)v[r]);
      all &= (unsigned)(v[r] >> 32) == call;
      bad |= f < 0.f;
      mx = fmaxf(mx, f);
    }
    if (__all(all)) {
      return __any(bad) ? -1.0f : wave_max(mx);
    }
    if (((polls & 63u) == 0 && __hip_atomic_load(tmo, __ATOMIC_RELAXED, __HIP_MEMORY_SCOPE_AGENT) == call) ||
        __builtin_amdgcn_s_memrealtime() - t0 > DEADLINE_TICKS) {
      __hip_atomic_store(tmo, call, __ATOMIC_RELAXED, __HIP_MEMORY_SCOPE_AGENT);
      return -1.0f;
    }
    __builtin_amdgcn_s_sleep(SLEEP);
  }
}
template <int SLEEP>
__device__ __forceinline__ Quant global_scale(const u64* gmax, unsigned call, unsigned* tmo, bool ok, int lane) {
  const float gam = ok ? sweep_max<1, 0, __HIP_MEMORY_SCOPE_SYSTEM, SLEEP>(gmax + (lane & 7), call, tmo) : -1.0f;
  Quant qz;
  qz.inv = gam > 0.f ? 127.0f / gam : 0.f;
  qz.lo = gam >= 0.f ? QBIAS - 127.0f : QBIAS - 128.0f;
  qz.hi = gam >= 0.f ? QBIAS + 127.0f : QBIAS - 128.0f;
  asm volatile("" : "+v"(qz.lo), "+v"(qz.hi));
  return qz;
}

__device__ __forceinline__ int line_of(int ct, int s) {
  return ct < 7 ? 8 * ct + 4 + s : ct < 11 ? 16 * (ct - 7) + (s >> 1) * 8 + (s & 1) : ct == 11 ? 60 + s : (ct & 1) * 32 + 8 * s + (ct >> 1) - 4;
}
__device__ __forceinline__ void tile_of(int j, int& pd, int& ct) {
  const int k = j / 7;
  pd = j < 112 ? j % 7 + 1 : 0;
  ct = j >= 112 ? j - 112 : k < 2 ? 12 + k : k < NE - 2 ? 14 - NE + k : k < NE ? 16 - NE + k : k - NE;
}

__global__ void __launch_bounds__(NTHR, 1) fused_kernel(Params p) {
  extern __shared__ __attribute__((aligned(1024))) char smem[];
  const int tid = threadIdx.x, lane = tid & 63;
  const int wave = __builtin_amdgcn_readfirstlane(tid >> 6);
  char* ws = p.myws;

  STAMP_DECL;
  STAMP(0);
  STAMPC(8);
  const int b = blockIdx.x, xcd = b & 7, idx = b >> 3;
  const int kh = xcd & 1, slot = idx * 4 + (xcd >> 1), unit = slot * 2 + kh;
  const int wn = wave & 1, wm = wave >> 1;
  const int q = lane >> 4, i16 = lane & 15;
  const unsigned lane16 = lane * 16;
  const unsigned t_start = (unsigned)__builtin_amdgcn_s_memrealtime();

  Ctx c;
  c.xrsrc = make_rsrc(ws + WS_XT, (unsigned)XT_BYTES);
  unsigned* xflag = (unsigned*)(ws + WS_XFLAG);
  const int hs = kh * NT + slot;
  const unsigned xto = (unsigned)(hs * 16384 + wave * 4096);
  f32x4 raw[8];
  unsigned prev, warm;
  {
    const int st = hs >> 1, h = hs & 1;
    const i32x4 xin = make_rsrc(p.x, (unsigned)((size_t)ML * KD * 4));
#pragma unroll
    for (int j = 0; j < 4; ++j) {
      const unsigned o = (unsigned)((h * 16 + ((wave * 4 + j) ^ ((kh ^ 1) << 3))) * 16 + i16) * (KD * 4) + ((st ^ h) * 32 + q * 8) * 4;
      asm volatile("buffer_load_dwordx4 %0, %2, %3, 0 offen\n\tbuffer_load_dwordx4 %1, %2, %3, 0 offen offset:16"
                   : "=&v"(raw[2 * j]), "=&v"(raw[2 * j + 1]) : "v"(o), "s"(xin) : "memory");
    }
    asm volatile("global_load_dword %0, %1, off sc1" : "=v"(prev) : "v"(xflag + hs * (FS / 4)) : "memory");
    asm volatile("buffer_load_dword %0, %1, %2, %3 offen sc1" : "=v"(warm) : "v"(0), "s"(c.xrsrc), "s"(xto) : "memory");
  }

  const bool early = slot < NEARLY;
  int pd, ct;
  tile_of(slot, pd, ct);
  const int rank = p.rank;
  const unsigned dest = (unsigned)(rank + pd) & 7u;
  const int k0 = kh * KH;
  const int ln0 = line_of(ct, 2 * wn), ln1 = line_of(ct, 2 * wn + 1);
  unsigned* tmo = (unsigned*)(ws + WS_TMO);
  GAS char* dws = (GAS char*)p.ws[dest];
  GAS char* dout = (GAS char*)p.out[early ? rank : dest];
  GAS char* myout = (GAS char*)p.out[rank];
  asm volatile("" : "+s"(dws), "+s"(dout), "+s"(myout));
  GAS char* anyws;
  i32x2 stp;
  asm volatile("global_load_dwordx2 %0, %2, %4 offset:%5\n\tglobal_load_dwordx2 %1, %3, %4 offset:%6"
               : "=&v"(anyws), "=&v"(stp) : "v"((lane & 7) * 8), "v"(((KH - 1 - lane) & 31) * 4), "s"(__builtin_amdgcn_kernarg_segment_ptr()),
                 "i"(offsetof(Params, ws)), "i"(offsetof(Params, step)) : "memory");

  c.wrsrc = make_rsrc(p.w, (unsigned)W_BYTES);
#pragma unroll
  for (int j = 0; j < 8; ++j) {
    c.wvoff[j] = ((unsigned)(wm * 32 + q * 8 + j) << 16) + (i16 < 8 ? ln0 : ln1) * 128 + (i16 & 7) * 16;
    asm volatile("" : "+v"(c.wvoff[j]));
  }
  c.xvoff0 = wave * 8192 + lane16;
  c.xvoff1 = c.xvoff0 + 4096;
  asm volatile("" : "+v"(c.xvoff1));
  c.m0base = (unsigned)(size_t)(LAS char*)smem + wave * 8192;
  constexpr int nblk = KH;
  const unsigned wsb = ((unsigned)k0 << 22) + dest * (NL * 4u);
  c.left = nblk;
  c.wlim = 2;
  c.xso = (unsigned)(k0 + 1) * 65536u;
  c.wso = wsb + (2u << 22);
  const bool steady = early || ct == 11 || ct >= 14;
  const int cls = early ? slot / 7 : steady ? NE - 1 : NE, te = p.te[steady ? cls : 0], phase = cls < p.nwide ? 0 : (slot * 74 + kh * 8) & 255;
  c.due = t_start + p.start[cls] + phase * (steady ? te : p.step[0]) / 256;
  c.slack = p.slack;
  int wide = (p.wmask >> cls) & 1;
  asm volatile("" : "+s"(wide));
  c.lrd0 = (LAS const char*)smem + wm * 16384 + lane16;
  c.lrd1 = c.lrd0 + 65536;
  asm volatile("" : "+v"(c.lrd1));
  c.wmine = (LAS char*)smem + LDS_X + wave * 4096 + lane16;
  c.wother = (LAS const char*)smem + LDS_X + (wave ^ 2) * 4096 + lane16;

  f32x4 acc[16][4];
  f32x4 wr[2][8];
  u32x4 a[2][4], xf[8];
#pragma unroll
  for (int m = 0; m < 16; ++m)
#pragma unroll
    for (int t = 0; t < 4; ++t) {
      acc[m][t] = f32x4{0.f, 0.f, 0.f, 0.f};
      asm volatile("" : "+a"(acc[m][t]));
    }
  asm volatile("s_waitcnt vmcnt(2)" : "+v"(prev), "+v"(warm), "+v"(raw[0]), "+v"(raw[1]), "+v"(raw[2]), "+v"(raw[3]), "+v"(raw[4]), "+v"(raw[5]), "+v"(raw[6]), "+v"(raw[7]) :: "memory");
  STAMP(11);
#pragma unroll
  for (int j = 0; j < 4; ++j) {
    const f32x4 v0 = raw[2 * j], v1 = raw[2 * j + 1];
    const u32x4 d = {pk_bf16(v0[0], v0[1]), pk_bf16(v0[2], v0[3]), pk_bf16(v1[0], v1[1]), pk_bf16(v1[2], v1[3])};
    asm volatile("buffer_store_dwordx4 %0, %1, %2, %3 offen offset:%4 sc1\n\ts_nop 1" :: "v"(d), "v"(lane16), "s"(c.xrsrc), "s"(xto), "i"(j * 1024) : "memory");
  }
  const unsigned call = __builtin_amdgcn_readfirstlane(prev) + 1;
  asm volatile("s_waitcnt vmcnt(0)\n\ts_barrier" : "+v"(anyws), "+v"(stp) :: "memory");
  c.steps = steady ? te : stp[1] + phase * (stp[1] - stp[0]) / 256;
  STAMP(12);
  if (tid == 0) __hip_atomic_store(xflag + hs * (FS / 4), call, __ATOMIC_RELAXED, __HIP_MEMORY_SCOPE_AGENT);
  if (b == 0 && wave == 1 && lane < NR)
    __hip_atomic_store((GAS unsigned*)(anyws + WS_START + rank * FS), call, __ATOMIC_RELAXED, __HIP_MEMORY_SCOPE_SYSTEM);
  c.fbase = (const char*)xflag + kh * NT * FS;
  c.frsrc = make_rsrc(c.fbase, NT * FS);
  c.fvoff = (lane & 3) * FS;
  c.fso = 8 * FS;
  c.call = call;
  c.tmo = tmo;
  c.ok = true;
  if ((unsigned)(idx - PF0) < (1u << PFL)) {
    constexpr int NP = 16 >> PFL;
    const int pw = (idx - PF0) * 4 + wave;
    const unsigned pv = (unsigned)k0 * 65536u + pw * NP * 1024 + lane16;
    u32x4 pf = {0u, 0u, 0u, 0u};
#pragma unroll
    for (int j = 1; j < KH; ++j) {
      if (j == 1 || j == 5)
        c.ok &= wait_flag<__HIP_MEMORY_SCOPE_AGENT, 1>((const unsigned*)(c.fbase + ((j == 1 ? (lane & 3) + 1 : lane & 31) * 4 + (pw >> PFL)) * FS), call, tmo);
      const unsigned v = pv + j * 65536;
      asm volatile("s_waitcnt vmcnt(32)\n\tbuffer_load_dwordx4 %0, %1, %2, 0 offen sc1\n\tbuffer_load_dwordx4 %0, %1, %2, 0 offen offset:1024 sc1" : "+v"(pf) : "v"(v), "s"(c.xrsrc) : "memory");
      if (NP == 4) asm volatile("buffer_load_dwordx4 %0, %1, %2, 0 offen offset:2048 sc1\n\tbuffer_load_dwordx4 %0, %1, %2, 0 offen offset:3072 sc1" : "+v"(pf) : "v"(v), "s"(c.xrsrc) : "memory");
    }
    asm volatile("s_waitcnt vmcnt(0)" : "+v"(pf) :: "memory");
  }
#pragma unroll
  for (int j = 0; j < 8; ++j) w_load(wr[0][j], c.wvoff[j], c.wrsrc, wsb);
  if (wave == 0) c.ok &= wait_flag<__HIP_MEMORY_SCOPE_AGENT, 1>((const unsigned*)(c.fbase + (lane & 7) * FS), call, tmo);
  asm volatile("s_waitcnt vmcnt(0)" : "+v"(wr[0][0]), "+v"(wr[0][1]), "+v"(wr[0][2]), "+v"(wr[0][3]), "+v"(wr[0][4]), "+v"(wr[0][5]), "+v"(wr[0][6]), "+v"(wr[0][7]) :: "memory");
#pragma unroll
  for (int t = 0; t < 4; ++t) {
    a[0][t] = u32x4{pk_bf16(wr[0][0][t], wr[0][1][t]), pk_bf16(wr[0][2][t], wr[0][3][t]), pk_bf16(wr[0][4][t], wr[0][5][t]), pk_bf16(wr[0][6][t], wr[0][7][t])};
    *(LAS u32x4*)(c.wmine + t * 1024) = a[0][t];
  }
  wait_bar<0>();
  {
    const unsigned xso = (unsigned)k0 * 65536u;
    set_m0(c.m0base); asm volatile("s_nop 0");
    x_piece<0>(c, c.xrsrc, xso); x_piece<1>(c, c.xrsrc, xso); x_piece<2>(c, c.xrsrc, xso); x_piece<3>(c, c.xrsrc, xso);
    set_m0(c.m0base + 4096); asm volatile("s_nop 0");
    x_piece<4>(c, c.xrsrc, xso); x_piece<5>(c, c.xrsrc, xso); x_piece<6>(c, c.xrsrc, xso); x_piece<7>(c, c.xrsrc, xso);
    set_m0(c.m0base + 32768); asm volatile("s_nop 0");
    x_piece<8>(c, c.xrsrc, xso); x_piece<9>(c, c.xrsrc, xso); x_piece<10>(c, c.xrsrc, xso); x_piece<11>(c, c.xrsrc, xso);
    set_m0(c.m0base + 32768 + 4096); asm volatile("s_nop 0");
    x_piece<12>(c, c.xrsrc, xso); x_piece<13>(c, c.xrsrc, xso); x_piece<14>(c, c.xrsrc, xso); x_piece<15>(c, c.xrsrc, xso);
  }
#pragma unroll
  for (int j = 0; j < 8; ++j) w_load(wr[1][j], c.wvoff[j], c.wrsrc, wsb + (1u << 22));
  __builtin_amdgcn_s_waitcnt(0xC07F);
  wait_bar<8>();
  STAMP(10);
  SCHED();
  set_m0(c.m0base + 2 * 32768);
#pragma unroll
  for (int t = 0; t < 7; ++t) xf[t] = x_frag<0>(c, t);
  SCHED();
  while ((int)((unsigned)__builtin_amdgcn_s_memrealtime() + 100u - c.due) < 0) __builtin_amdgcn_s_sleep(4);
  __builtin_amdgcn_s_waitcnt(0xC07F);
  STAMP(1);

#define TRACE_PT()
  TRACE_PT()
  kstep<0, true>(c, acc, wr, a, xf);
  kstep<1, true>(c, acc, wr, a, xf);
  c.fvoff = (lane & 31) * FS;
  c.fso = wave * 32 * FS;
  kstep<2, true>(c, acc, wr, a, xf);
  kstep<3, true>(c, acc, wr, a, xf);
  kstep<0, false>(c, acc, wr, a, xf);
  kstep<1, false>(c, acc, wr, a, xf);
  u32x4 held[NH][4] = {};
  const int ltail = steady ? 1 : NH + 1;
  if (!steady) {
    c.wlim = 2 + NH;
    f32x4 raw2[2][8];
#pragma unroll
    for (int i = 0; i <= NH; ++i) {
      if (i < NH) {
#pragma unroll
        for (int j = 0; j < 8; ++j) w_load(raw2[i & 1][j], c.wvoff[j], c.wrsrc, wsb + ((unsigned)(nblk - NH + i) << 22));
      }
      if (i > 0) {
        f32x4 (&r)[8] = raw2[(i - 1) & 1];
        asm volatile("s_waitcnt vmcnt(%8)" : "+v"(r[0]), "+v"(r[1]), "+v"(r[2]), "+v"(r[3]), "+v"(r[4]), "+v"(r[5]), "+v"(r[6]), "+v"(r[7]) : "n"(i < NH ? 8 : 0) : "memory");
#pragma unroll
        for (int t = 0; t < 4; ++t) {
          held[i - 1][t] = u32x4{pk_bf16(r[0][t], r[1][t]), pk_bf16(r[2][t], r[3][t]), pk_bf16(r[4][t], r[5][t]), pk_bf16(r[6][t], r[7][t])};
          asm volatile("" : "+v"(held[i - 1][t]));
        }
      }
    }
    SCHED();
  }
  do {
    TRACE_PT()
    kstep<2, false>(c, acc, wr, a, xf);
    kstep<3, false>(c, acc, wr, a, xf);
    kstep<0, false>(c, acc, wr, a, xf);
    kstep<1, false>(c, acc, wr, a, xf);
  } while (c.left > ltail);
  while (c.left > 1) {
    TRACE_PT()
    kstep<2, false, true>(c, acc, wr, a, xf, held[0]);
    kstep<3, false, true>(c, acc, wr, a, xf, held[0]);
    kstep<0, false, true>(c, acc, wr, a, xf, held[1]);
    kstep<1, false, true>(c, acc, wr, a, xf, held[1]);
#pragma unroll
    for (int i = 0; i + 2 < NH; ++i)
#pragma unroll
      for (int t = 0; t < 4; ++t) held[i][t] = held[i + 2][t];
    asm volatile("s_nop 1");
  }
  LAS float* red = (LAS float*)(smem + LDS_X);
  const size_t gmax_off = WS_GMAX + (size_t)(call & 1) * FS;
  const size_t wave_off = (size_t)unit * HALF_B + wave * WAVE_B;
  const unsigned pov = (PC > 1024 && (lane >> 3) == 3) ? lane16 + 5 * 128 : lane16;
  unsigned* pflag = (unsigned*)(ws + WS_PFLAG + wave * FS);
  last_block(c, acc, a, xf, make_rsrc(ws + WS_PART + wave_off, WAVE_B), pov);
  bool ok = c.ok;
  STAMP(2);
  STAMPC(9);
  asm volatile("s_waitcnt vmcnt(0)"
               : "+v"(wr[0][0]), "+v"(wr[0][1]), "+v"(wr[0][2]), "+v"(wr[0][3]), "+v"(wr[0][4]), "+v"(wr[0][5]), "+v"(wr[0][6]), "+v"(wr[0][7])
               :: "memory");
  if (lane == 0) __hip_atomic_store(pflag + unit * FS, call, __ATOMIC_RELAXED, __HIP_MEMORY_SCOPE_AGENT);
  ok &= wait_flag<__HIP_MEMORY_SCOPE_SYSTEM, 2>((lane & 1) ? (const unsigned*)(ws + WS_START + dest * FS) : pflag + (unit ^ 1) * FS, call, tmo);
  const i32x4 srs = make_rsrc((const void*)(dws + WS_STAGE + wave_off), ok ? WAVE_B : 0);
  f32x4 y[8][4];
  {
    const i32x4 prs = make_rsrc(ws + WS_PART + (wave_off ^ HALF_B), WAVE_B);
    u32x4 pv[16];
#pragma unroll
    for (int j = 0; j < 16; ++j)
      asm volatile("buffer_load_dwordx4 %0, %1, %2, %3 offen offset:%4 sc1" : "=v"(pv[j]) : "v"(pov), "s"(prs), "s"(j * PC & ~4095), "i"(j * PC & 4095) : "memory");
    const unsigned one = (i16 >> 2) == q ? ((i16 & 1) ? 0x3F800000u : 0x00003F80u) : 0u;
    const unsigned e0 = (i16 & 2) ? 0u : one, e1 = (i16 & 2) ? one : 0u;
    u32x4 sel0 = {e0, e1, 0u, 0u}, sel1 = {0u, 0u, e0, e1};
    asm volatile("s_nop 1" : "+v"(sel0), "+v"(sel1));
    float am = 0.f;
#pragma unroll
    for (int j = 0; j < 16 + 4; ++j) {
      if (j < 16) {
        asm volatile("s_waitcnt vmcnt(%1)" : "+v"(pv[j]) : "n"(15 - j) : "memory");
        mfma(acc[8 + (j >> 1)][(j & 1) * 2], sel0, pv[j]);
        mfma(acc[8 + (j >> 1)][(j & 1) * 2 + 1], sel1, pv[j]);
      }
      if (j == 16) asm volatile("s_nop 7\n\ts_nop 7" ::: "memory");
      if (j >= 4) {
        const int m = (j - 4) >> 1, u = (j - 4) & 1;
        asm volatile("" : "+a"(acc[8 + m][2 * u]), "+a"(acc[8 + m][2 * u + 1]), "+v"(am));
        y[m][2 * u] = acc[8 + m][2 * u]; y[m][2 * u + 1] = acc[8 + m][2 * u + 1];
        asm volatile("" : "+v"(y[m][2 * u]), "+v"(y[m][2 * u + 1]));
        const f32x4 lo = y[m][2 * u], hi = y[m][2 * u + 1];
        am = fmaxf(fmaxf(am, fabsf(lo[0])), fabsf(lo[1])); am = fmaxf(fmaxf(am, fabsf(lo[2])), fabsf(lo[3]));
        am = fmaxf(fmaxf(am, fabsf(hi[0])), fabsf(hi[1])); am = fmaxf(fmaxf(am, fabsf(hi[2])), fabsf(hi[3]));
        asm volatile("" : "+v"(am));
      }
    }
    if (early) {
#pragma unroll
      for (int j = 0; j < SEND_WIN; ++j) send_piece(y, j, srs, pov);
    }
    am = wave_max(am);
    if (lane == 0) red[wave] = ok ? am : -1.0f;
    __syncthreads();
    if (tid == 0) {
      const float r0 = red[0], r1 = red[1], r2 = red[2], r3 = red[3];
      const float wam = fminf(fminf(r0, r1), fminf(r2, r3)) < 0.f ? -1.0f : fmaxf(fmaxf(r0, r1), fmaxf(r2, r3));
      __hip_atomic_store((u64*)(ws + WS_AMAX + (unit >> 4) * FS) + (unit & 15), ((u64)call << 32) | __builtin_bit_cast(unsigned, wam), __ATOMIC_RELAXED, __HIP_MEMORY_SCOPE_AGENT);
    }
  }
  STAMP(3);
  const int row0 = wm * 256 + kh * 128 + i16, col0 = (q < 2 ? ln0 : ln1) * 32 + (q & 1) * 16;
  const int srank = (rank - pd) & 7;

  if (early) {
#pragma unroll
    for (int j = SEND_WIN; j < 16; ++j) {
      if (wide > 0) asm volatile("s_waitcnt vmcnt(%0)" :: "n"(2 * SEND_WIN - 1) : "memory");
      else asm volatile("s_waitcnt vmcnt(%0)" :: "n"(SEND_WIN - 1) : "memory");
      send_piece(y, j, srs, pov);
    }
    asm volatile("s_waitcnt vmcnt(0)" ::: "memory");
    if (lane == 0)
      __hip_atomic_store((GAS unsigned*)(dws + WS_EFLAG + (size_t)(unit * 4 + wave) * FS), call, __ATOMIC_RELAXED, __HIP_MEMORY_SCOPE_SYSTEM);
    STAMP(5);
    ok &= wait_flag<__HIP_MEMORY_SCOPE_SYSTEM, 8>((const unsigned*)(ws + WS_EFLAG + (size_t)(unit * 4 + wave) * FS), call, tmo);
    u32x4 st[16];
    {
      const i32x4 rs = make_rsrc(ws + WS_STAGE + wave_off, WAVE_B);
#pragma unroll
      for (int j = 0; j < 16; ++j)
        asm volatile("buffer_load_dwordx4 %0, %1, %2, %3 offen offset:%4 sc0 sc1" : "=v"(st[j]) : "v"(pov), "s"(rs), "s"(j * PC & ~4095), "i"(j * PC & 4095) : "memory");
      asm volatile("s_waitcnt vmcnt(0)" ::: "memory");
#pragma unroll
      for (int j = 0; j < 16; ++j) asm volatile("" : "+v"(st[j]));
    }
    STAMP(6);
    const Quant qz = global_scale<ESLEEP>((const u64*)(ws + gmax_off), call, tmo, ok, lane);
    STAMP(4);
    GAS char* ob = dout + (size_t)(srank * ML + row0) * NL + col0;
#pragma unroll
    for (int m = 0; m < 8; ++m) {
      const u32x4 s0 = st[2 * m], s1 = st[2 * m + 1];
      u32x4 o;
#pragma unroll
      for (int r = 0; r < 4; ++r) {
        const int h = r >> 1;
        const unsigned w0 = s0[h], w1 = s0[2 + h], w2 = s1[h], w3 = s1[2 + h];
        const float v0 = __builtin_bit_cast(float, (r & 1) ? (w0 & 0xFFFF0000u) : (w0 << 16));
        const float v1 = __builtin_bit_cast(float, (r & 1) ? (w1 & 0xFFFF0000u) : (w1 << 16));
        const float v2 = __builtin_bit_cast(float, (r & 1) ? (w2 & 0xFFFF0000u) : (w2 << 16));
        const float v3 = __builtin_bit_cast(float, (r & 1) ? (w3 & 0xFFFF0000u) : (w3 << 16));
        o[r] = quant4(v0, v1, v2, v3, qz);
      }
      asm volatile("global_store_dwordx4 %0, %1, off sc0 sc1\n\ts_nop 1" :: "v"(ob + (size_t)m * 16 * NL), "v"(o) : "memory");
    }
    STAMP(7);
    STAMP_OUT();
    return;
  }

  if (unit == NWG - 1 && wave >= AGG0) {
    const float rm = sweep_max<4, 4 * FS / 8, __HIP_MEMORY_SCOPE_AGENT, 1>((const u64*)(ws + WS_AMAX + (lane >> 4) * FS) + (lane & 15), call, tmo);
    if (lane < NR)
      __hip_atomic_store((GAS u64*)(anyws + gmax_off) + rank, ((u64)call << 32) | __builtin_bit_cast(unsigned, rm), __ATOMIC_RELAXED, __HIP_MEMORY_SCOPE_SYSTEM);
  }
  const Quant qz = global_scale<LSLEEP>((const u64*)(ws + gmax_off), call, tmo, ok, lane);
  STAMP(4);
  {
    GAS char* ob = dout + (size_t)(rank * ML + row0) * NL + col0;
#pragma unroll
    for (int m = 0; m < 8; ++m) {
      u32x4 o;
#pragma unroll
      for (int r = 0; r < 4; ++r) o[r] = quant4(y[m][0][r], y[m][1][r], y[m][2][r], y[m][3][r], qz);
      asm volatile("s_waitcnt vmcnt(%2)\n\tglobal_store_dwordx4 %0, %1, off sc0 sc1\n\ts_nop 1" :: "v"(ob + (size_t)m * 16 * NL), "v"(o), "n"(LATE_WIN - 1) : "memory");
    }
  }
  asm volatile("s_waitcnt vmcnt(0)" ::: "memory");
  STAMP(5);
  if (lane == 0)
    __hip_atomic_store((GAS unsigned*)(dws + WS_EFLAG + (size_t)(unit * 4 + wave) * FS), call, __ATOMIC_RELAXED, __HIP_MEMORY_SCOPE_SYSTEM);
  if (!wait_flag<__HIP_MEMORY_SCOPE_SYSTEM, 2>((const unsigned*)(ws + WS_EFLAG + (size_t)(unit * 4 + wave) * FS), call, tmo) && lane == 0)
    myout[(size_t)(srank * ML + row0) * NL + col0] = (char)-128;
  STAMP(7);
  STAMP_OUT();
}

static void make_pace(Params& p, const int* pa) {
  const double T0 = pa[0], TH = pa[1], T1 = pa[2], TR = pa[3], TC = pa[6];
  const int TE = pa[4], TS = pa[5], A = pa[7], DS = pa[8];
  p.slack = pa[9];
  p.nwide = pa[10]; p.wmask = pa[14];
  for (int k = 0; k < NE; ++k) { p.te[k] = k < NE - 3 ? TE : k == NE - 3 ? TS : pa[13]; p.start[k] = A + (k * DS > pa[11] ? k * DS - pa[11] : 0) - KH * (p.te[k] - TE); if (p.start[k] < A) p.start[k] = A; }
  for (int k = 0; k < pa[10]; ++k) p.te[k] = pa[12];
  p.start[NE] = A;
  double t = A;
  for (int v = 0; v <= KH; ++v) {
    double done = 0, u = t;
    while (done < 1.0) {
      double r = NWG / (u < TH ? T0 : u < TR ? T0 + (T1 - T0) * (u - TH) / (TR - TH) : T1);
      for (int k = 0; k < NE; ++k) if (u >= p.start[k] - p.te[k] / 2 && u < p.start[k] + (KH - 0.5) * p.te[k]) r -= (k == NE - 1 ? 20.0 : 14.0) / p.te[k];
      r /= NWG - 2 * NEARLY - 6;
      if (r > 1.0 / TC) r = 1.0 / TC;
      if (r < 0.001) r = 0.001;
      done += r; u += 1.0;
    }
    p.step[v] = (int)(u - t);
    if (v) t = u; else t += (u - t) * 0.5;
  }
}

extern "C" void kernel_launch(int rank, int nranks, ncclComm_t comm,
                              hipStream_t stream,
                              void* const* d_in, const int* in_sizes, int n_in,
                              void* d_out, int out_size,
                              const int* in_dtypes, int out_dtype,
                              void* const* d_out_all,
                              void* const* d_ws_all, size_t ws_bytes) {
  (void)comm; (void)in_dtypes; (void)out_dtype; (void)d_out;
  if (nranks != NR || n_in != 2 || in_sizes[0] != ML * KD || in_sizes[1] != KD * ND || out_size != NR * ML * NL ||
      ws_bytes < WS_END || (unsigned)rank >= (unsigned)NR) {
    fprintf(stderr, "unexpected problem shape\n");
    return;
  }
  HIP_CHECK(hipFuncSetAttribute((const void*)fused_kernel, hipFuncAttributeMaxDynamicSharedMemorySize, LDS_BYTES));
  Params p;
  p.x = (const float*)d_in[0];
  p.w = (const float*)d_in[1];
  for (int r = 0; r < NR; ++r) { p.out[r] = (char*)d_out_all[r]; p.ws[r] = (char*)d_ws_all[r]; }
  p.rank = rank;
  p.myws = (char*)d_ws_all[rank];
  int pa[15] = {PACE_ARGS};
  make_pace(p, pa);
  fused_kernel<<<NWG, NTHR, LDS_BYTES, stream>>>(p);
  HIP_CHECK(hipGetLastError());
}
```

```cpp
#include <hip/hip_runtime.h>
#include <rccl/rccl.h>
#include <cstdio>
#include <cstdint>
#include <cstdlib>
#include <cstddef>

#define HIP_CHECK(expr) do { hipError_t _e = (expr); if (_e != hipSuccess) { \
    fprintf(stderr, "HIP error %s at %s:%d\n", hipGetErrorString(_e), __FILE__, __LINE__); \
    return; } } while (0)

#define LAS __attribute__((address_space(3)))
#define GAS __attribute__((address_space(1)))
typedef float f32x4 __attribute__((ext_vector_type(4)));
typedef float f32x2 __attribute__((ext_vector_type(2)));
typedef __bf16 bf16x2 __attribute__((ext_vector_type(2)));
typedef unsigned u32x4 __attribute__((ext_vector_type(4)));
typedef int i32x4 __attribute__((ext_vector_type(4)));
typedef int i32x2 __attribute__((ext_vector_type(2)));
typedef unsigned long long u64;

constexpr int NR = 8;
constexpr int KD = 4096, ND = 16384;
constexpr int ML = 512;
constexpr int NL = ND / NR;
constexpr int NWG = 256, NTHR = 256;
constexpr int NBLK = KD / 64;
constexpr int NT = ND / 128;
#define NE 9
constexpr int NEARLY = 7 * NE;
#define PF0 4
#define PFL 3
static_assert(NE >= 7 && NE <= 12, "the workgroups that prefetch x belong to early tiles that start late");
constexpr int KH = NBLK / 2;
#define NH 2
#define PACE_ARGS 212, 2250, 160, 4200, 125, 150, 115, 600, 245, 200, 2, 100, 122, 175, 3
constexpr int LDS_X = 4 * 32768;
constexpr int LDS_BYTES = LDS_X + 32768;
#define PC 2048
constexpr int FS = 256, WAVE_B = 16 * PC;
constexpr size_t HALF_B = 4 * WAVE_B;

constexpr size_t WS_TMO = 128;
constexpr size_t WS_GMAX = 512;
constexpr size_t WS_START = 1024;
constexpr size_t WS_AMAX = 4096;
constexpr size_t WS_XFLAG = WS_AMAX + NWG / 16 * FS;
constexpr size_t WS_EFLAG = WS_XFLAG + NWG * FS;
constexpr size_t WS_PFLAG = WS_EFLAG + NWG * 4 * FS;
constexpr size_t WS_STAMP = WS_PFLAG + NWG * 4 * FS;
constexpr size_t WS_XT = 1 << 20;
constexpr size_t XT_BYTES = (size_t)ML * KD * 2, W_BYTES = (size_t)KD * ND * 4;
constexpr size_t WS_PART = WS_XT + XT_BYTES;
constexpr size_t WS_STAGE = WS_PART + NWG * HALF_B;
constexpr size_t WS_END = WS_STAGE + 2 * NEARLY * HALF_B;
static_assert(WS_STAMP + 65536 + NWG * 512 <= WS_XT, "control block overlaps x");
constexpr u64 DEADLINE_TICKS = 50000000ull;

struct Params {
  const float* x;
  const float* w;
  char* myws;
  int rank;
  int slack;
  char* out[NR];
  char* ws[NR];
  int nwide, wmask;
  int start[NE + 1], te[NE];
  int step[KH + 1];
};

#define ESLEEP 32
#define LSLEEP 4
#define AGG0 2
#define WARMUP 14
#define LATE_WIN 2
#define SEND_WIN 3
#define STAMP(i) do {} while (0)
#define STAMPC(i) do {} while (0)
#define STAMP_DECL do {} while (0)
#define STAMP_OUT() do {} while (0)

__device__ __forceinline__ unsigned pk_bf16(float lo, float hi) {
  f32x2 v = {lo, hi};
  return __builtin_bit_cast(unsigned, __builtin_convertvector(v, bf16x2));
}
__device__ __forceinline__ float wave_max(float f) {
  int v = __builtin_bit_cast(int, f);
  v = max(v, __builtin_amdgcn_update_dpp(0, v, 0x111, 0xf, 0xf, true));
  v = max(v, __builtin_amdgcn_update_dpp(0, v, 0x112, 0xf, 0xf, true));
  v = max(v, __builtin_amdgcn_update_dpp(0, v, 0x114, 0xf, 0xf, true));
  v = max(v, __builtin_amdgcn_update_dpp(0, v, 0x118, 0xf, 0xf, true));
  const int r = max(max(__builtin_amdgcn_readlane(v, 15), __builtin_amdgcn_readlane(v, 31)), max(__builtin_amdgcn_readlane(v, 47), __builtin_amdgcn_readlane(v, 63)));
  return __builtin_bit_cast(float, r);
}
__device__ __forceinline__ i32x4 make_rsrc(const void* base, unsigned bytes) {
  const u64 a = (u64)base;
  return i32x4{__builtin_amdgcn_readfirstlane((int)(unsigned)a), __builtin_amdgcn_readfirstlane((int)(unsigned)(a >> 32)), (int)bytes, 0x00020000};
}

__device__ __forceinline__ void w_load(f32x4& d, unsigned voff, i32x4 rsrc, unsigned soff) {
  asm volatile("buffer_load_dwordx4 %0, %1, %2, %3 offen nt" : "=v"(d) : "v"(voff), "s"(rsrc), "s"(soff) : "memory");
}
__device__ __forceinline__ void set_m0(unsigned m0v) { asm volatile("s_mov_b32 m0, %0" :: "s"(m0v) : "memory"); }
template <int IMM>
__device__ __forceinline__ void x_load(unsigned voff, i32x4 rsrc, unsigned soff) {
  asm volatile("buffer_load_dwordx4 %0, %1, %2 offen offset:%3 sc1 lds" :: "v"(voff), "s"(rsrc), "s"(soff), "i"(IMM) : "memory");
}
template <int N>
__device__ __forceinline__ void wait_bar() { asm volatile("s_waitcnt vmcnt(%0) lgkmcnt(0)\n\ts_barrier" :: "n"(N) : "memory"); }
__device__ __forceinline__ void mfma(f32x4& c, u32x4 a, u32x4 b) {
  asm volatile("v_mfma_f32_16x16x32_bf16 %0, %1, %2, %0" : "+a"(c) : "v"(a), "v"(b));
}
#define SCHED() __builtin_amdgcn_sched_barrier(0)
__device__ __forceinline__ void send_piece(const f32x4 (&y)[8][4], int j, i32x4 srs, unsigned lane16) {
  const f32x4 lo = y[j >> 1][(j & 1) * 2], hi = y[j >> 1][(j & 1) * 2 + 1];
  const u32x4 d = {pk_bf16(lo[0], lo[1]), pk_bf16(lo[2], lo[3]), pk_bf16(hi[0], hi[1]), pk_bf16(hi[2], hi[3])};
  asm volatile("buffer_store_dwordx4 %0, %1, %2, %3 offen sc0 sc1\n\ts_nop 1" :: "v"(d), "v"(lane16), "s"(srs), "s"(j * PC) : "memory");
}
__device__ __forceinline__ void touch(const GAS char* t) {
  unsigned v;
  asm volatile("global_load_dword %0, %1, off nt\n\ts_waitcnt vmcnt(0)" : "=&v"(v) : "v"(t) : "memory");
}
constexpr float QBIAS = 12582912.0f;
struct Quant { float inv, lo, hi; };
__device__ __forceinline__ unsigned quant4(float v0, float v1, float v2, float v3, const Quant& qz) {
  const unsigned e0 = __builtin_bit_cast(unsigned, __builtin_amdgcn_fmed3f(__builtin_fmaf(v0, qz.inv, QBIAS), qz.lo, qz.hi));
  const unsigned e1 = __builtin_bit_cast(unsigned, __builtin_amdgcn_fmed3f(__builtin_fmaf(v1, qz.inv, QBIAS), qz.lo, qz.hi));
  const unsigned e2 = __builtin_bit_cast(unsigned, __builtin_amdgcn_fmed3f(__builtin_fmaf(v2, qz.inv, QBIAS), qz.lo, qz.hi));
  const unsigned e3 = __builtin_bit_cast(unsigned, __builtin_amdgcn_fmed3f(__builtin_fmaf(v3, qz.inv, QBIAS), qz.lo, qz.hi));
  return __builtin_amdgcn_perm(e1, e0, 0x0c0c0400u) | __builtin_amdgcn_perm(e3, e2, 0x04000c0cu);
}

template <int SCOPE, int SLEEP, int N = 1, int STRIDE = 0>
__device__ __forceinline__ bool wait_flag(const unsigned* flag, unsigned call, unsigned* tmo) {
  const u64 t0 = __builtin_amdgcn_s_memrealtime();
  for (unsigned polls = 1;; ++polls) {
    unsigned v[N];
    bool pend = false;
#pragma unroll
    for (int i = 0; i < N; ++i) v[i] = __hip_atomic_load(flag + i * STRIDE, __ATOMIC_RELAXED, SCOPE);
#pragma unroll
    for (int i = 0; i < N; ++i) pend |= v[i] < call;
    if (!__any(pend)) return true;
    if (((polls & 63u) == 0 && __hip_atomic_load(tmo, __ATOMIC_RELAXED, __HIP_MEMORY_SCOPE_AGENT) == call) ||
        __builtin_amdgcn_s_memrealtime() - t0 > DEADLINE_TICKS) {
      __hip_atomic_store(tmo, call, __ATOMIC_RELAXED, __HIP_MEMORY_SCOPE_AGENT);
      return false;
    }
    __builtin_amdgcn_s_sleep(SLEEP);
  }
}

struct Ctx {
  i32x4 wrsrc, xrsrc;
  unsigned wvoff[8], xvoff0, xvoff1, m0base;
  int wlim;
  int left;
  unsigned xso, wso;
  unsigned due, slack;
  int steps;
  i32x4 frsrc;
  const char* fbase;
  unsigned fvoff, fso, call, fl;
  unsigned* tmo;
  bool ok;
  LAS const char* lrd0;
  LAS const char* lrd1;
  LAS char* wmine;
  LAS const char* wother;
};

template <int SLOT>
__device__ __forceinline__ u32x4 x_frag(const Ctx& c, int mt) {
  return *(LAS const u32x4*)((SLOT < 2 ? c.lrd0 : c.lrd1) + (SLOT & 1) * 32768 + mt * 1024);
}
template <int G>
__device__ __forceinline__ void x_piece(const Ctx& c, i32x4 xr, unsigned xso) {
  x_load<(G & 3) * 1024>((G & 4) ? c.xvoff1 : c.xvoff0, xr, (G & 8) ? xso + 32768 : xso);
}
template <int SL, bool CHK, bool HELD = false>
__device__ __forceinline__ void kstep(Ctx& c, f32x4 (&acc)[16][4], f32x4 (&wr)[2][8], u32x4 (&a)[2][4], u32x4 (&xf)[8], const u32x4* held = nullptr) {
  constexpr int E = SL & 1, WS = SL >> 1, NX = (SL + 1) & 3;
  u64 now;
  unsigned lateby;
  const i32x4 xr = c.xrsrc, wrs = c.wrsrc;
  const unsigned xso = c.xso, wso = c.wso;
  const unsigned m0x = c.m0base + ((SL + 2) & 3) * 32768;
  f32x4 (&r)[8] = wr[WS ^ 1];
#pragma unroll
  for (int g = 0; g < 16; ++g) {
    if (E == 1 && g == 12) {
      if (CHK) {
        asm volatile("s_waitcnt vmcnt(8)" : "+v"(c.fl) :: "memory");
        if (__any(c.fl < c.call)) {
          c.ok &= wait_flag<__HIP_MEMORY_SCOPE_AGENT, 1>((const unsigned*)(c.fbase + c.fso + c.fvoff), c.call, c.tmo);
          __builtin_amdgcn_s_waitcnt(0xC07F);
        }
      }
      wait_bar<HELD ? 0 : 8>();
      SCHED();
    }
    mfma(acc[g][0], a[E][0], xf[g & 7]);
    SCHED();
    if (g < 9) xf[(g + 7) & 7] = x_frag<SL>(c, g + 7);
    else if (E == 0) xf[(g + 7) & 7] = x_frag<NX>(c, g - 9);
    else if (g >= 12) xf[(2 * (g - 12)) & 7] = x_frag<NX>(c, 2 * (g - 12));
    if (CHK && E == 0 && g == 0) asm volatile("buffer_load_dword %0, %1, %2, %3 offen sc1" : "=v"(c.fl) : "v"(c.fvoff), "s"(c.frsrc), "s"(c.fso) : "memory");
    if (E == 1 && g == 9) *(LAS u32x4*)(c.wmine + (WS ^ 1) * 16384 + 1024) = a[0][1];
    if (E == 1 && g == 10) *(LAS u32x4*)(c.wmine + (WS ^ 1) * 16384 + 3072) = a[0][3];
    if (E == 1 && g == 11) {
      asm volatile("s_waitcnt lgkmcnt(0)" : "+s"(now) :: "memory");
      lateby = (unsigned)now - c.due;
      asm volatile("" : "+s"(lateby));
    }
    SCHED();
    mfma(acc[g][1], a[E][1], xf[g & 7]);
    SCHED();
    if (E == 0) {
      switch (g) {
        case 0: x_piece<0>(c, xr, xso); break;   case 1: x_piece<1>(c, xr, xso); break;
        case 2: x_piece<2>(c, xr, xso); break;   case 3: x_piece<3>(c, xr, xso); break;
        case 4: x_piece<4>(c, xr, xso); break;   case 5: x_piece<5>(c, xr, xso); break;
        case 6: x_piece<6>(c, xr, xso); break;   case 7: x_piece<7>(c, xr, xso); break;
        case 8: x_piece<8>(c, xr, xso); break;   case 9: x_piece<9>(c, xr, xso); break;
        case 10: x_piece<10>(c, xr, xso); break; case 11: x_piece<11>(c, xr, xso); break;
        case 12: x_piece<12>(c, xr, xso); break; case 13: x_piece<13>(c, xr, xso); break;
        case 14: x_piece<14>(c, xr, xso); break; default: x_piece<15>(c, xr, xso); break;
      }
    } else {
      if (!HELD && g < 8) w_load(wr[WS][g & 7], c.wvoff[g & 7], wrs, wso);
      if (HELD && g >= 4 && g < 8) { a[0][g & 3] = held[g & 3]; asm volatile("" : "+v"(a[0][g & 3])); }
      if (!HELD && g == 5)
        asm volatile("s_waitcnt vmcnt(%8)" : "+v"(r[0]), "+v"(r[1]), "+v"(r[2]), "+v"(r[3]), "+v"(r[4]), "+v"(r[5]), "+v"(r[6]), "+v"(r[7]) : "n"(22 + CHK) : "memory");
      if (g == 8) *(LAS u32x4*)(c.wmine + (WS ^ 1) * 16384) = a[0][0];
      if (g == 9) { c.left -= 1; c.xso += 65536u; asm volatile("" : "+s"(c.left), "+s"(c.xso)); }
      if (g == 10) { c.xrsrc[2] = c.left > 1 ? (int)XT_BYTES : 0; asm volatile("" : "+s"(c.xrsrc)); }
      if (g == 11) { c.wrsrc[2] = c.left > c.wlim ? (int)W_BYTES : 0; asm volatile("" : "+s"(c.wrsrc)); }
      if (g == 12) { lateby = (int)lateby > (int)c.slack ? lateby - c.slack : 0u; asm volatile("" : "+s"(lateby)); }
      if (g == 13) { c.due += lateby; lateby = __builtin_amdgcn_readlane(c.steps, c.left); asm volatile("" : "+s"(c.due), "+s"(lateby)); }
      if (g == 14) { c.due += lateby; asm volatile("" : "+s"(c.due)); }
    }
    SCHED();
    mfma(acc[g][2], a[E][2], xf[g & 7]);
    SCHED();
    if (E == 0 && g < 4) a[1][g] = *(LAS const u32x4*)(c.wother + WS * 16384 + g * 1024);
    if (!HELD && E == 1 && g >= 5 && g < 9) { const int t = (g - 5) & 3; a[0][t][0] = pk_bf16(r[0][t], r[1][t]); a[0][t][1] = pk_bf16(r[2][t], r[3][t]); }
    if (E == 1 && g == 9) *(LAS u32x4*)(c.wmine + (WS ^ 1) * 16384 + 2048) = a[0][2];
    if (E == 1 && g == 11) {
      if (__builtin_expect((int)lateby < 0, 0))
        do __builtin_amdgcn_s_sleep(1); while ((int)((unsigned)__builtin_amdgcn_s_memrealtime() - c.due) < 0);
    }
    if (E == 1 && g >= 12 && g < 15) xf[(2 * (g - 12) + 1) & 7] = x_frag<NX>(c, 2 * (g - 12) + 1);
    SCHED();
    mfma(acc[g][3], a[E][3], xf[g & 7]);
    SCHED();
    if (!HELD && E == 1 && g >= 5 && g < 9) { const int t = (g - 5) & 3; a[0][t][2] = pk_bf16(r[4][t], r[5][t]); a[0][t][3] = pk_bf16(r[6][t], r[7][t]); }
    if (E == 0 && (g & 3) == 3 && g < 15) set_m0(m0x + ((g + 1) >> 3) * 32768 + (((g + 1) >> 2) & 1) * 4096);
    if (E == 1 && g == 9) asm volatile("s_memrealtime %0" : "=s"(now) :: "memory");
    if (E == 1 && g == 10) { c.wso += 1u << 22; asm volatile("" : "+s"(c.wso)); }
    if (E == 1 && g == 15) set_m0(c.m0base + ((SL + 3) & 3) * 32768);
    SCHED();
  }
}

__device__ __forceinline__ void last_block(const Ctx& c, f32x4 (&acc)[16][4], u32x4 (&a)[2][4], u32x4 (&xf)[8], i32x4 prs, unsigned pov) {
  f32x4 lo, hi;
#pragma unroll
  for (int i = 0; i < 32; ++i) {
    const int e = (i >> 3) & 1, m = (i & 7) + (i >> 4) * 8;
    const int i7 = i + 7, e7 = (i7 >> 3) & 1, m7 = (i7 & 7) + (i7 >> 4) * 8;
    const int j = i - 16;
    mfma(acc[m][0], a[e][0], xf[i & 7]);
    SCHED();
    if (i7 < 32) xf[i7 & 7] = e7 ? x_frag<3>(c, m7) : x_frag<2>(c, m7);
    if (j >= 0) {
      asm volatile("" : "+a"(acc[j >> 1][(j & 1) * 2]));
      lo = acc[j >> 1][(j & 1) * 2];
      asm volatile("" : "+v"(lo));
    }
    SCHED();
    mfma(acc[m][1], a[e][1], xf[i & 7]);
    SCHED();
    if (i < 4) a[1][i] = *(LAS const u32x4*)(c.wother + 16384 + i * 1024);
    if (j >= 0) {
      asm volatile("" : "+a"(acc[j >> 1][(j & 1) * 2 + 1]));
      hi = acc[j >> 1][(j & 1) * 2 + 1];
      asm volatile("" : "+v"(hi));
    }
    SCHED();
    mfma(acc[m][2], a[e][2], xf[i & 7]);
    SCHED();
    u32x4 d;
    if (j >= 0) { d = u32x4{pk_bf16(lo[0], lo[1]), pk_bf16(lo[2], lo[3]), pk_bf16(hi[0], hi[1]), pk_bf16(hi[2], hi[3])}; asm volatile("" : "+v"(d)); }
    SCHED();
    mfma(acc[m][3], a[e][3], xf[i & 7]);
    SCHED();
    if (j >= 0)
      asm volatile("buffer_store_dwordx4 %0, %1, %2, %3 offen offset:%4 sc1\n\ts_nop 1"
                   :: "v"(d), "v"(pov), "s"(prs), "s"(j * PC & ~4095), "i"(j * PC & 4095) : "memory");
    SCHED();
  }
}

template <int N, int STRIDE, int SCOPE, int SLEEP>
__device__ __forceinline__ float sweep_max(const u64* slots, unsigned call, unsigned* tmo) {
  const u64 t0 = __builtin_amdgcn_s_memrealtime();
  for (unsigned polls = 1;; ++polls) {
    bool all = true, bad = false;
    float mx = 0.f;
    u64 v[N];
#pragma unroll
    for (int r = 0; r < N; ++r) v[r] = __hip_atomic_load(slots + r * STRIDE, __ATOMIC_RELAXED, SCOPE);
#pragma unroll
    for (int r = 0; r < N; ++r) {
      const float f = __builtin_bit_cast(float, (unsigned)v[r]);
      all &= (unsigned)(v[r] >> 32) == call;
      bad |= f < 0.f;
      mx = fmaxf(mx, f);
    }
    if (__all(all)) {
      return __any(bad) ? -1.0f : wave_max(mx);
    }
    if (((polls & 63u) == 0 && __hip_atomic_load(tmo, __ATOMIC_RELAXED, __HIP_MEMORY_SCOPE_AGENT) == call) ||
        __builtin_amdgcn_s_memrealtime() - t0 > DEADLINE_TICKS) {
      __hip_atomic_store(tmo, call, __ATOMIC_RELAXED, __HIP_MEMORY_SCOPE_AGENT);
      return -1.0f;
    }
    __builtin_amdgcn_s_sleep(SLEEP);
  }
}
template <int SLEEP>
__device__ __forceinline__ Quant global_scale(const u64* gmax, unsigned call, unsigned* tmo, bool ok, int lane) {
  const float gam = ok ? sweep_max<1, 0, __HIP_MEMORY_SCOPE_SYSTEM, SLEEP>(gmax + (lane & 7), call, tmo) : -1.0f;
  Quant qz;
  qz.inv = gam > 0.f ? 127.0f / gam : 0.f;
  qz.lo = gam >= 0.f ? QBIAS - 127.0f : QBIAS - 128.0f;
  qz.hi = gam >= 0.f ? QBIAS + 127.0f : QBIAS - 128.0f;
  asm volatile("" : "+v"(qz.lo), "+v"(qz.hi));
  return qz;
}

__device__ __forceinline__ int line_of(int ct, int s) {
  return ct < 7 ? 8 * ct + 4 + s : ct < 11 ? 16 * (ct - 7) + (s >> 1) * 8 + (s & 1) : ct == 11 ? 60 + s : (ct & 1) * 32 + 8 * s + (ct >> 1) - 4;
}
__device__ __forceinline__ void tile_of(int j, int& pd, int& ct) {
  const int k = j / 7;
  pd = j < 112 ? j % 7 + 1 : 0;
  ct = j >= 112 ? j - 112 : k < 2 ? 12 + k : k < NE - 2 ? 14 - NE + k : k < NE ? 16 - NE + k : k - NE;
}

__global__ void __launch_bounds__(NTHR, 1) fused_kernel(Params p) {
  extern __shared__ __attribute__((aligned(1024))) char smem[];
  const int tid = threadIdx.x, lane = tid & 63;
  const int wave = __builtin_amdgcn_readfirstlane(tid >> 6);
  char* ws = p.myws;

  STAMP_DECL;
  STAMP(0);
  STAMPC(8);
  const int b = blockIdx.x, xcd = b & 7, idx = b >> 3;
  const int kh = xcd & 1, slot = idx * 4 + (xcd >> 1), unit = slot * 2 + kh;
  const int wn = wave & 1, wm = wave >> 1;
  const int q = lane >> 4, i16 = lane & 15;
  const unsigned lane16 = lane * 16;
  const unsigned t_start = (unsigned)__builtin_amdgcn_s_memrealtime();

  Ctx c;
  c.xrsrc = make_rsrc(ws + WS_XT, (unsigned)XT_BYTES);
  unsigned* xflag = (unsigned*)(ws + WS_XFLAG);
  const int hs = kh * NT + slot;
  const unsigned xto = (unsigned)(hs * 16384 + wave * 4096);
  f32x4 raw[8];
  unsigned prev, warm;
  {
    const int st = hs >> 1, h = hs & 1;
    const i32x4 xin = make_rsrc(p.x, (unsigned)((size_t)ML * KD * 4));
#pragma unroll
    for (int j = 0; j < 4; ++j) {
      const unsigned o = (unsigned)((h * 16 + ((wave * 4 + j) ^ ((kh ^ 1) << 3))) * 16 + i16) * (KD * 4) + ((st ^ h) * 32 + q * 8) * 4;
      asm volatile("buffer_load_dwordx4 %0, %2, %3, 0 offen\n\tbuffer_load_dwordx4 %1, %2, %3, 0 offen offset:16"
                   : "=&v"(raw[2 * j]), "=&v"(raw[2 * j + 1]) : "v"(o), "s"(xin) : "memory");
    }
    asm volatile("global_load_dword %0, %1, off sc1" : "=v"(prev) : "v"(xflag + hs * (FS / 4)) : "memory");
    asm volatile("buffer_load_dword %0, %1, %2, %3 offen sc1" : "=v"(warm) : "v"(0), "s"(c.xrsrc), "s"(xto) : "memory");
  }

  unsigned cpf = 0;
  {
    u64 pc; unsigned len1, len2;
    asm volatile("s_getpc_b64 %0\n.Lq8_code_begin:\n\ts_mov_b32 %1, .Lq8_code_end1-.Lq8_code_begin\n\ts_mov_b32 %2, .Lq8_code_end2-.Lq8_code_begin" : "=s"(pc), "=s"(len1), "=s"(len2));
    const int wv = (idx - 8) * 4 + wave;
    const i32x4 cr = (unsigned)wv < 8u ? make_rsrc((const void*)pc, (WARMUP & 4) ? (len1 > len2 ? len1 : len2) : 0)
                                       : make_rsrc(p.w, ((WARMUP & 8) && (unsigned)(wv - 8) < 2u) ? (unsigned)W_BYTES : 0);
    asm volatile("buffer_load_dword %0, %1, %2, 0 offen" : "+v"(cpf)
                 : "v"((unsigned)wv < 8u ? (unsigned)(wv * 64 + lane) * 128u : (unsigned)(kh * 128 + (wv & 1) * 64 + lane) << 20), "s"(cr) : "memory");
  }
  const bool early = slot < NEARLY;
  int pd, ct;
  tile_of(slot, pd, ct);
  const int rank = p.rank;
  const unsigned dest = (unsigned)(rank + pd) & 7u;
  const int k0 = kh * KH;
  const int ln0 = line_of(ct, 2 * wn), ln1 = line_of(ct, 2 * wn + 1);
  unsigned* tmo = (unsigned*)(ws + WS_TMO);
  GAS char* dws = (GAS char*)p.ws[dest];
  GAS char* dout = (GAS char*)p.out[early ? rank : dest];
  GAS char* myout = (GAS char*)p.out[rank];
  asm volatile("" : "+s"(dws), "+s"(dout), "+s"(myout));
  GAS char* anyws;
  i32x2 stp;
  asm volatile("global_load_dwordx2 %0, %2, %4 offset:%5\n\tglobal_load_dwordx2 %1, %3, %4 offset:%6"
               : "=&v"(anyws), "=&v"(stp) : "v"((lane & 7) * 8), "v"(((KH - 1 - lane) & 31) * 4), "s"(__builtin_amdgcn_kernarg_segment_ptr()),
                 "i"(offsetof(Params, ws)), "i"(offsetof(Params, step)) : "memory");

  c.wrsrc = make_rsrc(p.w, (unsigned)W_BYTES);
#pragma unroll
  for (int j = 0; j < 8; ++j) {
    c.wvoff[j] = ((unsigned)(wm * 32 + q * 8 + j) << 16) + (i16 < 8 ? ln0 : ln1) * 128 + (i16 & 7) * 16;
    asm volatile("" : "+v"(c.wvoff[j]));
  }
  c.xvoff0 = wave * 8192 + lane16;
  c.xvoff1 = c.xvoff0 + 4096;
  asm volatile("" : "+v"(c.xvoff1));
  c.m0base = (unsigned)(size_t)(LAS char*)smem + wave * 8192;
  constexpr int nblk = KH;
  const unsigned wsb = ((unsigned)k0 << 22) + dest * (NL * 4u);
  c.left = nblk;
  c.wlim = 2;
  c.xso = (unsigned)(k0 + 1) * 65536u;
  c.wso = wsb + (2u << 22);
  const bool steady = early || ct == 11 || ct >= 14;
  const int cls = early ? slot / 7 : steady ? NE - 1 : NE, te = p.te[steady ? cls : 0], phase = cls < p.nwide ? 0 : (slot * 74 + kh * 8) & 255;
  c.due = t_start + p.start[cls] + phase * (steady ? te : p.step[0]) / 256;
  c.slack = p.slack;
  int wide = (p.wmask >> cls) & 1;
  asm volatile("" : "+s"(wide));
  c.lrd0 = (LAS const char*)smem + wm * 16384 + lane16;
  c.lrd1 = c.lrd0 + 65536;
  asm volatile("" : "+v"(c.lrd1));
  c.wmine = (LAS char*)smem + LDS_X + wave * 4096 + lane16;
  c.wother = (LAS const char*)smem + LDS_X + (wave ^ 2) * 4096 + lane16;

  f32x4 acc[16][4];
  f32x4 wr[2][8];
  u32x4 a[2][4], xf[8];
#pragma unroll
  for (int m = 0; m < 16; ++m)
#pragma unroll
    for (int t = 0; t < 4; ++t) {
      acc[m][t] = f32x4{0.f, 0.f, 0.f, 0.f};
      asm volatile("" : "+a"(acc[m][t]));
    }
  asm volatile("s_waitcnt vmcnt(3)" : "+v"(prev), "+v"(warm), "+v"(raw[0]), "+v"(raw[1]), "+v"(raw[2]), "+v"(raw[3]), "+v"(raw[4]), "+v"(raw[5]), "+v"(raw[6]), "+v"(raw[7]) :: "memory");
  STAMP(11);
#pragma unroll
  for (int j = 0; j < 4; ++j) {
    const f32x4 v0 = raw[2 * j], v1 = raw[2 * j + 1];
    const u32x4 d = {pk_bf16(v0[0], v0[1]), pk_bf16(v0[2], v0[3]), pk_bf16(v1[0], v1[1]), pk_bf16(v1[2], v1[3])};
    asm volatile("buffer_store_dwordx4 %0, %1, %2, %3 offen offset:%4 sc1\n\ts_nop 1" :: "v"(d), "v"(lane16), "s"(c.xrsrc), "s"(xto), "i"(j * 1024) : "memory");
  }
  const unsigned call = __builtin_amdgcn_readfirstlane(prev) + 1;
  asm volatile("s_waitcnt vmcnt(0)\n\ts_barrier" : "+v"(anyws), "+v"(stp), "+v"(cpf) :: "memory");
  c.steps = steady ? te : stp[1] + phase * (stp[1] - stp[0]) / 256;
  STAMP(12);
  if (tid == 0) __hip_atomic_store(xflag + hs * (FS / 4), call, __ATOMIC_RELAXED, __HIP_MEMORY_SCOPE_AGENT);
  if (b == 0 && wave == 1 && lane < NR)
    __hip_atomic_store((GAS unsigned*)(anyws + WS_START + rank * FS), call, __ATOMIC_RELAXED, __HIP_MEMORY_SCOPE_SYSTEM);
  c.fbase = (const char*)xflag + kh * NT * FS;
  c.frsrc = make_rsrc(c.fbase, NT * FS);
  c.fvoff = (lane & 3) * FS;
  c.fso = 8 * FS;
  c.call = call;
  c.tmo = tmo;
  c.ok = true;
  if ((unsigned)(idx - PF0) < (1u << PFL)) {
    constexpr int NP = 16 >> PFL;
    const int pw = (idx - PF0) * 4 + wave;
    const unsigned pv = (unsigned)k0 * 65536u + pw * NP * 1024 + lane16;
    u32x4 pf = {0u, 0u, 0u, 0u};
#pragma unroll
    for (int j = 1; j < KH; ++j) {
      if (j == 1 || j == 5)
        c.ok &= wait_flag<__HIP_MEMORY_SCOPE_AGENT, 1>((const unsigned*)(c.fbase + ((j == 1 ? (lane & 3) + 1 : lane & 31) * 4 + (pw >> PFL)) * FS), call, tmo);
      const unsigned v = pv + j * 65536;
      asm volatile("s_waitcnt vmcnt(32)\n\tbuffer_load_dwordx4 %0, %1, %2, 0 offen sc1\n\tbuffer_load_dwordx4 %0, %1, %2, 0 offen offset:1024 sc1" : "+v"(pf) : "v"(v), "s"(c.xrsrc) : "memory");
      if (NP == 4) asm volatile("buffer_load_dwordx4 %0, %1, %2, 0 offen offset:2048 sc1\n\tbuffer_load_dwordx4 %0, %1, %2, 0 offen offset:3072 sc1" : "+v"(pf) : "v"(v), "s"(c.xrsrc) : "memory");
    }
    asm volatile("s_waitcnt vmcnt(0)" : "+v"(pf) :: "memory");
    if (WARMUP & 2) {
      const int u1 = unit + ((lane & 3) == 1 ? -56 : (lane & 3) == 2 ? 56 : 0), u2 = (unsigned)u1 < 2u * NEARLY ? u1 : unit;
      const size_t o = (size_t)u2 * HALF_B + wave * WAVE_B;
      const int k = (lane >> 2) & 7;
      touch(k == 0 ? dws + WS_STAGE + o : k == 1 ? (GAS char*)ws + WS_PART + o : k == 2 ? (GAS char*)ws + WS_PART + (o ^ HALF_B)
            : k == 3 ? (GAS char*)ws + WS_STAGE + o : k == 4 ? dws + WS_EFLAG + (size_t)u2 * 4 * FS + 128
            : myout + (size_t)(((rank - pd) & 7) * ML + wm * 256 + kh * 128 + (lane & 3) * 32) * NL);
    }
  }
#pragma unroll
  for (int j = 0; j < 8; ++j) w_load(wr[0][j], c.wvoff[j], c.wrsrc, wsb);
  if (wave == 0) c.ok &= wait_flag<__HIP_MEMORY_SCOPE_AGENT, 1>((const unsigned*)(c.fbase + (lane & 7) * FS), call, tmo);
  asm volatile("s_waitcnt vmcnt(0)" : "+v"(wr[0][0]), "+v"(wr[0][1]), "+v"(wr[0][2]), "+v"(wr[0][3]), "+v"(wr[0][4]), "+v"(wr[0][5]), "+v"(wr[0][6]), "+v"(wr[0][7]) :: "memory");
#pragma unroll
  for (int t = 0; t < 4; ++t) {
    a[0][t] = u32x4{pk_bf16(wr[0][0][t], wr[0][1][t]), pk_bf16(wr[0][2][t], wr[0][3][t]), pk_bf16(wr[0][4][t], wr[0][5][t]), pk_bf16(wr[0][6][t], wr[0][7][t])};
    *(LAS u32x4*)(c.wmine + t * 1024) = a[0][t];
  }
  wait_bar<0>();
  {
    const unsigned xso = (unsigned)k0 * 65536u;
    set_m0(c.m0base); asm volatile("s_nop 0");
    x_piece<0>(c, c.xrsrc, xso); x_piece<1>(c, c.xrsrc, xso); x_piece<2>(c, c.xrsrc, xso); x_piece<3>(c, c.xrsrc, xso);
    set_m0(c.m0base + 4096); asm volatile("s_nop 0");
    x_piece<4>(c, c.xrsrc, xso); x_piece<5>(c, c.xrsrc, xso); x_piece<6>(c, c.xrsrc, xso); x_piece<7>(c, c.xrsrc, xso);
    set_m0(c.m0base + 32768); asm volatile("s_nop 0");
    x_piece<8>(c, c.xrsrc, xso); x_piece<9>(c, c.xrsrc, xso); x_piece<10>(c, c.xrsrc, xso); x_piece<11>(c, c.xrsrc, xso);
    set_m0(c.m0base + 32768 + 4096); asm volatile("s_nop 0");
    x_piece<12>(c, c.xrsrc, xso); x_piece<13>(c, c.xrsrc, xso); x_piece<14>(c, c.xrsrc, xso); x_piece<15>(c, c.xrsrc, xso);
  }
#pragma unroll
  for (int j = 0; j < 8; ++j) w_load(wr[1][j], c.wvoff[j], c.wrsrc, wsb + (1u << 22));
  __builtin_amdgcn_s_waitcnt(0xC07F);
  wait_bar<8>();
  STAMP(10);
  SCHED();
  set_m0(c.m0base + 2 * 32768);
#pragma unroll
  for (int t = 0; t < 7; ++t) xf[t] = x_frag<0>(c, t);
  SCHED();
  while ((int)((unsigned)__builtin_amdgcn_s_memrealtime() + 100u - c.due) < 0) __builtin_amdgcn_s_sleep(4);
  __builtin_amdgcn_s_waitcnt(0xC07F);
  STAMP(1);

#define TRACE_PT()
  TRACE_PT()
  kstep<0, true>(c, acc, wr, a, xf);
  kstep<1, true>(c, acc, wr, a, xf);
  c.fvoff = (lane & 31) * FS;
  c.fso = wave * 32 * FS;
  kstep<2, true>(c, acc, wr, a, xf);
  kstep<3, true>(c, acc, wr, a, xf);
  kstep<0, false>(c, acc, wr, a, xf);
  kstep<1, false>(c, acc, wr, a, xf);
  u32x4 held[NH][4] = {};
  const int ltail = steady ? 1 : NH + 1;
  if (!steady) {
    c.wlim = 2 + NH;
    f32x4 raw2[2][8];
#pragma unroll
    for (int i = 0; i <= NH; ++i) {
      if (i < NH) {
#pragma unroll
        for (int j = 0; j < 8; ++j) w_load(raw2[i & 1][j], c.wvoff[j], c.wrsrc, wsb + ((unsigned)(nblk - NH + i) << 22));
      }
      if (i > 0) {
        f32x4 (&r)[8] = raw2[(i - 1) & 1];
        asm volatile("s_waitcnt vmcnt(%8)" : "+v"(r[0]), "+v"(r[1]), "+v"(r[2]), "+v"(r[3]), "+v"(r[4]), "+v"(r[5]), "+v"(r[6]), "+v"(r[7]) : "n"(i < NH ? 8 : 0) : "memory");
#pragma unroll
        for (int t = 0; t < 4; ++t) {
          held[i - 1][t] = u32x4{pk_bf16(r[0][t], r[1][t]), pk_bf16(r[2][t], r[3][t]), pk_bf16(r[4][t], r[5][t]), pk_bf16(r[6][t], r[7][t])};
          asm volatile("" : "+v"(held[i - 1][t]));
        }
      }
    }
    SCHED();
  }
  do {
    TRACE_PT()
    kstep<2, false>(c, acc, wr, a, xf);
    kstep<3, false>(c, acc, wr, a, xf);
    kstep<0, false>(c, acc, wr, a, xf);
    kstep<1, false>(c, acc, wr, a, xf);
  } while (c.left > ltail);
  while (c.left > 1) {
    TRACE_PT()
    kstep<2, false, true>(c, acc, wr, a, xf, held[0]);
    kstep<3, false, true>(c, acc, wr, a, xf, held[0]);
    kstep<0, false, true>(c, acc, wr, a, xf, held[1]);
    kstep<1, false, true>(c, acc, wr, a, xf, held[1]);
#pragma unroll
    for (int i = 0; i + 2 < NH; ++i)
#pragma unroll
      for (int t = 0; t < 4; ++t) held[i][t] = held[i + 2][t];
    asm volatile("s_nop 1");
  }
  LAS float* red = (LAS float*)(smem + LDS_X);
  const size_t gmax_off = WS_GMAX + (size_t)(call & 1) * FS;
  const size_t wave_off = (size_t)unit * HALF_B + wave * WAVE_B;
  const unsigned pov = (PC > 1024 && (lane >> 3) == 3) ? lane16 + 5 * 128 : lane16;
  unsigned* pflag = (unsigned*)(ws + WS_PFLAG + wave * FS);
  last_block(c, acc, a, xf, make_rsrc(ws + WS_PART + wave_off, WAVE_B), pov);
  bool ok = c.ok;
  STAMP(2);
  STAMPC(9);
  asm volatile("s_waitcnt vmcnt(0)"
               : "+v"(wr[0][0]), "+v"(wr[0][1]), "+v"(wr[0][2]), "+v"(wr[0][3]), "+v"(wr[0][4]), "+v"(wr[0][5]), "+v"(wr[0][6]), "+v"(wr[0][7])
               :: "memory");
  if (lane == 0) __hip_atomic_store(pflag + unit * FS, call, __ATOMIC_RELAXED, __HIP_MEMORY_SCOPE_AGENT);
  ok &= wait_flag<__HIP_MEMORY_SCOPE_SYSTEM, 2>((lane & 1) ? (const unsigned*)(ws + WS_START + dest * FS) : pflag + (unit ^ 1) * FS, call, tmo);
  const i32x4 srs = make_rsrc((const void*)(dws + WS_STAGE + wave_off), ok ? WAVE_B : 0);
  f32x4 y[8][4];
  {
    const i32x4 prs = make_rsrc(ws + WS_PART + (wave_off ^ HALF_B), WAVE_B);
    u32x4 pv[16];
#pragma unroll
    for (int j = 0; j < 16; ++j)
      asm volatile("buffer_load_dwordx4 %0, %1, %2, %3 offen offset:%4 sc1" : "=v"(pv[j]) : "v"(pov), "s"(prs), "s"(j * PC & ~4095), "i"(j * PC & 4095) : "memory");
    const unsigned one = (i16 >> 2) == q ? ((i16 & 1) ? 0x3F800000u : 0x00003F80u) : 0u;
    const unsigned e0 = (i16 & 2) ? 0u : one, e1 = (i16 & 2) ? one : 0u;
    u32x4 sel0 = {e0, e1, 0u, 0u}, sel1 = {0u, 0u, e0, e1};
    asm volatile("s_nop 1" : "+v"(sel0), "+v"(sel1));
    float am = 0.f;
#pragma unroll
    for (int j = 0; j < 16 + 4; ++j) {
      if (j < 16) {
        asm volatile("s_waitcnt vmcnt(%1)" : "+v"(pv[j]) : "n"(15 - j) : "memory");
        mfma(acc[8 + (j >> 1)][(j & 1) * 2], sel0, pv[j]);
        mfma(acc[8 + (j >> 1)][(j & 1) * 2 + 1], sel1, pv[j]);
      }
      if (j == 16) asm volatile("s_nop 7\n\ts_nop 7" ::: "memory");
      if (j >= 4) {
        const int m = (j - 4) >> 1, u = (j - 4) & 1;
        asm volatile("" : "+a"(acc[8 + m][2 * u]), "+a"(acc[8 + m][2 * u + 1]), "+v"(am));
        y[m][2 * u] = acc[8 + m][2 * u]; y[m][2 * u + 1] = acc[8 + m][2 * u + 1];
        asm volatile("" : "+v"(y[m][2 * u]), "+v"(y[m][2 * u + 1]));
        const f32x4 lo = y[m][2 * u], hi = y[m][2 * u + 1];
        am = fmaxf(fmaxf(am, fabsf(lo[0])), fabsf(lo[1])); am = fmaxf(fmaxf(am, fabsf(lo[2])), fabsf(lo[3]));
        am = fmaxf(fmaxf(am, fabsf(hi[0])), fabsf(hi[1])); am = fmaxf(fmaxf(am, fabsf(hi[2])), fabsf(hi[3]));
        asm volatile("" : "+v"(am));
      }
    }
    if (early) {
#pragma unroll
      for (int j = 0; j < SEND_WIN; ++j) send_piece(y, j, srs, pov);
    }
    am = wave_max(am);
    if (lane == 0) red[wave] = ok ? am : -1.0f;
    __syncthreads();
    if (tid == 0) {
      const float r0 = red[0], r1 = red[1], r2 = red[2], r3 = red[3];
      const float wam = fminf(fminf(r0, r1), fminf(r2, r3)) < 0.f ? -1.0f : fmaxf(fmaxf(r0, r1), fmaxf(r2, r3));
      __hip_atomic_store((u64*)(ws + WS_AMAX + (unit >> 4) * FS) + (unit & 15), ((u64)call << 32) | __builtin_bit_cast(unsigned, wam), __ATOMIC_RELAXED, __HIP_MEMORY_SCOPE_AGENT);
    }
  }
  STAMP(3);
  const int row0 = wm * 256 + kh * 128 + i16, col0 = (q < 2 ? ln0 : ln1) * 32 + (q & 1) * 16;
  const int srank = (rank - pd) & 7;

  if (early) {
#pragma unroll
    for (int j = SEND_WIN; j < 16; ++j) {
      if (wide > 0) asm volatile("s_waitcnt vmcnt(%0)" :: "n"(2 * SEND_WIN - 1) : "memory");
      else asm volatile("s_waitcnt vmcnt(%0)" :: "n"(SEND_WIN - 1) : "memory");
      send_piece(y, j, srs, pov);
    }
    asm volatile("s_waitcnt vmcnt(0)" ::: "memory");
    if (lane == 0)
      __hip_atomic_store((GAS unsigned*)(dws + WS_EFLAG + (size_t)(unit * 4 + wave) * FS), call, __ATOMIC_RELAXED, __HIP_MEMORY_SCOPE_SYSTEM);
    STAMP(5);
    ok &= wait_flag<__HIP_MEMORY_SCOPE_SYSTEM, 8>((const unsigned*)(ws + WS_EFLAG + (size_t)(unit * 4 + wave) * FS), call, tmo);
    u32x4 st[16];
    {
      const i32x4 rs = make_rsrc(ws + WS_STAGE + wave_off, WAVE_B);
#pragma unroll
      for (int j = 0; j < 16; ++j)
        asm volatile("buffer_load_dwordx4 %0, %1, %2, %3 offen offset:%4 sc0 sc1" : "=v"(st[j]) : "v"(pov), "s"(rs), "s"(j * PC & ~4095), "i"(j * PC & 4095) : "memory");
      asm volatile("s_waitcnt vmcnt(0)" ::: "memory");
#pragma unroll
      for (int j = 0; j < 16; ++j) asm volatile("" : "+v"(st[j]));
    }
    STAMP(6);
    const Quant qz = global_scale<ESLEEP>((const u64*)(ws + gmax_off), call, tmo, ok, lane);
    STAMP(4);
    GAS char* ob = dout + (size_t)(srank * ML + row0) * NL + col0;
#pragma unroll
    for (int m = 0; m < 8; ++m) {
      const u32x4 s0 = st[2 * m], s1 = st[2 * m + 1];
      u32x4 o;
#pragma unroll
      for (int r = 0; r < 4; ++r) {
        const int h = r >> 1;
        const unsigned w0 = s0[h], w1 = s0[2 + h], w2 = s1[h], w3 = s1[2 + h];
        const float v0 = __builtin_bit_cast(float, (r & 1) ? (w0 & 0xFFFF0000u) : (w0 << 16));
        const float v1 = __builtin_bit_cast(float, (r & 1) ? (w1 & 0xFFFF0000u) : (w1 << 16));
        const float v2 = __builtin_bit_cast(float, (r & 1) ? (w2 & 0xFFFF0000u) : (w2 << 16));
        const float v3 = __builtin_bit_cast(float, (r & 1) ? (w3 & 0xFFFF0000u) : (w3 << 16));
        o[r] = quant4(v0, v1, v2, v3, qz);
      }
      asm volatile("global_store_dwordx4 %0, %1, off sc0 sc1\n\ts_nop 1" :: "v"(ob + (size_t)m * 16 * NL), "v"(o) : "memory");
    }
    asm volatile(".Lq8_code_end1:");
    STAMP(7);
    STAMP_OUT();
    return;
  }

  if (unit == NWG - 1 && wave >= AGG0) {
    const float rm = sweep_max<4, 4 * FS / 8, __HIP_MEMORY_SCOPE_AGENT, 1>((const u64*)(ws + WS_AMAX + (lane >> 4) * FS) + (lane & 15), call, tmo);
    if (lane < NR)
      __hip_atomic_store((GAS u64*)(anyws + gmax_off) + rank, ((u64)call << 32) | __builtin_bit_cast(unsigned, rm), __ATOMIC_RELAXED, __HIP_MEMORY_SCOPE_SYSTEM);
  }
  const Quant qz = global_scale<LSLEEP>((const u64*)(ws + gmax_off), call, tmo, ok, lane);
  STAMP(4);
  {
    GAS char* ob = dout + (size_t)(rank * ML + row0) * NL + col0;
#pragma unroll
    for (int m = 0; m < 8; ++m) {
      u32x4 o;
#pragma unroll
      for (int r = 0; r < 4; ++r) o[r] = quant4(y[m][0][r], y[m][1][r], y[m][2][r], y[m][3][r], qz);
      asm volatile("s_waitcnt vmcnt(%2)\n\tglobal_store_dwordx4 %0, %1, off sc0 sc1\n\ts_nop 1" :: "v"(ob + (size_t)m * 16 * NL), "v"(o), "n"(LATE_WIN - 1) : "memory");
    }
  }
  asm volatile("s_waitcnt vmcnt(0)\n.Lq8_code_end2:" ::: "memory");
  STAMP(5);
  if (lane == 0)
    __hip_atomic_store((GAS unsigned*)(dws + WS_EFLAG + (size_t)(unit * 4 + wave) * FS), call, __ATOMIC_RELAXED, __HIP_MEMORY_SCOPE_SYSTEM);
  if (!wait_flag<__HIP_MEMORY_SCOPE_SYSTEM, 2>((const unsigned*)(ws + WS_EFLAG + (size_t)(unit * 4 + wave) * FS), call, tmo) && lane == 0)
    myout[(size_t)(srank * ML + row0) * NL + col0] = (char)-128;
  STAMP(7);
  STAMP_OUT();
}

static void make_pace(Params& p, const int* pa) {
  const double T0 = pa[0], TH = pa[1], T1 = pa[2], TR = pa[3], TC = pa[6];
  const int TE = pa[4], TS = pa[5], A = pa[7], DS = pa[8];
  p.slack = pa[9];
  p.nwide = pa[10]; p.wmask = pa[14];
  for (int k = 0; k < NE; ++k) { p.te[k] = k < NE - 3 ? TE : k == NE - 3 ? TS : pa[13]; p.start[k] = A + (k * DS > pa[11] ? k * DS - pa[11] : 0) - KH * (p.te[k] - TE); if (p.start[k] < A) p.start[k] = A; }
  for (int k = 0; k < pa[10]; ++k) p.te[k] = pa[12];
  p.start[NE] = A;
  double t = A;
  for (int v = 0; v <= KH; ++v) {
    double done = 0, u = t;
    while (done < 1.0) {
      double r = NWG / (u < TH ? T0 : u < TR ? T0 + (T1 - T0) * (u - TH) / (TR - TH) : T1);
      for (int k = 0; k < NE; ++k) if (u >= p.start[k] - p.te[k] / 2 && u < p.start[k] + (KH - 0.5) * p.te[k]) r -= (k == NE - 1 ? 20.0 : 14.0) / p.te[k];
      r /= NWG - 2 * NEARLY - 6;
      if (r > 1.0 / TC) r = 1.0 / TC;
      if (r < 0.001) r = 0.001;
      done += r; u += 1.0;
    }
    p.step[v] = (int)(u - t);
    if (v) t = u; else t += (u - t) * 0.5;
  }
}

extern "C" void kernel_launch(int rank, int nranks, ncclComm_t comm,
                              hipStream_t stream,
                              void* const* d_in, const int* in_sizes, int n_in,
                              void* d_out, int out_size,
                              const int* in_dtypes, int out_dtype,
                              void* const* d_out_all,
                              void* const* d_ws_all, size_t ws_bytes) {
  (void)comm; (void)in_dtypes; (void)out_dtype; (void)d_out;
  if (nranks != NR || n_in != 2 || in_sizes[0] != ML * KD || in_sizes[1] != KD * ND || out_size != NR * ML * NL ||
      ws_bytes < WS_END || (unsigned)rank >= (unsigned)NR) {
    fprintf(stderr, "unexpected problem shape\n");
    return;
  }
  HIP_CHECK(hipFuncSetAttribute((const void*)fused_kernel, hipFuncAttributeMaxDynamicSharedMemorySize, LDS_BYTES));
  Params p;
  p.x = (const float*)d_in[0];
  p.w = (const float*)d_in[1];
  for (int r = 0; r < NR; ++r) { p.out[r] = (char*)d_out_all[r]; p.ws[r] = (char*)d_ws_all[r]; }
  p.rank = rank;
  p.myws = (char*)d_ws_all[rank];
  int pa[15] = {PACE_ARGS};
  make_pace(p, pa);
  fused_kernel<<<NWG, NTHR, LDS_BYTES, stream>>>(p);
  HIP_CHECK(hipGetLastError());
}
```

```cpp
#include <hip/hip_runtime.h>
#include <rccl/rccl.h>
#include <cstdio>
#include <cstdint>
#include <cstdlib>
#include <cstddef>

#define HIP_CHECK(expr) do { hipError_t _e = (expr); if (_e != hipSuccess) { \
    fprintf(stderr, "HIP error %s at %s:%d\n", hipGetErrorString(_e), __FILE__, __LINE__); \
    return; } } while (0)

#define LAS __attribute__((address_space(3)))
#define GAS __attribute__((address_space(1)))
typedef float f32x4 __attribute__((ext_vector_type(4)));
typedef float f32x2 __attribute__((ext_vector_type(2)));
typedef __bf16 bf16x2 __attribute__((ext_vector_type(2)));
typedef unsigned u32x4 __attribute__((ext_vector_type(4)));
typedef int i32x4 __attribute__((ext_vector_type(4)));
typedef int i32x2 __attribute__((ext_vector_type(2)));
typedef unsigned long long u64;

constexpr int NR = 8;
constexpr int KD = 4096, ND = 16384;
constexpr int ML = 512;
constexpr int NL = ND / NR;
constexpr int NWG = 256, NTHR = 256;
constexpr int NBLK = KD / 64;
constexpr int NT = ND / 128;
#define NE 9
constexpr int NEARLY = 7 * NE;
#define PF0 4
#define PFL 3
static_assert(NE >= 7 && NE <= 12, "the workgroups that prefetch x belong to early tiles that start late");
constexpr int KH = NBLK / 2;
#define NH 2
#define PACE_ARGS 212, 2100, 157, 4300, 125, 150, 115, 600, 245, 100, 2, 100, 122, 175, 3, 100, 150, 228, 2300, 163, 4400
constexpr int LDS_X = 4 * 32768;
constexpr int LDS_BYTES = LDS_X + 32768;
#define PC 2048
constexpr int FS = 256, WAVE_B = 16 * PC;
constexpr size_t HALF_B = 4 * WAVE_B;

constexpr size_t WS_TMO = 128;
constexpr size_t WS_GMAX = 512;
constexpr size_t WS_START = 1024;
constexpr size_t WS_AMAX = 4096;
constexpr size_t WS_XFLAG = WS_AMAX + NWG / 16 * FS;
constexpr size_t WS_EFLAG = WS_XFLAG + NWG * FS;
constexpr size_t WS_PFLAG = WS_EFLAG + NWG * 4 * FS;
constexpr size_t WS_STAMP = WS_PFLAG + NWG * 4 * FS;
constexpr size_t WS_XT = 1 << 20;
constexpr size_t XT_BYTES = (size_t)ML * KD * 2, W_BYTES = (size_t)KD * ND * 4;
constexpr size_t WS_PART = WS_XT + XT_BYTES;
constexpr size_t WS_STAGE = WS_PART + NWG * HALF_B;
constexpr size_t WS_END = WS_STAGE + 2 * NEARLY * HALF_B;
static_assert(WS_STAMP + 65536 + NWG * 512 <= WS_XT, "control block overlaps x");
constexpr unsigned BUSY_TICKS = 50000;
constexpr u64 DEADLINE_TICKS = 50000000ull;

struct Params {
  const float* x;
  const float* w;
  char* myws;
  int rank;
  int slack;
  char* out[NR];
  char* ws[NR];
  int nwide, wmask;
  int start[NE + 1], te[NE];
  int step[KH + 1];
};

#define ESLEEP 32
#define LSLEEP 4
#define AGG0 2
#define WARMUP 14
#define LATE_WIN 2
#define SEND_WIN 3
#define STAMP(i) do {} while (0)
#define STAMPC(i) do {} while (0)
#define STAMP_DECL do {} while (0)
#define STAMP_OUT() do {} while (0)

__device__ __forceinline__ unsigned pk_bf16(float lo, float hi) {
  f32x2 v = {lo, hi};
  return __builtin_bit_cast(unsigned, __builtin_convertvector(v, bf16x2));
}
__device__ __forceinline__ float wave_max(float f) {
  int v = __builtin_bit_cast(int, f);
  v = max(v, __builtin_amdgcn_update_dpp(0, v, 0x111, 0xf, 0xf, true));
  v = max(v, __builtin_amdgcn_update_dpp(0, v, 0x112, 0xf, 0xf, true));
  v = max(v, __builtin_amdgcn_update_dpp(0, v, 0x114, 0xf, 0xf, true));
  v = max(v, __builtin_amdgcn_update_dpp(0, v, 0x118, 0xf, 0xf, true));
  const int r = max(max(__builtin_amdgcn_readlane(v, 15), __builtin_amdgcn_readlane(v, 31)), max(__builtin_amdgcn_readlane(v, 47), __builtin_amdgcn_readlane(v, 63)));
  return __builtin_bit_cast(float, r);
}
__device__ __forceinline__ i32x4 make_rsrc(const void* base, unsigned bytes) {
  const u64 a = (u64)base;
  return i32x4{__builtin_amdgcn_readfirstlane((int)(unsigned)a), __builtin_amdgcn_readfirstlane((int)(unsigned)(a >> 32)), (int)bytes, 0x00020000};
}

__device__ __forceinline__ void w_load(f32x4& d, unsigned voff, i32x4 rsrc, unsigned soff) {
  asm volatile("buffer_load_dwordx4 %0, %1, %2, %3 offen nt" : "=v"(d) : "v"(voff), "s"(rsrc), "s"(soff) : "memory");
}
__device__ __forceinline__ void set_m0(unsigned m0v) { asm volatile("s_mov_b32 m0, %0" :: "s"(m0v) : "memory"); }
template <int IMM>
__device__ __forceinline__ void x_load(unsigned voff, i32x4 rsrc, unsigned soff) {
  asm volatile("buffer_load_dwordx4 %0, %1, %2 offen offset:%3 sc1 lds" :: "v"(voff), "s"(rsrc), "s"(soff), "i"(IMM) : "memory");
}
template <int N>
__device__ __forceinline__ void wait_bar() { asm volatile("s_waitcnt vmcnt(%0) lgkmcnt(0)\n\ts_barrier" :: "n"(N) : "memory"); }
__device__ __forceinline__ void mfma(f32x4& c, u32x4 a, u32x4 b) {
  asm volatile("v_mfma_f32_16x16x32_bf16 %0, %1, %2, %0" : "+a"(c) : "v"(a), "v"(b));
}
#define SCHED() __builtin_amdgcn_sched_barrier(0)
__device__ __forceinline__ void send_piece(const f32x4 (&y)[8][4], int j, i32x4 srs, unsigned lane16) {
  const f32x4 lo = y[j >> 1][(j & 1) * 2], hi = y[j >> 1][(j & 1) * 2 + 1];
  const u32x4 d = {pk_bf16(lo[0], lo[1]), pk_bf16(lo[2], lo[3]), pk_bf16(hi[0], hi[1]), pk_bf16(hi[2], hi[3])};
  asm volatile("buffer_store_dwordx4 %0, %1, %2, %3 offen sc0 sc1\n\ts_nop 1" :: "v"(d), "v"(lane16), "s"(srs), "s"(j * PC) : "memory");
}
__device__ __forceinline__ void touch(const GAS char* t) {
  unsigned v;
  asm volatile("global_load_dword %0, %1, off nt\n\ts_waitcnt vmcnt(0)" : "=&v"(v) : "v"(t) : "memory");
}
constexpr float QBIAS = 12582912.0f;
struct Quant { float inv, lo, hi; };
__device__ __forceinline__ unsigned quant4(float v0, float v1, float v2, float v3, const Quant& qz) {
  const unsigned e0 = __builtin_bit_cast(unsigned, __builtin_amdgcn_fmed3f(__builtin_fmaf(v0, qz.inv, QBIAS), qz.lo, qz.hi));
  const unsigned e1 = __builtin_bit_cast(unsigned, __builtin_amdgcn_fmed3f(__builtin_fmaf(v1, qz.inv, QBIAS), qz.lo, qz.hi));
  const unsigned e2 = __builtin_bit_cast(unsigned, __builtin_amdgcn_fmed3f(__builtin_fmaf(v2, qz.inv, QBIAS), qz.lo, qz.hi));
  const unsigned e3 = __builtin_bit_cast(unsigned, __builtin_amdgcn_fmed3f(__builtin_fmaf(v3, qz.inv, QBIAS), qz.lo, qz.hi));
  return __builtin_amdgcn_perm(e1, e0, 0x0c0c0400u) | __builtin_amdgcn_perm(e3, e2, 0x04000c0cu);
}

template <int SCOPE, int SLEEP, int N = 1, int STRIDE = 0>
__device__ __forceinline__ bool wait_flag(const unsigned* flag, unsigned call, unsigned* tmo) {
  const u64 t0 = __builtin_amdgcn_s_memrealtime();
  for (unsigned polls = 1;; ++polls) {
    unsigned v[N];
    bool pend = false;
#pragma unroll
    for (int i = 0; i < N; ++i) v[i] = __hip_atomic_load(flag + i * STRIDE, __ATOMIC_RELAXED, SCOPE);
#pragma unroll
    for (int i = 0; i < N; ++i) pend |= v[i] < call;
    if (!__any(pend)) return true;
    if (((polls & 63u) == 0 && __hip_atomic_load(tmo, __ATOMIC_RELAXED, __HIP_MEMORY_SCOPE_AGENT) == call) ||
        __builtin_amdgcn_s_memrealtime() - t0 > DEADLINE_TICKS) {
      __hip_atomic_store(tmo, call, __ATOMIC_RELAXED, __HIP_MEMORY_SCOPE_AGENT);
      return false;
    }
    __builtin_amdgcn_s_sleep(SLEEP);
  }
}

struct Ctx {
  i32x4 wrsrc, xrsrc;
  unsigned wvoff[8], xvoff0, xvoff1, m0base;
  int wlim;
  int left;
  unsigned xso, wso;
  unsigned due, slack;
  int steps;
  i32x4 frsrc;
  const char* fbase;
  unsigned fvoff, fso, call, fl;
  unsigned* tmo;
  bool ok;
  LAS const char* lrd0;
  LAS const char* lrd1;
  LAS char* wmine;
  LAS const char* wother;
};

template <int SLOT>
__device__ __forceinline__ u32x4 x_frag(const Ctx& c, int mt) {
  return *(LAS const u32x4*)((SLOT < 2 ? c.lrd0 : c.lrd1) + (SLOT & 1) * 32768 + mt * 1024);
}
template <int G>
__device__ __forceinline__ void x_piece(const Ctx& c, i32x4 xr, unsigned xso) {
  x_load<(G & 3) * 1024>((G & 4) ? c.xvoff1 : c.xvoff0, xr, (G & 8) ? xso + 32768 : xso);
}
template <int SL, bool CHK, bool HELD = false>
__device__ __forceinline__ void kstep(Ctx& c, f32x4 (&acc)[16][4], f32x4 (&wr)[2][8], u32x4 (&a)[2][4], u32x4 (&xf)[8], const u32x4* held = nullptr) {
  constexpr int E = SL & 1, WS = SL >> 1, NX = (SL + 1) & 3;
  u64 now;
  unsigned lateby;
  const i32x4 xr = c.xrsrc, wrs = c.wrsrc;
  const unsigned xso = c.xso, wso = c.wso;
  const unsigned m0x = c.m0base + ((SL + 2) & 3) * 32768;
  f32x4 (&r)[8] = wr[WS ^ 1];
#pragma unroll
  for (int g = 0; g < 16; ++g) {
    if (E == 1 && g == 12) {
      if (CHK) {
        asm volatile("s_waitcnt vmcnt(8)" : "+v"(c.fl) :: "memory");
        if (__any(c.fl < c.call)) {
          c.ok &= wait_flag<__HIP_MEMORY_SCOPE_AGENT, 1>((const unsigned*)(c.fbase + c.fso + c.fvoff), c.call, c.tmo);
          __builtin_amdgcn_s_waitcnt(0xC07F);
        }
      }
      wait_bar<HELD ? 0 : 8>();
      SCHED();
    }
    mfma(acc[g][0], a[E][0], xf[g & 7]);
    SCHED();
    if (g < 9) xf[(g + 7) & 7] = x_frag<SL>(c, g + 7);
    else if (E == 0) xf[(g + 7) & 7] = x_frag<NX>(c, g - 9);
    else if (g >= 12) xf[(2 * (g - 12)) & 7] = x_frag<NX>(c, 2 * (g - 12));
    if (CHK && E == 0 && g == 0) asm volatile("buffer_load_dword %0, %1, %2, %3 offen sc1" : "=v"(c.fl) : "v"(c.fvoff), "s"(c.frsrc), "s"(c.fso) : "memory");
    if (E == 1 && g == 9) *(LAS u32x4*)(c.wmine + (WS ^ 1) * 16384 + 1024) = a[0][1];
    if (E == 1 && g == 10) *(LAS u32x4*)(c.wmine + (WS ^ 1) * 16384 + 3072) = a[0][3];
    if (E == 1 && g == 11) {
      asm volatile("s_waitcnt lgkmcnt(0)" : "+s"(now) :: "memory");
      lateby = (unsigned)now - c.due;
      asm volatile("" : "+s"(lateby));
    }
    SCHED();
    mfma(acc[g][1], a[E][1], xf[g & 7]);
    SCHED();
    if (E == 0) {
      switch (g) {
        case 0: x_piece<0>(c, xr, xso); break;   case 1: x_piece<1>(c, xr, xso); break;
        case 2: x_piece<2>(c, xr, xso); break;   case 3: x_piece<3>(c, xr, xso); break;
        case 4: x_piece<4>(c, xr, xso); break;   case 5: x_piece<5>(c, xr, xso); break;
        case 6: x_piece<6>(c, xr, xso); break;   case 7: x_piece<7>(c, xr, xso); break;
        case 8: x_piece<8>(c, xr, xso); break;   case 9: x_piece<9>(c, xr, xso); break;
        case 10: x_piece<10>(c, xr, xso); break; case 11: x_piece<11>(c, xr, xso); break;
        case 12: x_piece<12>(c, xr, xso); break; case 13: x_piece<13>(c, xr, xso); break;
        case 14: x_piece<14>(c, xr, xso); break; default: x_piece<15>(c, xr, xso); break;
      }
    } else {
      if (!HELD && g < 8) w_load(wr[WS][g & 7], c.wvoff[g & 7], wrs, wso);
      if (HELD && g >= 4 && g < 8) { a[0][g & 3] = held[g & 3]; asm volatile("" : "+v"(a[0][g & 3])); }
      if (!HELD && g == 5)
        asm volatile("s_waitcnt vmcnt(%8)" : "+v"(r[0]), "+v"(r[1]), "+v"(r[2]), "+v"(r[3]), "+v"(r[4]), "+v"(r[5]), "+v"(r[6]), "+v"(r[7]) : "n"(22 + CHK) : "memory");
      if (g == 8) *(LAS u32x4*)(c.wmine + (WS ^ 1) * 16384) = a[0][0];
      if (g == 9) { c.left -= 1; c.xso += 65536u; asm volatile("" : "+s"(c.left), "+s"(c.xso)); }
      if (g == 10) { c.xrsrc[2] = c.left > 1 ? (int)XT_BYTES : 0; asm volatile("" : "+s"(c.xrsrc)); }
      if (g == 11) { c.wrsrc[2] = c.left > c.wlim ? (int)W_BYTES : 0; asm volatile("" : "+s"(c.wrsrc)); }
      if (g == 12) { lateby = (int)lateby > (int)c.slack ? lateby - c.slack : 0u; asm volatile("" : "+s"(lateby)); }
      if (g == 13) { c.due += lateby; lateby = __builtin_amdgcn_readlane(c.steps, c.left); asm volatile("" : "+s"(c.due), "+s"(lateby)); }
      if (g == 14) { c.due += lateby; asm volatile("" : "+s"(c.due)); }
    }
    SCHED();
    mfma(acc[g][2], a[E][2], xf[g & 7]);
    SCHED();
    if (E == 0 && g < 4) a[1][g] = *(LAS const u32x4*)(c.wother + WS * 16384 + g * 1024);
    if (!HELD && E == 1 && g >= 5 && g < 9) { const int t = (g - 5) & 3; a[0][t][0] = pk_bf16(r[0][t], r[1][t]); a[0][t][1] = pk_bf16(r[2][t], r[3][t]); }
    if (E == 1 && g == 9) *(LAS u32x4*)(c.wmine + (WS ^ 1) * 16384 + 2048) = a[0][2];
    if (E == 1 && g == 11) {
      if (__builtin_expect((int)lateby < 0, 0))
        do __builtin_amdgcn_s_sleep(1); while ((int)((unsigned)__builtin_amdgcn_s_memrealtime() - c.due) < 0);
    }
    if (E == 1 && g >= 12 && g < 15) xf[(2 * (g - 12) + 1) & 7] = x_frag<NX>(c, 2 * (g - 12) + 1);
    SCHED();
    mfma(acc[g][3], a[E][3], xf[g & 7]);
    SCHED();
    if (!HELD && E == 1 && g >= 5 && g < 9) { const int t = (g - 5) & 3; a[0][t][2] = pk_bf16(r[4][t], r[5][t]); a[0][t][3] = pk_bf16(r[6][t], r[7][t]); }
    if (E == 0 && (g & 3) == 3 && g < 15) set_m0(m0x + ((g + 1) >> 3) * 32768 + (((g + 1) >> 2) & 1) * 4096);
    if (E == 1 && g == 9) asm volatile("s_memrealtime %0" : "=s"(now) :: "memory");
    if (E == 1 && g == 10) { c.wso += 1u << 22; asm volatile("" : "+s"(c.wso)); }
    if (E == 1 && g == 15) set_m0(c.m0base + ((SL + 3) & 3) * 32768);
    SCHED();
  }
}

__device__ __forceinline__ void last_block(const Ctx& c, f32x4 (&acc)[16][4], u32x4 (&a)[2][4], u32x4 (&xf)[8], i32x4 prs, unsigned pov) {
  f32x4 lo, hi;
#pragma unroll
  for (int i = 0; i < 32; ++i) {
    const int e = (i >> 3) & 1, m = (i & 7) + (i >> 4) * 8;
    const int i7 = i + 7, e7 = (i7 >> 3) & 1, m7 = (i7 & 7) + (i7 >> 4) * 8;
    const int j = i - 16;
    mfma(acc[m][0], a[e][0], xf[i & 7]);
    SCHED();
    if (i7 < 32) xf[i7 & 7] = e7 ? x_frag<3>(c, m7) : x_frag<2>(c, m7);
    if (j >= 0) {
      asm volatile("" : "+a"(acc[j >> 1][(j & 1) * 2]));
      lo = acc[j >> 1][(j & 1) * 2];
      asm volatile("" : "+v"(lo));
    }
    SCHED();
    mfma(acc[m][1], a[e][1], xf[i & 7]);
    SCHED();
    if (i < 4) a[1][i] = *(LAS const u32x4*)(c.wother + 16384 + i * 1024);
    if (j >= 0) {
      asm volatile("" : "+a"(acc[j >> 1][(j & 1) * 2 + 1]));
      hi = acc[j >> 1][(j & 1) * 2 + 1];
      asm volatile("" : "+v"(hi));
    }
    SCHED();
    mfma(acc[m][2], a[e][2], xf[i & 7]);
    SCHED();
    u32x4 d;
    if (j >= 0) { d = u32x4{pk_bf16(lo[0], lo[1]), pk_bf16(lo[2], lo[3]), pk_bf16(hi[0], hi[1]), pk_bf16(hi[2], hi[3])}; asm volatile("" : "+v"(d)); }
    SCHED();
    mfma(acc[m][3], a[e][3], xf[i & 7]);
    SCHED();
    if (j >= 0)
      asm volatile("buffer_store_dwordx4 %0, %1, %2, %3 offen offset:%4 sc1\n\ts_nop 1"
                   :: "v"(d), "v"(pov), "s"(prs), "s"(j * PC & ~4095), "i"(j * PC & 4095) : "memory");
    SCHED();
  }
}

template <int N, int STRIDE, int SCOPE, int SLEEP>
__device__ __forceinline__ float sweep_max(const u64* slots, unsigned call, unsigned* tmo) {
  const u64 t0 = __builtin_amdgcn_s_memrealtime();
  for (unsigned polls = 1;; ++polls) {
    bool all = true, bad = false;
    float mx = 0.f;
    u64 v[N];
#pragma unroll
    for (int r = 0; r < N; ++r) v[r] = __hip_atomic_load(slots + r * STRIDE, __ATOMIC_RELAXED, SCOPE);
#pragma unroll
    for (int r = 0; r < N; ++r) {
      const float f = __builtin_bit_cast(float, (unsigned)v[r]);
      all &= (unsigned)(v[r] >> 32) == call;
      bad |= f < 0.f;
      mx = fmaxf(mx, f);
    }
    if (__all(all)) {
      return __any(bad) ? -1.0f : wave_max(mx);
    }
    if (((polls & 63u) == 0 && __hip_atomic_load(tmo, __ATOMIC_RELAXED, __HIP_MEMORY_SCOPE_AGENT) == call) ||
        __builtin_amdgcn_s_memrealtime() - t0 > DEADLINE_TICKS) {
      __hip_atomic_store(tmo, call, __ATOMIC_RELAXED, __HIP_MEMORY_SCOPE_AGENT);
      return -1.0f;
    }
    __builtin_amdgcn_s_sleep(SLEEP);
  }
}
template <int SLEEP>
__device__ __forceinline__ Quant global_scale(const u64* gmax, unsigned call, unsigned* tmo, bool ok, int lane) {
  const float gam = ok ? sweep_max<1, 0, __HIP_MEMORY_SCOPE_SYSTEM, SLEEP>(gmax + (lane & 7), call, tmo) : -1.0f;
  Quant qz;
  qz.inv = gam > 0.f ? 127.0f / gam : 0.f;
  qz.lo = gam >= 0.f ? QBIAS - 127.0f : QBIAS - 128.0f;
  qz.hi = gam >= 0.f ? QBIAS + 127.0f : QBIAS - 128.0f;
  asm volatile("" : "+v"(qz.lo), "+v"(qz.hi));
  return qz;
}

__device__ __forceinline__ int line_of(int ct, int s) {
  return ct < 7 ? 8 * ct + 4 + s : ct < 11 ? 16 * (ct - 7) + (s >> 1) * 8 + (s & 1) : ct == 11 ? 60 + s : (ct & 1) * 32 + 8 * s + (ct >> 1) - 4;
}
__device__ __forceinline__ void tile_of(int j, int& pd, int& ct) {
  const int k = j / 7;
  pd = j < 112 ? j % 7 + 1 : 0;
  ct = j >= 112 ? j - 112 : k < 2 ? 12 + k : k < NE - 2 ? 14 - NE + k : k < NE ? 16 - NE + k : k - NE;
}

__global__ void __launch_bounds__(NTHR, 1) fused_kernel(Params p) {
  extern __shared__ __attribute__((aligned(1024))) char smem[];
  const int tid = threadIdx.x, lane = tid & 63;
  const int wave = __builtin_amdgcn_readfirstlane(tid >> 6);
  char* ws = p.myws;

  STAMP_DECL;
  STAMP(0);
  STAMPC(8);
  const int b = blockIdx.x, xcd = b & 7, idx = b >> 3;
  const int kh = xcd & 1, slot = idx * 4 + (xcd >> 1), unit = slot * 2 + kh;
  const int wn = wave & 1, wm = wave >> 1;
  const int q = lane >> 4, i16 = lane & 15;
  const unsigned lane16 = lane * 16;
  const unsigned t_start = (unsigned)__builtin_amdgcn_s_memrealtime();

  Ctx c;
  c.xrsrc = make_rsrc(ws + WS_XT, (unsigned)XT_BYTES);
  unsigned* xflag = (unsigned*)(ws + WS_XFLAG);
  const int hs = kh * NT + slot;
  const unsigned xto = (unsigned)(hs * 16384 + wave * 4096);
  f32x4 raw[8];
  i32x2 prev;
  unsigned warm;
  {
    const int st = hs >> 1, h = hs & 1;
    const i32x4 xin = make_rsrc(p.x, (unsigned)((size_t)ML * KD * 4));
#pragma unroll
    for (int j = 0; j < 4; ++j) {
      const unsigned o = (unsigned)((h * 16 + ((wave * 4 + j) ^ ((kh ^ 1) << 3))) * 16 + i16) * (KD * 4) + ((st ^ h) * 32 + q * 8) * 4;
      asm volatile("buffer_load_dwordx4 %0, %2, %3, 0 offen\n\tbuffer_load_dwordx4 %1, %2, %3, 0 offen offset:16"
                   : "=&v"(raw[2 * j]), "=&v"(raw[2 * j + 1]) : "v"(o), "s"(xin) : "memory");
    }
    asm volatile("global_load_dwordx2 %0, %1, off sc1" : "=v"(prev) : "v"(xflag + hs * (FS / 4)) : "memory");
    asm volatile("buffer_load_dword %0, %1, %2, %3 offen sc1" : "=v"(warm) : "v"(0), "s"(c.xrsrc), "s"(xto) : "memory");
  }

  unsigned cpf = 0;
  {
    u64 pc; unsigned len1, len2;
    asm volatile("s_getpc_b64 %0\n.Lq8_code_begin:\n\ts_mov_b32 %1, .Lq8_code_end1-.Lq8_code_begin\n\ts_mov_b32 %2, .Lq8_code_end2-.Lq8_code_begin" : "=s"(pc), "=s"(len1), "=s"(len2));
    const int wv = (idx - 8) * 4 + wave;
    const i32x4 cr = (unsigned)wv < 8u ? make_rsrc((const void*)pc, (WARMUP & 4) ? (len1 > len2 ? len1 : len2) : 0)
                                       : make_rsrc(p.w, ((WARMUP & 8) && (unsigned)(wv - 8) < 2u) ? (unsigned)W_BYTES : 0);
    asm volatile("buffer_load_dword %0, %1, %2, 0 offen" : "+v"(cpf)
                 : "v"((unsigned)wv < 8u ? (unsigned)(wv * 64 + lane) * 128u : (unsigned)(kh * 128 + (wv & 1) * 64 + lane) << 20), "s"(cr) : "memory");
  }
  const bool early = slot < NEARLY;
  int pd, ct;
  tile_of(slot, pd, ct);
  const int rank = p.rank;
  const unsigned dest = (unsigned)(rank + pd) & 7u;
  const int k0 = kh * KH;
  const int ln0 = line_of(ct, 2 * wn), ln1 = line_of(ct, 2 * wn + 1);
  unsigned* tmo = (unsigned*)(ws + WS_TMO);
  GAS char* dws = (GAS char*)p.ws[dest];
  GAS char* dout = (GAS char*)p.out[early ? rank : dest];
  GAS char* myout = (GAS char*)p.out[rank];
  asm volatile("" : "+s"(dws), "+s"(dout), "+s"(myout));
  GAS char* anyws;
  i32x2 stp;
  asm volatile("global_load_dwordx2 %0, %2, %4 offset:%5\n\tglobal_load_dwordx2 %1, %3, %4 offset:%6"
               : "=&v"(anyws), "=&v"(stp) : "v"((lane & 7) * 8), "v"(((KH - 1 - lane) & 31) * 4), "s"(__builtin_amdgcn_kernarg_segment_ptr()),
                 "i"(offsetof(Params, ws)), "i"(offsetof(Params, step)) : "memory");

  c.wrsrc = make_rsrc(p.w, (unsigned)W_BYTES);
#pragma unroll
  for (int j = 0; j < 8; ++j) {
    c.wvoff[j] = ((unsigned)(wm * 32 + q * 8 + j) << 16) + (i16 < 8 ? ln0 : ln1) * 128 + (i16 & 7) * 16;
    asm volatile("" : "+v"(c.wvoff[j]));
  }
  c.xvoff0 = wave * 8192 + lane16;
  c.xvoff1 = c.xvoff0 + 4096;
  asm volatile("" : "+v"(c.xvoff1));
  c.m0base = (unsigned)(size_t)(LAS char*)smem + wave * 8192;
  constexpr int nblk = KH;
  const unsigned wsb = ((unsigned)k0 << 22) + dest * (NL * 4u);
  c.left = nblk;
  c.wlim = 2;
  c.xso = (unsigned)(k0 + 1) * 65536u;
  c.wso = wsb + (2u << 22);
  const bool steady = early || ct == 11 || ct >= 14;
  const int cls = early ? slot / 7 : steady ? NE - 1 : NE, te = p.te[steady ? cls : 0], phase = cls < p.nwide ? 0 : (slot * 74 + kh * 8) & 255;
  c.due = t_start + p.start[cls] + phase * (steady ? te : p.step[0] & 0xFFFF) / 256;
  c.slack = p.slack;
  int wide = (p.wmask >> cls) & 1;
  asm volatile("" : "+s"(wide));
  c.lrd0 = (LAS const char*)smem + wm * 16384 + lane16;
  c.lrd1 = c.lrd0 + 65536;
  asm volatile("" : "+v"(c.lrd1));
  c.wmine = (LAS char*)smem + LDS_X + wave * 4096 + lane16;
  c.wother = (LAS const char*)smem + LDS_X + (wave ^ 2) * 4096 + lane16;

  f32x4 acc[16][4];
  f32x4 wr[2][8];
  u32x4 a[2][4], xf[8];
#pragma unroll
  for (int m = 0; m < 16; ++m)
#pragma unroll
    for (int t = 0; t < 4; ++t) {
      acc[m][t] = f32x4{0.f, 0.f, 0.f, 0.f};
      asm volatile("" : "+a"(acc[m][t]));
    }
  asm volatile("s_waitcnt vmcnt(3)" : "+v"(prev), "+v"(warm), "+v"(raw[0]), "+v"(raw[1]), "+v"(raw[2]), "+v"(raw[3]), "+v"(raw[4]), "+v"(raw[5]), "+v"(raw[6]), "+v"(raw[7]) :: "memory");
  STAMP(11);
#pragma unroll
  for (int j = 0; j < 4; ++j) {
    const f32x4 v0 = raw[2 * j], v1 = raw[2 * j + 1];
    const u32x4 d = {pk_bf16(v0[0], v0[1]), pk_bf16(v0[2], v0[3]), pk_bf16(v1[0], v1[1]), pk_bf16(v1[2], v1[3])};
    asm volatile("buffer_store_dwordx4 %0, %1, %2, %3 offen offset:%4 sc1\n\ts_nop 1" :: "v"(d), "v"(lane16), "s"(c.xrsrc), "s"(xto), "i"(j * 1024) : "memory");
  }
  const unsigned call = __builtin_amdgcn_readfirstlane(prev[0]) + 1;
  const int half = (call > 1 && t_start - (unsigned)__builtin_amdgcn_readfirstlane(prev[1]) < BUSY_TICKS) ? 16 : 0;
  asm volatile("s_waitcnt vmcnt(0)\n\ts_barrier" : "+v"(anyws), "+v"(stp), "+v"(cpf) :: "memory");
  {
    const int s0 = (stp[0] >> half) & 0xFFFF, s1 = (stp[1] >> half) & 0xFFFF;
    c.steps = steady ? te : s0 + phase * (s1 - s0) / 256;
  }
  STAMP(12);
  if (tid == 0) __hip_atomic_store((u64*)(xflag + hs * (FS / 4)), ((u64)t_start << 32) | call, __ATOMIC_RELAXED, __HIP_MEMORY_SCOPE_AGENT);
  if (b == 0 && wave == 1 && lane < NR)
    __hip_atomic_store((GAS unsigned*)(anyws + WS_START + rank * FS), call, __ATOMIC_RELAXED, __HIP_MEMORY_SCOPE_SYSTEM);
  c.fbase = (const char*)xflag + kh * NT * FS;
  c.frsrc = make_rsrc(c.fbase, NT * FS);
  c.fvoff = (lane & 3) * FS;
  c.fso = 8 * FS;
  c.call = call;
  c.tmo = tmo;
  c.ok = true;
  if ((unsigned)(idx - PF0) < (1u << PFL)) {
    constexpr int NP = 16 >> PFL;
    const int pw = (idx - PF0) * 4 + wave;
    const unsigned pv = (unsigned)k0 * 65536u + pw * NP * 1024 + lane16;
    u32x4 pf = {0u, 0u, 0u, 0u};
#pragma unroll
    for (int j = 1; j < KH; ++j) {
      if (j == 1 || j == 5)
        c.ok &= wait_flag<__HIP_MEMORY_SCOPE_AGENT, 1>((const unsigned*)(c.fbase + ((j == 1 ? (lane & 3) + 1 : lane & 31) * 4 + (pw >> PFL)) * FS), call, tmo);
      const unsigned v = pv + j * 65536;
      asm volatile("s_waitcnt vmcnt(32)\n\tbuffer_load_dwordx4 %0, %1, %2, 0 offen sc1\n\tbuffer_load_dwordx4 %0, %1, %2, 0 offen offset:1024 sc1" : "+v"(pf) : "v"(v), "s"(c.xrsrc) : "memory");
      if (NP == 4) asm volatile("buffer_load_dwordx4 %0, %1, %2, 0 offen offset:2048 sc1\n\tbuffer_load_dwordx4 %0, %1, %2, 0 offen offset:3072 sc1" : "+v"(pf) : "v"(v), "s"(c.xrsrc) : "memory");
    }
    asm volatile("s_waitcnt vmcnt(0)" : "+v"(pf) :: "memory");
    if ((WARMUP & 2) && cls >= 3) {
      const int u1 = unit + ((lane & 3) == 1 ? -56 : (lane & 3) == 2 ? 56 : 0), u2 = (unsigned)u1 < 2u * NEARLY ? u1 : unit;
      const size_t o = (size_t)u2 * HALF_B + wave * WAVE_B;
      const int k = (lane >> 2) & 7;
      touch(k == 0 ? dws + WS_STAGE + o : k == 1 ? (GAS char*)ws + WS_PART + o : k == 2 ? (GAS char*)ws + WS_PART + (o ^ HALF_B)
            : k == 3 ? (GAS char*)ws + WS_STAGE + o : k == 4 ? dws + WS_EFLAG + (size_t)u2 * 4 * FS + 128
            : myout + (size_t)(((rank - pd) & 7) * ML + wm * 256 + kh * 128 + (lane & 3) * 32) * NL);
    }
  }
#pragma unroll
  for (int j = 0; j < 8; ++j) w_load(wr[0][j], c.wvoff[j], c.wrsrc, wsb);
  if (wave == 0) c.ok &= wait_flag<__HIP_MEMORY_SCOPE_AGENT, 1>((const unsigned*)(c.fbase + (lane & 7) * FS), call, tmo);
  asm volatile("s_waitcnt vmcnt(0)" : "+v"(wr[0][0]), "+v"(wr[0][1]), "+v"(wr[0][2]), "+v"(wr[0][3]), "+v"(wr[0][4]), "+v"(wr[0][5]), "+v"(wr[0][6]), "+v"(wr[0][7]) :: "memory");
#pragma unroll
  for (int t = 0; t < 4; ++t) {
    a[0][t] = u32x4{pk_bf16(wr[0][0][t], wr[0][1][t]), pk_bf16(wr[0][2][t], wr[0][3][t]), pk_bf16(wr[0][4][t], wr[0][5][t]), pk_bf16(wr[0][6][t], wr[0][7][t])};
    *(LAS u32x4*)(c.wmine + t * 1024) = a[0][t];
  }
  wait_bar<0>();
  {
    const unsigned xso = (unsigned)k0 * 65536u;
    set_m0(c.m0base); asm volatile("s_nop 0");
    x_piece<0>(c, c.xrsrc, xso); x_piece<1>(c, c.xrsrc, xso); x_piece<2>(c, c.xrsrc, xso); x_piece<3>(c, c.xrsrc, xso);
    set_m0(c.m0base + 4096); asm volatile("s_nop 0");
    x_piece<4>(c, c.xrsrc, xso); x_piece<5>(c, c.xrsrc, xso); x_piece<6>(c, c.xrsrc, xso); x_piece<7>(c, c.xrsrc, xso);
    set_m0(c.m0base + 32768); asm volatile("s_nop 0");
    x_piece<8>(c, c.xrsrc, xso); x_piece<9>(c, c.xrsrc, xso); x_piece<10>(c, c.xrsrc, xso); x_piece<11>(c, c.xrsrc, xso);
    set_m0(c.m0base + 32768 + 4096); asm volatile("s_nop 0");
    x_piece<12>(c, c.xrsrc, xso); x_piece<13>(c, c.xrsrc, xso); x_piece<14>(c, c.xrsrc, xso); x_piece<15>(c, c.xrsrc, xso);
  }
#pragma unroll
  for (int j = 0; j < 8; ++j) w_load(wr[1][j], c.wvoff[j], c.wrsrc, wsb + (1u << 22));
  __builtin_amdgcn_s_waitcnt(0xC07F);
  wait_bar<8>();
  STAMP(10);
  SCHED();
  set_m0(c.m0base + 2 * 32768);
#pragma unroll
  for (int t = 0; t < 7; ++t) xf[t] = x_frag<0>(c, t);
  SCHED();
  while ((int)((unsigned)__builtin_amdgcn_s_memrealtime() + 100u - c.due) < 0) __builtin_amdgcn_s_sleep(4);
  __builtin_amdgcn_s_waitcnt(0xC07F);
  STAMP(1);

#define TRACE_PT()
  TRACE_PT()
  kstep<0, true>(c, acc, wr, a, xf);
  kstep<1, true>(c, acc, wr, a, xf);
  c.fvoff = (lane & 31) * FS;
  c.fso = wave * 32 * FS;
  kstep<2, true>(c, acc, wr, a, xf);
  kstep<3, true>(c, acc, wr, a, xf);
  kstep<0, false>(c, acc, wr, a, xf);
  kstep<1, false>(c, acc, wr, a, xf);
  u32x4 held[NH][4] = {};
  const int ltail = steady ? 1 : NH + 1;
  if (!steady) {
    c.wlim = 2 + NH;
    f32x4 raw2[2][8];
#pragma unroll
    for (int i = 0; i <= NH; ++i) {
      if (i < NH) {
#pragma unroll
        for (int j = 0; j < 8; ++j) w_load(raw2[i & 1][j], c.wvoff[j], c.wrsrc, wsb + ((unsigned)(nblk - NH + i) << 22));
      }
      if (i > 0) {
        f32x4 (&r)[8] = raw2[(i - 1) & 1];
        asm volatile("s_waitcnt vmcnt(%8)" : "+v"(r[0]), "+v"(r[1]), "+v"(r[2]), "+v"(r[3]), "+v"(r[4]), "+v"(r[5]), "+v"(r[6]), "+v"(r[7]) : "n"(i < NH ? 8 : 0) : "memory");
#pragma unroll
        for (int t = 0; t < 4; ++t) {
          held[i - 1][t] = u32x4{pk_bf16(r[0][t], r[1][t]), pk_bf16(r[2][t], r[3][t]), pk_bf16(r[4][t], r[5][t]), pk_bf16(r[6][t], r[7][t])};
          asm volatile("" : "+v"(held[i - 1][t]));
        }
      }
    }
    SCHED();
  }
  do {
    TRACE_PT()
    kstep<2, false>(c, acc, wr, a, xf);
    kstep<3, false>(c, acc, wr, a, xf);
    kstep<0, false>(c, acc, wr, a, xf);
    kstep<1, false>(c, acc, wr, a, xf);
  } while (c.left > ltail);
  while (c.left > 1) {
    TRACE_PT()
    kstep<2, false, true>(c, acc, wr, a, xf, held[0]);
    kstep<3, false, true>(c, acc, wr, a, xf, held[0]);
    kstep<0, false, true>(c, acc, wr, a, xf, held[1]);
    kstep<1, false, true>(c, acc, wr, a, xf, held[1]);
#pragma unroll
    for (int i = 0; i + 2 < NH; ++i)
#pragma unroll
      for (int t = 0; t < 4; ++t) held[i][t] = held[i + 2][t];
    asm volatile("s_nop 1");
  }
  LAS float* red = (LAS float*)(smem + LDS_X);
  const size_t gmax_off = WS_GMAX + (size_t)(call & 1) * FS;
  const size_t wave_off = (size_t)unit * HALF_B + wave * WAVE_B;
  const unsigned pov = (PC > 1024 && (lane >> 3) == 3) ? lane16 + 5 * 128 : lane16;
  unsigned* pflag = (unsigned*)(ws + WS_PFLAG + wave * FS);
  last_block(c, acc, a, xf, make_rsrc(ws + WS_PART + wave_off, WAVE_B), pov);
  bool ok = c.ok;
  STAMP(2);
  STAMPC(9);
  asm volatile("s_waitcnt vmcnt(0)"
               : "+v"(wr[0][0]), "+v"(wr[0][1]), "+v"(wr[0][2]), "+v"(wr[0][3]), "+v"(wr[0][4]), "+v"(wr[0][5]), "+v"(wr[0][6]), "+v"(wr[0][7])
               :: "memory");
  if (lane == 0) __hip_atomic_store(pflag + unit * FS, call, __ATOMIC_RELAXED, __HIP_MEMORY_SCOPE_AGENT);
  ok &= wait_flag<__HIP_MEMORY_SCOPE_SYSTEM, 2>((lane & 1) ? (const unsigned*)(ws + WS_START + dest * FS) : pflag + (unit ^ 1) * FS, call, tmo);
  const i32x4 srs = make_rsrc((const void*)(dws + WS_STAGE + wave_off), ok ? WAVE_B : 0);
  f32x4 y[8][4];
  {
    const i32x4 prs = make_rsrc(ws + WS_PART + (wave_off ^ HALF_B), WAVE_B);
    u32x4 pv[16];
#pragma unroll
    for (int j = 0; j < 16; ++j)
      asm volatile("buffer_load_dwordx4 %0, %1, %2, %3 offen offset:%4 sc1" : "=v"(pv[j]) : "v"(pov), "s"(prs), "s"(j * PC & ~4095), "i"(j * PC & 4095) : "memory");
    const unsigned one = (i16 >> 2) == q ? ((i16 & 1) ? 0x3F800000u : 0x00003F80u) : 0u;
    const unsigned e0 = (i16 & 2) ? 0u : one, e1 = (i16 & 2) ? one : 0u;
    u32x4 sel0 = {e0, e1, 0u, 0u}, sel1 = {0u, 0u, e0, e1};
    asm volatile("s_nop 1" : "+v"(sel0), "+v"(sel1));
    float am = 0.f;
#pragma unroll
    for (int j = 0; j < 16 + 4; ++j) {
      if (j < 16) {
        asm volatile("s_waitcnt vmcnt(%1)" : "+v"(pv[j]) : "n"(15 - j) : "memory");
        mfma(acc[8 + (j >> 1)][(j & 1) * 2], sel0, pv[j]);
        mfma(acc[8 + (j >> 1)][(j & 1) * 2 + 1], sel1, pv[j]);
      }
      if (j == 16) asm volatile("s_nop 7\n\ts_nop 7" ::: "memory");
      if (j >= 4) {
        const int m = (j - 4) >> 1, u = (j - 4) & 1;
        asm volatile("" : "+a"(acc[8 + m][2 * u]), "+a"(acc[8 + m][2 * u + 1]), "+v"(am));
        y[m][2 * u] = acc[8 + m][2 * u]; y[m][2 * u + 1] = acc[8 + m][2 * u + 1];
        asm volatile("" : "+v"(y[m][2 * u]), "+v"(y[m][2 * u + 1]));
        const f32x4 lo = y[m][2 * u], hi = y[m][2 * u + 1];
        am = fmaxf(fmaxf(am, fabsf(lo[0])), fabsf(lo[1])); am = fmaxf(fmaxf(am, fabsf(lo[2])), fabsf(lo[3]));
        am = fmaxf(fmaxf(am, fabsf(hi[0])), fabsf(hi[1])); am = fmaxf(fmaxf(am, fabsf(hi[2])), fabsf(hi[3]));
        asm volatile("" : "+v"(am));
      }
    }
    if (early) {
#pragma unroll
      for (int j = 0; j < SEND_WIN; ++j) send_piece(y, j, srs, pov);
    }
    am = wave_max(am);
    if (lane == 0) red[wave] = ok ? am : -1.0f;
    __syncthreads();
    if (tid == 0) {
      const float r0 = red[0], r1 = red[1], r2 = red[2], r3 = red[3];
      const float wam = fminf(fminf(r0, r1), fminf(r2, r3)) < 0.f ? -1.0f : fmaxf(fmaxf(r0, r1), fmaxf(r2, r3));
      __hip_atomic_store((u64*)(ws + WS_AMAX + (unit >> 4) * FS) + (unit & 15), ((u64)call << 32) | __builtin_bit_cast(unsigned, wam), __ATOMIC_RELAXED, __HIP_MEMORY_SCOPE_AGENT);
    }
  }
  STAMP(3);
  const int row0 = wm * 256 + kh * 128 + i16, col0 = (q < 2 ? ln0 : ln1) * 32 + (q & 1) * 16;
  const int srank = (rank - pd) & 7;

  if (early) {
#pragma unroll
    for (int j = SEND_WIN; j < 16; ++j) {
      if (wide > 0) asm volatile("s_waitcnt vmcnt(%0)" :: "n"(2 * SEND_WIN - 1) : "memory");
      else asm volatile("s_waitcnt vmcnt(%0)" :: "n"(SEND_WIN - 1) : "memory");
      send_piece(y, j, srs, pov);
    }
    asm volatile("s_waitcnt vmcnt(0)" ::: "memory");
    if (lane == 0)
      __hip_atomic_store((GAS unsigned*)(dws + WS_EFLAG + (size_t)(unit * 4 + wave) * FS), call, __ATOMIC_RELAXED, __HIP_MEMORY_SCOPE_SYSTEM);
    STAMP(5);
    ok &= wait_flag<__HIP_MEMORY_SCOPE_SYSTEM, 8>((const unsigned*)(ws + WS_EFLAG + (size_t)(unit * 4 + wave) * FS), call, tmo);
    u32x4 st[16];
    {
      const i32x4 rs = make_rsrc(ws + WS_STAGE + wave_off, WAVE_B);
#pragma unroll
      for (int j = 0; j < 16; ++j)
        asm volatile("buffer_load_dwordx4 %0, %1, %2, %3 offen offset:%4 sc0 sc1" : "=v"(st[j]) : "v"(pov), "s"(rs), "s"(j * PC & ~4095), "i"(j * PC & 4095) : "memory");
      asm volatile("s_waitcnt vmcnt(0)" ::: "memory");
#pragma unroll
      for (int j = 0; j < 16; ++j) asm volatile("" : "+v"(st[j]));
    }
    STAMP(6);
    const Quant qz = global_scale<ESLEEP>((const u64*)(ws + gmax_off), call, tmo, ok, lane);
    STAMP(4);
    GAS char* ob = dout + (size_t)(srank * ML + row0) * NL + col0;
#pragma unroll
    for (int m = 0; m < 8; ++m) {
      const u32x4 s0 = st[2 * m], s1 = st[2 * m + 1];
      u32x4 o;
#pragma unroll
      for (int r = 0; r < 4; ++r) {
        const int h = r >> 1;
        const unsigned w0 = s0[h], w1 = s0[2 + h], w2 = s1[h], w3 = s1[2 + h];
        const float v0 = __builtin_bit_cast(float, (r & 1) ? (w0 & 0xFFFF0000u) : (w0 << 16));
        const float v1 = __builtin_bit_cast(float, (r & 1) ? (w1 & 0xFFFF0000u) : (w1 << 16));
        const float v2 = __builtin_bit_cast(float, (r & 1) ? (w2 & 0xFFFF0000u) : (w2 << 16));
        const float v3 = __builtin_bit_cast(float, (r & 1) ? (w3 & 0xFFFF0000u) : (w3 << 16));
        o[r] = quant4(v0, v1, v2, v3, qz);
      }
      asm volatile("global_store_dwordx4 %0, %1, off sc0 sc1\n\ts_nop 1" :: "v"(ob + (size_t)m * 16 * NL), "v"(o) : "memory");
    }
    asm volatile(".Lq8_code_end1:");
    STAMP(7);
    STAMP_OUT();
    return;
  }

  if (unit == NWG - 1 && wave >= AGG0) {
    const float rm = sweep_max<4, 4 * FS / 8, __HIP_MEMORY_SCOPE_AGENT, 1>((const u64*)(ws + WS_AMAX + (lane >> 4) * FS) + (lane & 15), call, tmo);
    if (lane < NR)
      __hip_atomic_store((GAS u64*)(anyws + gmax_off) + rank, ((u64)call << 32) | __builtin_bit_cast(unsigned, rm), __ATOMIC_RELAXED, __HIP_MEMORY_SCOPE_SYSTEM);
  }
  const Quant qz = global_scale<LSLEEP>((const u64*)(ws + gmax_off), call, tmo, ok, lane);
  STAMP(4);
  {
    GAS char* ob = dout + (size_t)(rank * ML + row0) * NL + col0;
#pragma unroll
    for (int m = 0; m < 8; ++m) {
      u32x4 o;
#pragma unroll
      for (int r = 0; r < 4; ++r) o[r] = quant4(y[m][0][r], y[m][1][r], y[m][2][r], y[m][3][r], qz);
      asm volatile("s_waitcnt vmcnt(%2)\n\tglobal_store_dwordx4 %0, %1, off sc0 sc1\n\ts_nop 1" :: "v"(ob + (size_t)m * 16 * NL), "v"(o), "n"(LATE_WIN - 1) : "memory");
    }
  }
  asm volatile("s_waitcnt vmcnt(0)\n.Lq8_code_end2:" ::: "memory");
  STAMP(5);
  if (lane == 0)
    __hip_atomic_store((GAS unsigned*)(dws + WS_EFLAG + (size_t)(unit * 4 + wave) * FS), call, __ATOMIC_RELAXED, __HIP_MEMORY_SCOPE_SYSTEM);
  if (!wait_flag<__HIP_MEMORY_SCOPE_SYSTEM, 2>((const unsigned*)(ws + WS_EFLAG + (size_t)(unit * 4 + wave) * FS), call, tmo) && lane == 0)
    myout[(size_t)(srank * ML + row0) * NL + col0] = (char)-128;
  STAMP(7);
  STAMP_OUT();
}

static void make_pace(Params& p, const int* pa) {
  const int TE = pa[4], TS = pa[5], A = pa[7], DS = pa[8];
  p.slack = pa[9];
  p.nwide = pa[10]; p.wmask = pa[14];
  for (int k = 0; k < NE; ++k) {
    p.te[k] = k < NE - 3 ? TE : k == NE - 3 ? TS : pa[13];
    p.start[k] = A + (k * DS > pa[11] ? k * DS - pa[11] : 0) - KH * (p.te[k] - TE);
    if (p.start[k] < A) p.start[k] = A;
  }
  for (int k = 0; k < pa[10]; ++k) p.te[k] = pa[12];
  for (int busy = 1; busy >= 0; --busy) {
    const int* e = busy ? pa + 17 : pa;
    const double T0 = e[0], TH = e[1], T1 = e[2], TR = e[3], TC = pa[6], C = pa[15] * 0.01;
    double P = 0, last = 0;
    int v = 0, stall = 0;
    for (int u = A - 100; v <= KH + 1; ++u) {
      double r = NWG / (u < TH ? T0 : u < TR ? T0 + (T1 - T0) * (u - TH) / (TR - TH) : T1);
      for (int k = 0; k < NE; ++k) {
        const double s0 = p.start[k] - 100 + (k < pa[10] ? 0 : p.te[k] / 2);
        if (u >= s0 && u < s0 + KH * p.te[k]) r -= (k == NE - 1 ? 20.0 : 14.0) / p.te[k];
      }
      r /= NWG - 2 * NEARLY - 6;
      if (r > 1.0 / TC) r = 1.0 / TC;
      if (r < 0.001) r = 0.001;
      if (P >= 3.0 && stall < pa[16]) { ++stall; r = 0; }
      P += r;
      if (P >= v + C) {
        if (v) p.step[v - 1] = busy ? (int)(u - last) << 16 : p.step[v - 1] | (int)(u - last); else p.start[NE] = u;
        last = u; ++v;
      }
    }
  }
}

extern "C" void kernel_launch(int rank, int nranks, ncclComm_t comm,
                              hipStream_t stream,
                              void* const* d_in, const int* in_sizes, int n_in,
                              void* d_out, int out_size,
                              const int* in_dtypes, int out_dtype,
                              void* const* d_out_all,
                              void* const* d_ws_all, size_t ws_bytes) {
  (void)comm; (void)in_dtypes; (void)out_dtype; (void)d_out;
  if (nranks != NR || n_in != 2 || in_sizes[0] != ML * KD || in_sizes[1] != KD * ND || out_size != NR * ML * NL ||
      ws_bytes < WS_END || (unsigned)rank >= (unsigned)NR) {
    fprintf(stderr, "unexpected problem shape\n");
    return;
  }
  HIP_CHECK(hipFuncSetAttribute((const void*)fused_kernel, hipFuncAttributeMaxDynamicSharedMemorySize, LDS_BYTES));
  Params p;
  p.x = (const float*)d_in[0];
  p.w = (const float*)d_in[1];
  for (int r = 0; r < NR; ++r) { p.out[r] = (char*)d_out_all[r]; p.ws[r] = (char*)d_ws_all[r]; }
  p.rank = rank;
  p.myws = (char*)d_ws_all[rank];
  int pa[21] = {PACE_ARGS};
  make_pace(p, pa);
  fused_kernel<<<NWG, NTHR, LDS_BYTES, stream>>>(p);
  HIP_CHECK(hipGetLastError());
}
```

```cpp
#include <hip/hip_runtime.h>
#include <rccl/rccl.h>
#include <cstdio>
#include <cstdint>
#include <cstdlib>
#include <cstddef>

#define HIP_CHECK(expr) do { hipError_t _e = (expr); if (_e != hipSuccess) { \
    fprintf(stderr, "HIP error %s at %s:%d\n", hipGetErrorString(_e), __FILE__, __LINE__); \
    return; } } while (0)

#define LAS __attribute__((address_space(3)))
#define GAS __attribute__((address_space(1)))
typedef float f32x4 __attribute__((ext_vector_type(4)));
typedef float f32x2 __attribute__((ext_vector_type(2)));
typedef __bf16 bf16x2 __attribute__((ext_vector_type(2)));
typedef unsigned u32x4 __attribute__((ext_vector_type(4)));
typedef int i32x4 __attribute__((ext_vector_type(4)));
typedef int i32x2 __attribute__((ext_vector_type(2)));
typedef unsigned long long u64;

constexpr int NR = 8;
constexpr int KD = 4096, ND = 16384;
constexpr int ML = 512;
constexpr int NL = ND / NR;
constexpr int NWG = 256, NTHR = 256;
constexpr int NBLK = KD / 64;
constexpr int NT = ND / 128;
constexpr int KH = NBLK / 2;
constexpr int NE = 9;
constexpr int NEARLY = 7 * NE;
constexpr int PF0 = 4, PFL = 3;
constexpr int NH = 2;
constexpr int SEND_WIN = 3, LATE_WIN = 2;
#define PACE_ARGS 212, 2100, 157, 4300, 125, 150, 115, 600, 245, 100, 2, 100, 122, 175, 3, 100, 150, 228, 2300, 163, 4400
constexpr int LDS_X = 4 * 32768;
constexpr int LDS_BYTES = LDS_X + 32768;
constexpr int PC = 2048, FS = 256;
constexpr int WAVE_B = 16 * PC;
constexpr size_t HALF_B = 4 * WAVE_B;

constexpr size_t WS_TMO = 128;
constexpr size_t WS_GMAX = 512;
constexpr size_t WS_START = 1024;
constexpr size_t WS_AMAX = 4096;
constexpr size_t WS_XFLAG = WS_AMAX + NWG / 16 * FS;
constexpr size_t WS_EFLAG = WS_XFLAG + NWG * FS;
constexpr size_t WS_PFLAG = WS_EFLAG + NWG * 4 * FS;
constexpr size_t WS_STAMP = WS_PFLAG + NWG * 4 * FS;
constexpr size_t WS_XT = 1 << 20;
constexpr size_t XT_BYTES = (size_t)ML * KD * 2, W_BYTES = (size_t)KD * ND * 4;
constexpr size_t WS_PART = WS_XT + XT_BYTES;
constexpr size_t WS_STAGE = WS_PART + NWG * HALF_B;
constexpr size_t WS_END = WS_STAGE + 2 * NEARLY * HALF_B;
static_assert(WS_STAMP + 65536 + NWG * 512 <= WS_XT, "control block overlaps x");
constexpr unsigned BUSY_TICKS = 50000;
constexpr u64 DEADLINE_TICKS = 50000000ull;

struct Params {
  const float* x;
  const float* w;
  char* myws;
  int rank;
  int slack;
  char* out[NR];
  char* ws[NR];
  int nwide, wmask;
  int start[NE + 1], te[NE];
  int step[KH + 1];
};

#define STAMP(i) do {} while (0)
#define STAMPC(i) do {} while (0)
#define STAMP_DECL do {} while (0)
#define STAMP_OUT() do {} while (0)

__device__ __forceinline__ unsigned pk_bf16(float lo, float hi) {
  f32x2 v = {lo, hi};
  return __builtin_bit_cast(unsigned, __builtin_convertvector(v, bf16x2));
}
__device__ __forceinline__ float wave_max(float f) {
  int v = __builtin_bit_cast(int, f);
  v = max(v, __builtin_amdgcn_update_dpp(0, v, 0x111, 0xf, 0xf, true));
  v = max(v, __builtin_amdgcn_update_dpp(0, v, 0x112, 0xf, 0xf, true));
  v = max(v, __builtin_amdgcn_update_dpp(0, v, 0x114, 0xf, 0xf, true));
  v = max(v, __builtin_amdgcn_update_dpp(0, v, 0x118, 0xf, 0xf, true));
  const int r = max(max(__builtin_amdgcn_readlane(v, 15), __builtin_amdgcn_readlane(v, 31)), max(__builtin_amdgcn_readlane(v, 47), __builtin_amdgcn_readlane(v, 63)));
  return __builtin_bit_cast(float, r);
}
__device__ __forceinline__ i32x4 make_rsrc(const void* base, unsigned bytes) {
  const u64 a = (u64)base;
  return i32x4{__builtin_amdgcn_readfirstlane((int)(unsigned)a), __builtin_amdgcn_readfirstlane((int)(unsigned)(a >> 32)), (int)bytes, 0x00020000};
}

__device__ __forceinline__ void w_load(f32x4& d, unsigned voff, i32x4 rsrc, unsigned soff) {
  asm volatile("buffer_load_dwordx4 %0, %1, %2, %3 offen nt" : "=v"(d) : "v"(voff), "s"(rsrc), "s"(soff) : "memory");
}
__device__ __forceinline__ void set_m0(unsigned m0v) { asm volatile("s_mov_b32 m0, %0" :: "s"(m0v) : "memory"); }
template <int IMM>
__device__ __forceinline__ void x_load(unsigned voff, i32x4 rsrc, unsigned soff) {
  asm volatile("buffer_load_dwordx4 %0, %1, %2 offen offset:%3 sc1 lds" :: "v"(voff), "s"(rsrc), "s"(soff), "i"(IMM) : "memory");
}
template <int N>
__device__ __forceinline__ void wait_bar() { asm volatile("s_waitcnt vmcnt(%0) lgkmcnt(0)\n\ts_barrier" :: "n"(N) : "memory"); }
__device__ __forceinline__ void mfma(f32x4& c, u32x4 a, u32x4 b) {
  asm volatile("v_mfma_f32_16x16x32_bf16 %0, %1, %2, %0" : "+a"(c) : "v"(a), "v"(b));
}
#define SCHED() __builtin_amdgcn_sched_barrier(0)
__device__ __forceinline__ void send_piece(const f32x4 (&y)[8][4], int j, i32x4 srs, unsigned lane16) {
  const f32x4 lo = y[j >> 1][(j & 1) * 2], hi = y[j >> 1][(j & 1) * 2 + 1];
  const u32x4 d = {pk_bf16(lo[0], lo[1]), pk_bf16(lo[2], lo[3]), pk_bf16(hi[0], hi[1]), pk_bf16(hi[2], hi[3])};
  asm volatile("buffer_store_dwordx4 %0, %1, %2, %3 offen sc0 sc1\n\ts_nop 1" :: "v"(d), "v"(lane16), "s"(srs), "s"(j * PC) : "memory");
}
__device__ __forceinline__ void touch(const GAS char* t) {
  unsigned v;
  asm volatile("global_load_dword %0, %1, off nt\n\ts_waitcnt vmcnt(0)" : "=&v"(v) : "v"(t) : "memory");
}
constexpr float QBIAS = 12582912.0f;
struct Quant { float inv, lo, hi; };
__device__ __forceinline__ unsigned quant4(float v0, float v1, float v2, float v3, const Quant& qz) {
  const unsigned e0 = __builtin_bit_cast(unsigned, __builtin_amdgcn_fmed3f(__builtin_fmaf(v0, qz.inv, QBIAS), qz.lo, qz.hi));
  const unsigned e1 = __builtin_bit_cast(unsigned, __builtin_amdgcn_fmed3f(__builtin_fmaf(v1, qz.inv, QBIAS), qz.lo, qz.hi));
  const unsigned e2 = __builtin_bit_cast(unsigned, __builtin_amdgcn_fmed3f(__builtin_fmaf(v2, qz.inv, QBIAS), qz.lo, qz.hi));
  const unsigned e3 = __builtin_bit_cast(unsigned, __builtin_amdgcn_fmed3f(__builtin_fmaf(v3, qz.inv, QBIAS), qz.lo, qz.hi));
  return __builtin_amdgcn_perm(e1, e0, 0x0c0c0400u) | __builtin_amdgcn_perm(e3, e2, 0x04000c0cu);
}

template <int SCOPE, int SLEEP, int N = 1, int STRIDE = 0>
__device__ __forceinline__ bool wait_flag(const unsigned* flag, unsigned call, unsigned* tmo) {
  const u64 t0 = __builtin_amdgcn_s_memrealtime();
  for (unsigned polls = 1;; ++polls) {
    unsigned v[N];
    bool pend = false;
#pragma unroll
    for (int i = 0; i < N; ++i) v[i] = __hip_atomic_load(flag + i * STRIDE, __ATOMIC_RELAXED, SCOPE);
#pragma unroll
    for (int i = 0; i < N; ++i) pend |= v[i] < call;
    if (!__any(pend)) return true;
    if (((polls & 63u) == 0 && __hip_atomic_load(tmo, __ATOMIC_RELAXED, __HIP_MEMORY_SCOPE_AGENT) == call) ||
        __builtin_amdgcn_s_memrealtime() - t0 > DEADLINE_TICKS) {
      __hip_atomic_store(tmo, call, __ATOMIC_RELAXED, __HIP_MEMORY_SCOPE_AGENT);
      return false;
    }
    __builtin_amdgcn_s_sleep(SLEEP);
  }
}

struct Ctx {
  i32x4 wrsrc, xrsrc;
  unsigned wvoff[8], xvoff0, xvoff1, m0base;
  int wlim;
  int left;
  unsigned xso, wso;
  unsigned due, slack;
  int steps;
  i32x4 frsrc;
  const char* fbase;
  unsigned fvoff, fso, call, fl;
  unsigned* tmo;
  bool ok;
  LAS const char* lrd0;
  LAS const char* lrd1;
  LAS char* wmine;
  LAS const char* wother;
};

template <int SLOT>
__device__ __forceinline__ u32x4 x_frag(const Ctx& c, int mt) {
  return *(LAS const u32x4*)((SLOT < 2 ? c.lrd0 : c.lrd1) + (SLOT & 1) * 32768 + mt * 1024);
}
template <int G>
__device__ __forceinline__ void x_piece(const Ctx& c, i32x4 xr, unsigned xso) {
  x_load<(G & 3) * 1024>((G & 4) ? c.xvoff1 : c.xvoff0, xr, (G & 8) ? xso + 32768 : xso);
}
template <int SL, bool CHK, bool HELD = false>
__device__ __forceinline__ void kstep(Ctx& c, f32x4 (&acc)[16][4], f32x4 (&wr)[2][8], u32x4 (&a)[2][4], u32x4 (&xf)[8], const u32x4* held = nullptr) {
  constexpr int E = SL & 1, WS = SL >> 1, NX = (SL + 1) & 3;
  u64 now;
  unsigned lateby;
  const i32x4 xr = c.xrsrc, wrs = c.wrsrc;
  const unsigned xso = c.xso, wso = c.wso;
  const unsigned m0x = c.m0base + ((SL + 2) & 3) * 32768;
  f32x4 (&r)[8] = wr[WS ^ 1];
#pragma unroll
  for (int g = 0; g < 16; ++g) {
    if (E == 1 && g == 12) {
      if (CHK) {
        asm volatile("s_waitcnt vmcnt(8)" : "+v"(c.fl) :: "memory");
        if (__any(c.fl < c.call)) {
          c.ok &= wait_flag<__HIP_MEMORY_SCOPE_AGENT, 1>((const unsigned*)(c.fbase + c.fso + c.fvoff), c.call, c.tmo);
          __builtin_amdgcn_s_waitcnt(0xC07F);
        }
      }
      wait_bar<HELD ? 0 : 8>();
      SCHED();
    }
    mfma(acc[g][0], a[E][0], xf[g & 7]);
    SCHED();
    if (g < 9) xf[(g + 7) & 7] = x_frag<SL>(c, g + 7);
    else if (E == 0) xf[(g + 7) & 7] = x_frag<NX>(c, g - 9);
    else if (g >= 12) xf[(2 * (g - 12)) & 7] = x_frag<NX>(c, 2 * (g - 12));
    if (CHK && E == 0 && g == 0) asm volatile("buffer_load_dword %0, %1, %2, %3 offen sc1" : "=v"(c.fl) : "v"(c.fvoff), "s"(c.frsrc), "s"(c.fso) : "memory");
    if (E == 1 && g == 9) *(LAS u32x4*)(c.wmine + (WS ^ 1) * 16384 + 1024) = a[0][1];
    if (E == 1 && g == 10) *(LAS u32x4*)(c.wmine + (WS ^ 1) * 16384 + 3072) = a[0][3];
    if (E == 1 && g == 11) {
      asm volatile("s_waitcnt lgkmcnt(0)" : "+s"(now) :: "memory");
      lateby = (unsigned)now - c.due;
      asm volatile("" : "+s"(lateby));
    }
    SCHED();
    mfma(acc[g][1], a[E][1], xf[g & 7]);
    SCHED();
    if (E == 0) {
      switch (g) {
        case 0: x_piece<0>(c, xr, xso); break;   case 1: x_piece<1>(c, xr, xso); break;
        case 2: x_piece<2>(c, xr, xso); break;   case 3: x_piece<3>(c, xr, xso); break;
        case 4: x_piece<4>(c, xr, xso); break;   case 5: x_piece<5>(c, xr, xso); break;
        case 6: x_piece<6>(c, xr, xso); break;   case 7: x_piece<7>(c, xr, xso); break;
        case 8: x_piece<8>(c, xr, xso); break;   case 9: x_piece<9>(c, xr, xso); break;
        case 10: x_piece<10>(c, xr, xso); break; case 11: x_piece<11>(c, xr, xso); break;
        case 12: x_piece<12>(c, xr, xso); break; case 13: x_piece<13>(c, xr, xso); break;
        case 14: x_piece<14>(c, xr, xso); break; default: x_piece<15>(c, xr, xso); break;
      }
    } else {
      if (!HELD && g < 8) w_load(wr[WS][g & 7], c.wvoff[g & 7], wrs, wso);
      if (HELD && g >= 4 && g < 8) { a[0][g & 3] = held[g & 3]; asm volatile("" : "+v"(a[0][g & 3])); }
      if (!HELD && g == 5)
        asm volatile("s_waitcnt vmcnt(%8)" : "+v"(r[0]), "+v"(r[1]), "+v"(r[2]), "+v"(r[3]), "+v"(r[4]), "+v"(r[5]), "+v"(r[6]), "+v"(r[7]) : "n"(22 + CHK) : "memory");
      if (g == 8) *(LAS u32x4*)(c.wmine + (WS ^ 1) * 16384) = a[0][0];
      if (g == 9) { c.left -= 1; c.xso += 65536u; asm volatile("" : "+s"(c.left), "+s"(c.xso)); }
      if (g == 10) { c.xrsrc[2] = c.left > 1 ? (int)XT_BYTES : 0; asm volatile("" : "+s"(c.xrsrc)); }
      if (g == 11) { c.wrsrc[2] = c.left > c.wlim ? (int)W_BYTES : 0; asm volatile("" : "+s"(c.wrsrc)); }
      if (g == 12) { lateby = (int)lateby > (int)c.slack ? lateby - c.slack : 0u; asm volatile("" : "+s"(lateby)); }
      if (g == 13) { c.due += lateby; lateby = __builtin_amdgcn_readlane(c.steps, c.left); asm volatile("" : "+s"(c.due), "+s"(lateby)); }
      if (g == 14) { c.due += lateby; asm volatile("" : "+s"(c.due)); }
    }
    SCHED();
    mfma(acc[g][2], a[E][2], xf[g & 7]);
    SCHED();
    if (E == 0 && g < 4) a[1][g] = *(LAS const u32x4*)(c.wother + WS * 16384 + g * 1024);
    if (!HELD && E == 1 && g >= 5 && g < 9) { const int t = (g - 5) & 3; a[0][t][0] = pk_bf16(r[0][t], r[1][t]); a[0][t][1] = pk_bf16(r[2][t], r[3][t]); }
    if (E == 1 && g == 9) *(LAS u32x4*)(c.wmine + (WS ^ 1) * 16384 + 2048) = a[0][2];
    if (E == 1 && g == 11) {
      if (__builtin_expect((int)lateby < 0, 0))
        do __builtin_amdgcn_s_sleep(1); while ((int)((unsigned)__builtin_amdgcn_s_memrealtime() - c.due) < 0);
    }
    if (E == 1 && g >= 12 && g < 15) xf[(2 * (g - 12) + 1) & 7] = x_frag<NX>(c, 2 * (g - 12) + 1);
    SCHED();
    mfma(acc[g][3], a[E][3], xf[g & 7]);
    SCHED();
    if (!HELD && E == 1 && g >= 5 && g < 9) { const int t = (g - 5) & 3; a[0][t][2] = pk_bf16(r[4][t], r[5][t]); a[0][t][3] = pk_bf16(r[6][t], r[7][t]); }
    if (E == 0 && (g & 3) == 3 && g < 15) set_m0(m0x + ((g + 1) >> 3) * 32768 + (((g + 1) >> 2) & 1) * 4096);
    if (E == 1 && g == 9) asm volatile("s_memrealtime %0" : "=s"(now) :: "memory");
    if (E == 1 && g == 10) { c.wso += 1u << 22; asm volatile("" : "+s"(c.wso)); }
    if (E == 1 && g == 15) set_m0(c.m0base + ((SL + 3) & 3) * 32768);
    SCHED();
  }
}

__device__ __forceinline__ void last_block(const Ctx& c, f32x4 (&acc)[16][4], u32x4 (&a)[2][4], u32x4 (&xf)[8], i32x4 prs, unsigned pov) {
  f32x4 lo, hi;
#pragma unroll
  for (int i = 0; i < 32; ++i) {
    const int e = (i >> 3) & 1, m = (i & 7) + (i >> 4) * 8;
    const int i7 = i + 7, e7 = (i7 >> 3) & 1, m7 = (i7 & 7) + (i7 >> 4) * 8;
    const int j = i - 16;
    mfma(acc[m][0], a[e][0], xf[i & 7]);
    SCHED();
    if (i7 < 32) xf[i7 & 7] = e7 ? x_frag<3>(c, m7) : x_frag<2>(c, m7);
    if (j >= 0) {
      asm volatile("" : "+a"(acc[j >> 1][(j & 1) * 2]));
      lo = acc[j >> 1][(j & 1) * 2];
      asm volatile("" : "+v"(lo));
    }
    SCHED();
    mfma(acc[m][1], a[e][1], xf[i & 7]);
    SCHED();
    if (i < 4) a[1][i] = *(LAS const u32x4*)(c.wother + 16384 + i * 1024);
    if (j >= 0) {
      asm volatile("" : "+a"(acc[j >> 1][(j & 1) * 2 + 1]));
      hi = acc[j >> 1][(j & 1) * 2 + 1];
      asm volatile("" : "+v"(hi));
    }
    SCHED();
    mfma(acc[m][2], a[e][2], xf[i & 7]);
    SCHED();
    u32x4 d;
    if (j >= 0) { d = u32x4{pk_bf16(lo[0], lo[1]), pk_bf16(lo[2], lo[3]), pk_bf16(hi[0], hi[1]), pk_bf16(hi[2], hi[3])}; asm volatile("" : "+v"(d)); }
    SCHED();
    mfma(acc[m][3], a[e][3], xf[i & 7]);
    SCHED();
    if (j >= 0)
      asm volatile("buffer_store_dwordx4 %0, %1, %2, %3 offen offset:%4 sc1\n\ts_nop 1"
                   :: "v"(d), "v"(pov), "s"(prs), "s"(j * PC & ~4095), "i"(j * PC & 4095) : "memory");
    SCHED();
  }
}

template <int N, int STRIDE, int SCOPE, int SLEEP>
__device__ __forceinline__ float sweep_max(const u64* slots, unsigned call, unsigned* tmo) {
  const u64 t0 = __builtin_amdgcn_s_memrealtime();
  for (unsigned polls = 1;; ++polls) {
    bool all = true, bad = false;
    float mx = 0.f;
    u64 v[N];
#pragma unroll
    for (int r = 0; r < N; ++r) v[r] = __hip_atomic_load(slots + r * STRIDE, __ATOMIC_RELAXED, SCOPE);
#pragma unroll
    for (int r = 0; r < N; ++r) {
      const float f = __builtin_bit_cast(float, (unsigned)v[r]);
      all &= (unsigned)(v[r] >> 32) == call;
      bad |= f < 0.f;
      mx = fmaxf(mx, f);
    }
    if (__all(all)) {
      return __any(bad) ? -1.0f : wave_max(mx);
    }
    if (((polls & 63u) == 0 && __hip_atomic_load(tmo, __ATOMIC_RELAXED, __HIP_MEMORY_SCOPE_AGENT) == call) ||
        __builtin_amdgcn_s_memrealtime() - t0 > DEADLINE_TICKS) {
      __hip_atomic_store(tmo, call, __ATOMIC_RELAXED, __HIP_MEMORY_SCOPE_AGENT);
      return -1.0f;
    }
    __builtin_amdgcn_s_sleep(SLEEP);
  }
}
template <int SLEEP>
__device__ __forceinline__ Quant global_scale(const u64* gmax, unsigned call, unsigned* tmo, bool ok, int lane) {
  const float gam = ok ? sweep_max<1, 0, __HIP_MEMORY_SCOPE_SYSTEM, SLEEP>(gmax + (lane & 7), call, tmo) : -1.0f;
  Quant qz;
  qz.inv = gam > 0.f ? 127.0f / gam : 0.f;
  qz.lo = gam >= 0.f ? QBIAS - 127.0f : QBIAS - 128.0f;
  qz.hi = gam >= 0.f ? QBIAS + 127.0f : QBIAS - 128.0f;
  asm volatile("" : "+v"(qz.lo), "+v"(qz.hi));
  return qz;
}

__device__ __forceinline__ int line_of(int ct, int s) {
  return ct < 7 ? 8 * ct + 4 + s : ct < 11 ? 16 * (ct - 7) + (s >> 1) * 8 + (s & 1) : ct == 11 ? 60 + s : (ct & 1) * 32 + 8 * s + (ct >> 1) - 4;
}
__device__ __forceinline__ void tile_of(int j, int& pd, int& ct) {
  const int k = j / 7;
  pd = j < 112 ? j % 7 + 1 : 0;
  ct = j >= 112 ? j - 112 : k < 2 ? 12 + k : k < NE - 2 ? 14 - NE + k : k < NE ? 16 - NE + k : k - NE;
}

__global__ void __launch_bounds__(NTHR, 1) fused_kernel(Params p) {
  extern __shared__ __attribute__((aligned(1024))) char smem[];
  const int tid = threadIdx.x, lane = tid & 63;
  const int wave = __builtin_amdgcn_readfirstlane(tid >> 6);
  char* ws = p.myws;

  STAMP_DECL;
  STAMP(0);
  STAMPC(8);
  const int b = blockIdx.x, xcd = b & 7, idx = b >> 3;
  const int kh = xcd & 1, slot = idx * 4 + (xcd >> 1), unit = slot * 2 + kh;
  const int wn = wave & 1, wm = wave >> 1;
  const int q = lane >> 4, i16 = lane & 15;
  const unsigned lane16 = lane * 16;
  const unsigned t_start = (unsigned)__builtin_amdgcn_s_memrealtime();

  Ctx c;
  c.xrsrc = make_rsrc(ws + WS_XT, (unsigned)XT_BYTES);
  unsigned* xflag = (unsigned*)(ws + WS_XFLAG);
  const int hs = kh * NT + slot;
  const unsigned xto = (unsigned)(hs * 16384 + wave * 4096);
  f32x4 raw[8];
  i32x2 prev;
  unsigned warm;
  {
    const int st = hs >> 1, h = hs & 1;
    const i32x4 xin = make_rsrc(p.x, (unsigned)((size_t)ML * KD * 4));
#pragma unroll
    for (int j = 0; j < 4; ++j) {
      const unsigned o = (unsigned)((h * 16 + ((wave * 4 + j) ^ ((kh ^ 1) << 3))) * 16 + i16) * (KD * 4) + ((st ^ h) * 32 + q * 8) * 4;
      asm volatile("buffer_load_dwordx4 %0, %2, %3, 0 offen\n\tbuffer_load_dwordx4 %1, %2, %3, 0 offen offset:16"
                   : "=&v"(raw[2 * j]), "=&v"(raw[2 * j + 1]) : "v"(o), "s"(xin) : "memory");
    }
    asm volatile("global_load_dwordx2 %0, %1, off sc1" : "=v"(prev) : "v"(xflag + hs * (FS / 4)) : "memory");
    asm volatile("buffer_load_dword %0, %1, %2, %3 offen sc1" : "=v"(warm) : "v"(0), "s"(c.xrsrc), "s"(xto) : "memory");
  }

  unsigned cpf = 0;
  {
    u64 pc; unsigned len1, len2;
    asm volatile("s_getpc_b64 %0\n.Lq8_code_begin:\n\ts_mov_b32 %1, .Lq8_code_end1-.Lq8_code_begin\n\ts_mov_b32 %2, .Lq8_code_end2-.Lq8_code_begin" : "=s"(pc), "=s"(len1), "=s"(len2));
    const int wv = (idx - 8) * 4 + wave;
    const i32x4 cr = (unsigned)wv < 8u ? make_rsrc((const void*)pc, len1 > len2 ? len1 : len2)
                                       : make_rsrc(p.w, (unsigned)(wv - 8) < 2u ? (unsigned)W_BYTES : 0);
    asm volatile("buffer_load_dword %0, %1, %2, 0 offen" : "+v"(cpf)
                 : "v"((unsigned)wv < 8u ? (unsigned)(wv * 64 + lane) * 128u : (unsigned)(kh * 128 + (wv & 1) * 64 + lane) << 20), "s"(cr) : "memory");
  }
  const bool early = slot < NEARLY;
  int pd, ct;
  tile_of(slot, pd, ct);
  const int rank = p.rank;
  const unsigned dest = (unsigned)(rank + pd) & 7u;
  const int k0 = kh * KH;
  const int ln0 = line_of(ct, 2 * wn), ln1 = line_of(ct, 2 * wn + 1);
  unsigned* tmo = (unsigned*)(ws + WS_TMO);
  GAS char* dws = (GAS char*)p.ws[dest];
  GAS char* dout = (GAS char*)p.out[early ? rank : dest];
  GAS char* myout = (GAS char*)p.out[rank];
  asm volatile("" : "+s"(dws), "+s"(dout), "+s"(myout));
  GAS char* anyws;
  i32x2 stp;
  asm volatile("global_load_dwordx2 %0, %2, %4 offset:%5\n\tglobal_load_dwordx2 %1, %3, %4 offset:%6"
               : "=&v"(anyws), "=&v"(stp) : "v"((lane & 7) * 8), "v"(((KH - 1 - lane) & 31) * 4), "s"(__builtin_amdgcn_kernarg_segment_ptr()),
                 "i"(offsetof(Params, ws)), "i"(offsetof(Params, step)) : "memory");

  c.wrsrc = make_rsrc(p.w, (unsigned)W_BYTES);
#pragma unroll
  for (int j = 0; j < 8; ++j) {
    c.wvoff[j] = ((unsigned)(wm * 32 + q * 8 + j) << 16) + (i16 < 8 ? ln0 : ln1) * 128 + (i16 & 7) * 16;
    asm volatile("" : "+v"(c.wvoff[j]));
  }
  c.xvoff0 = wave * 8192 + lane16;
  c.xvoff1 = c.xvoff0 + 4096;
  asm volatile("" : "+v"(c.xvoff1));
  c.m0base = (unsigned)(size_t)(LAS char*)smem + wave * 8192;
  constexpr int nblk = KH;
  const unsigned wsb = ((unsigned)k0 << 22) + dest * (NL * 4u);
  c.left = nblk;
  c.wlim = 2;
  c.xso = (unsigned)(k0 + 1) * 65536u;
  c.wso = wsb + (2u << 22);
  const bool steady = early || ct == 11 || ct >= 14;
  const int cls = early ? slot / 7 : steady ? NE - 1 : NE, te = p.te[steady ? cls : 0], phase = cls < p.nwide ? 0 : (slot * 74 + kh * 8) & 255;
  c.due = t_start + p.start[cls] + phase * (steady ? te : p.step[0] & 0xFFFF) / 256;
  c.slack = p.slack;
  int wide = (p.wmask >> cls) & 1;
  asm volatile("" : "+s"(wide));
  c.lrd0 = (LAS const char*)smem + wm * 16384 + lane16;
  c.lrd1 = c.lrd0 + 65536;
  asm volatile("" : "+v"(c.lrd1));
  c.wmine = (LAS char*)smem + LDS_X + wave * 4096 + lane16;
  c.wother = (LAS const char*)smem + LDS_X + (wave ^ 2) * 4096 + lane16;

  f32x4 acc[16][4];
  f32x4 wr[2][8];
  u32x4 a[2][4], xf[8];
#pragma unroll
  for (int m = 0; m < 16; ++m)
#pragma unroll
    for (int t = 0; t < 4; ++t) {
      acc[m][t] = f32x4{0.f, 0.f, 0.f, 0.f};
      asm volatile("" : "+a"(acc[m][t]));
    }
  asm volatile("s_waitcnt vmcnt(3)" : "+v"(prev), "+v"(warm), "+v"(raw[0]), "+v"(raw[1]), "+v"(raw[2]), "+v"(raw[3]), "+v"(raw[4]), "+v"(raw[5]), "+v"(raw[6]), "+v"(raw[7]) :: "memory");
  STAMP(11);
#pragma unroll
  for (int j = 0; j < 4; ++j) {
    const f32x4 v0 = raw[2 * j], v1 = raw[2 * j + 1];
    const u32x4 d = {pk_bf16(v0[0], v0[1]), pk_bf16(v0[2], v0[3]), pk_bf16(v1[0], v1[1]), pk_bf16(v1[2], v1[3])};
    asm volatile("buffer_store_dwordx4 %0, %1, %2, %3 offen offset:%4 sc1\n\ts_nop 1" :: "v"(d), "v"(lane16), "s"(c.xrsrc), "s"(xto), "i"(j * 1024) : "memory");
  }
  const unsigned call = __builtin_amdgcn_readfirstlane(prev[0]) + 1;
  const int half = (call > 1 && t_start - (unsigned)__builtin_amdgcn_readfirstlane(prev[1]) < BUSY_TICKS) ? 16 : 0;
  asm volatile("s_waitcnt vmcnt(0)\n\ts_barrier" : "+v"(anyws), "+v"(stp), "+v"(cpf) :: "memory");
  {
    const int s0 = (stp[0] >> half) & 0xFFFF, s1 = (stp[1] >> half) & 0xFFFF;
    c.steps = steady ? te : s0 + phase * (s1 - s0) / 256;
  }
  STAMP(12);
  if (tid == 0) __hip_atomic_store((u64*)(xflag + hs * (FS / 4)), ((u64)t_start << 32) | call, __ATOMIC_RELAXED, __HIP_MEMORY_SCOPE_AGENT);
  if (b == 0 && wave == 1 && lane < NR)
    __hip_atomic_store((GAS unsigned*)(anyws + WS_START + rank * FS), call, __ATOMIC_RELAXED, __HIP_MEMORY_SCOPE_SYSTEM);
  c.fbase = (const char*)xflag + kh * NT * FS;
  c.frsrc = make_rsrc(c.fbase, NT * FS);
  c.fvoff = (lane & 3) * FS;
  c.fso = 8 * FS;
  c.call = call;
  c.tmo = tmo;
  c.ok = true;
  if ((unsigned)(idx - PF0) < (1u << PFL)) {
    constexpr int NP = 16 >> PFL;
    const int pw = (idx - PF0) * 4 + wave;
    const unsigned pv = (unsigned)k0 * 65536u + pw * NP * 1024 + lane16;
    u32x4 pf = {0u, 0u, 0u, 0u};
#pragma unroll
    for (int j = 1; j < KH; ++j) {
      if (j == 1 || j == 5)
        c.ok &= wait_flag<__HIP_MEMORY_SCOPE_AGENT, 1>((const unsigned*)(c.fbase + ((j == 1 ? (lane & 3) + 1 : lane & 31) * 4 + (pw >> PFL)) * FS), call, tmo);
      const unsigned v = pv + j * 65536;
      asm volatile("s_waitcnt vmcnt(32)\n\tbuffer_load_dwordx4 %0, %1, %2, 0 offen sc1\n\tbuffer_load_dwordx4 %0, %1, %2, 0 offen offset:1024 sc1" : "+v"(pf) : "v"(v), "s"(c.xrsrc) : "memory");
      if (NP == 4) asm volatile("buffer_load_dwordx4 %0, %1, %2, 0 offen offset:2048 sc1\n\tbuffer_load_dwordx4 %0, %1, %2, 0 offen offset:3072 sc1" : "+v"(pf) : "v"(v), "s"(c.xrsrc) : "memory");
    }
    asm volatile("s_waitcnt vmcnt(0)" : "+v"(pf) :: "memory");
    if (cls >= 3) {
      const int u1 = unit + ((lane & 3) == 1 ? -56 : (lane & 3) == 2 ? 56 : 0), u2 = (unsigned)u1 < 2u * NEARLY ? u1 : unit;
      const size_t o = (size_t)u2 * HALF_B + wave * WAVE_B;
      const int k = (lane >> 2) & 7;
      touch(k == 0 ? dws + WS_STAGE + o : k == 1 ? (GAS char*)ws + WS_PART + o : k == 2 ? (GAS char*)ws + WS_PART + (o ^ HALF_B)
            : k == 3 ? (GAS char*)ws + WS_STAGE + o : k == 4 ? dws + WS_EFLAG + (size_t)u2 * 4 * FS + 128
            : myout + (size_t)(((rank - pd) & 7) * ML + wm * 256 + kh * 128 + (lane & 3) * 32) * NL);
    }
  }
#pragma unroll
  for (int j = 0; j < 8; ++j) w_load(wr[0][j], c.wvoff[j], c.wrsrc, wsb);
  if (wave == 0) c.ok &= wait_flag<__HIP_MEMORY_SCOPE_AGENT, 1>((const unsigned*)(c.fbase + (lane & 7) * FS), call, tmo);
  asm volatile("s_waitcnt vmcnt(0)" : "+v"(wr[0][0]), "+v"(wr[0][1]), "+v"(wr[0][2]), "+v"(wr[0][3]), "+v"(wr[0][4]), "+v"(wr[0][5]), "+v"(wr[0][6]), "+v"(wr[0][7]) :: "memory");
#pragma unroll
  for (int t = 0; t < 4; ++t) {
    a[0][t] = u32x4{pk_bf16(wr[0][0][t], wr[0][1][t]), pk_bf16(wr[0][2][t], wr[0][3][t]), pk_bf16(wr[0][4][t], wr[0][5][t]), pk_bf16(wr[0][6][t], wr[0][7][t])};
    *(LAS u32x4*)(c.wmine + t * 1024) = a[0][t];
  }
  wait_bar<0>();
  {
    const unsigned xso = (unsigned)k0 * 65536u;
    set_m0(c.m0base); asm volatile("s_nop 0");
    x_piece<0>(c, c.xrsrc, xso); x_piece<1>(c, c.xrsrc, xso); x_piece<2>(c, c.xrsrc, xso); x_piece<3>(c, c.xrsrc, xso);
    set_m0(c.m0base + 4096); asm volatile("s_nop 0");
    x_piece<4>(c, c.xrsrc, xso); x_piece<5>(c, c.xrsrc, xso); x_piece<6>(c, c.xrsrc, xso); x_piece<7>(c, c.xrsrc, xso);
    set_m0(c.m0base + 32768); asm volatile("s_nop 0");
    x_piece<8>(c, c.xrsrc, xso); x_piece<9>(c, c.xrsrc, xso); x_piece<10>(c, c.xrsrc, xso); x_piece<11>(c, c.xrsrc, xso);
    set_m0(c.m0base + 32768 + 4096); asm volatile("s_nop 0");
    x_piece<12>(c, c.xrsrc, xso); x_piece<13>(c, c.xrsrc, xso); x_piece<14>(c, c.xrsrc, xso); x_piece<15>(c, c.xrsrc, xso);
  }
#pragma unroll
  for (int j = 0; j < 8; ++j) w_load(wr[1][j], c.wvoff[j], c.wrsrc, wsb + (1u << 22));
  __builtin_amdgcn_s_waitcnt(0xC07F);
  wait_bar<8>();
  STAMP(10);
  SCHED();
  set_m0(c.m0base + 2 * 32768);
#pragma unroll
  for (int t = 0; t < 7; ++t) xf[t] = x_frag<0>(c, t);
  SCHED();
  while ((int)((unsigned)__builtin_amdgcn_s_memrealtime() + 100u - c.due) < 0) __builtin_amdgcn_s_sleep(4);
  __builtin_amdgcn_s_waitcnt(0xC07F);
  STAMP(1);

#define TRACE_PT()
  TRACE_PT()
  kstep<0, true>(c, acc, wr, a, xf);
  kstep<1, true>(c, acc, wr, a, xf);
  c.fvoff = (lane & 31) * FS;
  c.fso = wave * 32 * FS;
  kstep<2, true>(c, acc, wr, a, xf);
  kstep<3, true>(c, acc, wr, a, xf);
  kstep<0, false>(c, acc, wr, a, xf);
  kstep<1, false>(c, acc, wr, a, xf);
  u32x4 held[NH][4] = {};
  const int ltail = steady ? 1 : NH + 1;
  if (!steady) {
    c.wlim = 2 + NH;
    f32x4 raw2[2][8];
#pragma unroll
    for (int i = 0; i <= NH; ++i) {
      if (i < NH) {
#pragma unroll
        for (int j = 0; j < 8; ++j) w_load(raw2[i & 1][j], c.wvoff[j], c.wrsrc, wsb + ((unsigned)(nblk - NH + i) << 22));
      }
      if (i > 0) {
        f32x4 (&r)[8] = raw2[(i - 1) & 1];
        asm volatile("s_waitcnt vmcnt(%8)" : "+v"(r[0]), "+v"(r[1]), "+v"(r[2]), "+v"(r[3]), "+v"(r[4]), "+v"(r[5]), "+v"(r[6]), "+v"(r[7]) : "n"(i < NH ? 8 : 0) : "memory");
#pragma unroll
        for (int t = 0; t < 4; ++t) {
          held[i - 1][t] = u32x4{pk_bf16(r[0][t], r[1][t]), pk_bf16(r[2][t], r[3][t]), pk_bf16(r[4][t], r[5][t]), pk_bf16(r[6][t], r[7][t])};
          asm volatile("" : "+v"(held[i - 1][t]));
        }
      }
    }
    SCHED();
  }
  do {
    TRACE_PT()
    kstep<2, false>(c, acc, wr, a, xf);
    kstep<3, false>(c, acc, wr, a, xf);
    kstep<0, false>(c, acc, wr, a, xf);
    kstep<1, false>(c, acc, wr, a, xf);
  } while (c.left > ltail);
  while (c.left > 1) {
    TRACE_PT()
    kstep<2, false, true>(c, acc, wr, a, xf, held[0]);
    kstep<3, false, true>(c, acc, wr, a, xf, held[0]);
    kstep<0, false, true>(c, acc, wr, a, xf, held[1]);
    kstep<1, false, true>(c, acc, wr, a, xf, held[1]);
#pragma unroll
    for (int i = 0; i + 2 < NH; ++i)
#pragma unroll
      for (int t = 0; t < 4; ++t) held[i][t] = held[i + 2][t];
    asm volatile("s_nop 1");
  }
  LAS float* red = (LAS float*)(smem + LDS_X);
  const size_t gmax_off = WS_GMAX + (size_t)(call & 1) * FS;
  const size_t wave_off = (size_t)unit * HALF_B + wave * WAVE_B;
  const unsigned pov = (lane >> 3) == 3 ? lane16 + 5 * 128 : lane16;
  unsigned* pflag = (unsigned*)(ws + WS_PFLAG + wave * FS);
  last_block(c, acc, a, xf, make_rsrc(ws + WS_PART + wave_off, WAVE_B), pov);
  bool ok = c.ok;
  STAMP(2);
  STAMPC(9);
  asm volatile("s_waitcnt vmcnt(0)"
               : "+v"(wr[0][0]), "+v"(wr[0][1]), "+v"(wr[0][2]), "+v"(wr[0][3]), "+v"(wr[0][4]), "+v"(wr[0][5]), "+v"(wr[0][6]), "+v"(wr[0][7])
               :: "memory");
  if (lane == 0) __hip_atomic_store(pflag + unit * FS, call, __ATOMIC_RELAXED, __HIP_MEMORY_SCOPE_AGENT);
  ok &= wait_flag<__HIP_MEMORY_SCOPE_SYSTEM, 2>((lane & 1) ? (const unsigned*)(ws + WS_START + dest * FS) : pflag + (unit ^ 1) * FS, call, tmo);
  const i32x4 srs = make_rsrc((const void*)(dws + WS_STAGE + wave_off), ok ? WAVE_B : 0);
  f32x4 y[8][4];
  {
    const i32x4 prs = make_rsrc(ws + WS_PART + (wave_off ^ HALF_B), WAVE_B);
    u32x4 pv[16];
#pragma unroll
    for (int j = 0; j < 16; ++j)
      asm volatile("buffer_load_dwordx4 %0, %1, %2, %3 offen offset:%4 sc1" : "=v"(pv[j]) : "v"(pov), "s"(prs), "s"(j * PC & ~4095), "i"(j * PC & 4095) : "memory");
    const unsigned one = (i16 >> 2) == q ? ((i16 & 1) ? 0x3F800000u : 0x00003F80u) : 0u;
    const unsigned e0 = (i16 & 2) ? 0u : one, e1 = (i16 & 2) ? one : 0u;
    u32x4 sel0 = {e0, e1, 0u, 0u}, sel1 = {0u, 0u, e0, e1};
    asm volatile("s_nop 1" : "+v"(sel0), "+v"(sel1));
    float am = 0.f;
#pragma unroll
    for (int j = 0; j < 16 + 4; ++j) {
      if (j < 16) {
        asm volatile("s_waitcnt vmcnt(%1)" : "+v"(pv[j]) : "n"(15 - j) : "memory");
        mfma(acc[8 + (j >> 1)][(j & 1) * 2], sel0, pv[j]);
        mfma(acc[8 + (j >> 1)][(j & 1) * 2 + 1], sel1, pv[j]);
      }
      if (j == 16) asm volatile("s_nop 7\n\ts_nop 7" ::: "memory");
      if (j >= 4) {
        const int m = (j - 4) >> 1, u = (j - 4) & 1;
        asm volatile("" : "+a"(acc[8 + m][2 * u]), "+a"(acc[8 + m][2 * u + 1]), "+v"(am));
        y[m][2 * u] = acc[8 + m][2 * u]; y[m][2 * u + 1] = acc[8 + m][2 * u + 1];
        asm volatile("" : "+v"(y[m][2 * u]), "+v"(y[m][2 * u + 1]));
        const f32x4 lo = y[m][2 * u], hi = y[m][2 * u + 1];
        am = fmaxf(fmaxf(am, fabsf(lo[0])), fabsf(lo[1])); am = fmaxf(fmaxf(am, fabsf(lo[2])), fabsf(lo[3]));
        am = fmaxf(fmaxf(am, fabsf(hi[0])), fabsf(hi[1])); am = fmaxf(fmaxf(am, fabsf(hi[2])), fabsf(hi[3]));
        asm volatile("" : "+v"(am));
      }
    }
    if (early) {
#pragma unroll
      for (int j = 0; j < SEND_WIN; ++j) send_piece(y, j, srs, pov);
    }
    am = wave_max(am);
    if (lane == 0) red[wave] = ok ? am : -1.0f;
    __syncthreads();
    if (tid == 0) {
      const float r0 = red[0], r1 = red[1], r2 = red[2], r3 = red[3];
      const float wam = fminf(fminf(r0, r1), fminf(r2, r3)) < 0.f ? -1.0f : fmaxf(fmaxf(r0, r1), fmaxf(r2, r3));
      __hip_atomic_store((u64*)(ws + WS_AMAX + (unit >> 4) * FS) + (unit & 15), ((u64)call << 32) | __builtin_bit_cast(unsigned, wam), __ATOMIC_RELAXED, __HIP_MEMORY_SCOPE_AGENT);
    }
  }
  STAMP(3);
  const int row0 = wm * 256 + kh * 128 + i16, col0 = (q < 2 ? ln0 : ln1) * 32 + (q & 1) * 16;
  const int srank = (rank - pd) & 7;

  if (early) {
#pragma unroll
    for (int j = SEND_WIN; j < 16; ++j) {
      if (wide > 0) asm volatile("s_waitcnt vmcnt(%0)" :: "n"(2 * SEND_WIN - 1) : "memory");
      else asm volatile("s_waitcnt vmcnt(%0)" :: "n"(SEND_WIN - 1) : "memory");
      send_piece(y, j, srs, pov);
    }
    asm volatile("s_waitcnt vmcnt(0)" ::: "memory");
    if (lane == 0)
      __hip_atomic_store((GAS unsigned*)(dws + WS_EFLAG + (size_t)(unit * 4 + wave) * FS), call, __ATOMIC_RELAXED, __HIP_MEMORY_SCOPE_SYSTEM);
    STAMP(5);
    ok &= wait_flag<__HIP_MEMORY_SCOPE_SYSTEM, 8>((const unsigned*)(ws + WS_EFLAG + (size_t)(unit * 4 + wave) * FS), call, tmo);
    u32x4 st[16];
    {
      const i32x4 rs = make_rsrc(ws + WS_STAGE + wave_off, WAVE_B);
#pragma unroll
      for (int j = 0; j < 16; ++j)
        asm volatile("buffer_load_dwordx4 %0, %1, %2, %3 offen offset:%4 sc0 sc1" : "=v"(st[j]) : "v"(pov), "s"(rs), "s"(j * PC & ~4095), "i"(j * PC & 4095) : "memory");
      asm volatile("s_waitcnt vmcnt(0)" ::: "memory");
#pragma unroll
      for (int j = 0; j < 16; ++j) asm volatile("" : "+v"(st[j]));
    }
    STAMP(6);
    const Quant qz = global_scale<32>((const u64*)(ws + gmax_off), call, tmo, ok, lane);
    STAMP(4);
    GAS char* ob = dout + (size_t)(srank * ML + row0) * NL + col0;
#pragma unroll
    for (int m = 0; m < 8; ++m) {
      const u32x4 s0 = st[2 * m], s1 = st[2 * m + 1];
      u32x4 o;
#pragma unroll
      for (int r = 0; r < 4; ++r) {
        const int h = r >> 1;
        const unsigned w0 = s0[h], w1 = s0[2 + h], w2 = s1[h], w3 = s1[2 + h];
        const float v0 = __builtin_bit_cast(float, (r & 1) ? (w0 & 0xFFFF0000u) : (w0 << 16));
        const float v1 = __builtin_bit_cast(float, (r & 1) ? (w1 & 0xFFFF0000u) : (w1 << 16));
        const float v2 = __builtin_bit_cast(float, (r & 1) ? (w2 & 0xFFFF0000u) : (w2 << 16));
        const float v3 = __builtin_bit_cast(float, (r & 1) ? (w3 & 0xFFFF0000u) : (w3 << 16));
        o[r] = quant4(v0, v1, v2, v3, qz);
      }
      asm volatile("global_store_dwordx4 %0, %1, off sc0 sc1\n\ts_nop 1" :: "v"(ob + (size_t)m * 16 * NL), "v"(o) : "memory");
    }
    asm volatile(".Lq8_code_end1:");
    STAMP(7);
    STAMP_OUT();
    return;
  }

  if (unit == NWG - 1 && wave >= 2) {
    const float rm = sweep_max<4, 4 * FS / 8, __HIP_MEMORY_SCOPE_AGENT, 1>((const u64*)(ws + WS_AMAX + (lane >> 4) * FS) + (lane & 15), call, tmo);
    if (lane < NR)
      __hip_atomic_store((GAS u64*)(anyws + gmax_off) + rank, ((u64)call << 32) | __builtin_bit_cast(unsigned, rm), __ATOMIC_RELAXED, __HIP_MEMORY_SCOPE_SYSTEM);
  }
  const Quant qz = global_scale<4>((const u64*)(ws + gmax_off), call, tmo, ok, lane);
  STAMP(4);
  {
    GAS char* ob = dout + (size_t)(rank * ML + row0) * NL + col0;
#pragma unroll
    for (int m = 0; m < 8; ++m) {
      u32x4 o;
#pragma unroll
      for (int r = 0; r < 4; ++r) o[r] = quant4(y[m][0][r], y[m][1][r], y[m][2][r], y[m][3][r], qz);
      asm volatile("s_waitcnt vmcnt(%2)\n\tglobal_store_dwordx4 %0, %1, off sc0 sc1\n\ts_nop 1" :: "v"(ob + (size_t)m * 16 * NL), "v"(o), "n"(LATE_WIN - 1) : "memory");
    }
  }
  asm volatile("s_waitcnt vmcnt(0)\n.Lq8_code_end2:" ::: "memory");
  STAMP(5);
  if (lane == 0)
    __hip_atomic_store((GAS unsigned*)(dws + WS_EFLAG + (size_t)(unit * 4 + wave) * FS), call, __ATOMIC_RELAXED, __HIP_MEMORY_SCOPE_SYSTEM);
  if (!wait_flag<__HIP_MEMORY_SCOPE_SYSTEM, 2>((const unsigned*)(ws + WS_EFLAG + (size_t)(unit * 4 + wave) * FS), call, tmo) && lane == 0)
    myout[(size_t)(srank * ML + row0) * NL + col0] = (char)-128;
  STAMP(7);
  STAMP_OUT();
}

static void make_pace(Params& p, const int* pa) {
  const int TE = pa[4], TS = pa[5], A = pa[7], DS = pa[8];
  p.slack = pa[9];
  p.nwide = pa[10]; p.wmask = pa[14];
  for (int k = 0; k < NE; ++k) {
    p.te[k] = k < NE - 3 ? TE : k == NE - 3 ? TS : pa[13];
    p.start[k] = A + (k * DS > pa[11] ? k * DS - pa[11] : 0) - KH * (p.te[k] - TE);
    if (p.start[k] < A) p.start[k] = A;
  }
  for (int k = 0; k < pa[10]; ++k) p.te[k] = pa[12];
  for (int busy = 1; busy >= 0; --busy) {
    const int* e = busy ? pa + 17 : pa;
    const double T0 = e[0], TH = e[1], T1 = e[2], TR = e[3], TC = pa[6], C = pa[15] * 0.01;
    double P = 0, last = 0;
    int v = 0, stall = 0;
    for (int u = A - 100; v <= KH + 1; ++u) {
      double r = NWG / (u < TH ? T0 : u < TR ? T0 + (T1 - T0) * (u - TH) / (TR - TH) : T1);
      for (int k = 0; k < NE; ++k) {
        const double s0 = p.start[k] - 100 + (k < pa[10] ? 0 : p.te[k] / 2);
        if (u >= s0 && u < s0 + KH * p.te[k]) r -= (k == NE - 1 ? 20.0 : 14.0) / p.te[k];
      }
      r /= NWG - 2 * NEARLY - 6;
      if (r > 1.0 / TC) r = 1.0 / TC;
      if (r < 0.001) r = 0.001;
      if (P >= 3.0 && stall < pa[16]) { ++stall; r = 0; }
      P += r;
      if (P >= v + C) {
        if (v) p.step[v - 1] = busy ? (int)(u - last) << 16 : p.step[v - 1] | (int)(u - last); else p.start[NE] = u;
        last = u; ++v;
      }
    }
  }
}

extern "C" void kernel_launch(int rank, int nranks, ncclComm_t comm,
                              hipStream_t stream,
                              void* const* d_in, const int* in_sizes, int n_in,
                              void* d_out, int out_size,
                              const int* in_dtypes, int out_dtype,
                              void* const* d_out_all,
                              void* const* d_ws_all, size_t ws_bytes) {
  (void)comm; (void)in_dtypes; (void)out_dtype; (void)d_out;
  if (nranks != NR || n_in != 2 || in_sizes[0] != ML * KD || in_sizes[1] != KD * ND || out_size != NR * ML * NL ||
      ws_bytes < WS_END || (unsigned)rank >= (unsigned)NR) {
    fprintf(stderr, "unexpected problem shape\n");
    return;
  }
  int dev = 0, cus = 0;
  HIP_CHECK(hipGetDevice(&dev));
  HIP_CHECK(hipDeviceGetAttribute(&cus, hipDeviceAttributeMultiprocessorCount, dev));
  if (cus < NWG) {
    fprintf(stderr, "needs %d CUs: the workgroups wait for one another\n", NWG);
    return;
  }
  HIP_CHECK(hipFuncSetAttribute((const void*)fused_kernel, hipFuncAttributeMaxDynamicSharedMemorySize, LDS_BYTES));
  Params p;
  p.x = (const float*)d_in[0];
  p.w = (const float*)d_in[1];
  for (int r = 0; r < NR; ++r) { p.out[r] = (char*)d_out_all[r]; p.ws[r] = (char*)d_ws_all[r]; }
  p.rank = rank;
  p.myws = (char*)d_ws_all[rank];
  int pa[21] = {PACE_ARGS};
  make_pace(p, pa);
  fused_kernel<<<NWG, NTHR, LDS_BYTES, stream>>>(p);
  HIP_CHECK(hipGetLastError());
}
```

```cpp
#include <hip/hip_runtime.h>
#include <rccl/rccl.h>
#include <cstdio>
#include <cstdint>
#include <cstdlib>
#include <cstddef>

#define HIP_CHECK(expr) do { hipError_t _e = (expr); if (_e != hipSuccess) { \
    fprintf(stderr, "HIP error %s at %s:%d\n", hipGetErrorString(_e), __FILE__, __LINE__); \
    return; } } while (0)

#define LAS __attribute__((address_space(3)))
#define GAS __attribute__((address_space(1)))
typedef float f32x4 __attribute__((ext_vector_type(4)));
typedef float f32x2 __attribute__((ext_vector_type(2)));
typedef __bf16 bf16x2 __attribute__((ext_vector_type(2)));
typedef unsigned u32x4 __attribute__((ext_vector_type(4)));
typedef int i32x4 __attribute__((ext_vector_type(4)));
typedef int i32x2 __attribute__((ext_vector_type(2)));
typedef unsigned long long u64;

constexpr int NR = 8;
constexpr int KD = 4096, ND = 16384;
constexpr int ML = 512;
constexpr int NL = ND / NR;
constexpr int NWG = 256, NTHR = 256;
constexpr int NBLK = KD / 64;
constexpr int NT = ND / 128;
constexpr int KH = NBLK / 2;
constexpr int NE = 9;
constexpr int NEARLY = 7 * NE;
constexpr int PF0 = 4, PFL = 3;
constexpr int NH = 2;
constexpr int SEND_WIN = 3, LATE_WIN = 2;
#define PACE_ARGS 212, 2100, 157, 4300, 125, 150, 115, 600, 245, 100, 2, 100, 122, 175, 3, 100, 175, 228, 2300, 163, 4400
constexpr int LDS_X = 4 * 32768;
constexpr int LDS_BYTES = LDS_X + 32768;
constexpr int PC = 2048, FS = 256;
constexpr int WAVE_B = 16 * PC;
constexpr size_t HALF_B = 4 * WAVE_B;

constexpr size_t WS_TMO = 128;
constexpr size_t WS_GMAX = 512;
constexpr size_t WS_START = 1024;
constexpr size_t WS_AMAX = 4096;
constexpr size_t WS_XFLAG = WS_AMAX + NWG / 16 * FS;
constexpr size_t WS_EFLAG = WS_XFLAG + NWG * FS;
constexpr size_t WS_PFLAG = WS_EFLAG + NWG * 4 * FS;
constexpr size_t WS_STAMP = WS_PFLAG + NWG * 4 * FS;
constexpr size_t WS_XT = 1 << 20;
constexpr size_t XT_BYTES = (size_t)ML * KD * 2, W_BYTES = (size_t)KD * ND * 4;
constexpr size_t WS_PART = WS_XT + XT_BYTES;
constexpr size_t WS_STAGE = WS_PART + NWG * HALF_B;
constexpr size_t WS_END = WS_STAGE + 2 * NEARLY * HALF_B;
static_assert(WS_STAMP + 65536 + NWG * 512 <= WS_XT, "control block overlaps x");
constexpr unsigned BUSY_TICKS = 50000;
constexpr u64 DEADLINE_TICKS = 50000000ull;

struct Params {
  const float* x;
  const float* w;
  char* myws;
  int rank;
  int slack;
  char* out[NR];
  char* ws[NR];
  int nwide, wmask;
  int start[NE + 1], te[NE];
  int step[KH + 1];
};

#define STAMP(i) do {} while (0)
#define STAMPC(i) do {} while (0)
#define STAMP_DECL do {} while (0)
#define STAMP_OUT() do {} while (0)

__device__ __forceinline__ unsigned pk_bf16(float lo, float hi) {
  f32x2 v = {lo, hi};
  return __builtin_bit_cast(unsigned, __builtin_convertvector(v, bf16x2));
}
__device__ __forceinline__ float wave_max(float f) {
  int v = __builtin_bit_cast(int, f);
  v = max(v, __builtin_amdgcn_update_dpp(0, v, 0x111, 0xf, 0xf, true));
  v = max(v, __builtin_amdgcn_update_dpp(0, v, 0x112, 0xf, 0xf, true));
  v = max(v, __builtin_amdgcn_update_dpp(0, v, 0x114, 0xf, 0xf, true));
  v = max(v, __builtin_amdgcn_update_dpp(0, v, 0x118, 0xf, 0xf, true));
  const int r = max(max(__builtin_amdgcn_readlane(v, 15), __builtin_amdgcn_readlane(v, 31)), max(__builtin_amdgcn_readlane(v, 47), __builtin_amdgcn_readlane(v, 63)));
  return __builtin_bit_cast(float, r);
}
__device__ __forceinline__ i32x4 make_rsrc(const void* base, unsigned bytes) {
  const u64 a = (u64)base;
  return i32x4{__builtin_amdgcn_readfirstlane((int)(unsigned)a), __builtin_amdgcn_readfirstlane((int)(unsigned)(a >> 32)), (int)bytes, 0x00020000};
}

__device__ __forceinline__ void w_load(f32x4& d, unsigned voff, i32x4 rsrc, unsigned soff) {
  asm volatile("buffer_load_dwordx4 %0, %1, %2, %3 offen nt" : "=v"(d) : "v"(voff), "s"(rsrc), "s"(soff) : "memory");
}
__device__ __forceinline__ void set_m0(unsigned m0v) { asm volatile("s_mov_b32 m0, %0" :: "s"(m0v) : "memory"); }
template <int IMM>
__device__ __forceinline__ void x_load(unsigned voff, i32x4 rsrc, unsigned soff) {
  asm volatile("buffer_load_dwordx4 %0, %1, %2 offen offset:%3 sc1 lds" :: "v"(voff), "s"(rsrc), "s"(soff), "i"(IMM) : "memory");
}
template <int N>
__device__ __forceinline__ void wait_bar() { asm volatile("s_waitcnt vmcnt(%0) lgkmcnt(0)\n\ts_barrier" :: "n"(N) : "memory"); }
__device__ __forceinline__ void mfma(f32x4& c, u32x4 a, u32x4 b) {
  asm volatile("v_mfma_f32_16x16x32_bf16 %0, %1, %2, %0" : "+a"(c) : "v"(a), "v"(b));
}
#define SCHED() __builtin_amdgcn_sched_barrier(0)
__device__ __forceinline__ void send_piece(const f32x4 (&y)[8][4], int j, i32x4 srs, unsigned lane16) {
  const f32x4 lo = y[j >> 1][(j & 1) * 2], hi = y[j >> 1][(j & 1) * 2 + 1];
  const u32x4 d = {pk_bf16(lo[0], lo[1]), pk_bf16(lo[2], lo[3]), pk_bf16(hi[0], hi[1]), pk_bf16(hi[2], hi[3])};
  asm volatile("buffer_store_dwordx4 %0, %1, %2, %3 offen sc0 sc1\n\ts_nop 1" :: "v"(d), "v"(lane16), "s"(srs), "s"(j * PC) : "memory");
}
__device__ __forceinline__ void touch(const GAS char* t) {
  unsigned v;
  asm volatile("global_load_dword %0, %1, off nt\n\ts_waitcnt vmcnt(0)" : "=&v"(v) : "v"(t) : "memory");
}
constexpr float QBIAS = 12582912.0f;
struct Quant { float inv, lo, hi; };
__device__ __forceinline__ unsigned quant4(float v0, float v1, float v2, float v3, const Quant& qz) {
  const unsigned e0 = __builtin_bit_cast(unsigned, __builtin_amdgcn_fmed3f(__builtin_fmaf(v0, qz.inv, QBIAS), qz.lo, qz.hi));
  const unsigned e1 = __builtin_bit_cast(unsigned, __builtin_amdgcn_fmed3f(__builtin_fmaf(v1, qz.inv, QBIAS), qz.lo, qz.hi));
  const unsigned e2 = __builtin_bit_cast(unsigned, __builtin_amdgcn_fmed3f(__builtin_fmaf(v2, qz.inv, QBIAS), qz.lo, qz.hi));
  const unsigned e3 = __builtin_bit_cast(unsigned, __builtin_amdgcn_fmed3f(__builtin_fmaf(v3, qz.inv, QBIAS), qz.lo, qz.hi));
  return __builtin_amdgcn_perm(e1, e0, 0x0c0c0400u) | __builtin_amdgcn_perm(e3, e2, 0x04000c0cu);
}

template <int SCOPE, int SLEEP, int N = 1, int STRIDE = 0>
__device__ __forceinline__ bool wait_flag(const unsigned* flag, unsigned call, unsigned* tmo) {
  const u64 t0 = __builtin_amdgcn_s_memrealtime();
  for (unsigned polls = 1;; ++polls) {
    unsigned v[N];
    bool pend = false;
#pragma unroll
    for (int i = 0; i < N; ++i) v[i] = __hip_atomic_load(flag + i * STRIDE, __ATOMIC_RELAXED, SCOPE);
#pragma unroll
    for (int i = 0; i < N; ++i) pend |= v[i] < call;
    if (!__any(pend)) return true;
    if (((polls & 63u) == 0 && __hip_atomic_load(tmo, __ATOMIC_RELAXED, __HIP_MEMORY_SCOPE_AGENT) == call) ||
        __builtin_amdgcn_s_memrealtime() - t0 > DEADLINE_TICKS) {
      __hip_atomic_store(tmo, call, __ATOMIC_RELAXED, __HIP_MEMORY_SCOPE_AGENT);
      return false;
    }
    __builtin_amdgcn_s_sleep(SLEEP);
  }
}

struct Ctx {
  i32x4 wrsrc, xrsrc;
  unsigned wvoff[8], xvoff0, xvoff1, m0base;
  int wlim;
  int left;
  unsigned xso, wso;
  unsigned due, slack;
  int steps;
  i32x4 frsrc;
  const char* fbase;
  unsigned fvoff, fso, call, fl;
  unsigned* tmo;
  bool ok;
  LAS const char* lrd0;
  LAS const char* lrd1;
  LAS char* wmine;
  LAS const char* wother;
};

template <int SLOT>
__device__ __forceinline__ u32x4 x_frag(const Ctx& c, int mt) {
  return *(LAS const u32x4*)((SLOT < 2 ? c.lrd0 : c.lrd1) + (SLOT & 1) * 32768 + mt * 1024);
}
template <int G>
__device__ __forceinline__ void x_piece(const Ctx& c, i32x4 xr, unsigned xso) {
  x_load<(G & 3) * 1024>((G & 4) ? c.xvoff1 : c.xvoff0, xr, (G & 8) ? xso + 32768 : xso);
}
template <int SL, bool CHK, bool HELD = false>
__device__ __forceinline__ void kstep(Ctx& c, f32x4 (&acc)[16][4], f32x4 (&wr)[2][8], u32x4 (&a)[2][4], u32x4 (&xf)[8], const u32x4* held = nullptr) {
  constexpr int E = SL & 1, WS = SL >> 1, NX = (SL + 1) & 3;
  u64 now;
  unsigned lateby;
  const i32x4 xr = c.xrsrc, wrs = c.wrsrc;
  const unsigned xso = c.xso, wso = c.wso;
  const unsigned m0x = c.m0base + ((SL + 2) & 3) * 32768;
  f32x4 (&r)[8] = wr[WS ^ 1];
#pragma unroll
  for (int g = 0; g < 16; ++g) {
    const int sn = (g & 1) * 3;
    if (E == 1 && g == 12) {
      if (CHK) {
        asm volatile("s_waitcnt vmcnt(8)" : "+v"(c.fl) :: "memory");
        if (__any(c.fl < c.call)) {
          c.ok &= wait_flag<__HIP_MEMORY_SCOPE_AGENT, 1>((const unsigned*)(c.fbase + c.fso + c.fvoff), c.call, c.tmo);
          __builtin_amdgcn_s_waitcnt(0xC07F);
        }
      }
      wait_bar<HELD ? 0 : 8>();
      SCHED();
    }
    mfma(acc[g][0 ^ sn], a[E][0 ^ sn], xf[g & 7]);
    SCHED();
    if (g < 9) xf[(g + 7) & 7] = x_frag<SL>(c, g + 7);
    else if (E == 0) xf[(g + 7) & 7] = x_frag<NX>(c, g - 9);
    else if (g >= 12) xf[(2 * (g - 12)) & 7] = x_frag<NX>(c, 2 * (g - 12));
    if (CHK && E == 0 && g == 0) asm volatile("buffer_load_dword %0, %1, %2, %3 offen sc1" : "=v"(c.fl) : "v"(c.fvoff), "s"(c.frsrc), "s"(c.fso) : "memory");
    if (E == 1 && g == 9) *(LAS u32x4*)(c.wmine + (WS ^ 1) * 16384 + 1024) = a[0][1];
    if (E == 1 && g == 10) *(LAS u32x4*)(c.wmine + (WS ^ 1) * 16384 + 3072) = a[0][3];
    if (E == 1 && g == 11) {
      asm volatile("s_waitcnt lgkmcnt(0)" : "+s"(now) :: "memory");
      lateby = (unsigned)now - c.due;
      asm volatile("" : "+s"(lateby));
    }
    SCHED();
    mfma(acc[g][1 ^ sn], a[E][1 ^ sn], xf[g & 7]);
    SCHED();
    if (E == 0) {
      switch (g) {
        case 0: x_piece<0>(c, xr, xso); break;   case 1: x_piece<1>(c, xr, xso); break;
        case 2: x_piece<2>(c, xr, xso); break;   case 3: x_piece<3>(c, xr, xso); break;
        case 4: x_piece<4>(c, xr, xso); break;   case 5: x_piece<5>(c, xr, xso); break;
        case 6: x_piece<6>(c, xr, xso); break;   case 7: x_piece<7>(c, xr, xso); break;
        case 8: x_piece<8>(c, xr, xso); break;   case 9: x_piece<9>(c, xr, xso); break;
        case 10: x_piece<10>(c, xr, xso); break; case 11: x_piece<11>(c, xr, xso); break;
        case 12: x_piece<12>(c, xr, xso); break; case 13: x_piece<13>(c, xr, xso); break;
        case 14: x_piece<14>(c, xr, xso); break; default: x_piece<15>(c, xr, xso); break;
      }
    } else {
      if (!HELD && g < 8) w_load(wr[WS][g & 7], c.wvoff[g & 7], wrs, wso);
      if (HELD && g >= 4 && g < 8) { a[0][g & 3] = held[g & 3]; asm volatile("" : "+v"(a[0][g & 3])); }
      if (!HELD && g == 5)
        asm volatile("s_waitcnt vmcnt(%8)" : "+v"(r[0]), "+v"(r[1]), "+v"(r[2]), "+v"(r[3]), "+v"(r[4]), "+v"(r[5]), "+v"(r[6]), "+v"(r[7]) : "n"(22 + CHK) : "memory");
      if (g == 8) *(LAS u32x4*)(c.wmine + (WS ^ 1) * 16384) = a[0][0];
      if (g == 9) { c.left -= 1; c.xso += 65536u; asm volatile("" : "+s"(c.left), "+s"(c.xso)); }
      if (g == 10) { c.xrsrc[2] = c.left > 1 ? (int)XT_BYTES : 0; asm volatile("" : "+s"(c.xrsrc)); }
      if (g == 11) { c.wrsrc[2] = c.left > c.wlim ? (int)W_BYTES : 0; asm volatile("" : "+s"(c.wrsrc)); }
      if (g == 12) { lateby = (int)lateby > (int)c.slack ? lateby - c.slack : 0u; asm volatile("" : "+s"(lateby)); }
      if (g == 13) { c.due += lateby; lateby = __builtin_amdgcn_readlane(c.steps, c.left); asm volatile("" : "+s"(c.due), "+s"(lateby)); }
      if (g == 14) { c.due += lateby; asm volatile("" : "+s"(c.due)); }
    }
    SCHED();
    mfma(acc[g][2 ^ sn], a[E][2 ^ sn], xf[g & 7]);
    SCHED();
    if (E == 0 && g < 4) a[1][g] = *(LAS const u32x4*)(c.wother + WS * 16384 + g * 1024);
    if (!HELD && E == 1 && g >= 5 && g < 9) { const int t = (g - 5) & 3; a[0][t][0] = pk_bf16(r[0][t], r[1][t]); a[0][t][1] = pk_bf16(r[2][t], r[3][t]); }
    if (E == 1 && g == 9) *(LAS u32x4*)(c.wmine + (WS ^ 1) * 16384 + 2048) = a[0][2];
    if (E == 1 && g == 11) {
      if (__builtin_expect((int)lateby < 0, 0))
        do __builtin_amdgcn_s_sleep(1); while ((int)((unsigned)__builtin_amdgcn_s_memrealtime() - c.due) < 0);
    }
    if (E == 1 && g >= 12 && g < 15) xf[(2 * (g - 12) + 1) & 7] = x_frag<NX>(c, 2 * (g - 12) + 1);
    SCHED();
    mfma(acc[g][3 ^ sn], a[E][3 ^ sn], xf[g & 7]);
    SCHED();
    if (!HELD && E == 1 && g >= 5 && g < 9) { const int t = (g - 5) & 3; a[0][t][2] = pk_bf16(r[4][t], r[5][t]); a[0][t][3] = pk_bf16(r[6][t], r[7][t]); }
    if (E == 0 && (g & 3) == 3 && g < 15) set_m0(m0x + ((g + 1) >> 3) * 32768 + (((g + 1) >> 2) & 1) * 4096);
    if (E == 1 && g == 9) asm volatile("s_memrealtime %0" : "=s"(now) :: "memory");
    if (E == 1 && g == 10) { c.wso += 1u << 22; asm volatile("" : "+s"(c.wso)); }
    if (E == 1 && g == 15) set_m0(c.m0base + ((SL + 3) & 3) * 32768);
    SCHED();
  }
}

__device__ __forceinline__ void last_block(const Ctx& c, f32x4 (&acc)[16][4], u32x4 (&a)[2][4], u32x4 (&xf)[8], i32x4 prs, unsigned pov) {
  f32x4 lo, hi;
#pragma unroll
  for (int i = 0; i < 32; ++i) {
    const int e = (i >> 3) & 1, m = (i & 7) + (i >> 4) * 8;
    const int i7 = i + 7, e7 = (i7 >> 3) & 1, m7 = (i7 & 7) + (i7 >> 4) * 8;
    const int sn = (i & 1) * 3;
    const int j = i - 16;
    mfma(acc[m][0 ^ sn], a[e][0 ^ sn], xf[i & 7]);
    SCHED();
    if (i7 < 32) xf[i7 & 7] = e7 ? x_frag<3>(c, m7) : x_frag<2>(c, m7);
    if (j >= 0) {
      asm volatile("" : "+a"(acc[j >> 1][(j & 1) * 2]));
      lo = acc[j >> 1][(j & 1) * 2];
      asm volatile("" : "+v"(lo));
    }
    SCHED();
    mfma(acc[m][1 ^ sn], a[e][1 ^ sn], xf[i & 7]);
    SCHED();
    if (i < 4) a[1][i] = *(LAS const u32x4*)(c.wother + 16384 + i * 1024);
    if (j >= 0) {
      asm volatile("" : "+a"(acc[j >> 1][(j & 1) * 2 + 1]));
      hi = acc[j >> 1][(j & 1) * 2 + 1];
      asm volatile("" : "+v"(hi));
    }
    SCHED();
    mfma(acc[m][2 ^ sn], a[e][2 ^ sn], xf[i & 7]);
    SCHED();
    u32x4 d;
    if (j >= 0) { d = u32x4{pk_bf16(lo[0], lo[1]), pk_bf16(lo[2], lo[3]), pk_bf16(hi[0], hi[1]), pk_bf16(hi[2], hi[3])}; asm volatile("" : "+v"(d)); }
    SCHED();
    mfma(acc[m][3 ^ sn], a[e][3 ^ sn], xf[i & 7]);
    SCHED();
    if (j >= 0)
      asm volatile("buffer_store_dwordx4 %0, %1, %2, %3 offen offset:%4 sc1\n\ts_nop 1"
                   :: "v"(d), "v"(pov), "s"(prs), "s"(j * PC & ~4095), "i"(j * PC & 4095) : "memory");
    SCHED();
  }
}

template <int N, int STRIDE, int SCOPE, int SLEEP>
__device__ __forceinline__ float sweep_max(const u64* slots, unsigned call, unsigned* tmo) {
  const u64 t0 = __builtin_amdgcn_s_memrealtime();
  for (unsigned polls = 1;; ++polls) {
    bool all = true, bad = false;
    float mx = 0.f;
    u64 v[N];
#pragma unroll
    for (int r = 0; r < N; ++r) v[r] = __hip_atomic_load(slots + r * STRIDE, __ATOMIC_RELAXED, SCOPE);
#pragma unroll
    for (int r = 0; r < N; ++r) {
      const float f = __builtin_bit_cast(float, (unsigned)v[r]);
      all &= (unsigned)(v[r] >> 32) == call;
      bad |= f < 0.f;
      mx = fmaxf(mx, f);
    }
    if (__all(all)) {
      return __any(bad) ? -1.0f : wave_max(mx);
    }
    if (((polls & 63u) == 0 && __hip_atomic_load(tmo, __ATOMIC_RELAXED, __HIP_MEMORY_SCOPE_AGENT) == call) ||
        __builtin_amdgcn_s_memrealtime() - t0 > DEADLINE_TICKS) {
      __hip_atomic_store(tmo, call, __ATOMIC_RELAXED, __HIP_MEMORY_SCOPE_AGENT);
      return -1.0f;
    }
    __builtin_amdgcn_s_sleep(SLEEP);
  }
}
template <int SLEEP>
__device__ __forceinline__ Quant global_scale(const u64* gmax, unsigned call, unsigned* tmo, bool ok, int lane) {
  const float gam = ok ? sweep_max<1, 0, __HIP_MEMORY_SCOPE_SYSTEM, SLEEP>(gmax + (lane & 7), call, tmo) : -1.0f;
  Quant qz;
  qz.inv = gam > 0.f ? 127.0f / gam : 0.f;
  qz.lo = gam >= 0.f ? QBIAS - 127.0f : QBIAS - 128.0f;
  qz.hi = gam >= 0.f ? QBIAS + 127.0f : QBIAS - 128.0f;
  asm volatile("" : "+v"(qz.lo), "+v"(qz.hi));
  return qz;
}

__device__ __forceinline__ int line_of(int ct, int s) {
  return ct < 7 ? 8 * ct + 4 + s : ct < 11 ? 16 * (ct - 7) + (s >> 1) * 8 + (s & 1) : ct == 11 ? 60 + s : (ct & 1) * 32 + 8 * s + (ct >> 1) - 4;
}
__device__ __forceinline__ void tile_of(int j, int& pd, int& ct) {
  const int k = j / 7;
  pd = j < 112 ? j % 7 + 1 : 0;
  ct = j >= 112 ? j - 112 : k < 2 ? 12 + k : k < NE - 2 ? 14 - NE + k : k < NE ? 16 - NE + k : k - NE;
}

__global__ void __launch_bounds__(NTHR, 1) fused_kernel(Params p) {
  extern __shared__ __attribute__((aligned(1024))) char smem[];
  const int tid = threadIdx.x, lane = tid & 63;
  const int wave = __builtin_amdgcn_readfirstlane(tid >> 6);
  char* ws = p.myws;

  STAMP_DECL;
  STAMP(0);
  STAMPC(8);
  const int b = blockIdx.x, xcd = b & 7, idx = b >> 3;
  const int kh = xcd & 1, slot = idx * 4 + (xcd >> 1), unit = slot * 2 + kh;
  const int wn = wave & 1, wm = wave >> 1;
  const int q = lane >> 4, i16 = lane & 15;
  const unsigned lane16 = lane * 16;
  const unsigned t_start = (unsigned)__builtin_amdgcn_s_memrealtime();

  Ctx c;
  c.xrsrc = make_rsrc(ws + WS_XT, (unsigned)XT_BYTES);
  unsigned* xflag = (unsigned*)(ws + WS_XFLAG);
  const int hs = kh * NT + slot;
  const unsigned xto = (unsigned)(hs * 16384 + wave * 4096);
  f32x4 raw[8];
  i32x2 prev;
  unsigned warm;
  {
    const int st = hs >> 1, h = hs & 1;
    const i32x4 xin = make_rsrc(p.x, (unsigned)((size_t)ML * KD * 4));
#pragma unroll
    for (int j = 0; j < 4; ++j) {
      const unsigned o = (unsigned)((h * 16 + ((wave * 4 + j) ^ ((kh ^ 1) << 3))) * 16 + i16) * (KD * 4) + ((st ^ h) * 32 + q * 8) * 4;
      asm volatile("buffer_load_dwordx4 %0, %2, %3, 0 offen\n\tbuffer_load_dwordx4 %1, %2, %3, 0 offen offset:16"
                   : "=&v"(raw[2 * j]), "=&v"(raw[2 * j + 1]) : "v"(o), "s"(xin) : "memory");
    }
    asm volatile("global_load_dwordx2 %0, %1, off sc1" : "=v"(prev) : "v"(xflag + hs * (FS / 4)) : "memory");
    asm volatile("buffer_load_dword %0, %1, %2, %3 offen sc1" : "=v"(warm) : "v"(0), "s"(c.xrsrc), "s"(xto) : "memory");
  }

  unsigned cpf = 0;
  {
    u64 pc; unsigned len1, len2;
    asm volatile("s_getpc_b64 %0\n.Lq8_code_begin:\n\ts_mov_b32 %1, .Lq8_code_end1-.Lq8_code_begin\n\ts_mov_b32 %2, .Lq8_code_end2-.Lq8_code_begin" : "=s"(pc), "=s"(len1), "=s"(len2));
    const int wv = (idx - 8) * 4 + wave;
    const i32x4 cr = (unsigned)wv < 8u ? make_rsrc((const void*)pc, len1 > len2 ? len1 : len2)
                                       : make_rsrc(p.w, (unsigned)(wv - 8) < 2u ? (unsigned)W_BYTES : 0);
    asm volatile("buffer_load_dword %0, %1, %2, 0 offen" : "+v"(cpf)
                 : "v"((unsigned)wv < 8u ? (unsigned)(wv * 64 + lane) * 128u : (unsigned)(kh * 128 + (wv & 1) * 64 + lane) << 20), "s"(cr) : "memory");
  }
  const bool early = slot < NEARLY;
  int pd, ct;
  tile_of(slot, pd, ct);
  const int rank = p.rank;
  const unsigned dest = (unsigned)(rank + pd) & 7u;
  const int k0 = kh * KH;
  const int ln0 = line_of(ct, 2 * wn), ln1 = line_of(ct, 2 * wn + 1);
  unsigned* tmo = (unsigned*)(ws + WS_TMO);
  GAS char* dws = (GAS char*)p.ws[dest];
  GAS char* dout = (GAS char*)p.out[early ? rank : dest];
  GAS char* myout = (GAS char*)p.out[rank];
  asm volatile("" : "+s"(dws), "+s"(dout), "+s"(myout));
  GAS char* anyws;
  i32x2 stp;
  asm volatile("global_load_dwordx2 %0, %2, %4 offset:%5\n\tglobal_load_dwordx2 %1, %3, %4 offset:%6"
               : "=&v"(anyws), "=&v"(stp) : "v"((lane & 7) * 8), "v"(((KH - 1 - lane) & 31) * 4), "s"(__builtin_amdgcn_kernarg_segment_ptr()),
                 "i"(offsetof(Params, ws)), "i"(offsetof(Params, step)) : "memory");

  c.wrsrc = make_rsrc(p.w, (unsigned)W_BYTES);
#pragma unroll
  for (int j = 0; j < 8; ++j) {
    c.wvoff[j] = ((unsigned)(wm * 32 + q * 8 + j) << 16) + (i16 < 8 ? ln0 : ln1) * 128 + (i16 & 7) * 16;
    asm volatile("" : "+v"(c.wvoff[j]));
  }
  c.xvoff0 = wave * 8192 + lane16;
  c.xvoff1 = c.xvoff0 + 4096;
  asm volatile("" : "+v"(c.xvoff1));
  c.m0base = (unsigned)(size_t)(LAS char*)smem + wave * 8192;
  constexpr int nblk = KH;
  const unsigned wsb = ((unsigned)k0 << 22) + dest * (NL * 4u);
  c.left = nblk;
  c.wlim = 2;
  c.xso = (unsigned)(k0 + 1) * 65536u;
  c.wso = wsb + (2u << 22);
  const bool steady = early || ct == 11 || ct >= 14;
  const int cls = early ? slot / 7 : steady ? NE - 1 : NE, te = p.te[steady ? cls : 0], phase = cls < p.nwide ? 0 : (slot * 74 + kh * 8) & 255;
  c.due = t_start + p.start[cls] + phase * (steady ? te : p.step[0] & 0xFFFF) / 256;
  c.slack = p.slack;
  int wide = (p.wmask >> cls) & 1;
  asm volatile("" : "+s"(wide));
  c.lrd0 = (LAS const char*)smem + wm * 16384 + lane16;
  c.lrd1 = c.lrd0 + 65536;
  asm volatile("" : "+v"(c.lrd1));
  c.wmine = (LAS char*)smem + LDS_X + wave * 4096 + lane16;
  c.wother = (LAS const char*)smem + LDS_X + (wave ^ 2) * 4096 + lane16;

  f32x4 acc[16][4];
  f32x4 wr[2][8];
  u32x4 a[2][4], xf[8];
#pragma unroll
  for (int m = 0; m < 16; ++m)
#pragma unroll
    for (int t = 0; t < 4; ++t) {
      acc[m][t] = f32x4{0.f, 0.f, 0.f, 0.f};
      asm volatile("" : "+a"(acc[m][t]));
    }
  asm volatile("s_waitcnt vmcnt(3)" : "+v"(prev), "+v"(warm), "+v"(raw[0]), "+v"(raw[1]), "+v"(raw[2]), "+v"(raw[3]), "+v"(raw[4]), "+v"(raw[5]), "+v"(raw[6]), "+v"(raw[7]) :: "memory");
  STAMP(11);
#pragma unroll
  for (int j = 0; j < 4; ++j) {
    const f32x4 v0 = raw[2 * j], v1 = raw[2 * j + 1];
    const u32x4 d = {pk_bf16(v0[0], v0[1]), pk_bf16(v0[2], v0[3]), pk_bf16(v1[0], v1[1]), pk_bf16(v1[2], v1[3])};
    asm volatile("buffer_store_dwordx4 %0, %1, %2, %3 offen offset:%4 sc1\n\ts_nop 1" :: "v"(d), "v"(lane16), "s"(c.xrsrc), "s"(xto), "i"(j * 1024) : "memory");
  }
  const unsigned call = __builtin_amdgcn_readfirstlane(prev[0]) + 1;
  const int half = (call > 1 && t_start - (unsigned)__builtin_amdgcn_readfirstlane(prev[1]) < BUSY_TICKS) ? 16 : 0;
  asm volatile("s_waitcnt vmcnt(0)\n\ts_barrier" : "+v"(anyws), "+v"(stp), "+v"(cpf) :: "memory");
  {
    const int s0 = (stp[0] >> half) & 0xFFFF, s1 = (stp[1] >> half) & 0xFFFF;
    c.steps = steady ? te : s0 + phase * (s1 - s0) / 256;
  }
  STAMP(12);
  if (tid == 0) __hip_atomic_store((u64*)(xflag + hs * (FS / 4)), ((u64)t_start << 32) | call, __ATOMIC_RELAXED, __HIP_MEMORY_SCOPE_AGENT);
  if (b == 0 && wave == 1 && lane < NR)
    __hip_atomic_store((GAS unsigned*)(anyws + WS_START + rank * FS), call, __ATOMIC_RELAXED, __HIP_MEMORY_SCOPE_SYSTEM);
  c.fbase = (const char*)xflag + kh * NT * FS;
  c.frsrc = make_rsrc(c.fbase, NT * FS);
  c.fvoff = (lane & 3) * FS;
  c.fso = 8 * FS;
  c.call = call;
  c.tmo = tmo;
  c.ok = true;
  if ((unsigned)(idx - PF0) < (1u << PFL)) {
    constexpr int NP = 16 >> PFL;
    const int pw = (idx - PF0) * 4 + wave;
    const unsigned pv = (unsigned)k0 * 65536u + pw * NP * 1024 + lane16;
    u32x4 pf = {0u, 0u, 0u, 0u};
#pragma unroll
    for (int j = 1; j < KH; ++j) {
      if (j == 1 || j == 5)
        c.ok &= wait_flag<__HIP_MEMORY_SCOPE_AGENT, 1>((const unsigned*)(c.fbase + ((j == 1 ? (lane & 3) + 1 : lane & 31) * 4 + (pw >> PFL)) * FS), call, tmo);
      const unsigned v = pv + j * 65536;
      asm volatile("s_waitcnt vmcnt(32)\n\tbuffer_load_dwordx4 %0, %1, %2, 0 offen sc1\n\tbuffer_load_dwordx4 %0, %1, %2, 0 offen offset:1024 sc1" : "+v"(pf) : "v"(v), "s"(c.xrsrc) : "memory");
      if (NP == 4) asm volatile("buffer_load_dwordx4 %0, %1, %2, 0 offen offset:2048 sc1\n\tbuffer_load_dwordx4 %0, %1, %2, 0 offen offset:3072 sc1" : "+v"(pf) : "v"(v), "s"(c.xrsrc) : "memory");
    }
    asm volatile("s_waitcnt vmcnt(0)" : "+v"(pf) :: "memory");
    if (cls >= 3) {
      const int u1 = unit + ((lane & 3) == 1 ? -56 : (lane & 3) == 2 ? 56 : 0), u2 = (unsigned)u1 < 2u * NEARLY ? u1 : unit;
      const size_t o = (size_t)u2 * HALF_B + wave * WAVE_B;
      const int k = (lane >> 2) & 7;
      touch(k == 0 ? dws + WS_STAGE + o : k == 1 ? (GAS char*)ws + WS_PART + o : k == 2 ? (GAS char*)ws + WS_PART + (o ^ HALF_B)
            : k == 3 ? (GAS char*)ws + WS_STAGE + o : k == 4 ? dws + WS_EFLAG + (size_t)u2 * 4 * FS + 128
            : myout + (size_t)(((rank - pd) & 7) * ML + wm * 256 + kh * 128 + (lane & 3) * 32) * NL);
    }
  }
#pragma unroll
  for (int j = 0; j < 8; ++j) w_load(wr[0][j], c.wvoff[j], c.wrsrc, wsb);
  if (wave == 0) c.ok &= wait_flag<__HIP_MEMORY_SCOPE_AGENT, 1>((const unsigned*)(c.fbase + (lane & 7) * FS), call, tmo);
  asm volatile("s_waitcnt vmcnt(0)" : "+v"(wr[0][0]), "+v"(wr[0][1]), "+v"(wr[0][2]), "+v"(wr[0][3]), "+v"(wr[0][4]), "+v"(wr[0][5]), "+v"(wr[0][6]), "+v"(wr[0][7]) :: "memory");
#pragma unroll
  for (int t = 0; t < 4; ++t) {
    a[0][t] = u32x4{pk_bf16(wr[0][0][t], wr[0][1][t]), pk_bf16(wr[0][2][t], wr[0][3][t]), pk_bf16(wr[0][4][t], wr[0][5][t]), pk_bf16(wr[0][6][t], wr[0][7][t])};
    *(LAS u32x4*)(c.wmine + t * 1024) = a[0][t];
  }
  wait_bar<0>();
  {
    const unsigned xso = (unsigned)k0 * 65536u;
    set_m0(c.m0base); asm volatile("s_nop 0");
    x_piece<0>(c, c.xrsrc, xso); x_piece<1>(c, c.xrsrc, xso); x_piece<2>(c, c.xrsrc, xso); x_piece<3>(c, c.xrsrc, xso);
    set_m0(c.m0base + 4096); asm volatile("s_nop 0");
    x_piece<4>(c, c.xrsrc, xso); x_piece<5>(c, c.xrsrc, xso); x_piece<6>(c, c.xrsrc, xso); x_piece<7>(c, c.xrsrc, xso);
    set_m0(c.m0base + 32768); asm volatile("s_nop 0");
    x_piece<8>(c, c.xrsrc, xso); x_piece<9>(c, c.xrsrc, xso); x_piece<10>(c, c.xrsrc, xso); x_piece<11>(c, c.xrsrc, xso);
    set_m0(c.m0base + 32768 + 4096); asm volatile("s_nop 0");
    x_piece<12>(c, c.xrsrc, xso); x_piece<13>(c, c.xrsrc, xso); x_piece<14>(c, c.xrsrc, xso); x_piece<15>(c, c.xrsrc, xso);
  }
#pragma unroll
  for (int j = 0; j < 8; ++j) w_load(wr[1][j], c.wvoff[j], c.wrsrc, wsb + (1u << 22));
  __builtin_amdgcn_s_waitcnt(0xC07F);
  wait_bar<8>();
  STAMP(10);
  SCHED();
  set_m0(c.m0base + 2 * 32768);
#pragma unroll
  for (int t = 0; t < 7; ++t) xf[t] = x_frag<0>(c, t);
  SCHED();
  while ((int)((unsigned)__builtin_amdgcn_s_memrealtime() + 100u - c.due) < 0) __builtin_amdgcn_s_sleep(4);
  __builtin_amdgcn_s_waitcnt(0xC07F);
  STAMP(1);

#define TRACE_PT()
  TRACE_PT()
  kstep<0, true>(c, acc, wr, a, xf);
  kstep<1, true>(c, acc, wr, a, xf);
  c.fvoff = (lane & 31) * FS;
  c.fso = wave * 32 * FS;
  kstep<2, true>(c, acc, wr, a, xf);
  kstep<3, true>(c, acc, wr, a, xf);
  kstep<0, false>(c, acc, wr, a, xf);
  kstep<1, false>(c, acc, wr, a, xf);
  u32x4 held[NH][4] = {};
  const int ltail = steady ? 1 : NH + 1;
  if (!steady) {
    c.wlim = 2 + NH;
    f32x4 raw2[2][8];
#pragma unroll
    for (int i = 0; i <= NH; ++i) {
      if (i < NH) {
#pragma unroll
        for (int j = 0; j < 8; ++j) w_load(raw2[i & 1][j], c.wvoff[j], c.wrsrc, wsb + ((unsigned)(nblk - NH + i) << 22));
      }
      if (i > 0) {
        f32x4 (&r)[8] = raw2[(i - 1) & 1];
        asm volatile("s_waitcnt vmcnt(%8)" : "+v"(r[0]), "+v"(r[1]), "+v"(r[2]), "+v"(r[3]), "+v"(r[4]), "+v"(r[5]), "+v"(r[6]), "+v"(r[7]) : "n"(i < NH ? 8 : 0) : "memory");
#pragma unroll
        for (int t = 0; t < 4; ++t) {
          held[i - 1][t] = u32x4{pk_bf16(r[0][t], r[1][t]), pk_bf16(r[2][t], r[3][t]), pk_bf16(r[4][t], r[5][t]), pk_bf16(r[6][t], r[7][t])};
          asm volatile("" : "+v"(held[i - 1][t]));
        }
      }
    }
    SCHED();
  }
  do {
    TRACE_PT()
    kstep<2, false>(c, acc, wr, a, xf);
    kstep<3, false>(c, acc, wr, a, xf);
    kstep<0, false>(c, acc, wr, a, xf);
    kstep<1, false>(c, acc, wr, a, xf);
  } while (c.left > ltail);
  while (c.left > 1) {
    TRACE_PT()
    kstep<2, false, true>(c, acc, wr, a, xf, held[0]);
    kstep<3, false, true>(c, acc, wr, a, xf, held[0]);
    kstep<0, false, true>(c, acc, wr, a, xf, held[1]);
    kstep<1, false, true>(c, acc, wr, a, xf, held[1]);
#pragma unroll
    for (int i = 0; i + 2 < NH; ++i)
#pragma unroll
      for (int t = 0; t < 4; ++t) held[i][t] = held[i + 2][t];
    asm volatile("s_nop 1");
  }
  LAS float* red = (LAS float*)(smem + LDS_X);
  const size_t gmax_off = WS_GMAX + (size_t)(call & 1) * FS;
  const size_t wave_off = (size_t)unit * HALF_B + wave * WAVE_B;
  const unsigned pov = (lane >> 3) == 3 ? lane16 + 5 * 128 : lane16;
  unsigned* pflag = (unsigned*)(ws + WS_PFLAG + wave * FS);
  last_block(c, acc, a, xf, make_rsrc(ws + WS_PART + wave_off, WAVE_B), pov);
  bool ok = c.ok;
  STAMP(2);
  STAMPC(9);
  asm volatile("s_waitcnt vmcnt(0)"
               : "+v"(wr[0][0]), "+v"(wr[0][1]), "+v"(wr[0][2]), "+v"(wr[0][3]), "+v"(wr[0][4]), "+v"(wr[0][5]), "+v"(wr[0][6]), "+v"(wr[0][7])
               :: "memory");
  if (lane == 0) __hip_atomic_store(pflag + unit * FS, call, __ATOMIC_RELAXED, __HIP_MEMORY_SCOPE_AGENT);
  ok &= wait_flag<__HIP_MEMORY_SCOPE_SYSTEM, 2>((lane & 1) ? (const unsigned*)(ws + WS_START + dest * FS) : pflag + (unit ^ 1) * FS, call, tmo);
  const i32x4 srs = make_rsrc((const void*)(dws + WS_STAGE + wave_off), ok ? WAVE_B : 0);
  f32x4 y[8][4];
  {
    const i32x4 prs = make_rsrc(ws + WS_PART + (wave_off ^ HALF_B), WAVE_B);
    u32x4 pv[16];
#pragma unroll
    for (int j = 0; j < 16; ++j)
      asm volatile("buffer_load_dwordx4 %0, %1, %2, %3 offen offset:%4 sc1" : "=v"(pv[j]) : "v"(pov), "s"(prs), "s"(j * PC & ~4095), "i"(j * PC & 4095) : "memory");
    const unsigned one = (i16 >> 2) == q ? ((i16 & 1) ? 0x3F800000u : 0x00003F80u) : 0u;
    const unsigned e0 = (i16 & 2) ? 0u : one, e1 = (i16 & 2) ? one : 0u;
    u32x4 sel0 = {e0, e1, 0u, 0u}, sel1 = {0u, 0u, e0, e1};
    asm volatile("s_nop 1" : "+v"(sel0), "+v"(sel1));
    float am = 0.f;
#pragma unroll
    for (int j = 0; j < 16 + 4; ++j) {
      if (j < 16) {
        asm volatile("s_waitcnt vmcnt(%1)" : "+v"(pv[j]) : "n"(15 - j) : "memory");
        mfma(acc[8 + (j >> 1)][(j & 1) * 2], sel0, pv[j]);
        mfma(acc[8 + (j >> 1)][(j & 1) * 2 + 1], sel1, pv[j]);
      }
      if (j == 16) asm volatile("s_nop 7\n\ts_nop 7" ::: "memory");
      if (j >= 4) {
        const int m = (j - 4) >> 1, u = (j - 4) & 1;
        asm volatile("" : "+a"(acc[8 + m][2 * u]), "+a"(acc[8 + m][2 * u + 1]), "+v"(am));
        y[m][2 * u] = acc[8 + m][2 * u]; y[m][2 * u + 1] = acc[8 + m][2 * u + 1];
        asm volatile("" : "+v"(y[m][2 * u]), "+v"(y[m][2 * u + 1]));
        const f32x4 lo = y[m][2 * u], hi = y[m][2 * u + 1];
        am = fmaxf(fmaxf(am, fabsf(lo[0])), fabsf(lo[1])); am = fmaxf(fmaxf(am, fabsf(lo[2])), fabsf(lo[3]));
        am = fmaxf(fmaxf(am, fabsf(hi[0])), fabsf(hi[1])); am = fmaxf(fmaxf(am, fabsf(hi[2])), fabsf(hi[3]));
        asm volatile("" : "+v"(am));
      }
    }
    if (early) {
#pragma unroll
      for (int j = 0; j < SEND_WIN; ++j) send_piece(y, j, srs, pov);
    }
    am = wave_max(am);
    if (lane == 0) red[wave] = ok ? am : -1.0f;
    __syncthreads();
    if (tid == 0) {
      const float r0 = red[0], r1 = red[1], r2 = red[2], r3 = red[3];
      const float wam = fminf(fminf(r0, r1), fminf(r2, r3)) < 0.f ? -1.0f : fmaxf(fmaxf(r0, r1), fmaxf(r2, r3));
      __hip_atomic_store((u64*)(ws + WS_AMAX + (unit >> 4) * FS) + (unit & 15), ((u64)call << 32) | __builtin_bit_cast(unsigned, wam), __ATOMIC_RELAXED, __HIP_MEMORY_SCOPE_AGENT);
    }
  }
  STAMP(3);
  const int row0 = wm * 256 + kh * 128 + i16, col0 = (q < 2 ? ln0 : ln1) * 32 + (q & 1) * 16;
  const int srank = (rank - pd) & 7;

  if (early) {
#pragma unroll
    for (int j = SEND_WIN; j < 16; ++j) {
      if (wide > 0) asm volatile("s_waitcnt vmcnt(%0)" :: "n"(2 * SEND_WIN - 1) : "memory");
      else asm volatile("s_waitcnt vmcnt(%0)" :: "n"(SEND_WIN - 1) : "memory");
      send_piece(y, j, srs, pov);
    }
    asm volatile("s_waitcnt vmcnt(0)" ::: "memory");
    if (lane == 0)
      __hip_atomic_store((GAS unsigned*)(dws + WS_EFLAG + (size_t)(unit * 4 + wave) * FS), call, __ATOMIC_RELAXED, __HIP_MEMORY_SCOPE_SYSTEM);
    STAMP(5);
    ok &= wait_flag<__HIP_MEMORY_SCOPE_SYSTEM, 8>((const unsigned*)(ws + WS_EFLAG + (size_t)(unit * 4 + wave) * FS), call, tmo);
    u32x4 st[16];
    {
      const i32x4 rs = make_rsrc(ws + WS_STAGE + wave_off, WAVE_B);
#pragma unroll
      for (int j = 0; j < 16; ++j)
        asm volatile("buffer_load_dwordx4 %0, %1, %2, %3 offen offset:%4 sc0 sc1" : "=v"(st[j]) : "v"(pov), "s"(rs), "s"(j * PC & ~4095), "i"(j * PC & 4095) : "memory");
      asm volatile("s_waitcnt vmcnt(0)" ::: "memory");
#pragma unroll
      for (int j = 0; j < 16; ++j) asm volatile("" : "+v"(st[j]));
    }
    STAMP(6);
    const Quant qz = global_scale<32>((const u64*)(ws + gmax_off), call, tmo, ok, lane);
    STAMP(4);
    GAS char* ob = dout + (size_t)(srank * ML + row0) * NL + col0;
#pragma unroll
    for (int m = 0; m < 8; ++m) {
      const u32x4 s0 = st[2 * m], s1 = st[2 * m + 1];
      u32x4 o;
#pragma unroll
      for (int r = 0; r < 4; ++r) {
        const int h = r >> 1;
        const unsigned w0 = s0[h], w1 = s0[2 + h], w2 = s1[h], w3 = s1[2 + h];
        const float v0 = __builtin_bit_cast(float, (r & 1) ? (w0 & 0xFFFF0000u) : (w0 << 16));
        const float v1 = __builtin_bit_cast(float, (r & 1) ? (w1 & 0xFFFF0000u) : (w1 << 16));
        const float v2 = __builtin_bit_cast(float, (r & 1) ? (w2 & 0xFFFF0000u) : (w2 << 16));
        const float v3 = __builtin_bit_cast(float, (r & 1) ? (w3 & 0xFFFF0000u) : (w3 << 16));
        o[r] = quant4(v0, v1, v2, v3, qz);
      }
      asm volatile("global_store_dwordx4 %0, %1, off sc0 sc1\n\ts_nop 1" :: "v"(ob + (size_t)m * 16 * NL), "v"(o) : "memory");
    }
    asm volatile(".Lq8_code_end1:");
    STAMP(7);
    STAMP_OUT();
    return;
  }

  if (unit == NWG - 1 && wave >= 2) {
    const float rm = sweep_max<4, 4 * FS / 8, __HIP_MEMORY_SCOPE_AGENT, 1>((const u64*)(ws + WS_AMAX + (lane >> 4) * FS) + (lane & 15), call, tmo);
    if (lane < NR)
      __hip_atomic_store((GAS u64*)(anyws + gmax_off) + rank, ((u64)call << 32) | __builtin_bit_cast(unsigned, rm), __ATOMIC_RELAXED, __HIP_MEMORY_SCOPE_SYSTEM);
  }
  const Quant qz = global_scale<4>((const u64*)(ws + gmax_off), call, tmo, ok, lane);
  STAMP(4);
  {
    GAS char* ob = dout + (size_t)(rank * ML + row0) * NL + col0;
#pragma unroll
    for (int m = 0; m < 8; ++m) {
      u32x4 o;
#pragma unroll
      for (int r = 0; r < 4; ++r) o[r] = quant4(y[m][0][r], y[m][1][r], y[m][2][r], y[m][3][r], qz);
      asm volatile("s_waitcnt vmcnt(%2)\n\tglobal_store_dwordx4 %0, %1, off sc0 sc1\n\ts_nop 1" :: "v"(ob + (size_t)m * 16 * NL), "v"(o), "n"(LATE_WIN - 1) : "memory");
    }
  }
  asm volatile("s_waitcnt vmcnt(0)\n.Lq8_code_end2:" ::: "memory");
  STAMP(5);
  if (lane == 0)
    __hip_atomic_store((GAS unsigned*)(dws + WS_EFLAG + (size_t)(unit * 4 + wave) * FS), call, __ATOMIC_RELAXED, __HIP_MEMORY_SCOPE_SYSTEM);
  if (!wait_flag<__HIP_MEMORY_SCOPE_SYSTEM, 2>((const unsigned*)(ws + WS_EFLAG + (size_t)(unit * 4 + wave) * FS), call, tmo) && lane == 0)
    myout[(size_t)rank * ML * NL] = (char)-128;
  STAMP(7);
  STAMP_OUT();
}

static void make_pace(Params& p, const int* pa) {
  const int TE = pa[4], TS = pa[5], A = pa[7], DS = pa[8];
  p.slack = pa[9];
  p.nwide = pa[10]; p.wmask = pa[14];
  for (int k = 0; k < NE; ++k) {
    p.te[k] = k < NE - 3 ? TE : k == NE - 3 ? TS : pa[13];
    p.start[k] = A + (k * DS > pa[11] ? k * DS - pa[11] : 0) - KH * (p.te[k] - TE);
    if (p.start[k] < A) p.start[k] = A;
  }
  for (int k = 0; k < pa[10]; ++k) p.te[k] = pa[12];
  for (int busy = 1; busy >= 0; --busy) {
    const int* e = busy ? pa + 17 : pa;
    const double T0 = e[0], TH = e[1], T1 = e[2], TR = e[3], TC = pa[6], C = pa[15] * 0.01;
    double P = 0, last = 0;
    int v = 0, stall = 0;
    for (int u = A - 100; v <= KH + 1; ++u) {
      double r = NWG / (u < TH ? T0 : u < TR ? T0 + (T1 - T0) * (u - TH) / (TR - TH) : T1);
      for (int k = 0; k < NE; ++k) {
        const double s0 = p.start[k] - 100 + (k < pa[10] ? 0 : p.te[k] / 2);
        if (u >= s0 && u < s0 + KH * p.te[k]) r -= (k == NE - 1 ? 20.0 : 14.0) / p.te[k];
      }
      r /= NWG - 2 * NEARLY - 6;
      if (r > 1.0 / TC) r = 1.0 / TC;
      if (r < 0.001) r = 0.001;
      if (P >= 3.0 && stall < pa[16]) { ++stall; r = 0; }
      P += r;
      if (P >= v + C) {
        if (v) p.step[v - 1] = busy ? (int)(u - last) << 16 : p.step[v - 1] | (int)(u - last); else p.start[NE] = u;
        last = u; ++v;
      }
    }
  }
}

extern "C" void kernel_launch(int rank, int nranks, ncclComm_t comm,
                              hipStream_t stream,
                              void* const* d_in, const int* in_sizes, int n_in,
                              void* d_out, int out_size,
                              const int* in_dtypes, int out_dtype,
                              void* const* d_out_all,
                              void* const* d_ws_all, size_t ws_bytes) {
  (void)comm; (void)in_dtypes; (void)out_dtype; (void)d_out;
  if (nranks != NR || n_in != 2 || in_sizes[0] != ML * KD || in_sizes[1] != KD * ND || out_size != NR * ML * NL ||
      ws_bytes < WS_END || (unsigned)rank >= (unsigned)NR) {
    fprintf(stderr, "unexpected problem shape\n");
    return;
  }
  int dev = 0, cus = 0;
  HIP_CHECK(hipGetDevice(&dev));
  HIP_CHECK(hipDeviceGetAttribute(&cus, hipDeviceAttributeMultiprocessorCount, dev));
  if (cus < NWG) {
    fprintf(stderr, "needs %d CUs: the workgroups wait for one another\n", NWG);
    return;
  }
  HIP_CHECK(hipFuncSetAttribute((const void*)fused_kernel, hipFuncAttributeMaxDynamicSharedMemorySize, LDS_BYTES));
  Params p;
  p.x = (const float*)d_in[0];
  p.w = (const float*)d_in[1];
  for (int r = 0; r < NR; ++r) { p.out[r] = (char*)d_out_all[r]; p.ws[r] = (char*)d_ws_all[r]; }
  p.rank = rank;
  p.myws = (char*)d_ws_all[rank];
  int pa[21] = {PACE_ARGS};
  make_pace(p, pa);
  fused_kernel<<<NWG, NTHR, LDS_BYTES, stream>>>(p);
  HIP_CHECK(hipGetLastError());
}
```
